```python
import numpy as np

kernel_name = "gpu_gemm_a2a_m4096_k4096_n16384_bf16_reluqe4m3"


ABSMAX_RTOL = 0.09

COMPUTE_DTYPE = "bfloat16"
EPILOGUE = "relu_quant_e4m3"
IN_QUANT = None
OUT_QUANT = "e4m3"
IN_DTYPES = {"x": "bfloat16", "w_mat": "bfloat16"}
WORLD_SIZE = 8

def _to_bf16(x):
    u = np.ascontiguousarray(x, dtype=np.float32).view(np.uint32)
    return ((u + 0x7FFF + ((u >> 16) & 1)) & 0xFFFF0000).view(np.float32)

def setup_inputs():

    x = _to_bf16(np.random.randn(4096, 4096).astype(np.float32))
    w_mat = _to_bf16((np.random.randn(4096, 16384) / np.sqrt(4096)).astype(np.float32))
    return {"x": x, "w_mat": w_mat}

def _snap_fp8(x, mant_bits, max_abs, min_normal):
    a = np.abs(x.astype(np.float64))
    _m, _e = np.frexp(a)
    q = np.where(a >= min_normal, np.ldexp(1.0, _e - 1 - mant_bits),
                 min_normal * 2.0 ** -mant_bits)
    snapped = np.minimum(np.round(a / q) * q, max_abs)
    return (np.sign(x) * snapped).astype(np.float32)
def _quant_dequant(y, amax):
    scale = amax / 448.0
    q = _snap_fp8(y / scale, 3, 448.0, 2.0 ** -6)
    return (q * scale).astype(np.float32)
def _epilogue(y):
    y = np.maximum(y, 0.0)
    return _quant_dequant(y, np.abs(y).max())

def reference(x, w_mat):
    y = (x @ w_mat).astype(np.float32)
    return _epilogue(y)
```

```cpp
DI void issue_a1(u32x4 rs, u32 v0, u32 v1, u32 mb, u32 sb) {
    u32 t;
    asm volatile(
        DMA(v0, 0, 0x0) DMA(v1, 1024, 0x10000) DMA(v0, 2048, 0x20000) DMA(v1, 3072, 0x30000)
        : [t] "=&s"(t) : [v0] "v"(v0), [v1] "v"(v1), [rs] "s"(rs), [mb] "s"(mb), [sb] "s"(sb) : "memory", "scc");
}
DI void issue_a2(u32x4 rs, u32 v0, u32 v1, u32 mb, u32 sb) {
    u32 t;
    asm volatile(
        DMA(v0, 4096, 0x40000) DMA(v1, 5120, 0x50000) DMA(v0, 6144, 0x60000) DMA(v1, 7168, 0x70000) DMA(v0, 8192, 0x80000) DMA(v1, 9216, 0x90000) DMA(v0, 10240, 0xa0000) DMA(v1, 11264, 0xb0000) DMA(v0, 12288, 0xc0000) DMA(v1, 13312, 0xd0000) DMA(v0, 14336, 0xe0000) DMA(v1, 15360, 0xf0000)
        : [t] "=&s"(t) : [v0] "v"(v0), [v1] "v"(v1), [rs] "s"(rs), [mb] "s"(mb), [sb] "s"(sb) : "memory", "scc");
}
DI void issue_b1(u32x4 rs, u32 v0, u32 v1, u32 mb, u32 sb) {
    u32 t;
    asm volatile(
        DMA(v0, 0, 0x0) DMA(v0, 2048, 0x20000) DMA(v1, 4096, 0x40000) DMA(v1, 6144, 0x60000)
        : [t] "=&s"(t) : [v0] "v"(v0), [v1] "v"(v1), [rs] "s"(rs), [mb] "s"(mb), [sb] "s"(sb) : "memory", "scc");
}
DI void issue_b2(u32x4 rs, u32 v0, u32 v1, u32 mb, u32 sb) {
    u32 t;
    asm volatile(
        DMA(v0, 8192, 0x80000) DMA(v0, 10240, 0xa0000) DMA(v1, 12288, 0xc0000) DMA(v1, 14336, 0xe0000) DMA(v0, 16384, 0x100000) DMA(v0, 18432, 0x120000) DMA(v1, 20480, 0x140000) DMA(v1, 22528, 0x160000) DMA(v0, 24576, 0x180000) DMA(v0, 26624, 0x1a0000) DMA(v1, 28672, 0x1c0000) DMA(v1, 30720, 0x1e0000)
        : [t] "=&s"(t) : [v0] "v"(v0), [v1] "v"(v1), [rs] "s"(rs), [mb] "s"(mb), [sb] "s"(sb) : "memory", "scc");
}

_m4096_k4096_n16384_bf16_reluqe4m3 — fused GEMM + ReLU + global amax + e4m3 quantize + all-to-all, one kernel.
#include <hip/hip_runtime.h>
#include <rccl/rccl.h>
#include <cstdio>
#include <cstdint>

#define HIP_CHECK(expr) do { hipError_t _e = (expr); if (_e != hipSuccess) { \
    fprintf(stderr, "HIP error %s at %s:%d\n", hipGetErrorString(_e), __FILE__, __LINE__); \
    return; } } while (0)

typedef unsigned u32;
typedef unsigned long long u64;
typedef float f32x16 __attribute__((ext_vector_type(16)));
typedef u32 u32x4 __attribute__((ext_vector_type(4)));
#define LAS __attribute__((address_space(3)))
#define GAS __attribute__((address_space(1)))
#define DI __device__ __forceinline__

constexpr int NR = 8, ML = 512, KD = 4096, ND = 16384, NC = 2048;
constexpr int BM = 128, BN = 256, NT = 64;
constexpr int NWG = 256;
constexpr u32 A_BASE = 0, A_ST = 16384, B_BASE = 49152, B_ST = 32768;
constexpr u32 F_FULL = 147456, F_DONE = F_FULL + 1024, F_MISC = F_DONE + 1024, LDS_BYTES = F_MISC + 1024;
constexpr u32 EP_PITCH = 272;
constexpr size_t WS_CALL = 0;
constexpr size_t WS_TMO = 4096;
constexpr size_t WS_LMAX = 8192;
constexpr size_t WS_GMAX = 12288;
constexpr size_t WS_DONE = 16384;
constexpr size_t WS_DBG = 65536;
constexpr size_t WS_NEED = WS_DBG + 256 * 64 * 8;
constexpr u64 DEADLINE = 50000000ull;

struct Params {
    const void* x; const void* w;
    unsigned char* out[NR];
    unsigned char* ws[NR];
    int rank; int pad;
};

DI u32 rfl(u32 v) { return __builtin_amdgcn_readfirstlane(v); }
DI u32x4 make_rsrc(const void* p, u32 bytes) {
    u64 a = (u64)p; u32x4 r; r.x = rfl((u32)a); r.y = rfl((u32)(a >> 32)); r.z = bytes; r.w = 0x00020000u; return r;
}
DI void lds_store(u32 addr, u32 v) { asm volatile("ds_write_b32 %0, %1" :: "v"(addr), "v"(v) : "memory"); }
DI u32 lds_min4(u32 addr) {
    u32x4 d; asm volatile("ds_read_b128 %0, %1\n\ts_waitcnt lgkmcnt(0)" : "=&v"(d) : "v"(addr) : "memory");
    return rfl(min(min(d.x, d.y), min(d.z, d.w)));
}
#define BAR() do { asm volatile("s_waitcnt lgkmcnt(0)\n\ts_barrier" ::: "memory"); } while (0)
#define NSTAMP 8
#define CPRIO 2
#define PPRIO 0
#define PSLEEP 1
#define STAMP(k) do {} while (0)

#define DMA(voff, LOFF, GOFF) \
    "s_add_u32 m0, %[mb], " #LOFF "\n\ts_add_u32 %[t], %[sb], " #GOFF "\n\tbuffer_load_dwordx4 %[" #voff "], %[rs], %[t] offen lds\n\t"

DI void issue_a(u32x4 rs, u32 v0, u32 v1, u32 mb, u32 sb) {
    u32 t;
    asm volatile(
        DMA(v0, 0, 0x00000) DMA(v1, 1024, 0x10000) DMA(v0, 2048, 0x20000) DMA(v1, 3072, 0x30000)
        DMA(v0, 4096, 0x40000) DMA(v1, 5120, 0x50000) DMA(v0, 6144, 0x60000) DMA(v1, 7168, 0x70000)
        DMA(v0, 8192, 0x80000) DMA(v1, 9216, 0x90000) DMA(v0, 10240, 0xa0000) DMA(v1, 11264, 0xb0000)
        DMA(v0, 12288, 0xc0000) DMA(v1, 13312, 0xd0000) DMA(v0, 14336, 0xe0000) DMA(v1, 15360, 0xf0000)
        : [t] "=&s"(t) : [v0] "v"(v0), [v1] "v"(v1), [rs] "s"(rs), [mb] "s"(mb), [sb] "s"(sb) : "memory", "scc");
}
DI void issue_b(u32x4 rs, u32 v0, u32 v1, u32 mb, u32 sb) {
    u32 t;
    asm volatile(
        DMA(v0, 0, 0x000000) DMA(v0, 2048, 0x020000) DMA(v1, 4096, 0x040000) DMA(v1, 6144, 0x060000)
        DMA(v0, 8192, 0x080000) DMA(v0, 10240, 0x0a0000) DMA(v1, 12288, 0x0c0000) DMA(v1, 14336, 0x0e0000)
        DMA(v0, 16384, 0x100000) DMA(v0, 18432, 0x120000) DMA(v1, 20480, 0x140000) DMA(v1, 22528, 0x160000)
        DMA(v0, 24576, 0x180000) DMA(v0, 26624, 0x1a0000) DMA(v1, 28672, 0x1c0000) DMA(v1, 30720, 0x1e0000)
        : [t] "=&s"(t) : [v0] "v"(v0), [v1] "v"(v1), [rs] "s"(rs), [mb] "s"(mb), [sb] "s"(sb) : "memory", "scc");
}

DI u64 ld_sys64(const unsigned char* p) { return __hip_atomic_load((const GAS u64*)p, __ATOMIC_RELAXED, __HIP_MEMORY_SCOPE_SYSTEM); }
DI void st_sys64(unsigned char* p, u64 v) { __hip_atomic_store((GAS u64*)p, v, __ATOMIC_RELAXED, __HIP_MEMORY_SCOPE_SYSTEM); }
DI u32 ld_sys32(const unsigned char* p) { return __hip_atomic_load((const GAS u32*)p, __ATOMIC_RELAXED, __HIP_MEMORY_SCOPE_SYSTEM); }
DI void st_sys32(unsigned char* p, u32 v) { __hip_atomic_store((GAS u32*)p, v, __ATOMIC_RELAXED, __HIP_MEMORY_SCOPE_SYSTEM); }
DI u32 ld_agt32(const unsigned char* p) { return __hip_atomic_load((const GAS u32*)p, __ATOMIC_RELAXED, __HIP_MEMORY_SCOPE_AGENT); }
DI void st_agt32(unsigned char* p, u32 v) { __hip_atomic_store((GAS u32*)p, v, __ATOMIC_RELAXED, __HIP_MEMORY_SCOPE_AGENT); }
DI u32 wave_umax(u32 v) {
#pragma unroll
    for (int o = 32; o >= 1; o >>= 1) v = max(v, (u32)__shfl_xor((int)v, o));
    return v;
}

DI u32 sweep(const unsigned char* g, int n, u32 call, unsigned char* tmo, int lane, bool& bad) {
    const u64 t0 = __builtin_amdgcn_s_memrealtime();
    const unsigned char* q = g + (lane < n ? lane : 0) * 8;
    for (;;) {
        const u64 v = ld_sys64(q);
        const u32 t = ld_agt32(tmo);
        if (t == call) { bad = true; return 0; }
        if (__all((u32)(v >> 32) == call)) return wave_umax((u32)v);
        if (__builtin_amdgcn_s_memrealtime() - t0 > DEADLINE) { if (lane == 0) st_agt32(tmo, call); bad = true; return 0; }
        __builtin_amdgcn_s_sleep(1);
    }
}

__global__ __launch_bounds__(512, 2) void fused_kernel(Params p) {
    extern __shared__ __attribute__((aligned(16))) unsigned char smem[];
    const int wid = rfl(threadIdx.x >> 6);
    const int b = blockIdx.x, grp = b & 7, idx = b >> 3;
    const int rowblk = idx & 3, ct = idx >> 2;
    const u32 lds0 = (u32)(uintptr_t)(LAS unsigned char*)smem;
    unsigned char* const myws = p.ws[p.rank];
    if (wid == 0) STAMP(0);

    if (wid < 4) {
        f32x16 c0, c1, c2, c3, c4, c5, c6, c7; u32 slowc; u64 tc0 = 0;
        {
            const u32 lane = threadIdx.x & 63, r = lane & 15, g = lane >> 4, q = r >> 2, pp = lane & 3;
            const u32 a0 = lds0 + A_BASE + r * 128 + ((g ^ (r >> 1)) << 4), a1 = a0 ^ 64;
            const u32 bb = lds0 + B_BASE + (8 * g + q) * 512 + (pp & 1) * 8, F = 2 * q + 8 * (g & 1), cb = 8 * wid + (pp >> 1);
            const u32 b0 = bb + (((cb + 0) ^ F) << 4), b1 = bb + (((cb + 2) ^ F) << 4), b2 = bb + (((cb + 4) ^ F) << 4), b3 = bb + (((cb + 6) ^ F) << 4);
            const u32 fa = lds0 + F_FULL, da = lds0 + F_DONE + lane * 16 + wid * 4;
            u32 si, sst, sneed, sdA, sdB;
            __builtin_amdgcn_s_setprio(CPRIO);
            BAR();
            asm volatile(
      "v_mov_b32 v16, %[a0]\n"
      "v_mov_b32 v17, %[a1]\n"
      "v_mov_b32 v18, %[b0]\n"
      "v_mov_b32 v19, %[b1]\n"
      "v_mov_b32 v20, %[b2]\n"
      "v_mov_b32 v21, %[b3]\n"
      "v_mov_b32 v30, %[fa]\n"
      "v_mov_b32 v31, %[da]\n"
      "v_mov_b32 v29, 0\n"
      "v_mov_b32 v128, 0\n"
      "v_mov_b32 v129, 0\n"
      "v_mov_b32 v130, 0\n"
      "v_mov_b32 v131, 0\n"
      "v_mov_b32 v132, 0\n"
      "v_mov_b32 v133, 0\n"
      "v_mov_b32 v134, 0\n"
      "v_mov_b32 v135, 0\n"
      "v_mov_b32 v136, 0\n"
      "v_mov_b32 v137, 0\n"
      "v_mov_b32 v138, 0\n"
      "v_mov_b32 v139, 0\n"
      "v_mov_b32 v140, 0\n"
      "v_mov_b32 v141, 0\n"
      "v_mov_b32 v142, 0\n"
      "v_mov_b32 v143, 0\n"
      "v_mov_b32 v144, 0\n"
      "v_mov_b32 v145, 0\n"
      "v_mov_b32 v146, 0\n"
      "v_mov_b32 v147, 0\n"
      "v_mov_b32 v148, 0\n"
      "v_mov_b32 v149, 0\n"
      "v_mov_b32 v150, 0\n"
      "v_mov_b32 v151, 0\n"
      "v_mov_b32 v152, 0\n"
      "v_mov_b32 v153, 0\n"
      "v_mov_b32 v154, 0\n"
      "v_mov_b32 v155, 0\n"
      "v_mov_b32 v156, 0\n"
      "v_mov_b32 v157, 0\n"
      "v_mov_b32 v158, 0\n"
      "v_mov_b32 v159, 0\n"
      "v_mov_b32 v160, 0\n"
      "v_mov_b32 v161, 0\n"
      "v_mov_b32 v162, 0\n"
      "v_mov_b32 v163, 0\n"
      "v_mov_b32 v164, 0\n"
      "v_mov_b32 v165, 0\n"
      "v_mov_b32 v166, 0\n"
      "v_mov_b32 v167, 0\n"
      "v_mov_b32 v168, 0\n"
      "v_mov_b32 v169, 0\n"
      "v_mov_b32 v170, 0\n"
      "v_mov_b32 v171, 0\n"
      "v_mov_b32 v172, 0\n"
      "v_mov_b32 v173, 0\n"
      "v_mov_b32 v174, 0\n"
      "v_mov_b32 v175, 0\n"
      "v_mov_b32 v176, 0\n"
      "v_mov_b32 v177, 0\n"
      "v_mov_b32 v178, 0\n"
      "v_mov_b32 v179, 0\n"
      "v_mov_b32 v180, 0\n"
      "v_mov_b32 v181, 0\n"
      "v_mov_b32 v182, 0\n"
      "v_mov_b32 v183, 0\n"
      "v_mov_b32 v184, 0\n"
      "v_mov_b32 v185, 0\n"
      "v_mov_b32 v186, 0\n"
      "v_mov_b32 v187, 0\n"
      "v_mov_b32 v188, 0\n"
      "v_mov_b32 v189, 0\n"
      "v_mov_b32 v190, 0\n"
      "v_mov_b32 v191, 0\n"
      "v_mov_b32 v192, 0\n"
      "v_mov_b32 v193, 0\n"
      "v_mov_b32 v194, 0\n"
      "v_mov_b32 v195, 0\n"
      "v_mov_b32 v196, 0\n"
      "v_mov_b32 v197, 0\n"
      "v_mov_b32 v198, 0\n"
      "v_mov_b32 v199, 0\n"
      "v_mov_b32 v200, 0\n"
      "v_mov_b32 v201, 0\n"
      "v_mov_b32 v202, 0\n"
      "v_mov_b32 v203, 0\n"
      "v_mov_b32 v204, 0\n"
      "v_mov_b32 v205, 0\n"
      "v_mov_b32 v206, 0\n"
      "v_mov_b32 v207, 0\n"
      "v_mov_b32 v208, 0\n"
      "v_mov_b32 v209, 0\n"
      "v_mov_b32 v210, 0\n"
      "v_mov_b32 v211, 0\n"
      "v_mov_b32 v212, 0\n"
      "v_mov_b32 v213, 0\n"
      "v_mov_b32 v214, 0\n"
      "v_mov_b32 v215, 0\n"
      "v_mov_b32 v216, 0\n"
      "v_mov_b32 v217, 0\n"
      "v_mov_b32 v218, 0\n"
      "v_mov_b32 v219, 0\n"
      "v_mov_b32 v220, 0\n"
      "v_mov_b32 v221, 0\n"
      "v_mov_b32 v222, 0\n"
      "v_mov_b32 v223, 0\n"
      "v_mov_b32 v224, 0\n"
      "v_mov_b32 v225, 0\n"
      "v_mov_b32 v226, 0\n"
      "v_mov_b32 v227, 0\n"
      "v_mov_b32 v228, 0\n"
      "v_mov_b32 v229, 0\n"
      "v_mov_b32 v230, 0\n"
      "v_mov_b32 v231, 0\n"
      "v_mov_b32 v232, 0\n"
      "v_mov_b32 v233, 0\n"
      "v_mov_b32 v234, 0\n"
      "v_mov_b32 v235, 0\n"
      "v_mov_b32 v236, 0\n"
      "v_mov_b32 v237, 0\n"
      "v_mov_b32 v238, 0\n"
      "v_mov_b32 v239, 0\n"
      "v_mov_b32 v240, 0\n"
      "v_mov_b32 v241, 0\n"
      "v_mov_b32 v242, 0\n"
      "v_mov_b32 v243, 0\n"
      "v_mov_b32 v244, 0\n"
      "v_mov_b32 v245, 0\n"
      "v_mov_b32 v246, 0\n"
      "v_mov_b32 v247, 0\n"
      "v_mov_b32 v248, 0\n"
      "v_mov_b32 v249, 0\n"
      "v_mov_b32 v250, 0\n"
      "v_mov_b32 v251, 0\n"
      "v_mov_b32 v252, 0\n"
      "v_mov_b32 v253, 0\n"
      "v_mov_b32 v254, 0\n"
      "v_mov_b32 v255, 0\n"
      "s_mov_b32 %[i], 0\n"
      "s_mov_b32 %[st], 0\n"
      "s_mov_b32 %[sc], 0\n"
      "L_w0_%=:\n"
      "ds_read_b128 v[24:27], v30\n"
      "s_waitcnt lgkmcnt(0)\n"
      "v_min3_u32 v28, v24, v25, v26\n"
      "v_cmp_gt_u32 vcc, 1, v28\n"
      "s_cbranch_vccnz L_w0_%=\n"
      "ds_read_b64_tr_b16 v[64:65], v18 offset:0\n"
      "ds_read_b64_tr_b16 v[66:67], v18 offset:2048\n"
      "ds_read_b64_tr_b16 v[68:69], v19 offset:0\n"
      "ds_read_b64_tr_b16 v[70:71], v19 offset:2048\n"
      "ds_read_b64_tr_b16 v[72:73], v20 offset:0\n"
      "ds_read_b64_tr_b16 v[74:75], v20 offset:2048\n"
      "ds_read_b64_tr_b16 v[76:77], v21 offset:0\n"
      "ds_read_b64_tr_b16 v[78:79], v21 offset:2048\n"
      "ds_read_b128 v[32:35], v16 offset:0\n"
      "ds_read_b128 v[36:39], v16 offset:2048\n"
      "ds_read_b128 v[40:43], v16 offset:4096\n"
      "ds_read_b128 v[44:47], v16 offset:6144\n"
      "ds_read_b128 v[48:51], v16 offset:8192\n"
      "ds_read_b128 v[52:55], v16 offset:10240\n"
      "ds_read_b128 v[56:59], v16 offset:12288\n"
      "ds_read_b128 v[60:63], v16 offset:14336\n"
      "s_waitcnt lgkmcnt(0)\n"
      "L_loop_%=:\n"
      "v_mfma_f32_16x16x32_bf16 v[128:131], v[64:67], v[32:35], v[128:131]\n"
      "ds_read_b64_tr_b16 v[112:113], v18 offset:16384\n"
      "s_add_u32 %[need], %[i], 2\n"
      "v_mfma_f32_16x16x32_bf16 v[132:135], v[68:71], v[32:35], v[132:135]\n"
      "ds_read_b64_tr_b16 v[114:115], v18 offset:18432\n"
      "s_min_u32 %[need], %[need], 64\n"
      "v_mfma_f32_16x16x32_bf16 v[136:139], v[72:75], v[32:35], v[136:139]\n"
      "ds_read_b64_tr_b16 v[116:117], v19 offset:16384\n"
      "s_cmp_eq_u32 %[st], 2\n"
      "v_mfma_f32_16x16x32_bf16 v[140:143], v[76:79], v[32:35], v[140:143]\n"
      "ds_read_b64_tr_b16 v[118:119], v19 offset:18432\n"
      "s_cselect_b32 %[dA], %[cm], %[cp]\n"
      "v_mfma_f32_16x16x32_bf16 v[144:147], v[64:67], v[36:39], v[144:147]\n"
      "ds_read_b64_tr_b16 v[120:121], v20 offset:16384\n"
      "s_cselect_b32 %[st], -1, %[st]\n"
      "v_mfma_f32_16x16x32_bf16 v[148:151], v[68:71], v[36:39], v[148:151]\n"
      "ds_read_b64_tr_b16 v[122:123], v20 offset:18432\n"
      "s_add_u32 %[st], %[st], 1\n"
      "v_mfma_f32_16x16x32_bf16 v[152:155], v[72:75], v[36:39], v[152:155]\n"
      "ds_read_b64_tr_b16 v[124:125], v21 offset:16384\n"
      "s_lshl_b32 %[dB], %[dA], 1\n"
      "v_mfma_f32_16x16x32_bf16 v[156:159], v[76:79], v[36:39], v[156:159]\n"
      "ds_read_b64_tr_b16 v[126:127], v21 offset:18432\n"
      "v_mfma_f32_16x16x32_bf16 v[160:163], v[64:67], v[40:43], v[160:163]\n"
      "ds_read_b128 v[80:83], v17 offset:0\n"
      "v_mfma_f32_16x16x32_bf16 v[164:167], v[68:71], v[40:43], v[164:167]\n"
      "ds_read_b128 v[84:87], v17 offset:2048\n"
      "v_mfma_f32_16x16x32_bf16 v[168:171], v[72:75], v[40:43], v[168:171]\n"
      "ds_read_b128 v[88:91], v17 offset:4096\n"
      "v_mfma_f32_16x16x32_bf16 v[172:175], v[76:79], v[40:43], v[172:175]\n"
      "ds_read_b128 v[92:95], v17 offset:6144\n"
      "v_mfma_f32_16x16x32_bf16 v[176:179], v[64:67], v[44:47], v[176:179]\n"
      "ds_read_b128 v[96:99], v17 offset:8192\n"
      "v_mfma_f32_16x16x32_bf16 v[180:183], v[68:71], v[44:47], v[180:183]\n"
      "ds_read_b128 v[100:103], v17 offset:10240\n"
      "v_mfma_f32_16x16x32_bf16 v[184:187], v[72:75], v[44:47], v[184:187]\n"
      "ds_read_b128 v[104:107], v17 offset:12288\n"
      "v_mfma_f32_16x16x32_bf16 v[188:191], v[76:79], v[44:47], v[188:191]\n"
      "ds_read_b128 v[108:111], v17 offset:14336\n"
      "v_mfma_f32_16x16x32_bf16 v[192:195], v[64:67], v[48:51], v[192:195]\n"
      "v_add_u32 v29, 1, v29\n"
      "v_mfma_f32_16x16x32_bf16 v[196:199], v[68:71], v[48:51], v[196:199]\n"
      "v_add_u32 v16, %[dA], v16\n"
      "v_mfma_f32_16x16x32_bf16 v[200:203], v[72:75], v[48:51], v[200:203]\n"
      "v_add_u32 v17, %[dA], v17\n"
      "v_mfma_f32_16x16x32_bf16 v[204:207], v[76:79], v[48:51], v[204:207]\n"
      "v_add_u32 v18, %[dB], v18\n"
      "v_mfma_f32_16x16x32_bf16 v[208:211], v[64:67], v[52:55], v[208:211]\n"
      "v_add_u32 v19, %[dB], v19\n"
      "v_mfma_f32_16x16x32_bf16 v[212:215], v[68:71], v[52:55], v[212:215]\n"
      "v_add_u32 v20, %[dB], v20\n"
      "v_mfma_f32_16x16x32_bf16 v[216:219], v[72:75], v[52:55], v[216:219]\n"
      "v_add_u32 v21, %[dB], v21\n"
      "v_mfma_f32_16x16x32_bf16 v[220:223], v[76:79], v[52:55], v[220:223]\n"
      "s_waitcnt lgkmcnt(0)\n"
      "ds_write_b32 v31, v29\n"
      "v_mfma_f32_16x16x32_bf16 v[224:227], v[64:67], v[56:59], v[224:227]\n"
      "ds_read_b128 v[24:27], v30\n"
      "v_mfma_f32_16x16x32_bf16 v[228:231], v[68:71], v[56:59], v[228:231]\n"
      "v_mfma_f32_16x16x32_bf16 v[232:235], v[72:75], v[56:59], v[232:235]\n"
      "v_mfma_f32_16x16x32_bf16 v[236:239], v[76:79], v[56:59], v[236:239]\n"
      "v_mfma_f32_16x16x32_bf16 v[240:243], v[64:67], v[60:63], v[240:243]\n"
      "v_mfma_f32_16x16x32_bf16 v[244:247], v[68:71], v[60:63], v[244:247]\n"
      "s_waitcnt lgkmcnt(0)\n"
      "v_min3_u32 v28, v24, v25, v26\n"
      "v_mfma_f32_16x16x32_bf16 v[248:251], v[72:75], v[60:63], v[248:251]\n"
      "v_cmp_gt_u32 vcc, %[need], v28\n"
      "v_mfma_f32_16x16x32_bf16 v[252:255], v[76:79], v[60:63], v[252:255]\n"
      "s_cbranch_vccnz L_slow_%=\n"
      "L_back_%=:\n"
      "v_mfma_f32_16x16x32_bf16 v[128:131], v[112:115], v[80:83], v[128:131]\n"
      "ds_read_b64_tr_b16 v[64:65], v18 offset:0\n"
      "v_mfma_f32_16x16x32_bf16 v[132:135], v[116:119], v[80:83], v[132:135]\n"
      "ds_read_b64_tr_b16 v[66:67], v18 offset:2048\n"
      "v_mfma_f32_16x16x32_bf16 v[136:139], v[120:123], v[80:83], v[136:139]\n"
      "ds_read_b64_tr_b16 v[68:69], v19 offset:0\n"
      "v_mfma_f32_16x16x32_bf16 v[140:143], v[124:127], v[80:83], v[140:143]\n"
      "ds_read_b64_tr_b16 v[70:71], v19 offset:2048\n"
      "v_mfma_f32_16x16x32_bf16 v[144:147], v[112:115], v[84:87], v[144:147]\n"
      "ds_read_b64_tr_b16 v[72:73], v20 offset:0\n"
      "v_mfma_f32_16x16x32_bf16 v[148:151], v[116:119], v[84:87], v[148:151]\n"
      "ds_read_b64_tr_b16 v[74:75], v20 offset:2048\n"
      "v_mfma_f32_16x16x32_bf16 v[152:155], v[120:123], v[84:87], v[152:155]\n"
      "ds_read_b64_tr_b16 v[76:77], v21 offset:0\n"
      "v_mfma_f32_16x16x32_bf16 v[156:159], v[124:127], v[84:87], v[156:159]\n"
      "ds_read_b64_tr_b16 v[78:79], v21 offset:2048\n"
      "v_mfma_f32_16x16x32_bf16 v[160:163], v[112:115], v[88:91], v[160:163]\n"
      "ds_read_b128 v[32:35], v16 offset:0\n"
      "v_mfma_f32_16x16x32_bf16 v[164:167], v[116:119], v[88:91], v[164:167]\n"
      "ds_read_b128 v[36:39], v16 offset:2048\n"
      "v_mfma_f32_16x16x32_bf16 v[168:171], v[120:123], v[88:91], v[168:171]\n"
      "ds_read_b128 v[40:43], v16 offset:4096\n"
      "v_mfma_f32_16x16x32_bf16 v[172:175], v[124:127], v[88:91], v[172:175]\n"
      "ds_read_b128 v[44:47], v16 offset:6144\n"
      "v_mfma_f32_16x16x32_bf16 v[176:179], v[112:115], v[92:95], v[176:179]\n"
      "ds_read_b128 v[48:51], v16 offset:8192\n"
      "v_mfma_f32_16x16x32_bf16 v[180:183], v[116:119], v[92:95], v[180:183]\n"
      "ds_read_b128 v[52:55], v16 offset:10240\n"
      "v_mfma_f32_16x16x32_bf16 v[184:187], v[120:123], v[92:95], v[184:187]\n"
      "ds_read_b128 v[56:59], v16 offset:12288\n"
      "v_mfma_f32_16x16x32_bf16 v[188:191], v[124:127], v[92:95], v[188:191]\n"
      "ds_read_b128 v[60:63], v16 offset:14336\n"
      "v_mfma_f32_16x16x32_bf16 v[192:195], v[112:115], v[96:99], v[192:195]\n"
      "v_mfma_f32_16x16x32_bf16 v[196:199], v[116:119], v[96:99], v[196:199]\n"
      "v_mfma_f32_16x16x32_bf16 v[200:203], v[120:123], v[96:99], v[200:203]\n"
      "v_mfma_f32_16x16x32_bf16 v[204:207], v[124:127], v[96:99], v[204:207]\n"
      "v_mfma_f32_16x16x32_bf16 v[208:211], v[112:115], v[100:103], v[208:211]\n"
      "v_mfma_f32_16x16x32_bf16 v[212:215], v[116:119], v[100:103], v[212:215]\n"
      "v_mfma_f32_16x16x32_bf16 v[216:219], v[120:123], v[100:103], v[216:219]\n"
      "v_mfma_f32_16x16x32_bf16 v[220:223], v[124:127], v[100:103], v[220:223]\n"
      "v_mfma_f32_16x16x32_bf16 v[224:227], v[112:115], v[104:107], v[224:227]\n"
      "v_mfma_f32_16x16x32_bf16 v[228:231], v[116:119], v[104:107], v[228:231]\n"
      "v_mfma_f32_16x16x32_bf16 v[232:235], v[120:123], v[104:107], v[232:235]\n"
      "v_mfma_f32_16x16x32_bf16 v[236:239], v[124:127], v[104:107], v[236:239]\n"
      "v_mfma_f32_16x16x32_bf16 v[240:243], v[112:115], v[108:111], v[240:243]\n"
      "s_add_u32 %[i], %[i], 1\n"
      "v_mfma_f32_16x16x32_bf16 v[244:247], v[116:119], v[108:111], v[244:247]\n"
      "s_cmp_lt_u32 %[i], 64\n"
      "v_mfma_f32_16x16x32_bf16 v[248:251], v[120:123], v[108:111], v[248:251]\n"
      "s_waitcnt lgkmcnt(0)\n"
      "v_mfma_f32_16x16x32_bf16 v[252:255], v[124:127], v[108:111], v[252:255]\n"
      "s_cbranch_scc1 L_loop_%=\n"
      "s_branch L_end_%=\n"
      "L_slow_%=:\n"
      "s_add_u32 %[sc], %[sc], 1\n"
      "ds_read_b128 v[24:27], v30\n"
      "s_waitcnt lgkmcnt(0)\n"
      "v_min3_u32 v28, v24, v25, v26\n"
      "v_cmp_gt_u32 vcc, %[need], v28\n"
      "s_cbranch_vccnz L_slow_%=\n"
      "s_branch L_back_%=\n"
      "L_end_%=:\n"
      "s_nop 15\n"
                : "=&{v[128:143]}"(c0), "=&{v[144:159]}"(c1), "=&{v[160:175]}"(c2), "=&{v[176:191]}"(c3),
                  "=&{v[192:207]}"(c4), "=&{v[208:223]}"(c5), "=&{v[224:239]}"(c6), "=&{v[240:255]}"(c7),
                  [i] "=&s"(si), [st] "=&s"(sst), [need] "=&s"(sneed), [dA] "=&s"(sdA), [dB] "=&s"(sdB), [sc] "=&s"(slowc)
                : [a0] "v"(a0), [a1] "v"(a1), [b0] "v"(b0), [b1] "v"(b1), [b2] "v"(b2), [b3] "v"(b3), [fa] "v"(fa), [da] "v"(da),
                  [cm] "s"(0xffff8000u), [cp] "s"(16384u)
                : "v16", "v17", "v18", "v19", "v20", "v21", "v22", "v23", "v24", "v25", "v26", "v27", "v28", "v29", "v30", "v31", "v32", "v33", "v34", "v35", "v36", "v37", "v38", "v39", "v40", "v41", "v42", "v43", "v44", "v45", "v46", "v47", "v48", "v49", "v50", "v51", "v52", "v53", "v54", "v55", "v56", "v57", "v58", "v59", "v60", "v61", "v62", "v63", "v64", "v65", "v66", "v67", "v68", "v69", "v70", "v71", "v72", "v73", "v74", "v75", "v76", "v77", "v78", "v79", "v80", "v81", "v82", "v83", "v84", "v85", "v86", "v87", "v88", "v89", "v90", "v91", "v92", "v93", "v94", "v95", "v96", "v97", "v98", "v99", "v100", "v101", "v102", "v103", "v104", "v105", "v106", "v107", "v108", "v109", "v110", "v111", "v112", "v113", "v114", "v115", "v116", "v117", "v118", "v119", "v120", "v121", "v122", "v123", "v124", "v125", "v126", "v127", "vcc", "scc", "memory");
        }
        if (wid == 0) STAMP(2);
        u32 tz = threadIdx.x; asm volatile("" : "+v"(tz));
        const u32 lane = tz & 63;
        float mx = 0.f;
#define MX(c) _Pragma("unroll") for (int j = 0; j < 16; j += 2) mx = __builtin_fmaxf(mx, __builtin_fmaxf(c[j], c[j + 1]));
        MX(c0) MX(c1) MX(c2) MX(c3) MX(c4) MX(c5) MX(c6) MX(c7)
        const u32 wm = wave_umax(__float_as_uint(mx));
        lds_store(lds0 + F_MISC + wid * 4, wm);
        BAR();
        BAR();
        u32x4 am; asm volatile("ds_read_b128 %0, %1\n\ts_waitcnt lgkmcnt(0)" : "=&v"(am) : "v"(lds0 + F_MISC + 16) : "memory");
        const float amax = __uint_as_float(am.x);
        const float inv = amax > 0.f ? 1.0f / (amax / 448.0f) : 0.f;
        const u32 nanm = am.y ? 0x7f7f7f7fu : 0u;
        const u32 ea = lds0 + (lane & 15) * EP_PITCH + wid * 64 + (lane >> 4) * 4;
#define QS(c, m) _Pragma("unroll") for (int t = 0; t < 4; ++t) { \
            int w = __builtin_amdgcn_cvt_pk_fp8_f32(__builtin_fmaxf(c[4 * t], 0.f) * inv, __builtin_fmaxf(c[4 * t + 1], 0.f) * inv, 0, false); \
            w = __builtin_amdgcn_cvt_pk_fp8_f32(__builtin_fmaxf(c[4 * t + 2], 0.f) * inv, __builtin_fmaxf(c[4 * t + 3], 0.f) * inv, w, true); \
            *(LAS u32*)(uintptr_t)(ea + (m) * 16 * EP_PITCH + t * 16) = (u32)w | nanm; }
        QS(c0, 0) QS(c1, 1) QS(c2, 2) QS(c3, 3) QS(c4, 4) QS(c5, 5) QS(c6, 6) QS(c7, 7)
    } else if (wid < 7) {
        const u32 lane = threadIdx.x & 63;
        const int pb = wid - 5;
        __builtin_amdgcn_s_setprio(PPRIO);
        u64 tis = 0, tfr = 0, tvm = 0;
        u32x4 rs; u32 v0, v1, sb, sstep, mb0, mst;
        if (wid == 4) {
            rs = make_rsrc(p.x, ML * KD * 2);
            v0 = (lane >> 3) * (KD * 2) + (((lane & 7) ^ (lane >> 4)) << 4); v1 = v0 ^ 64;
            sb = rowblk * BM * KD * 2; sstep = 128; mb0 = lds0 + A_BASE; mst = A_ST;
        } else {
            rs = make_rsrc(p.w, (u32)KD * ND * 2);
            v0 = (lane >> 5) * (ND * 2) + (((lane & 31) ^ (2 * (2 * pb + (lane >> 5)))) << 4); v1 = v0 ^ 128;
            sb = pb * 2 * ND * 2 + (grp * NC + ct * BN) * 2; sstep = 64 * ND * 2; mb0 = lds0 + B_BASE + pb * 1024; mst = B_ST;
        }
        sb = rfl(sb); sstep = rfl(sstep); mb0 = rfl(mb0); mst = rfl(mst);
        const u32 fw = lds0 + F_FULL + (wid - 4) * 4, dn = lds0 + F_DONE;
#define ISSUE1(mb, s) do { if (wid == 4) issue_a1(rs, v0, v1, mb, s); else issue_b1(rs, v0, v1, mb, s); } while (0)
#define ISSUE2(mb, s) do { if (wid == 4) issue_a2(rs, v0, v1, mb, s); else issue_b2(rs, v0, v1, mb, s); } while (0)
#define ISSUE(mb, s) do { ISSUE1(mb, s); ISSUE2(mb, s); } while (0)
        ISSUE(mb0, sb); ISSUE(mb0 + mst, sb + sstep); ISSUE(mb0 + 2 * mst, sb + 2 * sstep);
        sb += 3 * sstep;
        BAR();
        asm volatile("s_waitcnt vmcnt(32)" ::: "memory"); lds_store(fw, 1);
        if (wid == 5) STAMP(1);
        asm volatile("s_waitcnt vmcnt(16)" ::: "memory"); lds_store(fw, 2);
        u32 st = 0;
        for (u32 n = 3; n < NT; ++n) {
#define TT(v) u64 v = 0;
            TT(ta)
            while (lds_min4(dn) < n - 2) __builtin_amdgcn_s_sleep(PSLEEP);
            TT(tb)
            ISSUE1(mb0 + st * mst, sb);
            asm volatile("s_waitcnt vmcnt(4)" ::: "memory"); if (lane == 0) lds_store(fw, n);
            TT(tc)
            ISSUE2(mb0 + st * mst, sb);
            sb += sstep; st = st == 2 ? 0 : st + 1;
            TT(td)
            tfr += tb - ta; tis += tc - tb; tvm += td - tc;
        }
        asm volatile("s_waitcnt vmcnt(0)" ::: "memory"); lds_store(fw, NT);
        BAR();
        BAR();
    } else {
        const u32 lane = threadIdx.x & 63;
        if (lane < 4) lds_store(lds0 + F_FULL + lane * 4, 0);
        if (lane < 4) lds_store(lds0 + F_DONE + lane * 4, 0);
        BAR();
        u32 call = 0;
        if (lane == 0) call = __hip_atomic_fetch_add((GAS u32*)(myws + WS_CALL) + b, 1u, __ATOMIC_RELAXED, __HIP_MEMORY_SCOPE_AGENT) + 1;
        call = rfl(call);
        if (lane == 0) lds_store(lds0 + F_MISC + 24, call);
        BAR();
        u32x4 wm; asm volatile("ds_read_b128 %0, %1\n\ts_waitcnt lgkmcnt(0)" : "=&v"(wm) : "v"(lds0 + F_MISC) : "memory");
        const u32 tm = max(max(wm.x, wm.y), max(wm.z, wm.w));
        bool bad = false;
        if (lane == 0) st_sys64(myws + WS_LMAX + (grp * 32 + idx) * 8, ((u64)call << 32) | tm);
        if (idx == 0) {
            const u32 gm = sweep(myws + WS_LMAX + grp * 256, 32, call, myws + WS_TMO, lane, bad);
            if (!bad && lane < NR) st_sys64(p.ws[lane] + WS_GMAX + (call & 1) * 512 + (p.rank * 8 + grp) * 8, ((u64)call << 32) | gm);
        }
        const u32 am = sweep(myws + WS_GMAX + (call & 1) * 512, 64, call, myws + WS_TMO, lane, bad);
        if (lane == 0) { lds_store(lds0 + F_MISC + 16, am); lds_store(lds0 + F_MISC + 20, bad ? 1u : 0u); }
        STAMP(3);
        BAR();
    }
    BAR();
    {
        u32 tz = threadIdx.x; asm volatile("" : "+v"(tz));
        const u32 lane = tz & 63;
        unsigned char* dst = p.out[grp] + (size_t)(p.rank * ML + rowblk * BM + wid * 16 + (lane >> 4)) * NC + ct * BN + (lane & 15) * 16;
        const u32 la = lds0 + (wid * 16 + (lane >> 4)) * EP_PITCH + (lane & 15) * 16;
        u32x4 d0, d1, d2, d3;
        asm volatile("ds_read_b128 %0, %4\n\tds_read_b128 %1, %4 offset:1088\n\tds_read_b128 %2, %4 offset:2176\n\tds_read_b128 %3, %4 offset:3264\n\ts_waitcnt lgkmcnt(0)"
                     : "=&v"(d0), "=&v"(d1), "=&v"(d2), "=&v"(d3) : "v"(la) : "memory");
        asm volatile("global_store_dwordx4 %0, %1, off sc0 sc1\n\ts_nop 1" :: "v"(dst), "v"(d0) : "memory");
        asm volatile("global_store_dwordx4 %0, %1, off sc0 sc1\n\ts_nop 1" :: "v"(dst + 4 * NC), "v"(d1) : "memory");
        asm volatile("global_store_dwordx4 %0, %1, off sc0 sc1\n\ts_nop 1" :: "v"(dst + 8 * NC), "v"(d2) : "memory");
        asm volatile("global_store_dwordx4 %0, %1, off sc0 sc1\n\ts_nop 1" :: "v"(dst + 12 * NC), "v"(d3) : "memory");
        asm volatile("s_waitcnt vmcnt(0)" ::: "memory");
        BAR();
        if (wid == 7) {
            STAMP(4);
            u32 call; asm volatile("ds_read_b32 %0, %1\n\ts_waitcnt lgkmcnt(0)" : "=&v"(call) : "v"(lds0 + F_MISC + 24) : "memory");
            call = rfl(call);
            if (lane == 0 && grp != p.rank) st_sys32(p.ws[grp] + WS_DONE + (p.rank * 32 + idx) * 4, call);
            if (grp == p.rank && idx < NR && idx != p.rank) {
                const u64 t0 = __builtin_amdgcn_s_memrealtime();
                const unsigned char* q = myws + WS_DONE + (idx * 32 + (lane & 31)) * 4;
                for (;;) {
                    const u32 v = ld_sys32(q);
                    const u32 t = ld_agt32(myws + WS_TMO);
                    if (t != call && __all((int)(v - call) >= 0)) break;
                    if (t == call || __builtin_amdgcn_s_memrealtime() - t0 > DEADLINE) {
                        if (lane == 0) { st_agt32(myws + WS_TMO, call); *dst = 0x7f; }
                        break;
                    }
                    __builtin_amdgcn_s_sleep(1);
                }
            }
            STAMP(5);
        }
    }
}

extern "C" void kernel_launch(int rank, int nranks, ncclComm_t comm, hipStream_t stream,
                              void* const* d_in, const int* in_sizes, int n_in,
                              void* d_out, int out_size, const int* in_dtypes, int out_dtype,
                              void* const* d_out_all, void* const* d_ws_all, size_t ws_bytes) {
    if (nranks != NR || n_in < 2 || in_sizes[0] != ML * KD || in_sizes[1] != KD * ND || out_size != NR * ML * NC || ws_bytes < WS_NEED) {
        fprintf(stderr, "kernel_launch: unexpected shapes\n"); return;
    }
    static bool attr[NR] = {};
    if (!attr[rank]) { HIP_CHECK(hipFuncSetAttribute((const void*)fused_kernel, hipFuncAttributeMaxDynamicSharedMemorySize, LDS_BYTES)); attr[rank] = true; }
    Params p;
    p.x = d_in[0]; p.w = d_in[1]; p.rank = rank; p.pad = 0;
    for (int i = 0; i < NR; ++i) { p.out[i] = (unsigned char*)d_out_all[i]; p.ws[i] = (unsigned char*)d_ws_all[i]; }
    hipLaunchKernelGGL(fused_kernel, dim3(NWG), dim3(512), LDS_BYTES, stream, p);
    HIP_CHECK(hipGetLastError());
}
```

```cpp
DI void issue_a1(u32x4 rs, u32 v0, u32 v1, u32 mb, u32 sb) {
    u32 t;
    asm volatile(
        DMA(v0, 0, 0x0) DMA(v1, 1024, 0x10000) DMA(v0, 2048, 0x20000) DMA(v1, 3072, 0x30000)
        : [t] "=&s"(t) : [v0] "v"(v0), [v1] "v"(v1), [rs] "s"(rs), [mb] "s"(mb), [sb] "s"(sb) : "memory", "scc");
}
DI void issue_a2(u32x4 rs, u32 v0, u32 v1, u32 mb, u32 sb) {
    u32 t;
    asm volatile(
        DMA(v0, 4096, 0x40000) DMA(v1, 5120, 0x50000) DMA(v0, 6144, 0x60000) DMA(v1, 7168, 0x70000) DMA(v0, 8192, 0x80000) DMA(v1, 9216, 0x90000) DMA(v0, 10240, 0xa0000) DMA(v1, 11264, 0xb0000) DMA(v0, 12288, 0xc0000) DMA(v1, 13312, 0xd0000) DMA(v0, 14336, 0xe0000) DMA(v1, 15360, 0xf0000)
        : [t] "=&s"(t) : [v0] "v"(v0), [v1] "v"(v1), [rs] "s"(rs), [mb] "s"(mb), [sb] "s"(sb) : "memory", "scc");
}
DI void issue_b1(u32x4 rs, u32 v0, u32 v1, u32 mb, u32 sb) {
    u32 t;
    asm volatile(
        DMA(v0, 0, 0x0) DMA(v0, 2048, 0x20000) DMA(v1, 4096, 0x40000) DMA(v1, 6144, 0x60000)
        : [t] "=&s"(t) : [v0] "v"(v0), [v1] "v"(v1), [rs] "s"(rs), [mb] "s"(mb), [sb] "s"(sb) : "memory", "scc");
}
DI void issue_b2(u32x4 rs, u32 v0, u32 v1, u32 mb, u32 sb) {
    u32 t;
    asm volatile(
        DMA(v0, 8192, 0x80000) DMA(v0, 10240, 0xa0000) DMA(v1, 12288, 0xc0000) DMA(v1, 14336, 0xe0000) DMA(v0, 16384, 0x100000) DMA(v0, 18432, 0x120000) DMA(v1, 20480, 0x140000) DMA(v1, 22528, 0x160000) DMA(v0, 24576, 0x180000) DMA(v0, 26624, 0x1a0000) DMA(v1, 28672, 0x1c0000) DMA(v1, 30720, 0x1e0000)
        : [t] "=&s"(t) : [v0] "v"(v0), [v1] "v"(v1), [rs] "s"(rs), [mb] "s"(mb), [sb] "s"(sb) : "memory", "scc");
}

_m4096_k4096_n16384_bf16_reluqe4m3 — fused GEMM + ReLU + global amax + e4m3 quantize + all-to-all, one kernel.
#include <hip/hip_runtime.h>
#include <rccl/rccl.h>
#include <cstdio>
#include <cstdint>

#define HIP_CHECK(expr) do { hipError_t _e = (expr); if (_e != hipSuccess) { \
    fprintf(stderr, "HIP error %s at %s:%d\n", hipGetErrorString(_e), __FILE__, __LINE__); \
    return; } } while (0)

typedef unsigned u32;
typedef unsigned long long u64;
typedef float f32x16 __attribute__((ext_vector_type(16)));
typedef u32 u32x4 __attribute__((ext_vector_type(4)));
#define LAS __attribute__((address_space(3)))
#define GAS __attribute__((address_space(1)))
#define DI __device__ __forceinline__

constexpr int NR = 8, ML = 512, KD = 4096, ND = 16384, NC = 2048;
constexpr int BM = 128, BN = 256, NT = 64;
constexpr int NWG = 256;
constexpr u32 A_BASE = 0, A_ST = 16384, B_BASE = 49152, B_ST = 32768;
constexpr u32 F_FULL = 147456, F_DONE = F_FULL + 1024, F_MISC = F_DONE + 1024, LDS_BYTES = F_MISC + 1024;
constexpr u32 EP_PITCH = 272;
constexpr size_t WS_CALL = 0;
constexpr size_t WS_TMO = 4096;
constexpr size_t WS_LMAX = 8192;
constexpr size_t WS_GMAX = 12288;
constexpr size_t WS_DONE = 16384;
constexpr size_t WS_DBG = 65536;
constexpr size_t WS_NEED = WS_DBG + 256 * 64 * 8;
constexpr u64 DEADLINE = 50000000ull;

struct Params {
    const void* x; const void* w;
    unsigned char* out[NR];
    unsigned char* ws[NR];
    int rank; int pad;
};

DI u32 rfl(u32 v) { return __builtin_amdgcn_readfirstlane(v); }
DI u32x4 make_rsrc(const void* p, u32 bytes) {
    u64 a = (u64)p; u32x4 r; r.x = rfl((u32)a); r.y = rfl((u32)(a >> 32)); r.z = bytes; r.w = 0x00020000u; return r;
}
DI void lds_store(u32 addr, u32 v) { asm volatile("ds_write_b32 %0, %1" :: "v"(addr), "v"(v) : "memory"); }
DI u32 lds_min4(u32 addr) {
    u32x4 d; asm volatile("ds_read_b128 %0, %1\n\ts_waitcnt lgkmcnt(0)" : "=&v"(d) : "v"(addr) : "memory");
    return rfl(min(min(d.x, d.y), min(d.z, d.w)));
}
#define BAR() do { asm volatile("s_waitcnt lgkmcnt(0)\n\ts_barrier" ::: "memory"); } while (0)
#define NSTAMP 8
#define CPRIO 2
#define PPRIO 0
#define ROTX pw
#define ROTW pw
#define PSLEEP 1
#define STAMP(k) do {} while (0)

DI u64 ld_sys64(const unsigned char* p) { return __hip_atomic_load((const GAS u64*)p, __ATOMIC_RELAXED, __HIP_MEMORY_SCOPE_SYSTEM); }
DI void st_sys64(unsigned char* p, u64 v) { __hip_atomic_store((GAS u64*)p, v, __ATOMIC_RELAXED, __HIP_MEMORY_SCOPE_SYSTEM); }
DI u32 ld_sys32(const unsigned char* p) { return __hip_atomic_load((const GAS u32*)p, __ATOMIC_RELAXED, __HIP_MEMORY_SCOPE_SYSTEM); }
DI void st_sys32(unsigned char* p, u32 v) { __hip_atomic_store((GAS u32*)p, v, __ATOMIC_RELAXED, __HIP_MEMORY_SCOPE_SYSTEM); }
DI u32 ld_agt32(const unsigned char* p) { return __hip_atomic_load((const GAS u32*)p, __ATOMIC_RELAXED, __HIP_MEMORY_SCOPE_AGENT); }
DI void st_agt32(unsigned char* p, u32 v) { __hip_atomic_store((GAS u32*)p, v, __ATOMIC_RELAXED, __HIP_MEMORY_SCOPE_AGENT); }
DI u32 wave_umax(u32 v) {
#pragma unroll
    for (int o = 32; o >= 1; o >>= 1) v = max(v, (u32)__shfl_xor((int)v, o));
    return v;
}

DI u32 sweep(const unsigned char* g, int n, u32 call, unsigned char* tmo, int lane, bool& bad) {
    const u64 t0 = __builtin_amdgcn_s_memrealtime();
    const unsigned char* q = g + (lane < n ? lane : 0) * 8;
    for (;;) {
        const u64 v = ld_sys64(q);
        const u32 t = ld_agt32(tmo);
        if (t == call) { bad = true; return 0; }
        if (__all((u32)(v >> 32) == call)) return wave_umax((u32)v);
        if (__builtin_amdgcn_s_memrealtime() - t0 > DEADLINE) { if (lane == 0) st_agt32(tmo, call); bad = true; return 0; }
        __builtin_amdgcn_s_sleep(1);
    }
}

__global__ __launch_bounds__(512, 2) void fused_kernel(Params p) {
    extern __shared__ __attribute__((aligned(16))) unsigned char smem[];
    const int wid = rfl(threadIdx.x >> 6);
    const int b = blockIdx.x, grp = b & 7, idx = b >> 3;
    const int rowblk = idx & 3, ct = idx >> 2;
    const u32 lds0 = (u32)(uintptr_t)(LAS unsigned char*)smem;
    unsigned char* const myws = p.ws[p.rank];
    if (wid == 0) STAMP(0);

    if (wid < 4) {
        f32x16 c0, c1, c2, c3, c4, c5, c6, c7; u32 slowc; u64 tc0 = 0;
        {
            const u32 lane = threadIdx.x & 63, r = lane & 15, g = lane >> 4, q = r >> 2, pp = lane & 3;
            const u32 a0 = lds0 + A_BASE + r * 128 + ((g ^ (r >> 1)) << 4), a1 = a0 ^ 64;
            const u32 bb = lds0 + B_BASE + (8 * g + q) * 512 + (pp & 1) * 8, F = 2 * q + 8 * (g & 1), cb = 8 * wid + (pp >> 1);
            const u32 b0 = bb + (((cb + 0) ^ F) << 4), b1 = bb + (((cb + 2) ^ F) << 4), b2 = bb + (((cb + 4) ^ F) << 4), b3 = bb + (((cb + 6) ^ F) << 4);
            u32 si, sst, sdA, sdB; slowc = 0;
            __builtin_amdgcn_s_setprio(CPRIO);
            asm volatile(
      "v_mov_b32 v16, %[a0]\n"
      "v_mov_b32 v17, %[a1]\n"
      "v_mov_b32 v18, %[b0]\n"
      "v_mov_b32 v19, %[b1]\n"
      "v_mov_b32 v20, %[b2]\n"
      "v_mov_b32 v21, %[b3]\n"
      "v_mov_b32 v128, 0\n"
      "v_mov_b32 v129, 0\n"
      "v_mov_b32 v130, 0\n"
      "v_mov_b32 v131, 0\n"
      "v_mov_b32 v132, 0\n"
      "v_mov_b32 v133, 0\n"
      "v_mov_b32 v134, 0\n"
      "v_mov_b32 v135, 0\n"
      "v_mov_b32 v136, 0\n"
      "v_mov_b32 v137, 0\n"
      "v_mov_b32 v138, 0\n"
      "v_mov_b32 v139, 0\n"
      "v_mov_b32 v140, 0\n"
      "v_mov_b32 v141, 0\n"
      "v_mov_b32 v142, 0\n"
      "v_mov_b32 v143, 0\n"
      "v_mov_b32 v144, 0\n"
      "v_mov_b32 v145, 0\n"
      "v_mov_b32 v146, 0\n"
      "v_mov_b32 v147, 0\n"
      "v_mov_b32 v148, 0\n"
      "v_mov_b32 v149, 0\n"
      "v_mov_b32 v150, 0\n"
      "v_mov_b32 v151, 0\n"
      "v_mov_b32 v152, 0\n"
      "v_mov_b32 v153, 0\n"
      "v_mov_b32 v154, 0\n"
      "v_mov_b32 v155, 0\n"
      "v_mov_b32 v156, 0\n"
      "v_mov_b32 v157, 0\n"
      "v_mov_b32 v158, 0\n"
      "v_mov_b32 v159, 0\n"
      "v_mov_b32 v160, 0\n"
      "v_mov_b32 v161, 0\n"
      "v_mov_b32 v162, 0\n"
      "v_mov_b32 v163, 0\n"
      "v_mov_b32 v164, 0\n"
      "v_mov_b32 v165, 0\n"
      "v_mov_b32 v166, 0\n"
      "v_mov_b32 v167, 0\n"
      "v_mov_b32 v168, 0\n"
      "v_mov_b32 v169, 0\n"
      "v_mov_b32 v170, 0\n"
      "v_mov_b32 v171, 0\n"
      "v_mov_b32 v172, 0\n"
      "v_mov_b32 v173, 0\n"
      "v_mov_b32 v174, 0\n"
      "v_mov_b32 v175, 0\n"
      "v_mov_b32 v176, 0\n"
      "v_mov_b32 v177, 0\n"
      "v_mov_b32 v178, 0\n"
      "v_mov_b32 v179, 0\n"
      "v_mov_b32 v180, 0\n"
      "v_mov_b32 v181, 0\n"
      "v_mov_b32 v182, 0\n"
      "v_mov_b32 v183, 0\n"
      "v_mov_b32 v184, 0\n"
      "v_mov_b32 v185, 0\n"
      "v_mov_b32 v186, 0\n"
      "v_mov_b32 v187, 0\n"
      "v_mov_b32 v188, 0\n"
      "v_mov_b32 v189, 0\n"
      "v_mov_b32 v190, 0\n"
      "v_mov_b32 v191, 0\n"
      "v_mov_b32 v192, 0\n"
      "v_mov_b32 v193, 0\n"
      "v_mov_b32 v194, 0\n"
      "v_mov_b32 v195, 0\n"
      "v_mov_b32 v196, 0\n"
      "v_mov_b32 v197, 0\n"
      "v_mov_b32 v198, 0\n"
      "v_mov_b32 v199, 0\n"
      "v_mov_b32 v200, 0\n"
      "v_mov_b32 v201, 0\n"
      "v_mov_b32 v202, 0\n"
      "v_mov_b32 v203, 0\n"
      "v_mov_b32 v204, 0\n"
      "v_mov_b32 v205, 0\n"
      "v_mov_b32 v206, 0\n"
      "v_mov_b32 v207, 0\n"
      "v_mov_b32 v208, 0\n"
      "v_mov_b32 v209, 0\n"
      "v_mov_b32 v210, 0\n"
      "v_mov_b32 v211, 0\n"
      "v_mov_b32 v212, 0\n"
      "v_mov_b32 v213, 0\n"
      "v_mov_b32 v214, 0\n"
      "v_mov_b32 v215, 0\n"
      "v_mov_b32 v216, 0\n"
      "v_mov_b32 v217, 0\n"
      "v_mov_b32 v218, 0\n"
      "v_mov_b32 v219, 0\n"
      "v_mov_b32 v220, 0\n"
      "v_mov_b32 v221, 0\n"
      "v_mov_b32 v222, 0\n"
      "v_mov_b32 v223, 0\n"
      "v_mov_b32 v224, 0\n"
      "v_mov_b32 v225, 0\n"
      "v_mov_b32 v226, 0\n"
      "v_mov_b32 v227, 0\n"
      "v_mov_b32 v228, 0\n"
      "v_mov_b32 v229, 0\n"
      "v_mov_b32 v230, 0\n"
      "v_mov_b32 v231, 0\n"
      "v_mov_b32 v232, 0\n"
      "v_mov_b32 v233, 0\n"
      "v_mov_b32 v234, 0\n"
      "v_mov_b32 v235, 0\n"
      "v_mov_b32 v236, 0\n"
      "v_mov_b32 v237, 0\n"
      "v_mov_b32 v238, 0\n"
      "v_mov_b32 v239, 0\n"
      "v_mov_b32 v240, 0\n"
      "v_mov_b32 v241, 0\n"
      "v_mov_b32 v242, 0\n"
      "v_mov_b32 v243, 0\n"
      "v_mov_b32 v244, 0\n"
      "v_mov_b32 v245, 0\n"
      "v_mov_b32 v246, 0\n"
      "v_mov_b32 v247, 0\n"
      "v_mov_b32 v248, 0\n"
      "v_mov_b32 v249, 0\n"
      "v_mov_b32 v250, 0\n"
      "v_mov_b32 v251, 0\n"
      "v_mov_b32 v252, 0\n"
      "v_mov_b32 v253, 0\n"
      "v_mov_b32 v254, 0\n"
      "v_mov_b32 v255, 0\n"
      "s_mov_b32 %[i], 0\n"
      "s_mov_b32 %[st], 0\n"
      "s_barrier\n"
      "ds_read_b64_tr_b16 v[64:65], v18 offset:0\n"
      "ds_read_b64_tr_b16 v[66:67], v18 offset:2048\n"
      "ds_read_b128 v[32:35], v16 offset:0\n"
      "ds_read_b64_tr_b16 v[68:69], v19 offset:0\n"
      "ds_read_b64_tr_b16 v[70:71], v19 offset:2048\n"
      "ds_read_b64_tr_b16 v[72:73], v20 offset:0\n"
      "ds_read_b64_tr_b16 v[74:75], v20 offset:2048\n"
      "ds_read_b64_tr_b16 v[76:77], v21 offset:0\n"
      "ds_read_b64_tr_b16 v[78:79], v21 offset:2048\n"
      "ds_read_b128 v[36:39], v16 offset:2048\n"
      "ds_read_b128 v[40:43], v16 offset:4096\n"
      "ds_read_b128 v[44:47], v16 offset:6144\n"
      "ds_read_b128 v[48:51], v16 offset:8192\n"
      "ds_read_b128 v[52:55], v16 offset:10240\n"
      "ds_read_b128 v[56:59], v16 offset:12288\n"
      "ds_read_b128 v[60:63], v16 offset:14336\n"
      "s_waitcnt lgkmcnt(0)\n"
      ".p2align 4\n"
      "L_loop_%=:\n"
      "s_waitcnt lgkmcnt(0)\n"
      "v_mfma_f32_16x16x32_bf16 v[128:131], v[64:67], v[32:35], v[128:131]\n"
      "ds_read_b64_tr_b16 v[112:113], v18 offset:16384\n"
      "v_mfma_f32_16x16x32_bf16 v[132:135], v[68:71], v[32:35], v[132:135]\n"
      "ds_read_b64_tr_b16 v[114:115], v18 offset:18432\n"
      "s_cmp_eq_u32 %[st], 2\n"
      "v_mfma_f32_16x16x32_bf16 v[136:139], v[72:75], v[32:35], v[136:139]\n"
      "ds_read_b128 v[80:83], v17 offset:0\n"
      "s_cselect_b32 %[dA], %[cm], %[cp]\n"
      "v_mfma_f32_16x16x32_bf16 v[140:143], v[76:79], v[32:35], v[140:143]\n"
      "ds_read_b64_tr_b16 v[116:117], v19 offset:16384\n"
      "s_cselect_b32 %[st], -1, %[st]\n"
      "v_mfma_f32_16x16x32_bf16 v[156:159], v[76:79], v[36:39], v[156:159]\n"
      "ds_read_b64_tr_b16 v[118:119], v19 offset:18432\n"
      "s_add_u32 %[st], %[st], 1\n"
      "v_mfma_f32_16x16x32_bf16 v[152:155], v[72:75], v[36:39], v[152:155]\n"
      "ds_read_b64_tr_b16 v[120:121], v20 offset:16384\n"
      "s_lshl_b32 %[dB], %[dA], 1\n"
      "v_mfma_f32_16x16x32_bf16 v[148:151], v[68:71], v[36:39], v[148:151]\n"
      "ds_read_b64_tr_b16 v[122:123], v20 offset:18432\n"
      "v_mfma_f32_16x16x32_bf16 v[144:147], v[64:67], v[36:39], v[144:147]\n"
      "ds_read_b64_tr_b16 v[124:125], v21 offset:16384\n"
      "v_mfma_f32_16x16x32_bf16 v[160:163], v[64:67], v[40:43], v[160:163]\n"
      "ds_read_b64_tr_b16 v[126:127], v21 offset:18432\n"
      "v_mfma_f32_16x16x32_bf16 v[164:167], v[68:71], v[40:43], v[164:167]\n"
      "ds_read_b128 v[84:87], v17 offset:2048\n"
      "v_mfma_f32_16x16x32_bf16 v[168:171], v[72:75], v[40:43], v[168:171]\n"
      "ds_read_b128 v[88:91], v17 offset:4096\n"
      "v_mfma_f32_16x16x32_bf16 v[172:175], v[76:79], v[40:43], v[172:175]\n"
      "ds_read_b128 v[92:95], v17 offset:6144\n"
      "v_mfma_f32_16x16x32_bf16 v[188:191], v[76:79], v[44:47], v[188:191]\n"
      "ds_read_b128 v[96:99], v17 offset:8192\n"
      "v_mfma_f32_16x16x32_bf16 v[184:187], v[72:75], v[44:47], v[184:187]\n"
      "ds_read_b128 v[100:103], v17 offset:10240\n"
      "v_mfma_f32_16x16x32_bf16 v[180:183], v[68:71], v[44:47], v[180:183]\n"
      "ds_read_b128 v[104:107], v17 offset:12288\n"
      "v_mfma_f32_16x16x32_bf16 v[176:179], v[64:67], v[44:47], v[176:179]\n"
      "ds_read_b128 v[108:111], v17 offset:14336\n"
      "v_mfma_f32_16x16x32_bf16 v[192:195], v[64:67], v[48:51], v[192:195]\n"
      "v_add_u32 v16, %[dA], v16\n"
      "v_mfma_f32_16x16x32_bf16 v[196:199], v[68:71], v[48:51], v[196:199]\n"
      "v_add_u32 v17, %[dA], v17\n"
      "v_mfma_f32_16x16x32_bf16 v[200:203], v[72:75], v[48:51], v[200:203]\n"
      "v_add_u32 v18, %[dB], v18\n"
      "v_mfma_f32_16x16x32_bf16 v[204:207], v[76:79], v[48:51], v[204:207]\n"
      "v_add_u32 v19, %[dB], v19\n"
      "v_mfma_f32_16x16x32_bf16 v[220:223], v[76:79], v[52:55], v[220:223]\n"
      "v_add_u32 v20, %[dB], v20\n"
      "v_mfma_f32_16x16x32_bf16 v[216:219], v[72:75], v[52:55], v[216:219]\n"
      "v_add_u32 v21, %[dB], v21\n"
      "v_mfma_f32_16x16x32_bf16 v[212:215], v[68:71], v[52:55], v[212:215]\n"
      "v_mfma_f32_16x16x32_bf16 v[208:211], v[64:67], v[52:55], v[208:211]\n"
      "v_mfma_f32_16x16x32_bf16 v[224:227], v[64:67], v[56:59], v[224:227]\n"
      "s_waitcnt lgkmcnt(0)\n"
      "s_barrier\n"
      "v_mfma_f32_16x16x32_bf16 v[228:231], v[68:71], v[56:59], v[228:231]\n"
      "v_mfma_f32_16x16x32_bf16 v[232:235], v[72:75], v[56:59], v[232:235]\n"
      "v_mfma_f32_16x16x32_bf16 v[236:239], v[76:79], v[56:59], v[236:239]\n"
      "v_mfma_f32_16x16x32_bf16 v[252:255], v[76:79], v[60:63], v[252:255]\n"
      "v_mfma_f32_16x16x32_bf16 v[248:251], v[72:75], v[60:63], v[248:251]\n"
      "v_mfma_f32_16x16x32_bf16 v[244:247], v[68:71], v[60:63], v[244:247]\n"
      "v_mfma_f32_16x16x32_bf16 v[240:243], v[64:67], v[60:63], v[240:243]\n"
      "v_mfma_f32_16x16x32_bf16 v[128:131], v[112:115], v[80:83], v[128:131]\n"
      "ds_read_b64_tr_b16 v[64:65], v18 offset:0\n"
      "v_mfma_f32_16x16x32_bf16 v[132:135], v[116:119], v[80:83], v[132:135]\n"
      "ds_read_b64_tr_b16 v[66:67], v18 offset:2048\n"
      "v_mfma_f32_16x16x32_bf16 v[136:139], v[120:123], v[80:83], v[136:139]\n"
      "ds_read_b128 v[32:35], v16 offset:0\n"
      "v_mfma_f32_16x16x32_bf16 v[140:143], v[124:127], v[80:83], v[140:143]\n"
      "ds_read_b64_tr_b16 v[68:69], v19 offset:0\n"
      "v_mfma_f32_16x16x32_bf16 v[156:159], v[124:127], v[84:87], v[156:159]\n"
      "ds_read_b64_tr_b16 v[70:71], v19 offset:2048\n"
      "v_mfma_f32_16x16x32_bf16 v[152:155], v[120:123], v[84:87], v[152:155]\n"
      "ds_read_b64_tr_b16 v[72:73], v20 offset:0\n"
      "v_mfma_f32_16x16x32_bf16 v[148:151], v[116:119], v[84:87], v[148:151]\n"
      "ds_read_b64_tr_b16 v[74:75], v20 offset:2048\n"
      "v_mfma_f32_16x16x32_bf16 v[144:147], v[112:115], v[84:87], v[144:147]\n"
      "ds_read_b64_tr_b16 v[76:77], v21 offset:0\n"
      "v_mfma_f32_16x16x32_bf16 v[160:163], v[112:115], v[88:91], v[160:163]\n"
      "ds_read_b64_tr_b16 v[78:79], v21 offset:2048\n"
      "v_mfma_f32_16x16x32_bf16 v[164:167], v[116:119], v[88:91], v[164:167]\n"
      "ds_read_b128 v[36:39], v16 offset:2048\n"
      "v_mfma_f32_16x16x32_bf16 v[168:171], v[120:123], v[88:91], v[168:171]\n"
      "ds_read_b128 v[40:43], v16 offset:4096\n"
      "v_mfma_f32_16x16x32_bf16 v[172:175], v[124:127], v[88:91], v[172:175]\n"
      "ds_read_b128 v[44:47], v16 offset:6144\n"
      "v_mfma_f32_16x16x32_bf16 v[188:191], v[124:127], v[92:95], v[188:191]\n"
      "ds_read_b128 v[48:51], v16 offset:8192\n"
      "v_mfma_f32_16x16x32_bf16 v[184:187], v[120:123], v[92:95], v[184:187]\n"
      "ds_read_b128 v[52:55], v16 offset:10240\n"
      "v_mfma_f32_16x16x32_bf16 v[180:183], v[116:119], v[92:95], v[180:183]\n"
      "ds_read_b128 v[56:59], v16 offset:12288\n"
      "v_mfma_f32_16x16x32_bf16 v[176:179], v[112:115], v[92:95], v[176:179]\n"
      "ds_read_b128 v[60:63], v16 offset:14336\n"
      "v_mfma_f32_16x16x32_bf16 v[192:195], v[112:115], v[96:99], v[192:195]\n"
      "v_mfma_f32_16x16x32_bf16 v[196:199], v[116:119], v[96:99], v[196:199]\n"
      "v_mfma_f32_16x16x32_bf16 v[200:203], v[120:123], v[96:99], v[200:203]\n"
      "v_mfma_f32_16x16x32_bf16 v[204:207], v[124:127], v[96:99], v[204:207]\n"
      "v_mfma_f32_16x16x32_bf16 v[220:223], v[124:127], v[100:103], v[220:223]\n"
      "v_mfma_f32_16x16x32_bf16 v[216:219], v[120:123], v[100:103], v[216:219]\n"
      "v_mfma_f32_16x16x32_bf16 v[212:215], v[116:119], v[100:103], v[212:215]\n"
      "v_mfma_f32_16x16x32_bf16 v[208:211], v[112:115], v[100:103], v[208:211]\n"
      "v_mfma_f32_16x16x32_bf16 v[224:227], v[112:115], v[104:107], v[224:227]\n"
      "v_mfma_f32_16x16x32_bf16 v[228:231], v[116:119], v[104:107], v[228:231]\n"
      "v_mfma_f32_16x16x32_bf16 v[232:235], v[120:123], v[104:107], v[232:235]\n"
      "v_mfma_f32_16x16x32_bf16 v[236:239], v[124:127], v[104:107], v[236:239]\n"
      "v_mfma_f32_16x16x32_bf16 v[252:255], v[124:127], v[108:111], v[252:255]\n"
      "s_add_u32 %[i], %[i], 1\n"
      "v_mfma_f32_16x16x32_bf16 v[248:251], v[120:123], v[108:111], v[248:251]\n"
      "s_cmp_lt_u32 %[i], 64\n"
      "v_mfma_f32_16x16x32_bf16 v[244:247], v[116:119], v[108:111], v[244:247]\n"
      "v_mfma_f32_16x16x32_bf16 v[240:243], v[112:115], v[108:111], v[240:243]\n"
      "s_cbranch_scc1 L_loop_%=\n"
      "s_waitcnt lgkmcnt(0)\n"
      "s_nop 15\n"
                : "=&{v[128:143]}"(c0), "=&{v[144:159]}"(c1), "=&{v[160:175]}"(c2), "=&{v[176:191]}"(c3),
                  "=&{v[192:207]}"(c4), "=&{v[208:223]}"(c5), "=&{v[224:239]}"(c6), "=&{v[240:255]}"(c7),
                  [i] "=&s"(si), [st] "=&s"(sst), [dA] "=&s"(sdA), [dB] "=&s"(sdB)
                : [a0] "v"(a0), [a1] "v"(a1), [b0] "v"(b0), [b1] "v"(b1), [b2] "v"(b2), [b3] "v"(b3),
                  [cm] "s"(0xffff8000u), [cp] "s"(16384u)
                : "v16", "v17", "v18", "v19", "v20", "v21", "v22", "v23", "v24", "v25", "v26", "v27", "v28", "v29", "v30", "v31", "v32", "v33", "v34", "v35", "v36", "v37", "v38", "v39", "v40", "v41", "v42", "v43", "v44", "v45", "v46", "v47", "v48", "v49", "v50", "v51", "v52", "v53", "v54", "v55", "v56", "v57", "v58", "v59", "v60", "v61", "v62", "v63", "v64", "v65", "v66", "v67", "v68", "v69", "v70", "v71", "v72", "v73", "v74", "v75", "v76", "v77", "v78", "v79", "v80", "v81", "v82", "v83", "v84", "v85", "v86", "v87", "v88", "v89", "v90", "v91", "v92", "v93", "v94", "v95", "v96", "v97", "v98", "v99", "v100", "v101", "v102", "v103", "v104", "v105", "v106", "v107", "v108", "v109", "v110", "v111", "v112", "v113", "v114", "v115", "v116", "v117", "v118", "v119", "v120", "v121", "v122", "v123", "v124", "v125", "v126", "v127", "vcc", "scc", "memory");
        }
        if (wid == 0) STAMP(2);
        u32 tz = threadIdx.x; asm volatile("" : "+v"(tz));
        const u32 lane = tz & 63;
        float mx = 0.f;
#define MX(c) _Pragma("unroll") for (int j = 0; j < 16; j += 2) mx = __builtin_fmaxf(mx, __builtin_fmaxf(c[j], c[j + 1]));
        MX(c0) MX(c1) MX(c2) MX(c3) MX(c4) MX(c5) MX(c6) MX(c7)
        const u32 wm = wave_umax(__float_as_uint(mx));
        lds_store(lds0 + F_MISC + wid * 4, wm);
        BAR();
        BAR();
        u32x4 am; asm volatile("ds_read_b128 %0, %1\n\ts_waitcnt lgkmcnt(0)" : "=&v"(am) : "v"(lds0 + F_MISC + 16) : "memory");
        const float amax = __uint_as_float(am.x);
        const float inv = amax > 0.f ? 1.0f / (amax / 448.0f) : 0.f;
        const u32 nanm = am.y ? 0x7f7f7f7fu : 0u;
        const u32 ea = lds0 + (lane & 15) * EP_PITCH + wid * 64 + (lane >> 4) * 4;
#define QS(c, m) _Pragma("unroll") for (int t = 0; t < 4; ++t) { \
            int w = __builtin_amdgcn_cvt_pk_fp8_f32(__builtin_fmaxf(c[4 * t], 0.f) * inv, __builtin_fmaxf(c[4 * t + 1], 0.f) * inv, 0, false); \
            w = __builtin_amdgcn_cvt_pk_fp8_f32(__builtin_fmaxf(c[4 * t + 2], 0.f) * inv, __builtin_fmaxf(c[4 * t + 3], 0.f) * inv, w, true); \
            *(LAS u32*)(uintptr_t)(ea + (m) * 16 * EP_PITCH + t * 16) = (u32)w | nanm; }
        QS(c0, 0) QS(c1, 1) QS(c2, 2) QS(c3, 3) QS(c4, 4) QS(c5, 5) QS(c6, 6) QS(c7, 7)
    } else {
        const u32 lane = threadIdx.x & 63, pw = wid - 4;
        __builtin_amdgcn_s_setprio(PPRIO);
        u32 call = 1;
        {
            const u32x4 ra = make_rsrc(p.x, ML * KD * 2), rb = make_rsrc(p.w, (u32)KD * ND * 2);
            const u32 a0 = (lane >> 3) * (KD * 2) + (((lane & 7) ^ (lane >> 4)) << 4), a1 = a0 ^ 64;
            const u32 b0 = (lane >> 5) * (ND * 2) + (((lane & 31) ^ (2 * (lane >> 5))) << 4), b1 = b0 ^ 64, b2 = b0 ^ 128, b3 = b0 ^ 192;
            const u32 px = ROTX, pq = ROTW;
            u32 sa = rfl((rowblk * BM + px * 32) * KD * 2), sb = rfl(pq * 16 * ND * 2 + (grp * NC + ct * BN) * 2);
            const u32 ma0 = rfl(lds0 + A_BASE + px * 4096), mb0 = rfl(lds0 + B_BASE + pq * 8192), mae = ma0 + 3 * A_ST;
            const u32 ssa = 128, ssb = 64 * ND * 2;
            u32 ma = ma0, mb = mb0, t, n;
            asm volatile(
      "s_cmp_lg_u32 %[pw], 3\n"
      "s_cbranch_scc1 L_s_%=\n"
      "s_mov_b64 exec, 1\n"
      "global_atomic_add %[call], %[ca], %[call], off sc0\n"
      "s_mov_b64 exec, -1\n"
      "L_s_%=:\n"
      "s_add_u32 m0, %[mb], 0\n"
      "s_add_u32 %[t], %[sb], 0x0\n"
      "buffer_load_dwordx4 %[b0], %[rb], %[t] offen  lds\n"
      "s_add_u32 m0, %[mb], 1024\n"
      "s_add_u32 %[t], %[sb], 0x10000\n"
      "buffer_load_dwordx4 %[b1], %[rb], %[t] offen  lds\n"
      "s_add_u32 m0, %[mb], 2048\n"
      "s_add_u32 %[t], %[sb], 0x20000\n"
      "buffer_load_dwordx4 %[b0], %[rb], %[t] offen  lds\n"
      "s_add_u32 m0, %[mb], 3072\n"
      "s_add_u32 %[t], %[sb], 0x30000\n"
      "buffer_load_dwordx4 %[b1], %[rb], %[t] offen  lds\n"
      "s_add_u32 m0, %[mb], 4096\n"
      "s_add_u32 %[t], %[sb], 0x40000\n"
      "buffer_load_dwordx4 %[b2], %[rb], %[t] offen  lds\n"
      "s_add_u32 m0, %[mb], 5120\n"
      "s_add_u32 %[t], %[sb], 0x50000\n"
      "buffer_load_dwordx4 %[b3], %[rb], %[t] offen  lds\n"
      "s_add_u32 m0, %[mb], 6144\n"
      "s_add_u32 %[t], %[sb], 0x60000\n"
      "buffer_load_dwordx4 %[b2], %[rb], %[t] offen  lds\n"
      "s_add_u32 m0, %[mb], 7168\n"
      "s_add_u32 %[t], %[sb], 0x70000\n"
      "buffer_load_dwordx4 %[b3], %[rb], %[t] offen  lds\n"
      "s_add_u32 m0, %[ma], 0\n"
      "s_add_u32 %[t], %[sa], 0x0\n"
      "buffer_load_dwordx4 %[a0], %[ra], %[t] offen  lds\n"
      "s_add_u32 m0, %[ma], 1024\n"
      "s_add_u32 %[t], %[sa], 0x10000\n"
      "buffer_load_dwordx4 %[a1], %[ra], %[t] offen  lds\n"
      "s_add_u32 m0, %[ma], 2048\n"
      "s_add_u32 %[t], %[sa], 0x20000\n"
      "buffer_load_dwordx4 %[a0], %[ra], %[t] offen  lds\n"
      "s_add_u32 m0, %[ma], 3072\n"
      "s_add_u32 %[t], %[sa], 0x30000\n"
      "buffer_load_dwordx4 %[a1], %[ra], %[t] offen  lds\n"
      "s_add_u32 %[sa], %[sa], %[ssa]\n"
      "s_add_u32 %[sb], %[sb], %[ssb]\n"
      "s_add_u32 %[ma], %[ma], 16384\n"
      "s_add_u32 %[mb], %[mb], 32768\n"
      "s_cmp_eq_u32 %[ma], %[mae]\n"
      "s_cselect_b32 %[ma], %[ma0], %[ma]\n"
      "s_cselect_b32 %[mb], %[mb0], %[mb]\n"
      "s_add_u32 m0, %[mb], 0\n"
      "s_add_u32 %[t], %[sb], 0x0\n"
      "buffer_load_dwordx4 %[b0], %[rb], %[t] offen  lds\n"
      "s_add_u32 m0, %[mb], 1024\n"
      "s_add_u32 %[t], %[sb], 0x10000\n"
      "buffer_load_dwordx4 %[b1], %[rb], %[t] offen  lds\n"
      "s_add_u32 m0, %[mb], 2048\n"
      "s_add_u32 %[t], %[sb], 0x20000\n"
      "buffer_load_dwordx4 %[b0], %[rb], %[t] offen  lds\n"
      "s_add_u32 m0, %[mb], 3072\n"
      "s_add_u32 %[t], %[sb], 0x30000\n"
      "buffer_load_dwordx4 %[b1], %[rb], %[t] offen  lds\n"
      "s_add_u32 m0, %[mb], 4096\n"
      "s_add_u32 %[t], %[sb], 0x40000\n"
      "buffer_load_dwordx4 %[b2], %[rb], %[t] offen  lds\n"
      "s_add_u32 m0, %[mb], 5120\n"
      "s_add_u32 %[t], %[sb], 0x50000\n"
      "buffer_load_dwordx4 %[b3], %[rb], %[t] offen  lds\n"
      "s_add_u32 m0, %[mb], 6144\n"
      "s_add_u32 %[t], %[sb], 0x60000\n"
      "buffer_load_dwordx4 %[b2], %[rb], %[t] offen  lds\n"
      "s_add_u32 m0, %[mb], 7168\n"
      "s_add_u32 %[t], %[sb], 0x70000\n"
      "buffer_load_dwordx4 %[b3], %[rb], %[t] offen  lds\n"
      "s_add_u32 m0, %[ma], 0\n"
      "s_add_u32 %[t], %[sa], 0x0\n"
      "buffer_load_dwordx4 %[a0], %[ra], %[t] offen  lds\n"
      "s_add_u32 m0, %[ma], 1024\n"
      "s_add_u32 %[t], %[sa], 0x10000\n"
      "buffer_load_dwordx4 %[a1], %[ra], %[t] offen  lds\n"
      "s_add_u32 m0, %[ma], 2048\n"
      "s_add_u32 %[t], %[sa], 0x20000\n"
      "buffer_load_dwordx4 %[a0], %[ra], %[t] offen  lds\n"
      "s_add_u32 m0, %[ma], 3072\n"
      "s_add_u32 %[t], %[sa], 0x30000\n"
      "buffer_load_dwordx4 %[a1], %[ra], %[t] offen  lds\n"
      "s_add_u32 %[sa], %[sa], %[ssa]\n"
      "s_add_u32 %[sb], %[sb], %[ssb]\n"
      "s_add_u32 %[ma], %[ma], 16384\n"
      "s_add_u32 %[mb], %[mb], 32768\n"
      "s_cmp_eq_u32 %[ma], %[mae]\n"
      "s_cselect_b32 %[ma], %[ma0], %[ma]\n"
      "s_cselect_b32 %[mb], %[mb0], %[mb]\n"
      "s_add_u32 m0, %[mb], 0\n"
      "s_add_u32 %[t], %[sb], 0x0\n"
      "buffer_load_dwordx4 %[b0], %[rb], %[t] offen  lds\n"
      "s_add_u32 m0, %[mb], 1024\n"
      "s_add_u32 %[t], %[sb], 0x10000\n"
      "buffer_load_dwordx4 %[b1], %[rb], %[t] offen  lds\n"
      "s_add_u32 m0, %[mb], 2048\n"
      "s_add_u32 %[t], %[sb], 0x20000\n"
      "buffer_load_dwordx4 %[b0], %[rb], %[t] offen  lds\n"
      "s_add_u32 m0, %[mb], 3072\n"
      "s_add_u32 %[t], %[sb], 0x30000\n"
      "buffer_load_dwordx4 %[b1], %[rb], %[t] offen  lds\n"
      "s_add_u32 m0, %[mb], 4096\n"
      "s_add_u32 %[t], %[sb], 0x40000\n"
      "buffer_load_dwordx4 %[b2], %[rb], %[t] offen  lds\n"
      "s_add_u32 m0, %[mb], 5120\n"
      "s_add_u32 %[t], %[sb], 0x50000\n"
      "buffer_load_dwordx4 %[b3], %[rb], %[t] offen  lds\n"
      "s_add_u32 m0, %[mb], 6144\n"
      "s_add_u32 %[t], %[sb], 0x60000\n"
      "buffer_load_dwordx4 %[b2], %[rb], %[t] offen  lds\n"
      "s_add_u32 m0, %[mb], 7168\n"
      "s_add_u32 %[t], %[sb], 0x70000\n"
      "buffer_load_dwordx4 %[b3], %[rb], %[t] offen  lds\n"
      "s_add_u32 m0, %[ma], 0\n"
      "s_add_u32 %[t], %[sa], 0x0\n"
      "buffer_load_dwordx4 %[a0], %[ra], %[t] offen  lds\n"
      "s_add_u32 m0, %[ma], 1024\n"
      "s_add_u32 %[t], %[sa], 0x10000\n"
      "buffer_load_dwordx4 %[a1], %[ra], %[t] offen  lds\n"
      "s_add_u32 m0, %[ma], 2048\n"
      "s_add_u32 %[t], %[sa], 0x20000\n"
      "buffer_load_dwordx4 %[a0], %[ra], %[t] offen  lds\n"
      "s_add_u32 m0, %[ma], 3072\n"
      "s_add_u32 %[t], %[sa], 0x30000\n"
      "buffer_load_dwordx4 %[a1], %[ra], %[t] offen  lds\n"
      "s_add_u32 %[sa], %[sa], %[ssa]\n"
      "s_add_u32 %[sb], %[sb], %[ssb]\n"
      "s_add_u32 %[ma], %[ma], 16384\n"
      "s_add_u32 %[mb], %[mb], 32768\n"
      "s_cmp_eq_u32 %[ma], %[mae]\n"
      "s_cselect_b32 %[ma], %[ma0], %[ma]\n"
      "s_cselect_b32 %[mb], %[mb0], %[mb]\n"
      "s_waitcnt vmcnt(24)\n"
      "s_barrier\n"
      "s_waitcnt vmcnt(12)\n"
      "s_barrier\n"
      "s_mov_b32 %[n], 3\n"
      "L_p_%=:\n"
      "s_add_u32 m0, %[mb], 0\n"
      "s_add_u32 %[t], %[sb], 0x0\n"
      "buffer_load_dwordx4 %[b0], %[rb], %[t] offen  lds\n"
      "s_add_u32 m0, %[mb], 1024\n"
      "s_add_u32 %[t], %[sb], 0x10000\n"
      "buffer_load_dwordx4 %[b1], %[rb], %[t] offen  lds\n"
      "s_add_u32 m0, %[mb], 2048\n"
      "s_add_u32 %[t], %[sb], 0x20000\n"
      "buffer_load_dwordx4 %[b0], %[rb], %[t] offen  lds\n"
      "s_add_u32 m0, %[mb], 3072\n"
      "s_add_u32 %[t], %[sb], 0x30000\n"
      "buffer_load_dwordx4 %[b1], %[rb], %[t] offen  lds\n"
      "s_add_u32 m0, %[mb], 4096\n"
      "s_add_u32 %[t], %[sb], 0x40000\n"
      "buffer_load_dwordx4 %[b2], %[rb], %[t] offen  lds\n"
      "s_add_u32 m0, %[mb], 5120\n"
      "s_add_u32 %[t], %[sb], 0x50000\n"
      "buffer_load_dwordx4 %[b3], %[rb], %[t] offen  lds\n"
      "s_add_u32 m0, %[mb], 6144\n"
      "s_add_u32 %[t], %[sb], 0x60000\n"
      "buffer_load_dwordx4 %[b2], %[rb], %[t] offen  lds\n"
      "s_add_u32 m0, %[mb], 7168\n"
      "s_add_u32 %[t], %[sb], 0x70000\n"
      "buffer_load_dwordx4 %[b3], %[rb], %[t] offen  lds\n"
      "s_add_u32 m0, %[ma], 0\n"
      "s_add_u32 %[t], %[sa], 0x0\n"
      "buffer_load_dwordx4 %[a0], %[ra], %[t] offen  lds\n"
      "s_add_u32 m0, %[ma], 1024\n"
      "s_add_u32 %[t], %[sa], 0x10000\n"
      "buffer_load_dwordx4 %[a1], %[ra], %[t] offen  lds\n"
      "s_add_u32 m0, %[ma], 2048\n"
      "s_add_u32 %[t], %[sa], 0x20000\n"
      "buffer_load_dwordx4 %[a0], %[ra], %[t] offen  lds\n"
      "s_add_u32 m0, %[ma], 3072\n"
      "s_add_u32 %[t], %[sa], 0x30000\n"
      "buffer_load_dwordx4 %[a1], %[ra], %[t] offen  lds\n"
      "s_add_u32 %[sa], %[sa], %[ssa]\n"
      "s_add_u32 %[sb], %[sb], %[ssb]\n"
      "s_add_u32 %[ma], %[ma], 16384\n"
      "s_add_u32 %[mb], %[mb], 32768\n"
      "s_cmp_eq_u32 %[ma], %[mae]\n"
      "s_cselect_b32 %[ma], %[ma0], %[ma]\n"
      "s_cselect_b32 %[mb], %[mb0], %[mb]\n"
      "s_add_u32 %[n], %[n], 1\n"
      "s_cmp_lt_u32 %[n], 64\n"
      "s_waitcnt vmcnt(12)\n"
      "s_barrier\n"
      "s_cbranch_scc1 L_p_%=\n"
      "s_waitcnt vmcnt(0)\n"
      "s_barrier\n"
      "s_barrier\n"
                : [t] "=&s"(t), [n] "=&s"(n), [sa] "+&s"(sa), [sb] "+&s"(sb), [ma] "+&s"(ma), [mb] "+&s"(mb), [call] "+&v"(call)
                : [a0] "v"(a0), [a1] "v"(a1), [b0] "v"(b0), [b1] "v"(b1), [b2] "v"(b2), [b3] "v"(b3), [ra] "s"(ra), [rb] "s"(rb),
                  [ssa] "s"(ssa), [ssb] "s"(ssb), [ma0] "s"(ma0), [mb0] "s"(mb0), [mae] "s"(mae), [pw] "s"(pw), [ca] "v"(myws + WS_CALL + b * 4)
                : "memory", "scc");
        }
        if (wid == 5) STAMP(1);
      if (wid < 7) {
        BAR();
        BAR();
      } else {
        call = rfl(call) + 1;
        if (lane == 0) lds_store(lds0 + F_MISC + 24, call);
        BAR();
        u32x4 wm; asm volatile("ds_read_b128 %0, %1\n\ts_waitcnt lgkmcnt(0)" : "=&v"(wm) : "v"(lds0 + F_MISC) : "memory");
        const u32 tm = max(max(wm.x, wm.y), max(wm.z, wm.w));
        bool bad = false;
        if (lane == 0) st_sys64(myws + WS_LMAX + (grp * 32 + idx) * 8, ((u64)call << 32) | tm);
        if (idx == 0) {
            const u32 gm = sweep(myws + WS_LMAX + grp * 256, 32, call, myws + WS_TMO, lane, bad);
            if (!bad && lane < NR) st_sys64(p.ws[lane] + WS_GMAX + (call & 1) * 512 + (p.rank * 8 + grp) * 8, ((u64)call << 32) | gm);
        }
        const u32 am = sweep(myws + WS_GMAX + (call & 1) * 512, 64, call, myws + WS_TMO, lane, bad);
        if (lane == 0) { lds_store(lds0 + F_MISC + 16, am); lds_store(lds0 + F_MISC + 20, bad ? 1u : 0u); }
        STAMP(3);
        BAR();
      }
    }
    BAR();
    {
        u32 tz = threadIdx.x; asm volatile("" : "+v"(tz));
        const u32 lane = tz & 63;
        unsigned char* dst = p.out[grp] + (size_t)(p.rank * ML + rowblk * BM + wid * 16 + (lane >> 4)) * NC + ct * BN + (lane & 15) * 16;
        const u32 la = lds0 + (wid * 16 + (lane >> 4)) * EP_PITCH + (lane & 15) * 16;
        u32x4 d0, d1, d2, d3;
        asm volatile("ds_read_b128 %0, %4\n\tds_read_b128 %1, %4 offset:1088\n\tds_read_b128 %2, %4 offset:2176\n\tds_read_b128 %3, %4 offset:3264\n\ts_waitcnt lgkmcnt(0)"
                     : "=&v"(d0), "=&v"(d1), "=&v"(d2), "=&v"(d3) : "v"(la) : "memory");
        asm volatile("global_store_dwordx4 %0, %1, off sc0 sc1\n\ts_nop 1" :: "v"(dst), "v"(d0) : "memory");
        asm volatile("global_store_dwordx4 %0, %1, off sc0 sc1\n\ts_nop 1" :: "v"(dst + 4 * NC), "v"(d1) : "memory");
        asm volatile("global_store_dwordx4 %0, %1, off sc0 sc1\n\ts_nop 1" :: "v"(dst + 8 * NC), "v"(d2) : "memory");
        asm volatile("global_store_dwordx4 %0, %1, off sc0 sc1\n\ts_nop 1" :: "v"(dst + 12 * NC), "v"(d3) : "memory");
        asm volatile("s_waitcnt vmcnt(0)" ::: "memory");
        BAR();
        if (wid == 7) {
            STAMP(4);
            u32 call; asm volatile("ds_read_b32 %0, %1\n\ts_waitcnt lgkmcnt(0)" : "=&v"(call) : "v"(lds0 + F_MISC + 24) : "memory");
            call = rfl(call);
            if (lane == 0 && grp != p.rank) st_sys32(p.ws[grp] + WS_DONE + (p.rank * 32 + idx) * 4, call);
            if (grp == p.rank && idx < NR && idx != p.rank) {
                const u64 t0 = __builtin_amdgcn_s_memrealtime();
                const unsigned char* q = myws + WS_DONE + (idx * 32 + (lane & 31)) * 4;
                for (;;) {
                    const u32 v = ld_sys32(q);
                    const u32 t = ld_agt32(myws + WS_TMO);
                    if (t != call && __all((int)(v - call) >= 0)) break;
                    if (t == call || __builtin_amdgcn_s_memrealtime() - t0 > DEADLINE) {
                        if (lane == 0) { st_agt32(myws + WS_TMO, call); *dst = 0x7f; }
                        break;
                    }
                    __builtin_amdgcn_s_sleep(1);
                }
            }
            STAMP(5);
        }
    }
}

extern "C" void kernel_launch(int rank, int nranks, ncclComm_t comm, hipStream_t stream,
                              void* const* d_in, const int* in_sizes, int n_in,
                              void* d_out, int out_size, const int* in_dtypes, int out_dtype,
                              void* const* d_out_all, void* const* d_ws_all, size_t ws_bytes) {
    if (nranks != NR || n_in < 2 || in_sizes[0] != ML * KD || in_sizes[1] != KD * ND || out_size != NR * ML * NC || ws_bytes < WS_NEED) {
        fprintf(stderr, "kernel_launch: unexpected shapes\n"); return;
    }
    static bool attr[NR] = {};
    if (!attr[rank]) { HIP_CHECK(hipFuncSetAttribute((const void*)fused_kernel, hipFuncAttributeMaxDynamicSharedMemorySize, LDS_BYTES)); attr[rank] = true; }
    Params p;
    p.x = d_in[0]; p.w = d_in[1]; p.rank = rank; p.pad = 0;
    for (int i = 0; i < NR; ++i) { p.out[i] = (unsigned char*)d_out_all[i]; p.ws[i] = (unsigned char*)d_ws_all[i]; }
    hipLaunchKernelGGL(fused_kernel, dim3(NWG), dim3(512), LDS_BYTES, stream, p);
    HIP_CHECK(hipGetLastError());
}
```

```cpp
DI void issue_a1(u32x4 rs, u32 v0, u32 v1, u32 mb, u32 sb) {
    u32 t;
    asm volatile(
        DMA(v0, 0, 0x0) DMA(v1, 1024, 0x10000) DMA(v0, 2048, 0x20000) DMA(v1, 3072, 0x30000)
        : [t] "=&s"(t) : [v0] "v"(v0), [v1] "v"(v1), [rs] "s"(rs), [mb] "s"(mb), [sb] "s"(sb) : "memory", "scc");
}
DI void issue_a2(u32x4 rs, u32 v0, u32 v1, u32 mb, u32 sb) {
    u32 t;
    asm volatile(
        DMA(v0, 4096, 0x40000) DMA(v1, 5120, 0x50000) DMA(v0, 6144, 0x60000) DMA(v1, 7168, 0x70000) DMA(v0, 8192, 0x80000) DMA(v1, 9216, 0x90000) DMA(v0, 10240, 0xa0000) DMA(v1, 11264, 0xb0000) DMA(v0, 12288, 0xc0000) DMA(v1, 13312, 0xd0000) DMA(v0, 14336, 0xe0000) DMA(v1, 15360, 0xf0000)
        : [t] "=&s"(t) : [v0] "v"(v0), [v1] "v"(v1), [rs] "s"(rs), [mb] "s"(mb), [sb] "s"(sb) : "memory", "scc");
}
DI void issue_b1(u32x4 rs, u32 v0, u32 v1, u32 mb, u32 sb) {
    u32 t;
    asm volatile(
        DMA(v0, 0, 0x0) DMA(v0, 2048, 0x20000) DMA(v1, 4096, 0x40000) DMA(v1, 6144, 0x60000)
        : [t] "=&s"(t) : [v0] "v"(v0), [v1] "v"(v1), [rs] "s"(rs), [mb] "s"(mb), [sb] "s"(sb) : "memory", "scc");
}
DI void issue_b2(u32x4 rs, u32 v0, u32 v1, u32 mb, u32 sb) {
    u32 t;
    asm volatile(
        DMA(v0, 8192, 0x80000) DMA(v0, 10240, 0xa0000) DMA(v1, 12288, 0xc0000) DMA(v1, 14336, 0xe0000) DMA(v0, 16384, 0x100000) DMA(v0, 18432, 0x120000) DMA(v1, 20480, 0x140000) DMA(v1, 22528, 0x160000) DMA(v0, 24576, 0x180000) DMA(v0, 26624, 0x1a0000) DMA(v1, 28672, 0x1c0000) DMA(v1, 30720, 0x1e0000)
        : [t] "=&s"(t) : [v0] "v"(v0), [v1] "v"(v1), [rs] "s"(rs), [mb] "s"(mb), [sb] "s"(sb) : "memory", "scc");
}

_m4096_k4096_n16384_bf16_reluqe4m3 — fused GEMM + ReLU + global amax + e4m3 quantize + all-to-all, one kernel.
#include <hip/hip_runtime.h>
#include <rccl/rccl.h>
#include <cstdio>
#include <cstdint>

#define HIP_CHECK(expr) do { hipError_t _e = (expr); if (_e != hipSuccess) { \
    fprintf(stderr, "HIP error %s at %s:%d\n", hipGetErrorString(_e), __FILE__, __LINE__); \
    return; } } while (0)

typedef unsigned u32;
typedef unsigned long long u64;
typedef float f32x16 __attribute__((ext_vector_type(16)));
typedef u32 u32x4 __attribute__((ext_vector_type(4)));
#define LAS __attribute__((address_space(3)))
#define GAS __attribute__((address_space(1)))
#define DI __device__ __forceinline__

constexpr int NR = 8, ML = 512, KD = 4096, ND = 16384, NC = 2048;
constexpr int BM = 128, BN = 256, NT = 64;
constexpr int NWG = 256;
constexpr u32 A_BASE = 0, A_ST = 16384, B_BASE = 49152, B_ST = 32768;
constexpr u32 F_FULL = 147456, F_DONE = F_FULL + 1024, F_MISC = F_DONE + 1024, LDS_BYTES = F_MISC + 1024;
constexpr u32 EP_PITCH = 272;
constexpr size_t WS_CALL = 0;
constexpr size_t WS_TMO = 4096;
constexpr size_t WS_LMAX = 8192;
constexpr size_t WS_GMAX = 12288;
constexpr size_t WS_DONE = 16384;
constexpr size_t WS_DBG = 65536;
constexpr size_t WS_DBG2 = WS_DBG + 256 * 64 * 8;
constexpr size_t WS_NEED = WS_DBG2 + 256 * 4 * 128 * 4;
constexpr u64 DEADLINE = 50000000ull;

struct Params {
    const void* x; const void* w;
    unsigned char* out[NR];
    unsigned char* ws[NR];
    int rank; int pad;
};

DI u32 rfl(u32 v) { return __builtin_amdgcn_readfirstlane(v); }
DI u32x4 make_rsrc(const void* p, u32 bytes) {
    u64 a = (u64)p; u32x4 r; r.x = rfl((u32)a); r.y = rfl((u32)(a >> 32)); r.z = bytes; r.w = 0x00020000u; return r;
}
DI void lds_store(u32 addr, u32 v) { asm volatile("ds_write_b32 %0, %1" :: "v"(addr), "v"(v) : "memory"); }
DI u32 lds_min4(u32 addr) {
    u32x4 d; asm volatile("ds_read_b128 %0, %1\n\ts_waitcnt lgkmcnt(0)" : "=&v"(d) : "v"(addr) : "memory");
    return rfl(min(min(d.x, d.y), min(d.z, d.w)));
}
#define BAR() do { asm volatile("s_waitcnt lgkmcnt(0)\n\ts_barrier" ::: "memory"); } while (0)
#define NSTAMP 8
#define CPRIO 2
#define PPRIO 0
#define ROWBLK (idx & 3)
#define COLT (idx >> 2)
#define PFW 0
#define PFX 0
#define PFS 0
#define ROTX pw
#define ROTW pw
#define WPAIR 1
#define LAG 0
#define RW ((rowblk >> 1) * 4)
#define RX ((ct >> 2) * 2)
#define KX ((rowblk ^ (ct >> 1)) & 1)
#define KP (ct & 1)
#define PSLEEP 1
#define STAMP(k) do {} while (0)

DI u64 ld_sys64(const unsigned char* p) { return __hip_atomic_load((const GAS u64*)p, __ATOMIC_RELAXED, __HIP_MEMORY_SCOPE_SYSTEM); }
DI void st_sys64(unsigned char* p, u64 v) { __hip_atomic_store((GAS u64*)p, v, __ATOMIC_RELAXED, __HIP_MEMORY_SCOPE_SYSTEM); }
DI u32 ld_sys32(const unsigned char* p) { return __hip_atomic_load((const GAS u32*)p, __ATOMIC_RELAXED, __HIP_MEMORY_SCOPE_SYSTEM); }
DI void st_sys32(unsigned char* p, u32 v) { __hip_atomic_store((GAS u32*)p, v, __ATOMIC_RELAXED, __HIP_MEMORY_SCOPE_SYSTEM); }
DI u32 ld_agt32(const unsigned char* p) { return __hip_atomic_load((const GAS u32*)p, __ATOMIC_RELAXED, __HIP_MEMORY_SCOPE_AGENT); }
DI void st_agt32(unsigned char* p, u32 v) { __hip_atomic_store((GAS u32*)p, v, __ATOMIC_RELAXED, __HIP_MEMORY_SCOPE_AGENT); }
DI u32 wave_umax(u32 v) {
#pragma unroll
    for (int o = 32; o >= 1; o >>= 1) v = max(v, (u32)__shfl_xor((int)v, o));
    return v;
}

DI u32 sweep(const unsigned char* g, int n, u32 call, unsigned char* tmo, int lane, bool& bad) {
    const u64 t0 = __builtin_amdgcn_s_memrealtime();
    const unsigned char* q = g + (lane < n ? lane : 0) * 8;
    for (;;) {
        const u64 v = ld_sys64(q);
        const u32 t = ld_agt32(tmo);
        if (t == call) { bad = true; return 0; }
        if (__all((u32)(v >> 32) == call)) return wave_umax((u32)v);
        if (__builtin_amdgcn_s_memrealtime() - t0 > DEADLINE) { if (lane == 0) st_agt32(tmo, call); bad = true; return 0; }
        __builtin_amdgcn_s_sleep(1);
    }
}

__global__ __launch_bounds__(512, 2) void fused_kernel(Params p) {
    extern __shared__ __attribute__((aligned(16))) unsigned char smem[];
    const int wid = rfl(threadIdx.x >> 6);
    const int b = blockIdx.x, grp = b & 7, idx = b >> 3;
    const int rowblk = ROWBLK, ct = COLT;
    const u32 lds0 = (u32)(uintptr_t)(LAS unsigned char*)smem;
    unsigned char* const myws = p.ws[p.rank];
    if (wid == 0) STAMP(0);

    if (wid < 4) {
        f32x16 c0, c1, c2, c3, c4, c5, c6, c7; u32 slowc; u64 tc0 = 0;
        {
            const u32 lane = threadIdx.x & 63, r = lane & 15, g = lane >> 4, q = r >> 2, pp = lane & 3;
            const u32 a0 = lds0 + A_BASE + r * 128 + ((g ^ (r >> 1)) << 4), a1 = a0 ^ 64;
            const u32 bb = lds0 + B_BASE + (8 * g + q) * 512 + (pp & 1) * 8, F = 2 * q + 8 * (g & 1), cb = 8 * wid + (pp >> 1);
            const u32 b0 = bb + (((cb + 0) ^ F) << 4), b1 = bb + (((cb + 2) ^ F) << 4), b2 = bb + (((cb + 4) ^ F) << 4), b3 = bb + (((cb + 6) ^ F) << 4);
            u32 si, sst, sdA, sdB; slowc = 0;
            __builtin_amdgcn_s_setprio(CPRIO);
            asm volatile(
      "v_mov_b32 v16, %[a0]\n"
      "v_mov_b32 v17, %[a1]\n"
      "v_mov_b32 v18, %[b0]\n"
      "v_mov_b32 v19, %[b1]\n"
      "v_mov_b32 v20, %[b2]\n"
      "v_mov_b32 v21, %[b3]\n"
      "v_mov_b32 v128, 0\n"
      "v_mov_b32 v129, 0\n"
      "v_mov_b32 v130, 0\n"
      "v_mov_b32 v131, 0\n"
      "v_mov_b32 v132, 0\n"
      "v_mov_b32 v133, 0\n"
      "v_mov_b32 v134, 0\n"
      "v_mov_b32 v135, 0\n"
      "v_mov_b32 v136, 0\n"
      "v_mov_b32 v137, 0\n"
      "v_mov_b32 v138, 0\n"
      "v_mov_b32 v139, 0\n"
      "v_mov_b32 v140, 0\n"
      "v_mov_b32 v141, 0\n"
      "v_mov_b32 v142, 0\n"
      "v_mov_b32 v143, 0\n"
      "v_mov_b32 v144, 0\n"
      "v_mov_b32 v145, 0\n"
      "v_mov_b32 v146, 0\n"
      "v_mov_b32 v147, 0\n"
      "v_mov_b32 v148, 0\n"
      "v_mov_b32 v149, 0\n"
      "v_mov_b32 v150, 0\n"
      "v_mov_b32 v151, 0\n"
      "v_mov_b32 v152, 0\n"
      "v_mov_b32 v153, 0\n"
      "v_mov_b32 v154, 0\n"
      "v_mov_b32 v155, 0\n"
      "v_mov_b32 v156, 0\n"
      "v_mov_b32 v157, 0\n"
      "v_mov_b32 v158, 0\n"
      "v_mov_b32 v159, 0\n"
      "v_mov_b32 v160, 0\n"
      "v_mov_b32 v161, 0\n"
      "v_mov_b32 v162, 0\n"
      "v_mov_b32 v163, 0\n"
      "v_mov_b32 v164, 0\n"
      "v_mov_b32 v165, 0\n"
      "v_mov_b32 v166, 0\n"
      "v_mov_b32 v167, 0\n"
      "v_mov_b32 v168, 0\n"
      "v_mov_b32 v169, 0\n"
      "v_mov_b32 v170, 0\n"
      "v_mov_b32 v171, 0\n"
      "v_mov_b32 v172, 0\n"
      "v_mov_b32 v173, 0\n"
      "v_mov_b32 v174, 0\n"
      "v_mov_b32 v175, 0\n"
      "v_mov_b32 v176, 0\n"
      "v_mov_b32 v177, 0\n"
      "v_mov_b32 v178, 0\n"
      "v_mov_b32 v179, 0\n"
      "v_mov_b32 v180, 0\n"
      "v_mov_b32 v181, 0\n"
      "v_mov_b32 v182, 0\n"
      "v_mov_b32 v183, 0\n"
      "v_mov_b32 v184, 0\n"
      "v_mov_b32 v185, 0\n"
      "v_mov_b32 v186, 0\n"
      "v_mov_b32 v187, 0\n"
      "v_mov_b32 v188, 0\n"
      "v_mov_b32 v189, 0\n"
      "v_mov_b32 v190, 0\n"
      "v_mov_b32 v191, 0\n"
      "v_mov_b32 v192, 0\n"
      "v_mov_b32 v193, 0\n"
      "v_mov_b32 v194, 0\n"
      "v_mov_b32 v195, 0\n"
      "v_mov_b32 v196, 0\n"
      "v_mov_b32 v197, 0\n"
      "v_mov_b32 v198, 0\n"
      "v_mov_b32 v199, 0\n"
      "v_mov_b32 v200, 0\n"
      "v_mov_b32 v201, 0\n"
      "v_mov_b32 v202, 0\n"
      "v_mov_b32 v203, 0\n"
      "v_mov_b32 v204, 0\n"
      "v_mov_b32 v205, 0\n"
      "v_mov_b32 v206, 0\n"
      "v_mov_b32 v207, 0\n"
      "v_mov_b32 v208, 0\n"
      "v_mov_b32 v209, 0\n"
      "v_mov_b32 v210, 0\n"
      "v_mov_b32 v211, 0\n"
      "v_mov_b32 v212, 0\n"
      "v_mov_b32 v213, 0\n"
      "v_mov_b32 v214, 0\n"
      "v_mov_b32 v215, 0\n"
      "v_mov_b32 v216, 0\n"
      "v_mov_b32 v217, 0\n"
      "v_mov_b32 v218, 0\n"
      "v_mov_b32 v219, 0\n"
      "v_mov_b32 v220, 0\n"
      "v_mov_b32 v221, 0\n"
      "v_mov_b32 v222, 0\n"
      "v_mov_b32 v223, 0\n"
      "v_mov_b32 v224, 0\n"
      "v_mov_b32 v225, 0\n"
      "v_mov_b32 v226, 0\n"
      "v_mov_b32 v227, 0\n"
      "v_mov_b32 v228, 0\n"
      "v_mov_b32 v229, 0\n"
      "v_mov_b32 v230, 0\n"
      "v_mov_b32 v231, 0\n"
      "v_mov_b32 v232, 0\n"
      "v_mov_b32 v233, 0\n"
      "v_mov_b32 v234, 0\n"
      "v_mov_b32 v235, 0\n"
      "v_mov_b32 v236, 0\n"
      "v_mov_b32 v237, 0\n"
      "v_mov_b32 v238, 0\n"
      "v_mov_b32 v239, 0\n"
      "v_mov_b32 v240, 0\n"
      "v_mov_b32 v241, 0\n"
      "v_mov_b32 v242, 0\n"
      "v_mov_b32 v243, 0\n"
      "v_mov_b32 v244, 0\n"
      "v_mov_b32 v245, 0\n"
      "v_mov_b32 v246, 0\n"
      "v_mov_b32 v247, 0\n"
      "v_mov_b32 v248, 0\n"
      "v_mov_b32 v249, 0\n"
      "v_mov_b32 v250, 0\n"
      "v_mov_b32 v251, 0\n"
      "v_mov_b32 v252, 0\n"
      "v_mov_b32 v253, 0\n"
      "v_mov_b32 v254, 0\n"
      "v_mov_b32 v255, 0\n"
      "s_mov_b32 %[i], 0\n"
      "s_mov_b32 %[st], 0\n"
      "s_barrier\n"
      "ds_read_b64_tr_b16 v[64:65], v18 offset:0\n"
      "ds_read_b64_tr_b16 v[66:67], v18 offset:2048\n"
      "ds_read_b128 v[32:35], v16 offset:0\n"
      "ds_read_b64_tr_b16 v[68:69], v19 offset:0\n"
      "ds_read_b64_tr_b16 v[70:71], v19 offset:2048\n"
      "ds_read_b64_tr_b16 v[72:73], v20 offset:0\n"
      "ds_read_b64_tr_b16 v[74:75], v20 offset:2048\n"
      "ds_read_b64_tr_b16 v[76:77], v21 offset:0\n"
      "ds_read_b64_tr_b16 v[78:79], v21 offset:2048\n"
      "ds_read_b128 v[36:39], v16 offset:2048\n"
      "ds_read_b128 v[40:43], v16 offset:4096\n"
      "ds_read_b128 v[44:47], v16 offset:6144\n"
      "ds_read_b128 v[48:51], v16 offset:8192\n"
      "ds_read_b128 v[52:55], v16 offset:10240\n"
      "ds_read_b128 v[56:59], v16 offset:12288\n"
      "ds_read_b128 v[60:63], v16 offset:14336\n"
      "s_waitcnt lgkmcnt(0)\n"
      ".p2align 4\n"
      "L_loop_%=:\n"
      "s_waitcnt lgkmcnt(0)\n"
      "v_mfma_f32_16x16x32_bf16 v[128:131], v[64:67], v[32:35], v[128:131]\n"
      "ds_read_b64_tr_b16 v[112:113], v18 offset:16384\n"
      "v_mfma_f32_16x16x32_bf16 v[132:135], v[68:71], v[32:35], v[132:135]\n"
      "ds_read_b64_tr_b16 v[114:115], v18 offset:18432\n"
      "s_cmp_eq_u32 %[st], 2\n"
      "v_mfma_f32_16x16x32_bf16 v[136:139], v[72:75], v[32:35], v[136:139]\n"
      "ds_read_b128 v[80:83], v17 offset:0\n"
      "s_cselect_b32 %[dA], %[cm], %[cp]\n"
      "v_mfma_f32_16x16x32_bf16 v[140:143], v[76:79], v[32:35], v[140:143]\n"
      "ds_read_b64_tr_b16 v[116:117], v19 offset:16384\n"
      "s_cselect_b32 %[st], -1, %[st]\n"
      "v_mfma_f32_16x16x32_bf16 v[156:159], v[76:79], v[36:39], v[156:159]\n"
      "ds_read_b64_tr_b16 v[118:119], v19 offset:18432\n"
      "s_add_u32 %[st], %[st], 1\n"
      "v_mfma_f32_16x16x32_bf16 v[152:155], v[72:75], v[36:39], v[152:155]\n"
      "ds_read_b64_tr_b16 v[120:121], v20 offset:16384\n"
      "s_lshl_b32 %[dB], %[dA], 1\n"
      "v_mfma_f32_16x16x32_bf16 v[148:151], v[68:71], v[36:39], v[148:151]\n"
      "ds_read_b64_tr_b16 v[122:123], v20 offset:18432\n"
      "v_mfma_f32_16x16x32_bf16 v[144:147], v[64:67], v[36:39], v[144:147]\n"
      "ds_read_b64_tr_b16 v[124:125], v21 offset:16384\n"
      "v_mfma_f32_16x16x32_bf16 v[160:163], v[64:67], v[40:43], v[160:163]\n"
      "ds_read_b64_tr_b16 v[126:127], v21 offset:18432\n"
      "v_mfma_f32_16x16x32_bf16 v[164:167], v[68:71], v[40:43], v[164:167]\n"
      "ds_read_b128 v[84:87], v17 offset:2048\n"
      "v_mfma_f32_16x16x32_bf16 v[168:171], v[72:75], v[40:43], v[168:171]\n"
      "ds_read_b128 v[88:91], v17 offset:4096\n"
      "v_mfma_f32_16x16x32_bf16 v[172:175], v[76:79], v[40:43], v[172:175]\n"
      "ds_read_b128 v[92:95], v17 offset:6144\n"
      "v_mfma_f32_16x16x32_bf16 v[188:191], v[76:79], v[44:47], v[188:191]\n"
      "ds_read_b128 v[96:99], v17 offset:8192\n"
      "v_mfma_f32_16x16x32_bf16 v[184:187], v[72:75], v[44:47], v[184:187]\n"
      "ds_read_b128 v[100:103], v17 offset:10240\n"
      "v_mfma_f32_16x16x32_bf16 v[180:183], v[68:71], v[44:47], v[180:183]\n"
      "ds_read_b128 v[104:107], v17 offset:12288\n"
      "v_mfma_f32_16x16x32_bf16 v[176:179], v[64:67], v[44:47], v[176:179]\n"
      "ds_read_b128 v[108:111], v17 offset:14336\n"
      "v_mfma_f32_16x16x32_bf16 v[192:195], v[64:67], v[48:51], v[192:195]\n"
      "v_add_u32 v16, %[dA], v16\n"
      "v_mfma_f32_16x16x32_bf16 v[196:199], v[68:71], v[48:51], v[196:199]\n"
      "v_add_u32 v17, %[dA], v17\n"
      "v_mfma_f32_16x16x32_bf16 v[200:203], v[72:75], v[48:51], v[200:203]\n"
      "v_add_u32 v18, %[dB], v18\n"
      "v_mfma_f32_16x16x32_bf16 v[204:207], v[76:79], v[48:51], v[204:207]\n"
      "v_add_u32 v19, %[dB], v19\n"
      "v_mfma_f32_16x16x32_bf16 v[220:223], v[76:79], v[52:55], v[220:223]\n"
      "v_add_u32 v20, %[dB], v20\n"
      "v_mfma_f32_16x16x32_bf16 v[216:219], v[72:75], v[52:55], v[216:219]\n"
      "v_add_u32 v21, %[dB], v21\n"
      "v_mfma_f32_16x16x32_bf16 v[212:215], v[68:71], v[52:55], v[212:215]\n"
      "v_mfma_f32_16x16x32_bf16 v[208:211], v[64:67], v[52:55], v[208:211]\n"
      "v_mfma_f32_16x16x32_bf16 v[224:227], v[64:67], v[56:59], v[224:227]\n"
      "s_waitcnt lgkmcnt(0)\n"
      "s_barrier\n"
      "v_mfma_f32_16x16x32_bf16 v[228:231], v[68:71], v[56:59], v[228:231]\n"
      "v_mfma_f32_16x16x32_bf16 v[232:235], v[72:75], v[56:59], v[232:235]\n"
      "v_mfma_f32_16x16x32_bf16 v[236:239], v[76:79], v[56:59], v[236:239]\n"
      "v_mfma_f32_16x16x32_bf16 v[252:255], v[76:79], v[60:63], v[252:255]\n"
      "v_mfma_f32_16x16x32_bf16 v[248:251], v[72:75], v[60:63], v[248:251]\n"
      "v_mfma_f32_16x16x32_bf16 v[244:247], v[68:71], v[60:63], v[244:247]\n"
      "v_mfma_f32_16x16x32_bf16 v[240:243], v[64:67], v[60:63], v[240:243]\n"
      "v_mfma_f32_16x16x32_bf16 v[128:131], v[112:115], v[80:83], v[128:131]\n"
      "ds_read_b64_tr_b16 v[64:65], v18 offset:0\n"
      "v_mfma_f32_16x16x32_bf16 v[132:135], v[116:119], v[80:83], v[132:135]\n"
      "ds_read_b64_tr_b16 v[66:67], v18 offset:2048\n"
      "v_mfma_f32_16x16x32_bf16 v[136:139], v[120:123], v[80:83], v[136:139]\n"
      "ds_read_b128 v[32:35], v16 offset:0\n"
      "v_mfma_f32_16x16x32_bf16 v[140:143], v[124:127], v[80:83], v[140:143]\n"
      "ds_read_b64_tr_b16 v[68:69], v19 offset:0\n"
      "v_mfma_f32_16x16x32_bf16 v[156:159], v[124:127], v[84:87], v[156:159]\n"
      "ds_read_b64_tr_b16 v[70:71], v19 offset:2048\n"
      "v_mfma_f32_16x16x32_bf16 v[152:155], v[120:123], v[84:87], v[152:155]\n"
      "ds_read_b64_tr_b16 v[72:73], v20 offset:0\n"
      "v_mfma_f32_16x16x32_bf16 v[148:151], v[116:119], v[84:87], v[148:151]\n"
      "ds_read_b64_tr_b16 v[74:75], v20 offset:2048\n"
      "v_mfma_f32_16x16x32_bf16 v[144:147], v[112:115], v[84:87], v[144:147]\n"
      "ds_read_b64_tr_b16 v[76:77], v21 offset:0\n"
      "v_mfma_f32_16x16x32_bf16 v[160:163], v[112:115], v[88:91], v[160:163]\n"
      "ds_read_b64_tr_b16 v[78:79], v21 offset:2048\n"
      "v_mfma_f32_16x16x32_bf16 v[164:167], v[116:119], v[88:91], v[164:167]\n"
      "ds_read_b128 v[36:39], v16 offset:2048\n"
      "v_mfma_f32_16x16x32_bf16 v[168:171], v[120:123], v[88:91], v[168:171]\n"
      "ds_read_b128 v[40:43], v16 offset:4096\n"
      "v_mfma_f32_16x16x32_bf16 v[172:175], v[124:127], v[88:91], v[172:175]\n"
      "ds_read_b128 v[44:47], v16 offset:6144\n"
      "v_mfma_f32_16x16x32_bf16 v[188:191], v[124:127], v[92:95], v[188:191]\n"
      "ds_read_b128 v[48:51], v16 offset:8192\n"
      "v_mfma_f32_16x16x32_bf16 v[184:187], v[120:123], v[92:95], v[184:187]\n"
      "ds_read_b128 v[52:55], v16 offset:10240\n"
      "v_mfma_f32_16x16x32_bf16 v[180:183], v[116:119], v[92:95], v[180:183]\n"
      "ds_read_b128 v[56:59], v16 offset:12288\n"
      "v_mfma_f32_16x16x32_bf16 v[176:179], v[112:115], v[92:95], v[176:179]\n"
      "ds_read_b128 v[60:63], v16 offset:14336\n"
      "v_mfma_f32_16x16x32_bf16 v[192:195], v[112:115], v[96:99], v[192:195]\n"
      "v_mfma_f32_16x16x32_bf16 v[196:199], v[116:119], v[96:99], v[196:199]\n"
      "v_mfma_f32_16x16x32_bf16 v[200:203], v[120:123], v[96:99], v[200:203]\n"
      "v_mfma_f32_16x16x32_bf16 v[204:207], v[124:127], v[96:99], v[204:207]\n"
      "v_mfma_f32_16x16x32_bf16 v[220:223], v[124:127], v[100:103], v[220:223]\n"
      "v_mfma_f32_16x16x32_bf16 v[216:219], v[120:123], v[100:103], v[216:219]\n"
      "v_mfma_f32_16x16x32_bf16 v[212:215], v[116:119], v[100:103], v[212:215]\n"
      "v_mfma_f32_16x16x32_bf16 v[208:211], v[112:115], v[100:103], v[208:211]\n"
      "v_mfma_f32_16x16x32_bf16 v[224:227], v[112:115], v[104:107], v[224:227]\n"
      "v_mfma_f32_16x16x32_bf16 v[228:231], v[116:119], v[104:107], v[228:231]\n"
      "v_mfma_f32_16x16x32_bf16 v[232:235], v[120:123], v[104:107], v[232:235]\n"
      "v_mfma_f32_16x16x32_bf16 v[236:239], v[124:127], v[104:107], v[236:239]\n"
      "v_mfma_f32_16x16x32_bf16 v[252:255], v[124:127], v[108:111], v[252:255]\n"
      "s_add_u32 %[i], %[i], 1\n"
      "v_mfma_f32_16x16x32_bf16 v[248:251], v[120:123], v[108:111], v[248:251]\n"
      "s_cmp_lt_u32 %[i], 64\n"
      "v_mfma_f32_16x16x32_bf16 v[244:247], v[116:119], v[108:111], v[244:247]\n"
      "v_mfma_f32_16x16x32_bf16 v[240:243], v[112:115], v[108:111], v[240:243]\n"
      "s_cbranch_scc1 L_loop_%=\n"
      "s_waitcnt lgkmcnt(0)\n"
      "s_nop 15\n"
                : "=&{v[128:143]}"(c0), "=&{v[144:159]}"(c1), "=&{v[160:175]}"(c2), "=&{v[176:191]}"(c3),
                  "=&{v[192:207]}"(c4), "=&{v[208:223]}"(c5), "=&{v[224:239]}"(c6), "=&{v[240:255]}"(c7),
                  [i] "=&s"(si), [st] "=&s"(sst), [dA] "=&s"(sdA), [dB] "=&s"(sdB)
                : [a0] "v"(a0), [a1] "v"(a1), [b0] "v"(b0), [b1] "v"(b1), [b2] "v"(b2), [b3] "v"(b3),
                  [cm] "s"(0xffff8000u), [cp] "s"(16384u)
                : "v16", "v17", "v18", "v19", "v20", "v21", "v22", "v23", "v24", "v25", "v26", "v27", "v28", "v29", "v30", "v31", "v32", "v33", "v34", "v35", "v36", "v37", "v38", "v39", "v40", "v41", "v42", "v43", "v44", "v45", "v46", "v47", "v48", "v49", "v50", "v51", "v52", "v53", "v54", "v55", "v56", "v57", "v58", "v59", "v60", "v61", "v62", "v63", "v64", "v65", "v66", "v67", "v68", "v69", "v70", "v71", "v72", "v73", "v74", "v75", "v76", "v77", "v78", "v79", "v80", "v81", "v82", "v83", "v84", "v85", "v86", "v87", "v88", "v89", "v90", "v91", "v92", "v93", "v94", "v95", "v96", "v97", "v98", "v99", "v100", "v101", "v102", "v103", "v104", "v105", "v106", "v107", "v108", "v109", "v110", "v111", "v112", "v113", "v114", "v115", "v116", "v117", "v118", "v119", "v120", "v121", "v122", "v123", "v124", "v125", "v126", "v127", "vcc", "scc", "memory");
        }
        if (wid == 0) STAMP(2);
        u32 tz = threadIdx.x; asm volatile("" : "+v"(tz));
        const u32 lane = tz & 63;
        float mx = 0.f;
#define MX(c) _Pragma("unroll") for (int j = 0; j < 16; j += 2) mx = __builtin_fmaxf(mx, __builtin_fmaxf(c[j], c[j + 1]));
        MX(c0) MX(c1) MX(c2) MX(c3) MX(c4) MX(c5) MX(c6) MX(c7)
        const u32 wm = wave_umax(__float_as_uint(mx));
        lds_store(lds0 + F_MISC + wid * 4, wm);
        BAR();
        BAR();
        u32x4 am; asm volatile("ds_read_b128 %0, %1\n\ts_waitcnt lgkmcnt(0)" : "=&v"(am) : "v"(lds0 + F_MISC + 16) : "memory");
        const float amax = __uint_as_float(am.x);
        const float inv = amax > 0.f ? 1.0f / (amax / 448.0f) : 0.f;
        const u32 nanm = am.y ? 0x7f7f7f7fu : 0u;
        const u32 ea = lds0 + (lane & 15) * EP_PITCH + wid * 64 + (lane >> 4) * 4;
#define QS(c, m) _Pragma("unroll") for (int t = 0; t < 4; ++t) { \
            int w = __builtin_amdgcn_cvt_pk_fp8_f32(__builtin_fmaxf(c[4 * t], 0.f) * inv, __builtin_fmaxf(c[4 * t + 1], 0.f) * inv, 0, false); \
            w = __builtin_amdgcn_cvt_pk_fp8_f32(__builtin_fmaxf(c[4 * t + 2], 0.f) * inv, __builtin_fmaxf(c[4 * t + 3], 0.f) * inv, w, true); \
            *(LAS u32*)(uintptr_t)(ea + (m) * 16 * EP_PITCH + t * 16) = (u32)w | nanm; }
        QS(c0, 0) QS(c1, 1) QS(c2, 2) QS(c3, 3) QS(c4, 4) QS(c5, 5) QS(c6, 6) QS(c7, 7)
    } else {
        const u32 lane = threadIdx.x & 63, pw = wid - 4;
        __builtin_amdgcn_s_setprio(PPRIO);
        u32 call = 1;
        {
            const u32x4 ra = make_rsrc(p.x, ML * KD * 2), rb = make_rsrc(p.w, (u32)KD * ND * 2);
            const u32 rw = RW, rx = RX;
            const u32 a0 = ((lane >> 3) * (KD * 2) + (((lane & 7) ^ (lane >> 4)) << 4)) ^ ((rx & 1) << 6), a1 = a0 ^ 64;
            const u32 b0 = ((lane >> 5) * (WPAIR * ND * 2) + (((lane & 31) ^ (2 * (lane >> 5))) << 4)) ^ ((rw & 1) << 6) ^ ((rw >> 2) << 7), b1 = b0 ^ 64, b2 = b0 ^ 128, b3 = b0 ^ 192;
            const u32 px = ROTX, pq = ROTW;
            const u32 sa0 = rfl((rowblk * BM + px * 32 + rx * 8) * KD * 2), sb0 = rfl(rw * (WPAIR == 1 ? 2 : 1) * ND * 2 + (WPAIR <= 8 ? pq * 16 : WPAIR == 16 ? (pq >> 1) * 32 + (pq & 1) * 8 : pq * 8) * ND * 2 + (grp * NC + ct * BN) * 2);
            const u32 ma0 = rfl(lds0 + A_BASE + px * 4096 + rx * 1024), mb0 = rfl(lds0 + B_BASE + pq * 8192 + rw * 1024), mae = ma0 + 3 * A_ST;
            const u32 ssa = 128, ssb = 64 * ND * 2;
            u32 ty = 0, pv = 0, sp = 0, ssp = 0, pd; u32x4 rp = rb;
            ty = rfl(ty); sp = rfl(sp); ssp = rfl(ssp); rp.x = rfl(rp.x); rp.y = rfl(rp.y); rp.z = rfl(rp.z);
            u32 ma = ma0, mb = mb0, t, t2, n, tv = 0, tw = 0;
            const u32 kx = rfl(KX), kp = rfl((KP) + (LAG));
            u32 kk = rfl(KP);
            u32 sa = sa0 + (((kk ^ kx) - kp) & 63) * ssa, sb = sb0 + (((kk ^ kx) - kp) & 63) * ssb;
            asm volatile(
      "s_cmp_lg_u32 %[pw], 3\n"
      "s_cbranch_scc1 L_s_%=\n"
      "s_mov_b64 exec, 1\n"
      "global_atomic_add %[call], %[ca], %[call], off sc0\n"
      "s_mov_b64 exec, -1\n"
      "L_s_%=:\n"
      "s_cmp_eq_u32 %[ty], 1\n"
      "s_cbranch_scc1 L_t1_%=\n"
      "s_cmp_eq_u32 %[ty], 2\n"
      "s_cbranch_scc1 L_t2_%=\n"
      "s_xor_b32 m0, %[ma], 0\n"
      "s_xor_b32 %[t], %[sa], 0x0\n"
      "buffer_load_dwordx4 %[a0], %[ra], %[t] offen  lds\n"
      "s_xor_b32 m0, %[ma], 1024\n"
      "s_xor_b32 %[t], %[sa], 0x10000\n"
      "buffer_load_dwordx4 %[a1], %[ra], %[t] offen  lds\n"
      "s_xor_b32 m0, %[ma], 2048\n"
      "s_xor_b32 %[t], %[sa], 0x20000\n"
      "buffer_load_dwordx4 %[a0], %[ra], %[t] offen  lds\n"
      "s_xor_b32 m0, %[ma], 3072\n"
      "s_xor_b32 %[t], %[sa], 0x30000\n"
      "buffer_load_dwordx4 %[a1], %[ra], %[t] offen  lds\n"
      "s_xor_b32 m0, %[mb], 0\n"
      "s_xor_b32 %[t], %[sb], 0x0\n"
      "buffer_load_dwordx4 %[b0], %[rb], %[t] offen  lds\n"
      "s_xor_b32 m0, %[mb], 1024\n"
      "s_xor_b32 %[t], %[sb], 0x10000\n"
      "buffer_load_dwordx4 %[b1], %[rb], %[t] offen  lds\n"
      "s_xor_b32 m0, %[mb], 2048\n"
      "s_xor_b32 %[t], %[sb], 0x20000\n"
      "buffer_load_dwordx4 %[b0], %[rb], %[t] offen  lds\n"
      "s_xor_b32 m0, %[mb], 3072\n"
      "s_xor_b32 %[t], %[sb], 0x30000\n"
      "buffer_load_dwordx4 %[b1], %[rb], %[t] offen  lds\n"
      "s_xor_b32 m0, %[mb], 4096\n"
      "s_xor_b32 %[t], %[sb], 0x40000\n"
      "buffer_load_dwordx4 %[b2], %[rb], %[t] offen  lds\n"
      "s_xor_b32 m0, %[mb], 5120\n"
      "s_xor_b32 %[t], %[sb], 0x50000\n"
      "buffer_load_dwordx4 %[b3], %[rb], %[t] offen  lds\n"
      "s_xor_b32 m0, %[mb], 6144\n"
      "s_xor_b32 %[t], %[sb], 0x60000\n"
      "buffer_load_dwordx4 %[b2], %[rb], %[t] offen  lds\n"
      "s_xor_b32 m0, %[mb], 7168\n"
      "s_xor_b32 %[t], %[sb], 0x70000\n"
      "buffer_load_dwordx4 %[b3], %[rb], %[t] offen  lds\n"
      "s_add_u32 %[kk], %[kk], 1\n"
      "s_xor_b32 %[t], %[kk], %[kx]\n"
      "s_sub_u32 %[t], %[t], %[kp]\n"
      "s_and_b32 %[t], %[t], 63\n"
      "s_mul_i32 %[t2], %[t], %[ssa]\n"
      "s_add_u32 %[sa], %[sa0], %[t2]\n"
      "s_mul_i32 %[t2], %[t], %[ssb]\n"
      "s_add_u32 %[sb], %[sb0], %[t2]\n"
      "s_add_u32 %[ma], %[ma], 16384\n"
      "s_add_u32 %[mb], %[mb], 32768\n"
      "s_cmp_eq_u32 %[ma], %[mae]\n"
      "s_cselect_b32 %[ma], %[ma0], %[ma]\n"
      "s_cselect_b32 %[mb], %[mb0], %[mb]\n"
      "s_xor_b32 m0, %[ma], 0\n"
      "s_xor_b32 %[t], %[sa], 0x0\n"
      "buffer_load_dwordx4 %[a0], %[ra], %[t] offen  lds\n"
      "s_xor_b32 m0, %[ma], 1024\n"
      "s_xor_b32 %[t], %[sa], 0x10000\n"
      "buffer_load_dwordx4 %[a1], %[ra], %[t] offen  lds\n"
      "s_xor_b32 m0, %[ma], 2048\n"
      "s_xor_b32 %[t], %[sa], 0x20000\n"
      "buffer_load_dwordx4 %[a0], %[ra], %[t] offen  lds\n"
      "s_xor_b32 m0, %[ma], 3072\n"
      "s_xor_b32 %[t], %[sa], 0x30000\n"
      "buffer_load_dwordx4 %[a1], %[ra], %[t] offen  lds\n"
      "s_xor_b32 m0, %[mb], 0\n"
      "s_xor_b32 %[t], %[sb], 0x0\n"
      "buffer_load_dwordx4 %[b0], %[rb], %[t] offen  lds\n"
      "s_xor_b32 m0, %[mb], 1024\n"
      "s_xor_b32 %[t], %[sb], 0x10000\n"
      "buffer_load_dwordx4 %[b1], %[rb], %[t] offen  lds\n"
      "s_xor_b32 m0, %[mb], 2048\n"
      "s_xor_b32 %[t], %[sb], 0x20000\n"
      "buffer_load_dwordx4 %[b0], %[rb], %[t] offen  lds\n"
      "s_xor_b32 m0, %[mb], 3072\n"
      "s_xor_b32 %[t], %[sb], 0x30000\n"
      "buffer_load_dwordx4 %[b1], %[rb], %[t] offen  lds\n"
      "s_xor_b32 m0, %[mb], 4096\n"
      "s_xor_b32 %[t], %[sb], 0x40000\n"
      "buffer_load_dwordx4 %[b2], %[rb], %[t] offen  lds\n"
      "s_xor_b32 m0, %[mb], 5120\n"
      "s_xor_b32 %[t], %[sb], 0x50000\n"
      "buffer_load_dwordx4 %[b3], %[rb], %[t] offen  lds\n"
      "s_xor_b32 m0, %[mb], 6144\n"
      "s_xor_b32 %[t], %[sb], 0x60000\n"
      "buffer_load_dwordx4 %[b2], %[rb], %[t] offen  lds\n"
      "s_xor_b32 m0, %[mb], 7168\n"
      "s_xor_b32 %[t], %[sb], 0x70000\n"
      "buffer_load_dwordx4 %[b3], %[rb], %[t] offen  lds\n"
      "s_add_u32 %[kk], %[kk], 1\n"
      "s_xor_b32 %[t], %[kk], %[kx]\n"
      "s_sub_u32 %[t], %[t], %[kp]\n"
      "s_and_b32 %[t], %[t], 63\n"
      "s_mul_i32 %[t2], %[t], %[ssa]\n"
      "s_add_u32 %[sa], %[sa0], %[t2]\n"
      "s_mul_i32 %[t2], %[t], %[ssb]\n"
      "s_add_u32 %[sb], %[sb0], %[t2]\n"
      "s_add_u32 %[ma], %[ma], 16384\n"
      "s_add_u32 %[mb], %[mb], 32768\n"
      "s_cmp_eq_u32 %[ma], %[mae]\n"
      "s_cselect_b32 %[ma], %[ma0], %[ma]\n"
      "s_cselect_b32 %[mb], %[mb0], %[mb]\n"
      "s_xor_b32 m0, %[ma], 0\n"
      "s_xor_b32 %[t], %[sa], 0x0\n"
      "buffer_load_dwordx4 %[a0], %[ra], %[t] offen  lds\n"
      "s_xor_b32 m0, %[ma], 1024\n"
      "s_xor_b32 %[t], %[sa], 0x10000\n"
      "buffer_load_dwordx4 %[a1], %[ra], %[t] offen  lds\n"
      "s_xor_b32 m0, %[ma], 2048\n"
      "s_xor_b32 %[t], %[sa], 0x20000\n"
      "buffer_load_dwordx4 %[a0], %[ra], %[t] offen  lds\n"
      "s_xor_b32 m0, %[ma], 3072\n"
      "s_xor_b32 %[t], %[sa], 0x30000\n"
      "buffer_load_dwordx4 %[a1], %[ra], %[t] offen  lds\n"
      "s_xor_b32 m0, %[mb], 0\n"
      "s_xor_b32 %[t], %[sb], 0x0\n"
      "buffer_load_dwordx4 %[b0], %[rb], %[t] offen  lds\n"
      "s_xor_b32 m0, %[mb], 1024\n"
      "s_xor_b32 %[t], %[sb], 0x10000\n"
      "buffer_load_dwordx4 %[b1], %[rb], %[t] offen  lds\n"
      "s_xor_b32 m0, %[mb], 2048\n"
      "s_xor_b32 %[t], %[sb], 0x20000\n"
      "buffer_load_dwordx4 %[b0], %[rb], %[t] offen  lds\n"
      "s_xor_b32 m0, %[mb], 3072\n"
      "s_xor_b32 %[t], %[sb], 0x30000\n"
      "buffer_load_dwordx4 %[b1], %[rb], %[t] offen  lds\n"
      "s_xor_b32 m0, %[mb], 4096\n"
      "s_xor_b32 %[t], %[sb], 0x40000\n"
      "buffer_load_dwordx4 %[b2], %[rb], %[t] offen  lds\n"
      "s_xor_b32 m0, %[mb], 5120\n"
      "s_xor_b32 %[t], %[sb], 0x50000\n"
      "buffer_load_dwordx4 %[b3], %[rb], %[t] offen  lds\n"
      "s_xor_b32 m0, %[mb], 6144\n"
      "s_xor_b32 %[t], %[sb], 0x60000\n"
      "buffer_load_dwordx4 %[b2], %[rb], %[t] offen  lds\n"
      "s_xor_b32 m0, %[mb], 7168\n"
      "s_xor_b32 %[t], %[sb], 0x70000\n"
      "buffer_load_dwordx4 %[b3], %[rb], %[t] offen  lds\n"
      "s_add_u32 %[kk], %[kk], 1\n"
      "s_xor_b32 %[t], %[kk], %[kx]\n"
      "s_sub_u32 %[t], %[t], %[kp]\n"
      "s_and_b32 %[t], %[t], 63\n"
      "s_mul_i32 %[t2], %[t], %[ssa]\n"
      "s_add_u32 %[sa], %[sa0], %[t2]\n"
      "s_mul_i32 %[t2], %[t], %[ssb]\n"
      "s_add_u32 %[sb], %[sb0], %[t2]\n"
      "s_add_u32 %[ma], %[ma], 16384\n"
      "s_add_u32 %[mb], %[mb], 32768\n"
      "s_cmp_eq_u32 %[ma], %[mae]\n"
      "s_cselect_b32 %[ma], %[ma0], %[ma]\n"
      "s_cselect_b32 %[mb], %[mb0], %[mb]\n"
      "s_waitcnt vmcnt(24)\n"
      "s_barrier\n"
      "s_waitcnt vmcnt(12)\n"
      "s_barrier\n"
      "s_mov_b32 %[n], 3\n"
      "L_pa_%=:\n"
      "s_xor_b32 m0, %[ma], 0\n"
      "s_xor_b32 %[t], %[sa], 0x0\n"
      "buffer_load_dwordx4 %[a0], %[ra], %[t] offen  lds\n"
      "s_xor_b32 m0, %[ma], 1024\n"
      "s_xor_b32 %[t], %[sa], 0x10000\n"
      "buffer_load_dwordx4 %[a1], %[ra], %[t] offen  lds\n"
      "s_xor_b32 m0, %[ma], 2048\n"
      "s_xor_b32 %[t], %[sa], 0x20000\n"
      "buffer_load_dwordx4 %[a0], %[ra], %[t] offen  lds\n"
      "s_xor_b32 m0, %[ma], 3072\n"
      "s_xor_b32 %[t], %[sa], 0x30000\n"
      "buffer_load_dwordx4 %[a1], %[ra], %[t] offen  lds\n"
      "s_xor_b32 m0, %[mb], 0\n"
      "s_xor_b32 %[t], %[sb], 0x0\n"
      "buffer_load_dwordx4 %[b0], %[rb], %[t] offen  lds\n"
      "s_xor_b32 m0, %[mb], 1024\n"
      "s_xor_b32 %[t], %[sb], 0x10000\n"
      "buffer_load_dwordx4 %[b1], %[rb], %[t] offen  lds\n"
      "s_xor_b32 m0, %[mb], 2048\n"
      "s_xor_b32 %[t], %[sb], 0x20000\n"
      "buffer_load_dwordx4 %[b0], %[rb], %[t] offen  lds\n"
      "s_xor_b32 m0, %[mb], 3072\n"
      "s_xor_b32 %[t], %[sb], 0x30000\n"
      "buffer_load_dwordx4 %[b1], %[rb], %[t] offen  lds\n"
      "s_xor_b32 m0, %[mb], 4096\n"
      "s_xor_b32 %[t], %[sb], 0x40000\n"
      "buffer_load_dwordx4 %[b2], %[rb], %[t] offen  lds\n"
      "s_xor_b32 m0, %[mb], 5120\n"
      "s_xor_b32 %[t], %[sb], 0x50000\n"
      "buffer_load_dwordx4 %[b3], %[rb], %[t] offen  lds\n"
      "s_xor_b32 m0, %[mb], 6144\n"
      "s_xor_b32 %[t], %[sb], 0x60000\n"
      "buffer_load_dwordx4 %[b2], %[rb], %[t] offen  lds\n"
      "s_xor_b32 m0, %[mb], 7168\n"
      "s_xor_b32 %[t], %[sb], 0x70000\n"
      "buffer_load_dwordx4 %[b3], %[rb], %[t] offen  lds\n"
      "s_add_u32 %[kk], %[kk], 1\n"
      "s_xor_b32 %[t], %[kk], %[kx]\n"
      "s_sub_u32 %[t], %[t], %[kp]\n"
      "s_and_b32 %[t], %[t], 63\n"
      "s_mul_i32 %[t2], %[t], %[ssa]\n"
      "s_add_u32 %[sa], %[sa0], %[t2]\n"
      "s_mul_i32 %[t2], %[t], %[ssb]\n"
      "s_add_u32 %[sb], %[sb0], %[t2]\n"
      "s_add_u32 %[ma], %[ma], 16384\n"
      "s_add_u32 %[mb], %[mb], 32768\n"
      "s_cmp_eq_u32 %[ma], %[mae]\n"
      "s_cselect_b32 %[ma], %[ma0], %[ma]\n"
      "s_cselect_b32 %[mb], %[mb0], %[mb]\n"
      "s_add_u32 %[n], %[n], 1\n"
      "s_waitcnt vmcnt(12)\n"
      "s_cmp_lt_u32 %[n], 64\n"
      "s_barrier\n"
      "s_cbranch_scc1 L_pa_%=\n"
      "s_branch L_e_%=\n"
      "L_t1_%=:\n"
      "s_xor_b32 m0, %[ma], 0\n"
      "s_xor_b32 %[t], %[sa], 0x0\n"
      "buffer_load_dwordx4 %[a0], %[ra], %[t] offen  lds\n"
      "s_xor_b32 m0, %[ma], 1024\n"
      "s_xor_b32 %[t], %[sa], 0x10000\n"
      "buffer_load_dwordx4 %[a1], %[ra], %[t] offen  lds\n"
      "s_xor_b32 m0, %[ma], 2048\n"
      "s_xor_b32 %[t], %[sa], 0x20000\n"
      "buffer_load_dwordx4 %[a0], %[ra], %[t] offen  lds\n"
      "s_xor_b32 m0, %[ma], 3072\n"
      "s_xor_b32 %[t], %[sa], 0x30000\n"
      "buffer_load_dwordx4 %[a1], %[ra], %[t] offen  lds\n"
      "s_xor_b32 m0, %[mb], 0\n"
      "s_xor_b32 %[t], %[sb], 0x0\n"
      "buffer_load_dwordx4 %[b0], %[rb], %[t] offen  lds\n"
      "s_xor_b32 m0, %[mb], 1024\n"
      "s_xor_b32 %[t], %[sb], 0x10000\n"
      "buffer_load_dwordx4 %[b1], %[rb], %[t] offen  lds\n"
      "s_xor_b32 m0, %[mb], 2048\n"
      "s_xor_b32 %[t], %[sb], 0x20000\n"
      "buffer_load_dwordx4 %[b0], %[rb], %[t] offen  lds\n"
      "s_xor_b32 m0, %[mb], 3072\n"
      "s_xor_b32 %[t], %[sb], 0x30000\n"
      "buffer_load_dwordx4 %[b1], %[rb], %[t] offen  lds\n"
      "s_xor_b32 m0, %[mb], 4096\n"
      "s_xor_b32 %[t], %[sb], 0x40000\n"
      "buffer_load_dwordx4 %[b2], %[rb], %[t] offen  lds\n"
      "s_xor_b32 m0, %[mb], 5120\n"
      "s_xor_b32 %[t], %[sb], 0x50000\n"
      "buffer_load_dwordx4 %[b3], %[rb], %[t] offen  lds\n"
      "s_xor_b32 m0, %[mb], 6144\n"
      "s_xor_b32 %[t], %[sb], 0x60000\n"
      "buffer_load_dwordx4 %[b2], %[rb], %[t] offen  lds\n"
      "s_xor_b32 m0, %[mb], 7168\n"
      "s_xor_b32 %[t], %[sb], 0x70000\n"
      "buffer_load_dwordx4 %[b3], %[rb], %[t] offen  lds\n"
      "s_add_u32 %[kk], %[kk], 1\n"
      "s_xor_b32 %[t], %[kk], %[kx]\n"
      "s_sub_u32 %[t], %[t], %[kp]\n"
      "s_and_b32 %[t], %[t], 63\n"
      "s_mul_i32 %[t2], %[t], %[ssa]\n"
      "s_add_u32 %[sa], %[sa0], %[t2]\n"
      "s_mul_i32 %[t2], %[t], %[ssb]\n"
      "s_add_u32 %[sb], %[sb0], %[t2]\n"
      "s_add_u32 %[ma], %[ma], 16384\n"
      "s_add_u32 %[mb], %[mb], 32768\n"
      "s_cmp_eq_u32 %[ma], %[mae]\n"
      "s_cselect_b32 %[ma], %[ma0], %[ma]\n"
      "s_cselect_b32 %[mb], %[mb0], %[mb]\n"
      "s_xor_b32 m0, %[ma], 0\n"
      "s_xor_b32 %[t], %[sa], 0x0\n"
      "buffer_load_dwordx4 %[a0], %[ra], %[t] offen  lds\n"
      "s_xor_b32 m0, %[ma], 1024\n"
      "s_xor_b32 %[t], %[sa], 0x10000\n"
      "buffer_load_dwordx4 %[a1], %[ra], %[t] offen  lds\n"
      "s_xor_b32 m0, %[ma], 2048\n"
      "s_xor_b32 %[t], %[sa], 0x20000\n"
      "buffer_load_dwordx4 %[a0], %[ra], %[t] offen  lds\n"
      "s_xor_b32 m0, %[ma], 3072\n"
      "s_xor_b32 %[t], %[sa], 0x30000\n"
      "buffer_load_dwordx4 %[a1], %[ra], %[t] offen  lds\n"
      "s_xor_b32 m0, %[mb], 0\n"
      "s_xor_b32 %[t], %[sb], 0x0\n"
      "buffer_load_dwordx4 %[b0], %[rb], %[t] offen  lds\n"
      "s_xor_b32 m0, %[mb], 1024\n"
      "s_xor_b32 %[t], %[sb], 0x10000\n"
      "buffer_load_dwordx4 %[b1], %[rb], %[t] offen  lds\n"
      "s_xor_b32 m0, %[mb], 2048\n"
      "s_xor_b32 %[t], %[sb], 0x20000\n"
      "buffer_load_dwordx4 %[b0], %[rb], %[t] offen  lds\n"
      "s_xor_b32 m0, %[mb], 3072\n"
      "s_xor_b32 %[t], %[sb], 0x30000\n"
      "buffer_load_dwordx4 %[b1], %[rb], %[t] offen  lds\n"
      "s_xor_b32 m0, %[mb], 4096\n"
      "s_xor_b32 %[t], %[sb], 0x40000\n"
      "buffer_load_dwordx4 %[b2], %[rb], %[t] offen  lds\n"
      "s_xor_b32 m0, %[mb], 5120\n"
      "s_xor_b32 %[t], %[sb], 0x50000\n"
      "buffer_load_dwordx4 %[b3], %[rb], %[t] offen  lds\n"
      "s_xor_b32 m0, %[mb], 6144\n"
      "s_xor_b32 %[t], %[sb], 0x60000\n"
      "buffer_load_dwordx4 %[b2], %[rb], %[t] offen  lds\n"
      "s_xor_b32 m0, %[mb], 7168\n"
      "s_xor_b32 %[t], %[sb], 0x70000\n"
      "buffer_load_dwordx4 %[b3], %[rb], %[t] offen  lds\n"
      "s_add_u32 %[kk], %[kk], 1\n"
      "s_xor_b32 %[t], %[kk], %[kx]\n"
      "s_sub_u32 %[t], %[t], %[kp]\n"
      "s_and_b32 %[t], %[t], 63\n"
      "s_mul_i32 %[t2], %[t], %[ssa]\n"
      "s_add_u32 %[sa], %[sa0], %[t2]\n"
      "s_mul_i32 %[t2], %[t], %[ssb]\n"
      "s_add_u32 %[sb], %[sb0], %[t2]\n"
      "s_add_u32 %[ma], %[ma], 16384\n"
      "s_add_u32 %[mb], %[mb], 32768\n"
      "s_cmp_eq_u32 %[ma], %[mae]\n"
      "s_cselect_b32 %[ma], %[ma0], %[ma]\n"
      "s_cselect_b32 %[mb], %[mb0], %[mb]\n"
      "buffer_load_dword %[pd], %[pv], %[rp], %[sp] offen \n"
      "s_add_u32 %[sp], %[sp], %[ssp]\n"
      "buffer_load_dword %[pd], %[pv], %[rp], %[sp] offen \n"
      "s_add_u32 %[sp], %[sp], %[ssp]\n"
      "buffer_load_dword %[pd], %[pv], %[rp], %[sp] offen \n"
      "s_add_u32 %[sp], %[sp], %[ssp]\n"
      "s_xor_b32 m0, %[ma], 0\n"
      "s_xor_b32 %[t], %[sa], 0x0\n"
      "buffer_load_dwordx4 %[a0], %[ra], %[t] offen  lds\n"
      "s_xor_b32 m0, %[ma], 1024\n"
      "s_xor_b32 %[t], %[sa], 0x10000\n"
      "buffer_load_dwordx4 %[a1], %[ra], %[t] offen  lds\n"
      "s_xor_b32 m0, %[ma], 2048\n"
      "s_xor_b32 %[t], %[sa], 0x20000\n"
      "buffer_load_dwordx4 %[a0], %[ra], %[t] offen  lds\n"
      "s_xor_b32 m0, %[ma], 3072\n"
      "s_xor_b32 %[t], %[sa], 0x30000\n"
      "buffer_load_dwordx4 %[a1], %[ra], %[t] offen  lds\n"
      "s_xor_b32 m0, %[mb], 0\n"
      "s_xor_b32 %[t], %[sb], 0x0\n"
      "buffer_load_dwordx4 %[b0], %[rb], %[t] offen  lds\n"
      "s_xor_b32 m0, %[mb], 1024\n"
      "s_xor_b32 %[t], %[sb], 0x10000\n"
      "buffer_load_dwordx4 %[b1], %[rb], %[t] offen  lds\n"
      "s_xor_b32 m0, %[mb], 2048\n"
      "s_xor_b32 %[t], %[sb], 0x20000\n"
      "buffer_load_dwordx4 %[b0], %[rb], %[t] offen  lds\n"
      "s_xor_b32 m0, %[mb], 3072\n"
      "s_xor_b32 %[t], %[sb], 0x30000\n"
      "buffer_load_dwordx4 %[b1], %[rb], %[t] offen  lds\n"
      "s_xor_b32 m0, %[mb], 4096\n"
      "s_xor_b32 %[t], %[sb], 0x40000\n"
      "buffer_load_dwordx4 %[b2], %[rb], %[t] offen  lds\n"
      "s_xor_b32 m0, %[mb], 5120\n"
      "s_xor_b32 %[t], %[sb], 0x50000\n"
      "buffer_load_dwordx4 %[b3], %[rb], %[t] offen  lds\n"
      "s_xor_b32 m0, %[mb], 6144\n"
      "s_xor_b32 %[t], %[sb], 0x60000\n"
      "buffer_load_dwordx4 %[b2], %[rb], %[t] offen  lds\n"
      "s_xor_b32 m0, %[mb], 7168\n"
      "s_xor_b32 %[t], %[sb], 0x70000\n"
      "buffer_load_dwordx4 %[b3], %[rb], %[t] offen  lds\n"
      "s_add_u32 %[kk], %[kk], 1\n"
      "s_xor_b32 %[t], %[kk], %[kx]\n"
      "s_sub_u32 %[t], %[t], %[kp]\n"
      "s_and_b32 %[t], %[t], 63\n"
      "s_mul_i32 %[t2], %[t], %[ssa]\n"
      "s_add_u32 %[sa], %[sa0], %[t2]\n"
      "s_mul_i32 %[t2], %[t], %[ssb]\n"
      "s_add_u32 %[sb], %[sb0], %[t2]\n"
      "s_add_u32 %[ma], %[ma], 16384\n"
      "s_add_u32 %[mb], %[mb], 32768\n"
      "s_cmp_eq_u32 %[ma], %[mae]\n"
      "s_cselect_b32 %[ma], %[ma0], %[ma]\n"
      "s_cselect_b32 %[mb], %[mb0], %[mb]\n"
      "buffer_load_dword %[pd], %[pv], %[rp], %[sp] offen \n"
      "s_add_u32 %[sp], %[sp], %[ssp]\n"
      "s_waitcnt vmcnt(28)\n"
      "s_barrier\n"
      "s_waitcnt vmcnt(16)\n"
      "s_barrier\n"
      "s_mov_b32 %[n], 3\n"
      "L_pb_%=:\n"
      "s_xor_b32 m0, %[ma], 0\n"
      "s_xor_b32 %[t], %[sa], 0x0\n"
      "buffer_load_dwordx4 %[a0], %[ra], %[t] offen  lds\n"
      "s_xor_b32 m0, %[ma], 1024\n"
      "s_xor_b32 %[t], %[sa], 0x10000\n"
      "buffer_load_dwordx4 %[a1], %[ra], %[t] offen  lds\n"
      "s_xor_b32 m0, %[ma], 2048\n"
      "s_xor_b32 %[t], %[sa], 0x20000\n"
      "buffer_load_dwordx4 %[a0], %[ra], %[t] offen  lds\n"
      "s_xor_b32 m0, %[ma], 3072\n"
      "s_xor_b32 %[t], %[sa], 0x30000\n"
      "buffer_load_dwordx4 %[a1], %[ra], %[t] offen  lds\n"
      "s_xor_b32 m0, %[mb], 0\n"
      "s_xor_b32 %[t], %[sb], 0x0\n"
      "buffer_load_dwordx4 %[b0], %[rb], %[t] offen  lds\n"
      "s_xor_b32 m0, %[mb], 1024\n"
      "s_xor_b32 %[t], %[sb], 0x10000\n"
      "buffer_load_dwordx4 %[b1], %[rb], %[t] offen  lds\n"
      "s_xor_b32 m0, %[mb], 2048\n"
      "s_xor_b32 %[t], %[sb], 0x20000\n"
      "buffer_load_dwordx4 %[b0], %[rb], %[t] offen  lds\n"
      "s_xor_b32 m0, %[mb], 3072\n"
      "s_xor_b32 %[t], %[sb], 0x30000\n"
      "buffer_load_dwordx4 %[b1], %[rb], %[t] offen  lds\n"
      "s_xor_b32 m0, %[mb], 4096\n"
      "s_xor_b32 %[t], %[sb], 0x40000\n"
      "buffer_load_dwordx4 %[b2], %[rb], %[t] offen  lds\n"
      "s_xor_b32 m0, %[mb], 5120\n"
      "s_xor_b32 %[t], %[sb], 0x50000\n"
      "buffer_load_dwordx4 %[b3], %[rb], %[t] offen  lds\n"
      "s_xor_b32 m0, %[mb], 6144\n"
      "s_xor_b32 %[t], %[sb], 0x60000\n"
      "buffer_load_dwordx4 %[b2], %[rb], %[t] offen  lds\n"
      "s_xor_b32 m0, %[mb], 7168\n"
      "s_xor_b32 %[t], %[sb], 0x70000\n"
      "buffer_load_dwordx4 %[b3], %[rb], %[t] offen  lds\n"
      "s_add_u32 %[kk], %[kk], 1\n"
      "s_xor_b32 %[t], %[kk], %[kx]\n"
      "s_sub_u32 %[t], %[t], %[kp]\n"
      "s_and_b32 %[t], %[t], 63\n"
      "s_mul_i32 %[t2], %[t], %[ssa]\n"
      "s_add_u32 %[sa], %[sa0], %[t2]\n"
      "s_mul_i32 %[t2], %[t], %[ssb]\n"
      "s_add_u32 %[sb], %[sb0], %[t2]\n"
      "s_add_u32 %[ma], %[ma], 16384\n"
      "s_add_u32 %[mb], %[mb], 32768\n"
      "s_cmp_eq_u32 %[ma], %[mae]\n"
      "s_cselect_b32 %[ma], %[ma0], %[ma]\n"
      "s_cselect_b32 %[mb], %[mb0], %[mb]\n"
      "buffer_load_dword %[pd], %[pv], %[rp], %[sp] offen \n"
      "s_add_u32 %[sp], %[sp], %[ssp]\n"
      "s_add_u32 %[n], %[n], 1\n"
      "s_waitcnt vmcnt(14)\n"
      "s_cmp_lt_u32 %[n], 64\n"
      "s_barrier\n"
      "s_cbranch_scc1 L_pb_%=\n"
      "s_branch L_e_%=\n"
      "L_t2_%=:\n"
      "s_xor_b32 m0, %[ma], 0\n"
      "s_xor_b32 %[t], %[sa], 0x0\n"
      "buffer_load_dwordx4 %[a0], %[ra], %[t] offen  lds\n"
      "s_xor_b32 m0, %[ma], 1024\n"
      "s_xor_b32 %[t], %[sa], 0x10000\n"
      "buffer_load_dwordx4 %[a1], %[ra], %[t] offen  lds\n"
      "s_xor_b32 m0, %[ma], 2048\n"
      "s_xor_b32 %[t], %[sa], 0x20000\n"
      "buffer_load_dwordx4 %[a0], %[ra], %[t] offen  lds\n"
      "s_xor_b32 m0, %[ma], 3072\n"
      "s_xor_b32 %[t], %[sa], 0x30000\n"
      "buffer_load_dwordx4 %[a1], %[ra], %[t] offen  lds\n"
      "s_xor_b32 m0, %[mb], 0\n"
      "s_xor_b32 %[t], %[sb], 0x0\n"
      "buffer_load_dwordx4 %[b0], %[rb], %[t] offen  lds\n"
      "s_xor_b32 m0, %[mb], 1024\n"
      "s_xor_b32 %[t], %[sb], 0x10000\n"
      "buffer_load_dwordx4 %[b1], %[rb], %[t] offen  lds\n"
      "s_xor_b32 m0, %[mb], 2048\n"
      "s_xor_b32 %[t], %[sb], 0x20000\n"
      "buffer_load_dwordx4 %[b0], %[rb], %[t] offen  lds\n"
      "s_xor_b32 m0, %[mb], 3072\n"
      "s_xor_b32 %[t], %[sb], 0x30000\n"
      "buffer_load_dwordx4 %[b1], %[rb], %[t] offen  lds\n"
      "s_xor_b32 m0, %[mb], 4096\n"
      "s_xor_b32 %[t], %[sb], 0x40000\n"
      "buffer_load_dwordx4 %[b2], %[rb], %[t] offen  lds\n"
      "s_xor_b32 m0, %[mb], 5120\n"
      "s_xor_b32 %[t], %[sb], 0x50000\n"
      "buffer_load_dwordx4 %[b3], %[rb], %[t] offen  lds\n"
      "s_xor_b32 m0, %[mb], 6144\n"
      "s_xor_b32 %[t], %[sb], 0x60000\n"
      "buffer_load_dwordx4 %[b2], %[rb], %[t] offen  lds\n"
      "s_xor_b32 m0, %[mb], 7168\n"
      "s_xor_b32 %[t], %[sb], 0x70000\n"
      "buffer_load_dwordx4 %[b3], %[rb], %[t] offen  lds\n"
      "s_add_u32 %[kk], %[kk], 1\n"
      "s_xor_b32 %[t], %[kk], %[kx]\n"
      "s_sub_u32 %[t], %[t], %[kp]\n"
      "s_and_b32 %[t], %[t], 63\n"
      "s_mul_i32 %[t2], %[t], %[ssa]\n"
      "s_add_u32 %[sa], %[sa0], %[t2]\n"
      "s_mul_i32 %[t2], %[t], %[ssb]\n"
      "s_add_u32 %[sb], %[sb0], %[t2]\n"
      "s_add_u32 %[ma], %[ma], 16384\n"
      "s_add_u32 %[mb], %[mb], 32768\n"
      "s_cmp_eq_u32 %[ma], %[mae]\n"
      "s_cselect_b32 %[ma], %[ma0], %[ma]\n"
      "s_cselect_b32 %[mb], %[mb0], %[mb]\n"
      "s_xor_b32 m0, %[ma], 0\n"
      "s_xor_b32 %[t], %[sa], 0x0\n"
      "buffer_load_dwordx4 %[a0], %[ra], %[t] offen  lds\n"
      "s_xor_b32 m0, %[ma], 1024\n"
      "s_xor_b32 %[t], %[sa], 0x10000\n"
      "buffer_load_dwordx4 %[a1], %[ra], %[t] offen  lds\n"
      "s_xor_b32 m0, %[ma], 2048\n"
      "s_xor_b32 %[t], %[sa], 0x20000\n"
      "buffer_load_dwordx4 %[a0], %[ra], %[t] offen  lds\n"
      "s_xor_b32 m0, %[ma], 3072\n"
      "s_xor_b32 %[t], %[sa], 0x30000\n"
      "buffer_load_dwordx4 %[a1], %[ra], %[t] offen  lds\n"
      "s_xor_b32 m0, %[mb], 0\n"
      "s_xor_b32 %[t], %[sb], 0x0\n"
      "buffer_load_dwordx4 %[b0], %[rb], %[t] offen  lds\n"
      "s_xor_b32 m0, %[mb], 1024\n"
      "s_xor_b32 %[t], %[sb], 0x10000\n"
      "buffer_load_dwordx4 %[b1], %[rb], %[t] offen  lds\n"
      "s_xor_b32 m0, %[mb], 2048\n"
      "s_xor_b32 %[t], %[sb], 0x20000\n"
      "buffer_load_dwordx4 %[b0], %[rb], %[t] offen  lds\n"
      "s_xor_b32 m0, %[mb], 3072\n"
      "s_xor_b32 %[t], %[sb], 0x30000\n"
      "buffer_load_dwordx4 %[b1], %[rb], %[t] offen  lds\n"
      "s_xor_b32 m0, %[mb], 4096\n"
      "s_xor_b32 %[t], %[sb], 0x40000\n"
      "buffer_load_dwordx4 %[b2], %[rb], %[t] offen  lds\n"
      "s_xor_b32 m0, %[mb], 5120\n"
      "s_xor_b32 %[t], %[sb], 0x50000\n"
      "buffer_load_dwordx4 %[b3], %[rb], %[t] offen  lds\n"
      "s_xor_b32 m0, %[mb], 6144\n"
      "s_xor_b32 %[t], %[sb], 0x60000\n"
      "buffer_load_dwordx4 %[b2], %[rb], %[t] offen  lds\n"
      "s_xor_b32 m0, %[mb], 7168\n"
      "s_xor_b32 %[t], %[sb], 0x70000\n"
      "buffer_load_dwordx4 %[b3], %[rb], %[t] offen  lds\n"
      "s_add_u32 %[kk], %[kk], 1\n"
      "s_xor_b32 %[t], %[kk], %[kx]\n"
      "s_sub_u32 %[t], %[t], %[kp]\n"
      "s_and_b32 %[t], %[t], 63\n"
      "s_mul_i32 %[t2], %[t], %[ssa]\n"
      "s_add_u32 %[sa], %[sa0], %[t2]\n"
      "s_mul_i32 %[t2], %[t], %[ssb]\n"
      "s_add_u32 %[sb], %[sb0], %[t2]\n"
      "s_add_u32 %[ma], %[ma], 16384\n"
      "s_add_u32 %[mb], %[mb], 32768\n"
      "s_cmp_eq_u32 %[ma], %[mae]\n"
      "s_cselect_b32 %[ma], %[ma0], %[ma]\n"
      "s_cselect_b32 %[mb], %[mb0], %[mb]\n"
      "buffer_load_dword %[pd], %[pv], %[rp], %[sp] offen \n"
      "s_add_u32 %[sp], %[sp], %[ssp]\n"
      "buffer_load_dword %[pd], %[pv], %[rp], %[sp] offen \n"
      "s_add_u32 %[sp], %[sp], %[ssp]\n"
      "buffer_load_dword %[pd], %[pv], %[rp], %[sp] offen \n"
      "s_add_u32 %[sp], %[sp], %[ssp]\n"
      "buffer_load_dword %[pd], %[pv], %[rp], %[sp] offen \n"
      "s_add_u32 %[sp], %[sp], %[ssp]\n"
      "s_xor_b32 m0, %[ma], 0\n"
      "s_xor_b32 %[t], %[sa], 0x0\n"
      "buffer_load_dwordx4 %[a0], %[ra], %[t] offen  lds\n"
      "s_xor_b32 m0, %[ma], 1024\n"
      "s_xor_b32 %[t], %[sa], 0x10000\n"
      "buffer_load_dwordx4 %[a1], %[ra], %[t] offen  lds\n"
      "s_xor_b32 m0, %[ma], 2048\n"
      "s_xor_b32 %[t], %[sa], 0x20000\n"
      "buffer_load_dwordx4 %[a0], %[ra], %[t] offen  lds\n"
      "s_xor_b32 m0, %[ma], 3072\n"
      "s_xor_b32 %[t], %[sa], 0x30000\n"
      "buffer_load_dwordx4 %[a1], %[ra], %[t] offen  lds\n"
      "s_xor_b32 m0, %[mb], 0\n"
      "s_xor_b32 %[t], %[sb], 0x0\n"
      "buffer_load_dwordx4 %[b0], %[rb], %[t] offen  lds\n"
      "s_xor_b32 m0, %[mb], 1024\n"
      "s_xor_b32 %[t], %[sb], 0x10000\n"
      "buffer_load_dwordx4 %[b1], %[rb], %[t] offen  lds\n"
      "s_xor_b32 m0, %[mb], 2048\n"
      "s_xor_b32 %[t], %[sb], 0x20000\n"
      "buffer_load_dwordx4 %[b0], %[rb], %[t] offen  lds\n"
      "s_xor_b32 m0, %[mb], 3072\n"
      "s_xor_b32 %[t], %[sb], 0x30000\n"
      "buffer_load_dwordx4 %[b1], %[rb], %[t] offen  lds\n"
      "s_xor_b32 m0, %[mb], 4096\n"
      "s_xor_b32 %[t], %[sb], 0x40000\n"
      "buffer_load_dwordx4 %[b2], %[rb], %[t] offen  lds\n"
      "s_xor_b32 m0, %[mb], 5120\n"
      "s_xor_b32 %[t], %[sb], 0x50000\n"
      "buffer_load_dwordx4 %[b3], %[rb], %[t] offen  lds\n"
      "s_xor_b32 m0, %[mb], 6144\n"
      "s_xor_b32 %[t], %[sb], 0x60000\n"
      "buffer_load_dwordx4 %[b2], %[rb], %[t] offen  lds\n"
      "s_xor_b32 m0, %[mb], 7168\n"
      "s_xor_b32 %[t], %[sb], 0x70000\n"
      "buffer_load_dwordx4 %[b3], %[rb], %[t] offen  lds\n"
      "s_add_u32 %[kk], %[kk], 1\n"
      "s_xor_b32 %[t], %[kk], %[kx]\n"
      "s_sub_u32 %[t], %[t], %[kp]\n"
      "s_and_b32 %[t], %[t], 63\n"
      "s_mul_i32 %[t2], %[t], %[ssa]\n"
      "s_add_u32 %[sa], %[sa0], %[t2]\n"
      "s_mul_i32 %[t2], %[t], %[ssb]\n"
      "s_add_u32 %[sb], %[sb0], %[t2]\n"
      "s_add_u32 %[ma], %[ma], 16384\n"
      "s_add_u32 %[mb], %[mb], 32768\n"
      "s_cmp_eq_u32 %[ma], %[mae]\n"
      "s_cselect_b32 %[ma], %[ma0], %[ma]\n"
      "s_cselect_b32 %[mb], %[mb0], %[mb]\n"
      "s_waitcnt vmcnt(28)\n"
      "s_barrier\n"
      "s_waitcnt vmcnt(16)\n"
      "s_barrier\n"
      "s_mov_b32 %[n], 3\n"
      "L_pc_%=:\n"
      "s_xor_b32 m0, %[ma], 0\n"
      "s_xor_b32 %[t], %[sa], 0x0\n"
      "buffer_load_dwordx4 %[a0], %[ra], %[t] offen  lds\n"
      "s_xor_b32 m0, %[ma], 1024\n"
      "s_xor_b32 %[t], %[sa], 0x10000\n"
      "buffer_load_dwordx4 %[a1], %[ra], %[t] offen  lds\n"
      "s_xor_b32 m0, %[ma], 2048\n"
      "s_xor_b32 %[t], %[sa], 0x20000\n"
      "buffer_load_dwordx4 %[a0], %[ra], %[t] offen  lds\n"
      "s_xor_b32 m0, %[ma], 3072\n"
      "s_xor_b32 %[t], %[sa], 0x30000\n"
      "buffer_load_dwordx4 %[a1], %[ra], %[t] offen  lds\n"
      "s_xor_b32 m0, %[mb], 0\n"
      "s_xor_b32 %[t], %[sb], 0x0\n"
      "buffer_load_dwordx4 %[b0], %[rb], %[t] offen  lds\n"
      "s_xor_b32 m0, %[mb], 1024\n"
      "s_xor_b32 %[t], %[sb], 0x10000\n"
      "buffer_load_dwordx4 %[b1], %[rb], %[t] offen  lds\n"
      "s_xor_b32 m0, %[mb], 2048\n"
      "s_xor_b32 %[t], %[sb], 0x20000\n"
      "buffer_load_dwordx4 %[b0], %[rb], %[t] offen  lds\n"
      "s_xor_b32 m0, %[mb], 3072\n"
      "s_xor_b32 %[t], %[sb], 0x30000\n"
      "buffer_load_dwordx4 %[b1], %[rb], %[t] offen  lds\n"
      "s_xor_b32 m0, %[mb], 4096\n"
      "s_xor_b32 %[t], %[sb], 0x40000\n"
      "buffer_load_dwordx4 %[b2], %[rb], %[t] offen  lds\n"
      "s_xor_b32 m0, %[mb], 5120\n"
      "s_xor_b32 %[t], %[sb], 0x50000\n"
      "buffer_load_dwordx4 %[b3], %[rb], %[t] offen  lds\n"
      "s_xor_b32 m0, %[mb], 6144\n"
      "s_xor_b32 %[t], %[sb], 0x60000\n"
      "buffer_load_dwordx4 %[b2], %[rb], %[t] offen  lds\n"
      "s_xor_b32 m0, %[mb], 7168\n"
      "s_xor_b32 %[t], %[sb], 0x70000\n"
      "buffer_load_dwordx4 %[b3], %[rb], %[t] offen  lds\n"
      "s_add_u32 %[kk], %[kk], 1\n"
      "s_xor_b32 %[t], %[kk], %[kx]\n"
      "s_sub_u32 %[t], %[t], %[kp]\n"
      "s_and_b32 %[t], %[t], 63\n"
      "s_mul_i32 %[t2], %[t], %[ssa]\n"
      "s_add_u32 %[sa], %[sa0], %[t2]\n"
      "s_mul_i32 %[t2], %[t], %[ssb]\n"
      "s_add_u32 %[sb], %[sb0], %[t2]\n"
      "s_add_u32 %[ma], %[ma], 16384\n"
      "s_add_u32 %[mb], %[mb], 32768\n"
      "s_cmp_eq_u32 %[ma], %[mae]\n"
      "s_cselect_b32 %[ma], %[ma0], %[ma]\n"
      "s_cselect_b32 %[mb], %[mb0], %[mb]\n"
      "s_add_u32 %[n], %[n], 1\n"
      "s_waitcnt vmcnt(12)\n"
      "s_cmp_lt_u32 %[n], 64\n"
      "s_barrier\n"
      "s_cbranch_scc1 L_pc_%=\n"
      "L_e_%=:\n"
      "s_waitcnt vmcnt(0)\n"
      "s_barrier\n"
      "s_barrier\n"
                : [t] "=&s"(t), [t2] "=&s"(t2), [kk] "+&s"(kk), [n] "=&s"(n), [sa] "+&s"(sa), [sb] "+&s"(sb), [ma] "+&s"(ma), [mb] "+&s"(mb), [call] "+&v"(call), [sp] "+&s"(sp), [pd] "=&v"(pd), [tv] "+&v"(tv), [tw] "+&v"(tw)
                : [a0] "v"(a0), [a1] "v"(a1), [b0] "v"(b0), [b1] "v"(b1), [b2] "v"(b2), [b3] "v"(b3), [ra] "s"(ra), [rb] "s"(rb),
                  [ssa] "s"(ssa), [ssb] "s"(ssb), [sa0] "s"(sa0), [sb0] "s"(sb0), [kx] "s"(kx), [kp] "s"(kp), [ma0] "s"(ma0), [mb0] "s"(mb0), [mae] "s"(mae), [pw] "s"(pw), [ty] "s"(ty), [pv] "v"(pv), [rp] "s"(rp), [ssp] "s"(ssp), [ca] "v"(myws + WS_CALL + b * 4)
                : "memory", "scc", "s40", "s41");
        }
        if (wid == 5) STAMP(1);
      if (wid < 7) {
        BAR();
        BAR();
      } else {
        call = rfl(call) + 1;
        if (lane == 0) lds_store(lds0 + F_MISC + 24, call);
        BAR();
        u32x4 wm; asm volatile("ds_read_b128 %0, %1\n\ts_waitcnt lgkmcnt(0)" : "=&v"(wm) : "v"(lds0 + F_MISC) : "memory");
        const u32 tm = max(max(wm.x, wm.y), max(wm.z, wm.w));
        bool bad = false;
        if (lane == 0) st_sys64(myws + WS_LMAX + (grp * 32 + idx) * 8, ((u64)call << 32) | tm);
        if (idx == 0) {
            const u32 gm = sweep(myws + WS_LMAX + grp * 256, 32, call, myws + WS_TMO, lane, bad);
            if (!bad && lane < NR) st_sys64(p.ws[lane] + WS_GMAX + (call & 1) * 512 + (p.rank * 8 + grp) * 8, ((u64)call << 32) | gm);
        }
        const u32 am = sweep(myws + WS_GMAX + (call & 1) * 512, 64, call, myws + WS_TMO, lane, bad);
        if (lane == 0) { lds_store(lds0 + F_MISC + 16, am); lds_store(lds0 + F_MISC + 20, bad ? 1u : 0u); }
        STAMP(3);
        BAR();
      }
    }
    BAR();
    {
        u32 tz = threadIdx.x; asm volatile("" : "+v"(tz));
        const u32 lane = tz & 63;
        unsigned char* dst = p.out[grp] + (size_t)(p.rank * ML + rowblk * BM + wid * 16 + (lane >> 4)) * NC + ct * BN + (lane & 15) * 16;
        const u32 la = lds0 + (wid * 16 + (lane >> 4)) * EP_PITCH + (lane & 15) * 16;
        u32x4 d0, d1, d2, d3;
        asm volatile("ds_read_b128 %0, %4\n\tds_read_b128 %1, %4 offset:1088\n\tds_read_b128 %2, %4 offset:2176\n\tds_read_b128 %3, %4 offset:3264\n\ts_waitcnt lgkmcnt(0)"
                     : "=&v"(d0), "=&v"(d1), "=&v"(d2), "=&v"(d3) : "v"(la) : "memory");
        asm volatile("global_store_dwordx4 %0, %1, off sc0 sc1\n\ts_nop 1" :: "v"(dst), "v"(d0) : "memory");
        asm volatile("global_store_dwordx4 %0, %1, off sc0 sc1\n\ts_nop 1" :: "v"(dst + 4 * NC), "v"(d1) : "memory");
        asm volatile("global_store_dwordx4 %0, %1, off sc0 sc1\n\ts_nop 1" :: "v"(dst + 8 * NC), "v"(d2) : "memory");
        asm volatile("global_store_dwordx4 %0, %1, off sc0 sc1\n\ts_nop 1" :: "v"(dst + 12 * NC), "v"(d3) : "memory");
        asm volatile("s_waitcnt vmcnt(0)" ::: "memory");
        BAR();
        if (wid == 7) {
            STAMP(4);
            u32 call; asm volatile("ds_read_b32 %0, %1\n\ts_waitcnt lgkmcnt(0)" : "=&v"(call) : "v"(lds0 + F_MISC + 24) : "memory");
            call = rfl(call);
            if (lane == 0 && grp != p.rank) st_sys32(p.ws[grp] + WS_DONE + (p.rank * 32 + idx) * 4, call);
            if (grp == p.rank && idx < NR && idx != p.rank) {
                const u64 t0 = __builtin_amdgcn_s_memrealtime();
                const unsigned char* q = myws + WS_DONE + (idx * 32 + (lane & 31)) * 4;
                for (;;) {
                    const u32 v = ld_sys32(q);
                    const u32 t = ld_agt32(myws + WS_TMO);
                    if (t != call && __all((int)(v - call) >= 0)) break;
                    if (t == call || __builtin_amdgcn_s_memrealtime() - t0 > DEADLINE) {
                        if (lane == 0) { st_agt32(myws + WS_TMO, call); *dst = 0x7f; }
                        break;
                    }
                    __builtin_amdgcn_s_sleep(1);
                }
            }
            STAMP(5);
        }
    }
}

extern "C" void kernel_launch(int rank, int nranks, ncclComm_t comm, hipStream_t stream,
                              void* const* d_in, const int* in_sizes, int n_in,
                              void* d_out, int out_size, const int* in_dtypes, int out_dtype,
                              void* const* d_out_all, void* const* d_ws_all, size_t ws_bytes) {
    if (nranks != NR || n_in < 2 || in_sizes[0] != ML * KD || in_sizes[1] != KD * ND || out_size != NR * ML * NC || ws_bytes < WS_NEED) {
        fprintf(stderr, "kernel_launch: unexpected shapes\n"); return;
    }
    static bool attr[NR] = {};
    if (!attr[rank]) { HIP_CHECK(hipFuncSetAttribute((const void*)fused_kernel, hipFuncAttributeMaxDynamicSharedMemorySize, LDS_BYTES)); attr[rank] = true; }
    Params p;
    p.x = d_in[0]; p.w = d_in[1]; p.rank = rank; p.pad = 0;
    for (int i = 0; i < NR; ++i) { p.out[i] = (unsigned char*)d_out_all[i]; p.ws[i] = (unsigned char*)d_ws_all[i]; }
    hipLaunchKernelGGL(fused_kernel, dim3(NWG), dim3(512), LDS_BYTES, stream, p);
    HIP_CHECK(hipGetLastError());
}
```

```cpp
DI void issue_a1(u32x4 rs, u32 v0, u32 v1, u32 mb, u32 sb) {
    u32 t;
    asm volatile(
        DMA(v0, 0, 0x0) DMA(v1, 1024, 0x10000) DMA(v0, 2048, 0x20000) DMA(v1, 3072, 0x30000)
        : [t] "=&s"(t) : [v0] "v"(v0), [v1] "v"(v1), [rs] "s"(rs), [mb] "s"(mb), [sb] "s"(sb) : "memory", "scc");
}
DI void issue_a2(u32x4 rs, u32 v0, u32 v1, u32 mb, u32 sb) {
    u32 t;
    asm volatile(
        DMA(v0, 4096, 0x40000) DMA(v1, 5120, 0x50000) DMA(v0, 6144, 0x60000) DMA(v1, 7168, 0x70000) DMA(v0, 8192, 0x80000) DMA(v1, 9216, 0x90000) DMA(v0, 10240, 0xa0000) DMA(v1, 11264, 0xb0000) DMA(v0, 12288, 0xc0000) DMA(v1, 13312, 0xd0000) DMA(v0, 14336, 0xe0000) DMA(v1, 15360, 0xf0000)
        : [t] "=&s"(t) : [v0] "v"(v0), [v1] "v"(v1), [rs] "s"(rs), [mb] "s"(mb), [sb] "s"(sb) : "memory", "scc");
}
DI void issue_b1(u32x4 rs, u32 v0, u32 v1, u32 mb, u32 sb) {
    u32 t;
    asm volatile(
        DMA(v0, 0, 0x0) DMA(v0, 2048, 0x20000) DMA(v1, 4096, 0x40000) DMA(v1, 6144, 0x60000)
        : [t] "=&s"(t) : [v0] "v"(v0), [v1] "v"(v1), [rs] "s"(rs), [mb] "s"(mb), [sb] "s"(sb) : "memory", "scc");
}
DI void issue_b2(u32x4 rs, u32 v0, u32 v1, u32 mb, u32 sb) {
    u32 t;
    asm volatile(
        DMA(v0, 8192, 0x80000) DMA(v0, 10240, 0xa0000) DMA(v1, 12288, 0xc0000) DMA(v1, 14336, 0xe0000) DMA(v0, 16384, 0x100000) DMA(v0, 18432, 0x120000) DMA(v1, 20480, 0x140000) DMA(v1, 22528, 0x160000) DMA(v0, 24576, 0x180000) DMA(v0, 26624, 0x1a0000) DMA(v1, 28672, 0x1c0000) DMA(v1, 30720, 0x1e0000)
        : [t] "=&s"(t) : [v0] "v"(v0), [v1] "v"(v1), [rs] "s"(rs), [mb] "s"(mb), [sb] "s"(sb) : "memory", "scc");
}

_m4096_k4096_n16384_bf16_reluqe4m3 — fused GEMM + ReLU + global amax + e4m3 quantize + all-to-all, one kernel.
#include <hip/hip_runtime.h>
#include <rccl/rccl.h>
#include <cstdio>
#include <cstdint>

#define HIP_CHECK(expr) do { hipError_t _e = (expr); if (_e != hipSuccess) { \
    fprintf(stderr, "HIP error %s at %s:%d\n", hipGetErrorString(_e), __FILE__, __LINE__); \
    return; } } while (0)

typedef unsigned u32;
typedef unsigned long long u64;
typedef float f32x16 __attribute__((ext_vector_type(16)));
typedef u32 u32x4 __attribute__((ext_vector_type(4)));
typedef u32 u32x2 __attribute__((ext_vector_type(2)));
typedef float f32x4 __attribute__((ext_vector_type(4)));
#define LAS __attribute__((address_space(3)))
#define GAS __attribute__((address_space(1)))
#define DI __device__ __forceinline__

constexpr int NR = 8, ML = 512, KD = 4096, ND = 16384, NC = 2048;
constexpr int BM = 128, BN = 256, NT = 64;
constexpr int NWG = 256;
constexpr u32 A_BASE = 0, A_ST = 16384, B_BASE = 49152, B_ST = 32768;
constexpr u32 F_LUT = 147456, F_MISC = F_LUT + 256, LDS_BYTES = F_MISC + 256;
constexpr u32 ST_PITCH = 33280;
constexpr u32 PAY_PITCH = 9216;
constexpr size_t WS_CALL = 0;
constexpr size_t WS_TMO = 4096;
constexpr size_t WS_LMAX = 8192;
constexpr size_t WS_GMAX = 12288;
constexpr size_t WS_HFLAG = 16384;
constexpr size_t WS_PFLAG = 20480;
constexpr size_t WS_DBG = 65536;
constexpr size_t WS_DBG2 = WS_DBG + 256 * 64 * 8;
constexpr size_t WS_HDR = 1u << 20;
constexpr size_t WS_PAY = 3u << 20;
constexpr size_t WS_NEED = WS_PAY + (size_t)8 * 32 * 4 * PAY_PITCH + 1024;
static_assert(WS_DBG2 + 256 * 4 * 128 * 4 <= WS_HDR, "debug area");
constexpr u64 DEADLINE = 50000000ull;

struct Params {
    const void* x; const void* w;
    unsigned char* out[NR];
    unsigned char* ws[NR];
    int rank; int pad;
};

DI u32 rfl(u32 v) { return __builtin_amdgcn_readfirstlane(v); }
DI u32x4 make_rsrc(const void* p, u32 bytes) {
    u64 a = (u64)p; u32x4 r; r.x = rfl((u32)a); r.y = rfl((u32)(a >> 32)); r.z = bytes; r.w = 0x00020000u; return r;
}
DI void lds_store(u32 addr, u32 v) { asm volatile("ds_write_b32 %0, %1" :: "v"(addr), "v"(v) : "memory"); }
DI u32 lds_min4(u32 addr) {
    u32x4 d; asm volatile("ds_read_b128 %0, %1\n\ts_waitcnt lgkmcnt(0)" : "=&v"(d) : "v"(addr) : "memory");
    return rfl(min(min(d.x, d.y), min(d.z, d.w)));
}
#define BAR() do { asm volatile("s_waitcnt lgkmcnt(0)\n\ts_barrier" ::: "memory"); } while (0)
#define NSTAMP 10
#define CPRIO 2
#define PPRIO 0
#define ROWBLK (idx & 3)
#define COLT (idx >> 2)
#define ROTX pw
#define ROTW pw
#define WPAIR 1
#define LAG 0
#define RW ((rowblk >> 1) * 4)
#define RX ((ct >> 2) * 2)
#define KX ((rowblk ^ (ct >> 1)) & 1)
#define KP (ct & 1)
#define STAGW 6
#define STAGD 0
#define PSLEEP 1
#define STAMP(k) do {} while (0)

DI u64 ld_sys64(const unsigned char* p) { return __hip_atomic_load((const GAS u64*)p, __ATOMIC_RELAXED, __HIP_MEMORY_SCOPE_SYSTEM); }
DI void st_sys64(unsigned char* p, u64 v) { __hip_atomic_store((GAS u64*)p, v, __ATOMIC_RELAXED, __HIP_MEMORY_SCOPE_SYSTEM); }
DI u32 ld_sys32(const unsigned char* p) { return __hip_atomic_load((const GAS u32*)p, __ATOMIC_RELAXED, __HIP_MEMORY_SCOPE_SYSTEM); }
DI void st_sys32(unsigned char* p, u32 v) { __hip_atomic_store((GAS u32*)p, v, __ATOMIC_RELAXED, __HIP_MEMORY_SCOPE_SYSTEM); }
DI u32 ld_agt32(const unsigned char* p) { return __hip_atomic_load((const GAS u32*)p, __ATOMIC_RELAXED, __HIP_MEMORY_SCOPE_AGENT); }
DI void st_agt32(unsigned char* p, u32 v) { __hip_atomic_store((GAS u32*)p, v, __ATOMIC_RELAXED, __HIP_MEMORY_SCOPE_AGENT); }
DI u32 wave_umax(u32 v) {
#pragma unroll
    for (int o = 32; o >= 1; o >>= 1) v = max(v, (u32)__shfl_xor((int)v, o));
    return v;
}

DI u32 wave_scan(u32 v) {
    v += __builtin_amdgcn_update_dpp(0u, v, 0x111, 0xf, 0xf, false);
    v += __builtin_amdgcn_update_dpp(0u, v, 0x112, 0xf, 0xf, false);
    v += __builtin_amdgcn_update_dpp(0u, v, 0x114, 0xf, 0xf, false);
    v += __builtin_amdgcn_update_dpp(0u, v, 0x118, 0xf, 0xf, false);
    v += __builtin_amdgcn_update_dpp(0u, v, 0x142, 0xa, 0xf, false);
    v += __builtin_amdgcn_update_dpp(0u, v, 0x143, 0xc, 0xf, false);
    return v;
}
DI u32 q4(float a, float b, float c, float d) {
    int w = __builtin_amdgcn_cvt_pk_fp8_f32(a, b, 0, false);
    return (u32)__builtin_amdgcn_cvt_pk_fp8_f32(c, d, w, true);
}
DI bool wait_flag(const unsigned char* f, u32 call, unsigned char* tmo, int lane) {
    const u64 t0 = __builtin_amdgcn_s_memrealtime();
    for (;;) {
        const u32 v = rfl(ld_sys32(f));
        const u32 t = rfl(ld_agt32(tmo));
        if (t == call) return true;
        if ((int)(v - call) >= 0) return false;
        if (__builtin_amdgcn_s_memrealtime() - t0 > DEADLINE) { if (lane == 0) st_agt32(tmo, call); return true; }
        __builtin_amdgcn_s_sleep(PSLEEP);
    }
}

DI u32 sweep(const unsigned char* g, int n, u32 call, unsigned char* tmo, int lane, bool& bad) {
    const u64 t0 = __builtin_amdgcn_s_memrealtime();
    const unsigned char* q = g + (lane < n ? lane : 0) * 8;
    for (;;) {
        const u64 v = ld_sys64(q);
        const u32 t = ld_agt32(tmo);
        if (t == call) { bad = true; return 0; }
        if (__all((u32)(v >> 32) == call)) return wave_umax((u32)v);
        if (__builtin_amdgcn_s_memrealtime() - t0 > DEADLINE) { if (lane == 0) st_agt32(tmo, call); bad = true; return 0; }
        __builtin_amdgcn_s_sleep(1);
    }
}

__global__ __launch_bounds__(512, 2) void fused_kernel(Params p) {
    extern __shared__ __attribute__((aligned(16))) unsigned char smem[];
    const int wid = rfl(threadIdx.x >> 6);
    const int b = blockIdx.x, grp = b & 7, idx = b >> 3;
    const int rowblk = ROWBLK, ct = COLT;
    const u32 lds0 = (u32)(uintptr_t)(LAS unsigned char*)smem;
    unsigned char* const myws = p.ws[p.rank];
    if (wid == 0) STAMP(0);
    unsigned char* const pws = p.ws[grp]; const u32 sslot = p.rank * 32 + idx;
    unsigned char* const rout = p.out[p.rank] + (size_t)(grp * ML + rowblk * BM) * NC + ct * BN;
    const u32 rslot = grp * 32 + idx;
    const bool remote = grp != p.rank;

    if (wid < 4) {
        f32x16 c0, c1, c2, c3, c4, c5, c6, c7; u32 slowc; u64 tc0 = 0;
        {
            const u32 lane = threadIdx.x & 63, r = lane & 15, g = lane >> 4, q = r >> 2, pp = lane & 3;
            const u32 a0 = lds0 + A_BASE + r * 128 + ((g ^ (r >> 1)) << 4), a1 = a0 ^ 64;
            const u32 bb = lds0 + B_BASE + (8 * g + q) * 512, F = q + 8 * (g & 1), cb = 8 * wid + 2 * pp, h = (pp & 1) * 8;
            const u32 b0 = bb + ((cb ^ F) << 4) + h, b1 = b0 ^ 8, b2 = b0 ^ 16, b3 = b0 ^ 24;
            u32 si, sst, sdA, sdB; slowc = 0;
            __builtin_amdgcn_s_setprio(CPRIO);
            asm volatile(
      "v_mov_b32 v16, %[a0]\n"
      "v_mov_b32 v17, %[a1]\n"
      "v_mov_b32 v18, %[b0]\n"
      "v_mov_b32 v19, %[b1]\n"
      "v_mov_b32 v20, %[b2]\n"
      "v_mov_b32 v21, %[b3]\n"
      "v_mov_b32 v128, 0\n"
      "v_mov_b32 v129, 0\n"
      "v_mov_b32 v130, 0\n"
      "v_mov_b32 v131, 0\n"
      "v_mov_b32 v132, 0\n"
      "v_mov_b32 v133, 0\n"
      "v_mov_b32 v134, 0\n"
      "v_mov_b32 v135, 0\n"
      "v_mov_b32 v136, 0\n"
      "v_mov_b32 v137, 0\n"
      "v_mov_b32 v138, 0\n"
      "v_mov_b32 v139, 0\n"
      "v_mov_b32 v140, 0\n"
      "v_mov_b32 v141, 0\n"
      "v_mov_b32 v142, 0\n"
      "v_mov_b32 v143, 0\n"
      "v_mov_b32 v144, 0\n"
      "v_mov_b32 v145, 0\n"
      "v_mov_b32 v146, 0\n"
      "v_mov_b32 v147, 0\n"
      "v_mov_b32 v148, 0\n"
      "v_mov_b32 v149, 0\n"
      "v_mov_b32 v150, 0\n"
      "v_mov_b32 v151, 0\n"
      "v_mov_b32 v152, 0\n"
      "v_mov_b32 v153, 0\n"
      "v_mov_b32 v154, 0\n"
      "v_mov_b32 v155, 0\n"
      "v_mov_b32 v156, 0\n"
      "v_mov_b32 v157, 0\n"
      "v_mov_b32 v158, 0\n"
      "v_mov_b32 v159, 0\n"
      "v_mov_b32 v160, 0\n"
      "v_mov_b32 v161, 0\n"
      "v_mov_b32 v162, 0\n"
      "v_mov_b32 v163, 0\n"
      "v_mov_b32 v164, 0\n"
      "v_mov_b32 v165, 0\n"
      "v_mov_b32 v166, 0\n"
      "v_mov_b32 v167, 0\n"
      "v_mov_b32 v168, 0\n"
      "v_mov_b32 v169, 0\n"
      "v_mov_b32 v170, 0\n"
      "v_mov_b32 v171, 0\n"
      "v_mov_b32 v172, 0\n"
      "v_mov_b32 v173, 0\n"
      "v_mov_b32 v174, 0\n"
      "v_mov_b32 v175, 0\n"
      "v_mov_b32 v176, 0\n"
      "v_mov_b32 v177, 0\n"
      "v_mov_b32 v178, 0\n"
      "v_mov_b32 v179, 0\n"
      "v_mov_b32 v180, 0\n"
      "v_mov_b32 v181, 0\n"
      "v_mov_b32 v182, 0\n"
      "v_mov_b32 v183, 0\n"
      "v_mov_b32 v184, 0\n"
      "v_mov_b32 v185, 0\n"
      "v_mov_b32 v186, 0\n"
      "v_mov_b32 v187, 0\n"
      "v_mov_b32 v188, 0\n"
      "v_mov_b32 v189, 0\n"
      "v_mov_b32 v190, 0\n"
      "v_mov_b32 v191, 0\n"
      "v_mov_b32 v192, 0\n"
      "v_mov_b32 v193, 0\n"
      "v_mov_b32 v194, 0\n"
      "v_mov_b32 v195, 0\n"
      "v_mov_b32 v196, 0\n"
      "v_mov_b32 v197, 0\n"
      "v_mov_b32 v198, 0\n"
      "v_mov_b32 v199, 0\n"
      "v_mov_b32 v200, 0\n"
      "v_mov_b32 v201, 0\n"
      "v_mov_b32 v202, 0\n"
      "v_mov_b32 v203, 0\n"
      "v_mov_b32 v204, 0\n"
      "v_mov_b32 v205, 0\n"
      "v_mov_b32 v206, 0\n"
      "v_mov_b32 v207, 0\n"
      "v_mov_b32 v208, 0\n"
      "v_mov_b32 v209, 0\n"
      "v_mov_b32 v210, 0\n"
      "v_mov_b32 v211, 0\n"
      "v_mov_b32 v212, 0\n"
      "v_mov_b32 v213, 0\n"
      "v_mov_b32 v214, 0\n"
      "v_mov_b32 v215, 0\n"
      "v_mov_b32 v216, 0\n"
      "v_mov_b32 v217, 0\n"
      "v_mov_b32 v218, 0\n"
      "v_mov_b32 v219, 0\n"
      "v_mov_b32 v220, 0\n"
      "v_mov_b32 v221, 0\n"
      "v_mov_b32 v222, 0\n"
      "v_mov_b32 v223, 0\n"
      "v_mov_b32 v224, 0\n"
      "v_mov_b32 v225, 0\n"
      "v_mov_b32 v226, 0\n"
      "v_mov_b32 v227, 0\n"
      "v_mov_b32 v228, 0\n"
      "v_mov_b32 v229, 0\n"
      "v_mov_b32 v230, 0\n"
      "v_mov_b32 v231, 0\n"
      "v_mov_b32 v232, 0\n"
      "v_mov_b32 v233, 0\n"
      "v_mov_b32 v234, 0\n"
      "v_mov_b32 v235, 0\n"
      "v_mov_b32 v236, 0\n"
      "v_mov_b32 v237, 0\n"
      "v_mov_b32 v238, 0\n"
      "v_mov_b32 v239, 0\n"
      "v_mov_b32 v240, 0\n"
      "v_mov_b32 v241, 0\n"
      "v_mov_b32 v242, 0\n"
      "v_mov_b32 v243, 0\n"
      "v_mov_b32 v244, 0\n"
      "v_mov_b32 v245, 0\n"
      "v_mov_b32 v246, 0\n"
      "v_mov_b32 v247, 0\n"
      "v_mov_b32 v248, 0\n"
      "v_mov_b32 v249, 0\n"
      "v_mov_b32 v250, 0\n"
      "v_mov_b32 v251, 0\n"
      "v_mov_b32 v252, 0\n"
      "v_mov_b32 v253, 0\n"
      "v_mov_b32 v254, 0\n"
      "v_mov_b32 v255, 0\n"
      "s_mov_b32 %[i], 0\n"
      "s_mov_b32 %[st], 0\n"
      "s_barrier\n"
      "ds_read_b64_tr_b16 v[64:65], v18 offset:0\n"
      "ds_read_b64_tr_b16 v[66:67], v18 offset:2048\n"
      "ds_read_b128 v[32:35], v16 offset:0\n"
      "ds_read_b64_tr_b16 v[68:69], v19 offset:0\n"
      "ds_read_b64_tr_b16 v[70:71], v19 offset:2048\n"
      "ds_read_b64_tr_b16 v[72:73], v20 offset:0\n"
      "ds_read_b64_tr_b16 v[74:75], v20 offset:2048\n"
      "ds_read_b64_tr_b16 v[76:77], v21 offset:0\n"
      "ds_read_b64_tr_b16 v[78:79], v21 offset:2048\n"
      "ds_read_b128 v[36:39], v16 offset:2048\n"
      "ds_read_b128 v[40:43], v16 offset:4096\n"
      "ds_read_b128 v[44:47], v16 offset:6144\n"
      "ds_read_b128 v[48:51], v16 offset:8192\n"
      "ds_read_b128 v[52:55], v16 offset:10240\n"
      "ds_read_b128 v[56:59], v16 offset:12288\n"
      "ds_read_b128 v[60:63], v16 offset:14336\n"
      "s_waitcnt lgkmcnt(0)\n"
      ".p2align 4\n"
      "L_loop_%=:\n"
      "s_waitcnt lgkmcnt(0)\n"
      "v_mfma_f32_16x16x32_bf16 v[128:131], v[64:67], v[32:35], v[128:131]\n"
      "ds_read_b64_tr_b16 v[112:113], v18 offset:16384\n"
      "v_mfma_f32_16x16x32_bf16 v[132:135], v[68:71], v[32:35], v[132:135]\n"
      "ds_read_b64_tr_b16 v[114:115], v18 offset:18432\n"
      "s_cmp_eq_u32 %[st], 2\n"
      "v_mfma_f32_16x16x32_bf16 v[136:139], v[72:75], v[32:35], v[136:139]\n"
      "ds_read_b128 v[80:83], v17 offset:0\n"
      "s_cselect_b32 %[dA], %[cm], %[cp]\n"
      "v_mfma_f32_16x16x32_bf16 v[140:143], v[76:79], v[32:35], v[140:143]\n"
      "ds_read_b64_tr_b16 v[116:117], v19 offset:16384\n"
      "s_cselect_b32 %[st], -1, %[st]\n"
      "v_mfma_f32_16x16x32_bf16 v[156:159], v[76:79], v[36:39], v[156:159]\n"
      "ds_read_b64_tr_b16 v[118:119], v19 offset:18432\n"
      "s_add_u32 %[st], %[st], 1\n"
      "v_mfma_f32_16x16x32_bf16 v[152:155], v[72:75], v[36:39], v[152:155]\n"
      "ds_read_b64_tr_b16 v[120:121], v20 offset:16384\n"
      "s_lshl_b32 %[dB], %[dA], 1\n"
      "v_mfma_f32_16x16x32_bf16 v[148:151], v[68:71], v[36:39], v[148:151]\n"
      "ds_read_b64_tr_b16 v[122:123], v20 offset:18432\n"
      "v_mfma_f32_16x16x32_bf16 v[144:147], v[64:67], v[36:39], v[144:147]\n"
      "ds_read_b64_tr_b16 v[124:125], v21 offset:16384\n"
      "v_mfma_f32_16x16x32_bf16 v[160:163], v[64:67], v[40:43], v[160:163]\n"
      "ds_read_b64_tr_b16 v[126:127], v21 offset:18432\n"
      "v_mfma_f32_16x16x32_bf16 v[164:167], v[68:71], v[40:43], v[164:167]\n"
      "ds_read_b128 v[84:87], v17 offset:2048\n"
      "v_mfma_f32_16x16x32_bf16 v[168:171], v[72:75], v[40:43], v[168:171]\n"
      "ds_read_b128 v[88:91], v17 offset:4096\n"
      "v_mfma_f32_16x16x32_bf16 v[172:175], v[76:79], v[40:43], v[172:175]\n"
      "ds_read_b128 v[92:95], v17 offset:6144\n"
      "v_mfma_f32_16x16x32_bf16 v[188:191], v[76:79], v[44:47], v[188:191]\n"
      "ds_read_b128 v[96:99], v17 offset:8192\n"
      "v_mfma_f32_16x16x32_bf16 v[184:187], v[72:75], v[44:47], v[184:187]\n"
      "ds_read_b128 v[100:103], v17 offset:10240\n"
      "v_mfma_f32_16x16x32_bf16 v[180:183], v[68:71], v[44:47], v[180:183]\n"
      "ds_read_b128 v[104:107], v17 offset:12288\n"
      "v_mfma_f32_16x16x32_bf16 v[176:179], v[64:67], v[44:47], v[176:179]\n"
      "ds_read_b128 v[108:111], v17 offset:14336\n"
      "v_mfma_f32_16x16x32_bf16 v[192:195], v[64:67], v[48:51], v[192:195]\n"
      "v_add_u32 v16, %[dA], v16\n"
      "v_mfma_f32_16x16x32_bf16 v[196:199], v[68:71], v[48:51], v[196:199]\n"
      "v_add_u32 v17, %[dA], v17\n"
      "v_mfma_f32_16x16x32_bf16 v[200:203], v[72:75], v[48:51], v[200:203]\n"
      "v_add_u32 v18, %[dB], v18\n"
      "v_mfma_f32_16x16x32_bf16 v[204:207], v[76:79], v[48:51], v[204:207]\n"
      "v_add_u32 v19, %[dB], v19\n"
      "v_mfma_f32_16x16x32_bf16 v[220:223], v[76:79], v[52:55], v[220:223]\n"
      "v_add_u32 v20, %[dB], v20\n"
      "v_mfma_f32_16x16x32_bf16 v[216:219], v[72:75], v[52:55], v[216:219]\n"
      "v_add_u32 v21, %[dB], v21\n"
      "v_mfma_f32_16x16x32_bf16 v[212:215], v[68:71], v[52:55], v[212:215]\n"
      "v_mfma_f32_16x16x32_bf16 v[208:211], v[64:67], v[52:55], v[208:211]\n"
      "v_mfma_f32_16x16x32_bf16 v[224:227], v[64:67], v[56:59], v[224:227]\n"
      "s_waitcnt lgkmcnt(0)\n"
      "s_barrier\n"
      "v_mfma_f32_16x16x32_bf16 v[228:231], v[68:71], v[56:59], v[228:231]\n"
      "v_mfma_f32_16x16x32_bf16 v[232:235], v[72:75], v[56:59], v[232:235]\n"
      "v_mfma_f32_16x16x32_bf16 v[236:239], v[76:79], v[56:59], v[236:239]\n"
      "v_mfma_f32_16x16x32_bf16 v[252:255], v[76:79], v[60:63], v[252:255]\n"
      "v_mfma_f32_16x16x32_bf16 v[248:251], v[72:75], v[60:63], v[248:251]\n"
      "v_mfma_f32_16x16x32_bf16 v[244:247], v[68:71], v[60:63], v[244:247]\n"
      "v_mfma_f32_16x16x32_bf16 v[240:243], v[64:67], v[60:63], v[240:243]\n"
      "v_mfma_f32_16x16x32_bf16 v[128:131], v[112:115], v[80:83], v[128:131]\n"
      "ds_read_b64_tr_b16 v[64:65], v18 offset:0\n"
      "v_mfma_f32_16x16x32_bf16 v[132:135], v[116:119], v[80:83], v[132:135]\n"
      "ds_read_b64_tr_b16 v[66:67], v18 offset:2048\n"
      "v_mfma_f32_16x16x32_bf16 v[136:139], v[120:123], v[80:83], v[136:139]\n"
      "ds_read_b128 v[32:35], v16 offset:0\n"
      "v_mfma_f32_16x16x32_bf16 v[140:143], v[124:127], v[80:83], v[140:143]\n"
      "ds_read_b64_tr_b16 v[68:69], v19 offset:0\n"
      "v_mfma_f32_16x16x32_bf16 v[156:159], v[124:127], v[84:87], v[156:159]\n"
      "ds_read_b64_tr_b16 v[70:71], v19 offset:2048\n"
      "v_mfma_f32_16x16x32_bf16 v[152:155], v[120:123], v[84:87], v[152:155]\n"
      "ds_read_b64_tr_b16 v[72:73], v20 offset:0\n"
      "v_mfma_f32_16x16x32_bf16 v[148:151], v[116:119], v[84:87], v[148:151]\n"
      "ds_read_b64_tr_b16 v[74:75], v20 offset:2048\n"
      "v_mfma_f32_16x16x32_bf16 v[144:147], v[112:115], v[84:87], v[144:147]\n"
      "ds_read_b64_tr_b16 v[76:77], v21 offset:0\n"
      "v_mfma_f32_16x16x32_bf16 v[160:163], v[112:115], v[88:91], v[160:163]\n"
      "ds_read_b64_tr_b16 v[78:79], v21 offset:2048\n"
      "v_mfma_f32_16x16x32_bf16 v[164:167], v[116:119], v[88:91], v[164:167]\n"
      "ds_read_b128 v[36:39], v16 offset:2048\n"
      "v_mfma_f32_16x16x32_bf16 v[168:171], v[120:123], v[88:91], v[168:171]\n"
      "ds_read_b128 v[40:43], v16 offset:4096\n"
      "v_mfma_f32_16x16x32_bf16 v[172:175], v[124:127], v[88:91], v[172:175]\n"
      "ds_read_b128 v[44:47], v16 offset:6144\n"
      "v_mfma_f32_16x16x32_bf16 v[188:191], v[124:127], v[92:95], v[188:191]\n"
      "ds_read_b128 v[48:51], v16 offset:8192\n"
      "v_mfma_f32_16x16x32_bf16 v[184:187], v[120:123], v[92:95], v[184:187]\n"
      "ds_read_b128 v[52:55], v16 offset:10240\n"
      "v_mfma_f32_16x16x32_bf16 v[180:183], v[116:119], v[92:95], v[180:183]\n"
      "ds_read_b128 v[56:59], v16 offset:12288\n"
      "v_mfma_f32_16x16x32_bf16 v[176:179], v[112:115], v[92:95], v[176:179]\n"
      "ds_read_b128 v[60:63], v16 offset:14336\n"
      "v_mfma_f32_16x16x32_bf16 v[192:195], v[112:115], v[96:99], v[192:195]\n"
      "v_mfma_f32_16x16x32_bf16 v[196:199], v[116:119], v[96:99], v[196:199]\n"
      "v_mfma_f32_16x16x32_bf16 v[200:203], v[120:123], v[96:99], v[200:203]\n"
      "v_mfma_f32_16x16x32_bf16 v[204:207], v[124:127], v[96:99], v[204:207]\n"
      "v_mfma_f32_16x16x32_bf16 v[220:223], v[124:127], v[100:103], v[220:223]\n"
      "v_mfma_f32_16x16x32_bf16 v[216:219], v[120:123], v[100:103], v[216:219]\n"
      "v_mfma_f32_16x16x32_bf16 v[212:215], v[116:119], v[100:103], v[212:215]\n"
      "v_mfma_f32_16x16x32_bf16 v[208:211], v[112:115], v[100:103], v[208:211]\n"
      "v_mfma_f32_16x16x32_bf16 v[224:227], v[112:115], v[104:107], v[224:227]\n"
      "v_mfma_f32_16x16x32_bf16 v[228:231], v[116:119], v[104:107], v[228:231]\n"
      "v_mfma_f32_16x16x32_bf16 v[232:235], v[120:123], v[104:107], v[232:235]\n"
      "v_mfma_f32_16x16x32_bf16 v[236:239], v[124:127], v[104:107], v[236:239]\n"
      "v_mfma_f32_16x16x32_bf16 v[252:255], v[124:127], v[108:111], v[252:255]\n"
      "s_add_u32 %[i], %[i], 1\n"
      "v_mfma_f32_16x16x32_bf16 v[248:251], v[120:123], v[108:111], v[248:251]\n"
      "s_cmp_lt_u32 %[i], 64\n"
      "v_mfma_f32_16x16x32_bf16 v[244:247], v[116:119], v[108:111], v[244:247]\n"
      "v_mfma_f32_16x16x32_bf16 v[240:243], v[112:115], v[108:111], v[240:243]\n"
      "s_cbranch_scc1 L_loop_%=\n"
      "s_waitcnt lgkmcnt(0)\n"
      "s_nop 15\n"
                : "=&{v[128:143]}"(c0), "=&{v[144:159]}"(c1), "=&{v[160:175]}"(c2), "=&{v[176:191]}"(c3),
                  "=&{v[192:207]}"(c4), "=&{v[208:223]}"(c5), "=&{v[224:239]}"(c6), "=&{v[240:255]}"(c7),
                  [i] "=&s"(si), [st] "=&s"(sst), [dA] "=&s"(sdA), [dB] "=&s"(sdB)
                : [a0] "v"(a0), [a1] "v"(a1), [b0] "v"(b0), [b1] "v"(b1), [b2] "v"(b2), [b3] "v"(b3),
                  [cm] "s"(0xffff8000u), [cp] "s"(16384u)
                : "v16", "v17", "v18", "v19", "v20", "v21", "v22", "v23", "v24", "v25", "v26", "v27", "v28", "v29", "v30", "v31", "v32", "v33", "v34", "v35", "v36", "v37", "v38", "v39", "v40", "v41", "v42", "v43", "v44", "v45", "v46", "v47", "v48", "v49", "v50", "v51", "v52", "v53", "v54", "v55", "v56", "v57", "v58", "v59", "v60", "v61", "v62", "v63", "v64", "v65", "v66", "v67", "v68", "v69", "v70", "v71", "v72", "v73", "v74", "v75", "v76", "v77", "v78", "v79", "v80", "v81", "v82", "v83", "v84", "v85", "v86", "v87", "v88", "v89", "v90", "v91", "v92", "v93", "v94", "v95", "v96", "v97", "v98", "v99", "v100", "v101", "v102", "v103", "v104", "v105", "v106", "v107", "v108", "v109", "v110", "v111", "v112", "v113", "v114", "v115", "v116", "v117", "v118", "v119", "v120", "v121", "v122", "v123", "v124", "v125", "v126", "v127", "vcc", "scc", "memory");
        }
        if (wid == 0) STAMP(2);
        u32 tz = threadIdx.x; asm volatile("" : "+v"(tz));
        const u32 lane = tz & 63;
#define ACC_IN "{v[128:143]}"(c0), "{v[144:159]}"(c1), "{v[160:175]}"(c2), "{v[176:191]}"(c3), "{v[192:207]}"(c4), "{v[208:223]}"(c5), "{v[224:239]}"(c6), "{v[240:255]}"(c7)
        u32 m0, m1;
        asm volatile(
      "v_max3_f32 %[m0], v128, v129, 0\n"
      "v_max3_f32 %[m1], v130, v131, 0\n"
      "v_max3_f32 %[m0], v132, v133, %[m0]\n"
      "v_max3_f32 %[m1], v134, v135, %[m1]\n"
      "v_max3_f32 %[m0], v136, v137, %[m0]\n"
      "v_max3_f32 %[m1], v138, v139, %[m1]\n"
      "v_max3_f32 %[m0], v140, v141, %[m0]\n"
      "v_max3_f32 %[m1], v142, v143, %[m1]\n"
      "v_max3_f32 %[m0], v144, v145, %[m0]\n"
      "v_max3_f32 %[m1], v146, v147, %[m1]\n"
      "v_max3_f32 %[m0], v148, v149, %[m0]\n"
      "v_max3_f32 %[m1], v150, v151, %[m1]\n"
      "v_max3_f32 %[m0], v152, v153, %[m0]\n"
      "v_max3_f32 %[m1], v154, v155, %[m1]\n"
      "v_max3_f32 %[m0], v156, v157, %[m0]\n"
      "v_max3_f32 %[m1], v158, v159, %[m1]\n"
      "v_max3_f32 %[m0], v160, v161, %[m0]\n"
      "v_max3_f32 %[m1], v162, v163, %[m1]\n"
      "v_max3_f32 %[m0], v164, v165, %[m0]\n"
      "v_max3_f32 %[m1], v166, v167, %[m1]\n"
      "v_max3_f32 %[m0], v168, v169, %[m0]\n"
      "v_max3_f32 %[m1], v170, v171, %[m1]\n"
      "v_max3_f32 %[m0], v172, v173, %[m0]\n"
      "v_max3_f32 %[m1], v174, v175, %[m1]\n"
      "v_max3_f32 %[m0], v176, v177, %[m0]\n"
      "v_max3_f32 %[m1], v178, v179, %[m1]\n"
      "v_max3_f32 %[m0], v180, v181, %[m0]\n"
      "v_max3_f32 %[m1], v182, v183, %[m1]\n"
      "v_max3_f32 %[m0], v184, v185, %[m0]\n"
      "v_max3_f32 %[m1], v186, v187, %[m1]\n"
      "v_max3_f32 %[m0], v188, v189, %[m0]\n"
      "v_max3_f32 %[m1], v190, v191, %[m1]\n"
      "v_max3_f32 %[m0], v192, v193, %[m0]\n"
      "v_max3_f32 %[m1], v194, v195, %[m1]\n"
      "v_max3_f32 %[m0], v196, v197, %[m0]\n"
      "v_max3_f32 %[m1], v198, v199, %[m1]\n"
      "v_max3_f32 %[m0], v200, v201, %[m0]\n"
      "v_max3_f32 %[m1], v202, v203, %[m1]\n"
      "v_max3_f32 %[m0], v204, v205, %[m0]\n"
      "v_max3_f32 %[m1], v206, v207, %[m1]\n"
      "v_max3_f32 %[m0], v208, v209, %[m0]\n"
      "v_max3_f32 %[m1], v210, v211, %[m1]\n"
      "v_max3_f32 %[m0], v212, v213, %[m0]\n"
      "v_max3_f32 %[m1], v214, v215, %[m1]\n"
      "v_max3_f32 %[m0], v216, v217, %[m0]\n"
      "v_max3_f32 %[m1], v218, v219, %[m1]\n"
      "v_max3_f32 %[m0], v220, v221, %[m0]\n"
      "v_max3_f32 %[m1], v222, v223, %[m1]\n"
      "v_max3_f32 %[m0], v224, v225, %[m0]\n"
      "v_max3_f32 %[m1], v226, v227, %[m1]\n"
      "v_max3_f32 %[m0], v228, v229, %[m0]\n"
      "v_max3_f32 %[m1], v230, v231, %[m1]\n"
      "v_max3_f32 %[m0], v232, v233, %[m0]\n"
      "v_max3_f32 %[m1], v234, v235, %[m1]\n"
      "v_max3_f32 %[m0], v236, v237, %[m0]\n"
      "v_max3_f32 %[m1], v238, v239, %[m1]\n"
      "v_max3_f32 %[m0], v240, v241, %[m0]\n"
      "v_max3_f32 %[m1], v242, v243, %[m1]\n"
      "v_max3_f32 %[m0], v244, v245, %[m0]\n"
      "v_max3_f32 %[m1], v246, v247, %[m1]\n"
      "v_max3_f32 %[m0], v248, v249, %[m0]\n"
      "v_max3_f32 %[m1], v250, v251, %[m1]\n"
      "v_max3_f32 %[m0], v252, v253, %[m0]\n"
      "v_max3_f32 %[m1], v254, v255, %[m1]\n"
            : [m0] "=&v"(m0), [m1] "=&v"(m1) : ACC_IN);
        const u32 wm = wave_umax(max(m0, m1));
        lds_store(lds0 + F_MISC + wid * 4, wm);
        BAR();
        u32 call, n = 0;
        asm volatile("ds_read_b32 %0, %1\n\ts_waitcnt lgkmcnt(0)" : "=&v"(call) : "v"(lds0 + F_MISC + 24) : "memory");
        call = rfl(call);
        const u32 sbase = lds0 + wid * ST_PITCH;
        if (remote) {
            u32x4 d;
            asm volatile(
      "v_lshrrev_b32 %[d0], 31, v159\n"
      "v_lshrrev_b32 %[d1], 31, v191\n"
      "v_lshrrev_b32 %[d2], 31, v223\n"
      "v_lshrrev_b32 %[d3], 31, v255\n"
      "v_alignbit_b32 %[d0], %[d0], v158, 31\n"
      "v_alignbit_b32 %[d1], %[d1], v190, 31\n"
      "v_alignbit_b32 %[d2], %[d2], v222, 31\n"
      "v_alignbit_b32 %[d3], %[d3], v254, 31\n"
      "v_alignbit_b32 %[d0], %[d0], v157, 31\n"
      "v_alignbit_b32 %[d1], %[d1], v189, 31\n"
      "v_alignbit_b32 %[d2], %[d2], v221, 31\n"
      "v_alignbit_b32 %[d3], %[d3], v253, 31\n"
      "v_alignbit_b32 %[d0], %[d0], v156, 31\n"
      "v_alignbit_b32 %[d1], %[d1], v188, 31\n"
      "v_alignbit_b32 %[d2], %[d2], v220, 31\n"
      "v_alignbit_b32 %[d3], %[d3], v252, 31\n"
      "v_alignbit_b32 %[d0], %[d0], v155, 31\n"
      "v_alignbit_b32 %[d1], %[d1], v187, 31\n"
      "v_alignbit_b32 %[d2], %[d2], v219, 31\n"
      "v_alignbit_b32 %[d3], %[d3], v251, 31\n"
      "v_alignbit_b32 %[d0], %[d0], v154, 31\n"
      "v_alignbit_b32 %[d1], %[d1], v186, 31\n"
      "v_alignbit_b32 %[d2], %[d2], v218, 31\n"
      "v_alignbit_b32 %[d3], %[d3], v250, 31\n"
      "v_alignbit_b32 %[d0], %[d0], v153, 31\n"
      "v_alignbit_b32 %[d1], %[d1], v185, 31\n"
      "v_alignbit_b32 %[d2], %[d2], v217, 31\n"
      "v_alignbit_b32 %[d3], %[d3], v249, 31\n"
      "v_alignbit_b32 %[d0], %[d0], v152, 31\n"
      "v_alignbit_b32 %[d1], %[d1], v184, 31\n"
      "v_alignbit_b32 %[d2], %[d2], v216, 31\n"
      "v_alignbit_b32 %[d3], %[d3], v248, 31\n"
      "v_alignbit_b32 %[d0], %[d0], v151, 31\n"
      "v_alignbit_b32 %[d1], %[d1], v183, 31\n"
      "v_alignbit_b32 %[d2], %[d2], v215, 31\n"
      "v_alignbit_b32 %[d3], %[d3], v247, 31\n"
      "v_alignbit_b32 %[d0], %[d0], v150, 31\n"
      "v_alignbit_b32 %[d1], %[d1], v182, 31\n"
      "v_alignbit_b32 %[d2], %[d2], v214, 31\n"
      "v_alignbit_b32 %[d3], %[d3], v246, 31\n"
      "v_alignbit_b32 %[d0], %[d0], v149, 31\n"
      "v_alignbit_b32 %[d1], %[d1], v181, 31\n"
      "v_alignbit_b32 %[d2], %[d2], v213, 31\n"
      "v_alignbit_b32 %[d3], %[d3], v245, 31\n"
      "v_alignbit_b32 %[d0], %[d0], v148, 31\n"
      "v_alignbit_b32 %[d1], %[d1], v180, 31\n"
      "v_alignbit_b32 %[d2], %[d2], v212, 31\n"
      "v_alignbit_b32 %[d3], %[d3], v244, 31\n"
      "v_alignbit_b32 %[d0], %[d0], v147, 31\n"
      "v_alignbit_b32 %[d1], %[d1], v179, 31\n"
      "v_alignbit_b32 %[d2], %[d2], v211, 31\n"
      "v_alignbit_b32 %[d3], %[d3], v243, 31\n"
      "v_alignbit_b32 %[d0], %[d0], v146, 31\n"
      "v_alignbit_b32 %[d1], %[d1], v178, 31\n"
      "v_alignbit_b32 %[d2], %[d2], v210, 31\n"
      "v_alignbit_b32 %[d3], %[d3], v242, 31\n"
      "v_alignbit_b32 %[d0], %[d0], v145, 31\n"
      "v_alignbit_b32 %[d1], %[d1], v177, 31\n"
      "v_alignbit_b32 %[d2], %[d2], v209, 31\n"
      "v_alignbit_b32 %[d3], %[d3], v241, 31\n"
      "v_alignbit_b32 %[d0], %[d0], v144, 31\n"
      "v_alignbit_b32 %[d1], %[d1], v176, 31\n"
      "v_alignbit_b32 %[d2], %[d2], v208, 31\n"
      "v_alignbit_b32 %[d3], %[d3], v240, 31\n"
      "v_alignbit_b32 %[d0], %[d0], v143, 31\n"
      "v_alignbit_b32 %[d1], %[d1], v175, 31\n"
      "v_alignbit_b32 %[d2], %[d2], v207, 31\n"
      "v_alignbit_b32 %[d3], %[d3], v239, 31\n"
      "v_alignbit_b32 %[d0], %[d0], v142, 31\n"
      "v_alignbit_b32 %[d1], %[d1], v174, 31\n"
      "v_alignbit_b32 %[d2], %[d2], v206, 31\n"
      "v_alignbit_b32 %[d3], %[d3], v238, 31\n"
      "v_alignbit_b32 %[d0], %[d0], v141, 31\n"
      "v_alignbit_b32 %[d1], %[d1], v173, 31\n"
      "v_alignbit_b32 %[d2], %[d2], v205, 31\n"
      "v_alignbit_b32 %[d3], %[d3], v237, 31\n"
      "v_alignbit_b32 %[d0], %[d0], v140, 31\n"
      "v_alignbit_b32 %[d1], %[d1], v172, 31\n"
      "v_alignbit_b32 %[d2], %[d2], v204, 31\n"
      "v_alignbit_b32 %[d3], %[d3], v236, 31\n"
      "v_alignbit_b32 %[d0], %[d0], v139, 31\n"
      "v_alignbit_b32 %[d1], %[d1], v171, 31\n"
      "v_alignbit_b32 %[d2], %[d2], v203, 31\n"
      "v_alignbit_b32 %[d3], %[d3], v235, 31\n"
      "v_alignbit_b32 %[d0], %[d0], v138, 31\n"
      "v_alignbit_b32 %[d1], %[d1], v170, 31\n"
      "v_alignbit_b32 %[d2], %[d2], v202, 31\n"
      "v_alignbit_b32 %[d3], %[d3], v234, 31\n"
      "v_alignbit_b32 %[d0], %[d0], v137, 31\n"
      "v_alignbit_b32 %[d1], %[d1], v169, 31\n"
      "v_alignbit_b32 %[d2], %[d2], v201, 31\n"
      "v_alignbit_b32 %[d3], %[d3], v233, 31\n"
      "v_alignbit_b32 %[d0], %[d0], v136, 31\n"
      "v_alignbit_b32 %[d1], %[d1], v168, 31\n"
      "v_alignbit_b32 %[d2], %[d2], v200, 31\n"
      "v_alignbit_b32 %[d3], %[d3], v232, 31\n"
      "v_alignbit_b32 %[d0], %[d0], v135, 31\n"
      "v_alignbit_b32 %[d1], %[d1], v167, 31\n"
      "v_alignbit_b32 %[d2], %[d2], v199, 31\n"
      "v_alignbit_b32 %[d3], %[d3], v231, 31\n"
      "v_alignbit_b32 %[d0], %[d0], v134, 31\n"
      "v_alignbit_b32 %[d1], %[d1], v166, 31\n"
      "v_alignbit_b32 %[d2], %[d2], v198, 31\n"
      "v_alignbit_b32 %[d3], %[d3], v230, 31\n"
      "v_alignbit_b32 %[d0], %[d0], v133, 31\n"
      "v_alignbit_b32 %[d1], %[d1], v165, 31\n"
      "v_alignbit_b32 %[d2], %[d2], v197, 31\n"
      "v_alignbit_b32 %[d3], %[d3], v229, 31\n"
      "v_alignbit_b32 %[d0], %[d0], v132, 31\n"
      "v_alignbit_b32 %[d1], %[d1], v164, 31\n"
      "v_alignbit_b32 %[d2], %[d2], v196, 31\n"
      "v_alignbit_b32 %[d3], %[d3], v228, 31\n"
      "v_alignbit_b32 %[d0], %[d0], v131, 31\n"
      "v_alignbit_b32 %[d1], %[d1], v163, 31\n"
      "v_alignbit_b32 %[d2], %[d2], v195, 31\n"
      "v_alignbit_b32 %[d3], %[d3], v227, 31\n"
      "v_alignbit_b32 %[d0], %[d0], v130, 31\n"
      "v_alignbit_b32 %[d1], %[d1], v162, 31\n"
      "v_alignbit_b32 %[d2], %[d2], v194, 31\n"
      "v_alignbit_b32 %[d3], %[d3], v226, 31\n"
      "v_alignbit_b32 %[d0], %[d0], v129, 31\n"
      "v_alignbit_b32 %[d1], %[d1], v161, 31\n"
      "v_alignbit_b32 %[d2], %[d2], v193, 31\n"
      "v_alignbit_b32 %[d3], %[d3], v225, 31\n"
      "v_alignbit_b32 %[d0], %[d0], v128, 31\n"
      "v_alignbit_b32 %[d1], %[d1], v160, 31\n"
      "v_alignbit_b32 %[d2], %[d2], v192, 31\n"
      "v_alignbit_b32 %[d3], %[d3], v224, 31\n"
                : [d0] "=&v"(d.x), [d1] "=&v"(d.y), [d2] "=&v"(d.z), [d3] "=&v"(d.w) : ACC_IN);
            asm volatile("global_store_dwordx4 %0, %1, off sc0 sc1\n\ts_nop 1" :: "v"(pws + WS_HDR + (size_t)(((call & 1) * 256 + sslot) * 4 + wid) * 1024 + lane * 16), "v"(d) : "memory");
            const u32 cnt = 129 - (__builtin_popcount(d.x) + __builtin_popcount(d.y) + __builtin_popcount(d.z) + __builtin_popcount(d.w));
            const u32 inc = wave_scan(cnt);
            n = __builtin_amdgcn_readlane(inc, 63);
            u32 A = sbase + (inc - cnt) * 4, t0, t1;
            asm volatile(
      "v_ashrrev_i32 %[t0], 31, v128\n"
      "ds_write_b32 %[A], v128 offset:0\n"
      "v_lshl_add_u32 %[A], %[t0], 2, %[A]\n"
      "v_ashrrev_i32 %[t1], 31, v129\n"
      "ds_write_b32 %[A], v129 offset:4\n"
      "v_lshl_add_u32 %[A], %[t1], 2, %[A]\n"
      "v_ashrrev_i32 %[t0], 31, v130\n"
      "ds_write_b32 %[A], v130 offset:8\n"
      "v_lshl_add_u32 %[A], %[t0], 2, %[A]\n"
      "v_ashrrev_i32 %[t1], 31, v131\n"
      "ds_write_b32 %[A], v131 offset:12\n"
      "v_lshl_add_u32 %[A], %[t1], 2, %[A]\n"
      "v_ashrrev_i32 %[t0], 31, v132\n"
      "ds_write_b32 %[A], v132 offset:16\n"
      "v_lshl_add_u32 %[A], %[t0], 2, %[A]\n"
      "v_ashrrev_i32 %[t1], 31, v133\n"
      "ds_write_b32 %[A], v133 offset:20\n"
      "v_lshl_add_u32 %[A], %[t1], 2, %[A]\n"
      "v_ashrrev_i32 %[t0], 31, v134\n"
      "ds_write_b32 %[A], v134 offset:24\n"
      "v_lshl_add_u32 %[A], %[t0], 2, %[A]\n"
      "v_ashrrev_i32 %[t1], 31, v135\n"
      "ds_write_b32 %[A], v135 offset:28\n"
      "v_lshl_add_u32 %[A], %[t1], 2, %[A]\n"
      "v_ashrrev_i32 %[t0], 31, v136\n"
      "ds_write_b32 %[A], v136 offset:32\n"
      "v_lshl_add_u32 %[A], %[t0], 2, %[A]\n"
      "v_ashrrev_i32 %[t1], 31, v137\n"
      "ds_write_b32 %[A], v137 offset:36\n"
      "v_lshl_add_u32 %[A], %[t1], 2, %[A]\n"
      "v_ashrrev_i32 %[t0], 31, v138\n"
      "ds_write_b32 %[A], v138 offset:40\n"
      "v_lshl_add_u32 %[A], %[t0], 2, %[A]\n"
      "v_ashrrev_i32 %[t1], 31, v139\n"
      "ds_write_b32 %[A], v139 offset:44\n"
      "v_lshl_add_u32 %[A], %[t1], 2, %[A]\n"
      "v_ashrrev_i32 %[t0], 31, v140\n"
      "ds_write_b32 %[A], v140 offset:48\n"
      "v_lshl_add_u32 %[A], %[t0], 2, %[A]\n"
      "v_ashrrev_i32 %[t1], 31, v141\n"
      "ds_write_b32 %[A], v141 offset:52\n"
      "v_lshl_add_u32 %[A], %[t1], 2, %[A]\n"
      "v_ashrrev_i32 %[t0], 31, v142\n"
      "ds_write_b32 %[A], v142 offset:56\n"
      "v_lshl_add_u32 %[A], %[t0], 2, %[A]\n"
      "v_ashrrev_i32 %[t1], 31, v143\n"
      "ds_write_b32 %[A], v143 offset:60\n"
      "v_lshl_add_u32 %[A], %[t1], 2, %[A]\n"
      "v_ashrrev_i32 %[t0], 31, v144\n"
      "ds_write_b32 %[A], v144 offset:64\n"
      "v_lshl_add_u32 %[A], %[t0], 2, %[A]\n"
      "v_ashrrev_i32 %[t1], 31, v145\n"
      "ds_write_b32 %[A], v145 offset:68\n"
      "v_lshl_add_u32 %[A], %[t1], 2, %[A]\n"
      "v_ashrrev_i32 %[t0], 31, v146\n"
      "ds_write_b32 %[A], v146 offset:72\n"
      "v_lshl_add_u32 %[A], %[t0], 2, %[A]\n"
      "v_ashrrev_i32 %[t1], 31, v147\n"
      "ds_write_b32 %[A], v147 offset:76\n"
      "v_lshl_add_u32 %[A], %[t1], 2, %[A]\n"
      "v_ashrrev_i32 %[t0], 31, v148\n"
      "ds_write_b32 %[A], v148 offset:80\n"
      "v_lshl_add_u32 %[A], %[t0], 2, %[A]\n"
      "v_ashrrev_i32 %[t1], 31, v149\n"
      "ds_write_b32 %[A], v149 offset:84\n"
      "v_lshl_add_u32 %[A], %[t1], 2, %[A]\n"
      "v_ashrrev_i32 %[t0], 31, v150\n"
      "ds_write_b32 %[A], v150 offset:88\n"
      "v_lshl_add_u32 %[A], %[t0], 2, %[A]\n"
      "v_ashrrev_i32 %[t1], 31, v151\n"
      "ds_write_b32 %[A], v151 offset:92\n"
      "v_lshl_add_u32 %[A], %[t1], 2, %[A]\n"
      "v_ashrrev_i32 %[t0], 31, v152\n"
      "ds_write_b32 %[A], v152 offset:96\n"
      "v_lshl_add_u32 %[A], %[t0], 2, %[A]\n"
      "v_ashrrev_i32 %[t1], 31, v153\n"
      "ds_write_b32 %[A], v153 offset:100\n"
      "v_lshl_add_u32 %[A], %[t1], 2, %[A]\n"
      "v_ashrrev_i32 %[t0], 31, v154\n"
      "ds_write_b32 %[A], v154 offset:104\n"
      "v_lshl_add_u32 %[A], %[t0], 2, %[A]\n"
      "v_ashrrev_i32 %[t1], 31, v155\n"
      "ds_write_b32 %[A], v155 offset:108\n"
      "v_lshl_add_u32 %[A], %[t1], 2, %[A]\n"
      "v_ashrrev_i32 %[t0], 31, v156\n"
      "ds_write_b32 %[A], v156 offset:112\n"
      "v_lshl_add_u32 %[A], %[t0], 2, %[A]\n"
      "v_ashrrev_i32 %[t1], 31, v157\n"
      "ds_write_b32 %[A], v157 offset:116\n"
      "v_lshl_add_u32 %[A], %[t1], 2, %[A]\n"
      "v_ashrrev_i32 %[t0], 31, v158\n"
      "ds_write_b32 %[A], v158 offset:120\n"
      "v_lshl_add_u32 %[A], %[t0], 2, %[A]\n"
      "v_ashrrev_i32 %[t1], 31, v159\n"
      "ds_write_b32 %[A], v159 offset:124\n"
      "v_lshl_add_u32 %[A], %[t1], 2, %[A]\n"
      "v_ashrrev_i32 %[t0], 31, v160\n"
      "ds_write_b32 %[A], v160 offset:128\n"
      "v_lshl_add_u32 %[A], %[t0], 2, %[A]\n"
      "v_ashrrev_i32 %[t1], 31, v161\n"
      "ds_write_b32 %[A], v161 offset:132\n"
      "v_lshl_add_u32 %[A], %[t1], 2, %[A]\n"
      "v_ashrrev_i32 %[t0], 31, v162\n"
      "ds_write_b32 %[A], v162 offset:136\n"
      "v_lshl_add_u32 %[A], %[t0], 2, %[A]\n"
      "v_ashrrev_i32 %[t1], 31, v163\n"
      "ds_write_b32 %[A], v163 offset:140\n"
      "v_lshl_add_u32 %[A], %[t1], 2, %[A]\n"
      "v_ashrrev_i32 %[t0], 31, v164\n"
      "ds_write_b32 %[A], v164 offset:144\n"
      "v_lshl_add_u32 %[A], %[t0], 2, %[A]\n"
      "v_ashrrev_i32 %[t1], 31, v165\n"
      "ds_write_b32 %[A], v165 offset:148\n"
      "v_lshl_add_u32 %[A], %[t1], 2, %[A]\n"
      "v_ashrrev_i32 %[t0], 31, v166\n"
      "ds_write_b32 %[A], v166 offset:152\n"
      "v_lshl_add_u32 %[A], %[t0], 2, %[A]\n"
      "v_ashrrev_i32 %[t1], 31, v167\n"
      "ds_write_b32 %[A], v167 offset:156\n"
      "v_lshl_add_u32 %[A], %[t1], 2, %[A]\n"
      "v_ashrrev_i32 %[t0], 31, v168\n"
      "ds_write_b32 %[A], v168 offset:160\n"
      "v_lshl_add_u32 %[A], %[t0], 2, %[A]\n"
      "v_ashrrev_i32 %[t1], 31, v169\n"
      "ds_write_b32 %[A], v169 offset:164\n"
      "v_lshl_add_u32 %[A], %[t1], 2, %[A]\n"
      "v_ashrrev_i32 %[t0], 31, v170\n"
      "ds_write_b32 %[A], v170 offset:168\n"
      "v_lshl_add_u32 %[A], %[t0], 2, %[A]\n"
      "v_ashrrev_i32 %[t1], 31, v171\n"
      "ds_write_b32 %[A], v171 offset:172\n"
      "v_lshl_add_u32 %[A], %[t1], 2, %[A]\n"
      "v_ashrrev_i32 %[t0], 31, v172\n"
      "ds_write_b32 %[A], v172 offset:176\n"
      "v_lshl_add_u32 %[A], %[t0], 2, %[A]\n"
      "v_ashrrev_i32 %[t1], 31, v173\n"
      "ds_write_b32 %[A], v173 offset:180\n"
      "v_lshl_add_u32 %[A], %[t1], 2, %[A]\n"
      "v_ashrrev_i32 %[t0], 31, v174\n"
      "ds_write_b32 %[A], v174 offset:184\n"
      "v_lshl_add_u32 %[A], %[t0], 2, %[A]\n"
      "v_ashrrev_i32 %[t1], 31, v175\n"
      "ds_write_b32 %[A], v175 offset:188\n"
      "v_lshl_add_u32 %[A], %[t1], 2, %[A]\n"
      "v_ashrrev_i32 %[t0], 31, v176\n"
      "ds_write_b32 %[A], v176 offset:192\n"
      "v_lshl_add_u32 %[A], %[t0], 2, %[A]\n"
      "v_ashrrev_i32 %[t1], 31, v177\n"
      "ds_write_b32 %[A], v177 offset:196\n"
      "v_lshl_add_u32 %[A], %[t1], 2, %[A]\n"
      "v_ashrrev_i32 %[t0], 31, v178\n"
      "ds_write_b32 %[A], v178 offset:200\n"
      "v_lshl_add_u32 %[A], %[t0], 2, %[A]\n"
      "v_ashrrev_i32 %[t1], 31, v179\n"
      "ds_write_b32 %[A], v179 offset:204\n"
      "v_lshl_add_u32 %[A], %[t1], 2, %[A]\n"
      "v_ashrrev_i32 %[t0], 31, v180\n"
      "ds_write_b32 %[A], v180 offset:208\n"
      "v_lshl_add_u32 %[A], %[t0], 2, %[A]\n"
      "v_ashrrev_i32 %[t1], 31, v181\n"
      "ds_write_b32 %[A], v181 offset:212\n"
      "v_lshl_add_u32 %[A], %[t1], 2, %[A]\n"
      "v_ashrrev_i32 %[t0], 31, v182\n"
      "ds_write_b32 %[A], v182 offset:216\n"
      "v_lshl_add_u32 %[A], %[t0], 2, %[A]\n"
      "v_ashrrev_i32 %[t1], 31, v183\n"
      "ds_write_b32 %[A], v183 offset:220\n"
      "v_lshl_add_u32 %[A], %[t1], 2, %[A]\n"
      "v_ashrrev_i32 %[t0], 31, v184\n"
      "ds_write_b32 %[A], v184 offset:224\n"
      "v_lshl_add_u32 %[A], %[t0], 2, %[A]\n"
      "v_ashrrev_i32 %[t1], 31, v185\n"
      "ds_write_b32 %[A], v185 offset:228\n"
      "v_lshl_add_u32 %[A], %[t1], 2, %[A]\n"
      "v_ashrrev_i32 %[t0], 31, v186\n"
      "ds_write_b32 %[A], v186 offset:232\n"
      "v_lshl_add_u32 %[A], %[t0], 2, %[A]\n"
      "v_ashrrev_i32 %[t1], 31, v187\n"
      "ds_write_b32 %[A], v187 offset:236\n"
      "v_lshl_add_u32 %[A], %[t1], 2, %[A]\n"
      "v_ashrrev_i32 %[t0], 31, v188\n"
      "ds_write_b32 %[A], v188 offset:240\n"
      "v_lshl_add_u32 %[A], %[t0], 2, %[A]\n"
      "v_ashrrev_i32 %[t1], 31, v189\n"
      "ds_write_b32 %[A], v189 offset:244\n"
      "v_lshl_add_u32 %[A], %[t1], 2, %[A]\n"
      "v_ashrrev_i32 %[t0], 31, v190\n"
      "ds_write_b32 %[A], v190 offset:248\n"
      "v_lshl_add_u32 %[A], %[t0], 2, %[A]\n"
      "v_ashrrev_i32 %[t1], 31, v191\n"
      "ds_write_b32 %[A], v191 offset:252\n"
      "v_lshl_add_u32 %[A], %[t1], 2, %[A]\n"
      "v_ashrrev_i32 %[t0], 31, v192\n"
      "ds_write_b32 %[A], v192 offset:256\n"
      "v_lshl_add_u32 %[A], %[t0], 2, %[A]\n"
      "v_ashrrev_i32 %[t1], 31, v193\n"
      "ds_write_b32 %[A], v193 offset:260\n"
      "v_lshl_add_u32 %[A], %[t1], 2, %[A]\n"
      "v_ashrrev_i32 %[t0], 31, v194\n"
      "ds_write_b32 %[A], v194 offset:264\n"
      "v_lshl_add_u32 %[A], %[t0], 2, %[A]\n"
      "v_ashrrev_i32 %[t1], 31, v195\n"
      "ds_write_b32 %[A], v195 offset:268\n"
      "v_lshl_add_u32 %[A], %[t1], 2, %[A]\n"
      "v_ashrrev_i32 %[t0], 31, v196\n"
      "ds_write_b32 %[A], v196 offset:272\n"
      "v_lshl_add_u32 %[A], %[t0], 2, %[A]\n"
      "v_ashrrev_i32 %[t1], 31, v197\n"
      "ds_write_b32 %[A], v197 offset:276\n"
      "v_lshl_add_u32 %[A], %[t1], 2, %[A]\n"
      "v_ashrrev_i32 %[t0], 31, v198\n"
      "ds_write_b32 %[A], v198 offset:280\n"
      "v_lshl_add_u32 %[A], %[t0], 2, %[A]\n"
      "v_ashrrev_i32 %[t1], 31, v199\n"
      "ds_write_b32 %[A], v199 offset:284\n"
      "v_lshl_add_u32 %[A], %[t1], 2, %[A]\n"
      "v_ashrrev_i32 %[t0], 31, v200\n"
      "ds_write_b32 %[A], v200 offset:288\n"
      "v_lshl_add_u32 %[A], %[t0], 2, %[A]\n"
      "v_ashrrev_i32 %[t1], 31, v201\n"
      "ds_write_b32 %[A], v201 offset:292\n"
      "v_lshl_add_u32 %[A], %[t1], 2, %[A]\n"
      "v_ashrrev_i32 %[t0], 31, v202\n"
      "ds_write_b32 %[A], v202 offset:296\n"
      "v_lshl_add_u32 %[A], %[t0], 2, %[A]\n"
      "v_ashrrev_i32 %[t1], 31, v203\n"
      "ds_write_b32 %[A], v203 offset:300\n"
      "v_lshl_add_u32 %[A], %[t1], 2, %[A]\n"
      "v_ashrrev_i32 %[t0], 31, v204\n"
      "ds_write_b32 %[A], v204 offset:304\n"
      "v_lshl_add_u32 %[A], %[t0], 2, %[A]\n"
      "v_ashrrev_i32 %[t1], 31, v205\n"
      "ds_write_b32 %[A], v205 offset:308\n"
      "v_lshl_add_u32 %[A], %[t1], 2, %[A]\n"
      "v_ashrrev_i32 %[t0], 31, v206\n"
      "ds_write_b32 %[A], v206 offset:312\n"
      "v_lshl_add_u32 %[A], %[t0], 2, %[A]\n"
      "v_ashrrev_i32 %[t1], 31, v207\n"
      "ds_write_b32 %[A], v207 offset:316\n"
      "v_lshl_add_u32 %[A], %[t1], 2, %[A]\n"
      "v_ashrrev_i32 %[t0], 31, v208\n"
      "ds_write_b32 %[A], v208 offset:320\n"
      "v_lshl_add_u32 %[A], %[t0], 2, %[A]\n"
      "v_ashrrev_i32 %[t1], 31, v209\n"
      "ds_write_b32 %[A], v209 offset:324\n"
      "v_lshl_add_u32 %[A], %[t1], 2, %[A]\n"
      "v_ashrrev_i32 %[t0], 31, v210\n"
      "ds_write_b32 %[A], v210 offset:328\n"
      "v_lshl_add_u32 %[A], %[t0], 2, %[A]\n"
      "v_ashrrev_i32 %[t1], 31, v211\n"
      "ds_write_b32 %[A], v211 offset:332\n"
      "v_lshl_add_u32 %[A], %[t1], 2, %[A]\n"
      "v_ashrrev_i32 %[t0], 31, v212\n"
      "ds_write_b32 %[A], v212 offset:336\n"
      "v_lshl_add_u32 %[A], %[t0], 2, %[A]\n"
      "v_ashrrev_i32 %[t1], 31, v213\n"
      "ds_write_b32 %[A], v213 offset:340\n"
      "v_lshl_add_u32 %[A], %[t1], 2, %[A]\n"
      "v_ashrrev_i32 %[t0], 31, v214\n"
      "ds_write_b32 %[A], v214 offset:344\n"
      "v_lshl_add_u32 %[A], %[t0], 2, %[A]\n"
      "v_ashrrev_i32 %[t1], 31, v215\n"
      "ds_write_b32 %[A], v215 offset:348\n"
      "v_lshl_add_u32 %[A], %[t1], 2, %[A]\n"
      "v_ashrrev_i32 %[t0], 31, v216\n"
      "ds_write_b32 %[A], v216 offset:352\n"
      "v_lshl_add_u32 %[A], %[t0], 2, %[A]\n"
      "v_ashrrev_i32 %[t1], 31, v217\n"
      "ds_write_b32 %[A], v217 offset:356\n"
      "v_lshl_add_u32 %[A], %[t1], 2, %[A]\n"
      "v_ashrrev_i32 %[t0], 31, v218\n"
      "ds_write_b32 %[A], v218 offset:360\n"
      "v_lshl_add_u32 %[A], %[t0], 2, %[A]\n"
      "v_ashrrev_i32 %[t1], 31, v219\n"
      "ds_write_b32 %[A], v219 offset:364\n"
      "v_lshl_add_u32 %[A], %[t1], 2, %[A]\n"
      "v_ashrrev_i32 %[t0], 31, v220\n"
      "ds_write_b32 %[A], v220 offset:368\n"
      "v_lshl_add_u32 %[A], %[t0], 2, %[A]\n"
      "v_ashrrev_i32 %[t1], 31, v221\n"
      "ds_write_b32 %[A], v221 offset:372\n"
      "v_lshl_add_u32 %[A], %[t1], 2, %[A]\n"
      "v_ashrrev_i32 %[t0], 31, v222\n"
      "ds_write_b32 %[A], v222 offset:376\n"
      "v_lshl_add_u32 %[A], %[t0], 2, %[A]\n"
      "v_ashrrev_i32 %[t1], 31, v223\n"
      "ds_write_b32 %[A], v223 offset:380\n"
      "v_lshl_add_u32 %[A], %[t1], 2, %[A]\n"
      "v_ashrrev_i32 %[t0], 31, v224\n"
      "ds_write_b32 %[A], v224 offset:384\n"
      "v_lshl_add_u32 %[A], %[t0], 2, %[A]\n"
      "v_ashrrev_i32 %[t1], 31, v225\n"
      "ds_write_b32 %[A], v225 offset:388\n"
      "v_lshl_add_u32 %[A], %[t1], 2, %[A]\n"
      "v_ashrrev_i32 %[t0], 31, v226\n"
      "ds_write_b32 %[A], v226 offset:392\n"
      "v_lshl_add_u32 %[A], %[t0], 2, %[A]\n"
      "v_ashrrev_i32 %[t1], 31, v227\n"
      "ds_write_b32 %[A], v227 offset:396\n"
      "v_lshl_add_u32 %[A], %[t1], 2, %[A]\n"
      "v_ashrrev_i32 %[t0], 31, v228\n"
      "ds_write_b32 %[A], v228 offset:400\n"
      "v_lshl_add_u32 %[A], %[t0], 2, %[A]\n"
      "v_ashrrev_i32 %[t1], 31, v229\n"
      "ds_write_b32 %[A], v229 offset:404\n"
      "v_lshl_add_u32 %[A], %[t1], 2, %[A]\n"
      "v_ashrrev_i32 %[t0], 31, v230\n"
      "ds_write_b32 %[A], v230 offset:408\n"
      "v_lshl_add_u32 %[A], %[t0], 2, %[A]\n"
      "v_ashrrev_i32 %[t1], 31, v231\n"
      "ds_write_b32 %[A], v231 offset:412\n"
      "v_lshl_add_u32 %[A], %[t1], 2, %[A]\n"
      "v_ashrrev_i32 %[t0], 31, v232\n"
      "ds_write_b32 %[A], v232 offset:416\n"
      "v_lshl_add_u32 %[A], %[t0], 2, %[A]\n"
      "v_ashrrev_i32 %[t1], 31, v233\n"
      "ds_write_b32 %[A], v233 offset:420\n"
      "v_lshl_add_u32 %[A], %[t1], 2, %[A]\n"
      "v_ashrrev_i32 %[t0], 31, v234\n"
      "ds_write_b32 %[A], v234 offset:424\n"
      "v_lshl_add_u32 %[A], %[t0], 2, %[A]\n"
      "v_ashrrev_i32 %[t1], 31, v235\n"
      "ds_write_b32 %[A], v235 offset:428\n"
      "v_lshl_add_u32 %[A], %[t1], 2, %[A]\n"
      "v_ashrrev_i32 %[t0], 31, v236\n"
      "ds_write_b32 %[A], v236 offset:432\n"
      "v_lshl_add_u32 %[A], %[t0], 2, %[A]\n"
      "v_ashrrev_i32 %[t1], 31, v237\n"
      "ds_write_b32 %[A], v237 offset:436\n"
      "v_lshl_add_u32 %[A], %[t1], 2, %[A]\n"
      "v_ashrrev_i32 %[t0], 31, v238\n"
      "ds_write_b32 %[A], v238 offset:440\n"
      "v_lshl_add_u32 %[A], %[t0], 2, %[A]\n"
      "v_ashrrev_i32 %[t1], 31, v239\n"
      "ds_write_b32 %[A], v239 offset:444\n"
      "v_lshl_add_u32 %[A], %[t1], 2, %[A]\n"
      "v_ashrrev_i32 %[t0], 31, v240\n"
      "ds_write_b32 %[A], v240 offset:448\n"
      "v_lshl_add_u32 %[A], %[t0], 2, %[A]\n"
      "v_ashrrev_i32 %[t1], 31, v241\n"
      "ds_write_b32 %[A], v241 offset:452\n"
      "v_lshl_add_u32 %[A], %[t1], 2, %[A]\n"
      "v_ashrrev_i32 %[t0], 31, v242\n"
      "ds_write_b32 %[A], v242 offset:456\n"
      "v_lshl_add_u32 %[A], %[t0], 2, %[A]\n"
      "v_ashrrev_i32 %[t1], 31, v243\n"
      "ds_write_b32 %[A], v243 offset:460\n"
      "v_lshl_add_u32 %[A], %[t1], 2, %[A]\n"
      "v_ashrrev_i32 %[t0], 31, v244\n"
      "ds_write_b32 %[A], v244 offset:464\n"
      "v_lshl_add_u32 %[A], %[t0], 2, %[A]\n"
      "v_ashrrev_i32 %[t1], 31, v245\n"
      "ds_write_b32 %[A], v245 offset:468\n"
      "v_lshl_add_u32 %[A], %[t1], 2, %[A]\n"
      "v_ashrrev_i32 %[t0], 31, v246\n"
      "ds_write_b32 %[A], v246 offset:472\n"
      "v_lshl_add_u32 %[A], %[t0], 2, %[A]\n"
      "v_ashrrev_i32 %[t1], 31, v247\n"
      "ds_write_b32 %[A], v247 offset:476\n"
      "v_lshl_add_u32 %[A], %[t1], 2, %[A]\n"
      "v_ashrrev_i32 %[t0], 31, v248\n"
      "ds_write_b32 %[A], v248 offset:480\n"
      "v_lshl_add_u32 %[A], %[t0], 2, %[A]\n"
      "v_ashrrev_i32 %[t1], 31, v249\n"
      "ds_write_b32 %[A], v249 offset:484\n"
      "v_lshl_add_u32 %[A], %[t1], 2, %[A]\n"
      "v_ashrrev_i32 %[t0], 31, v250\n"
      "ds_write_b32 %[A], v250 offset:488\n"
      "v_lshl_add_u32 %[A], %[t0], 2, %[A]\n"
      "v_ashrrev_i32 %[t1], 31, v251\n"
      "ds_write_b32 %[A], v251 offset:492\n"
      "v_lshl_add_u32 %[A], %[t1], 2, %[A]\n"
      "v_ashrrev_i32 %[t0], 31, v252\n"
      "ds_write_b32 %[A], v252 offset:496\n"
      "v_lshl_add_u32 %[A], %[t0], 2, %[A]\n"
      "v_ashrrev_i32 %[t1], 31, v253\n"
      "ds_write_b32 %[A], v253 offset:500\n"
      "v_lshl_add_u32 %[A], %[t1], 2, %[A]\n"
      "v_ashrrev_i32 %[t0], 31, v254\n"
      "ds_write_b32 %[A], v254 offset:504\n"
      "v_lshl_add_u32 %[A], %[t0], 2, %[A]\n"
      "v_ashrrev_i32 %[t1], 31, v255\n"
      "ds_write_b32 %[A], v255 offset:508\n"
      "v_lshl_add_u32 %[A], %[t1], 2, %[A]\n"
      "s_waitcnt vmcnt(0)\n"
                : [A] "+&v"(A), [t0] "=&v"(t0), [t1] "=&v"(t1) : ACC_IN : "memory");
            if (lane == 0) st_sys32(pws + WS_HFLAG + (sslot * 4 + wid) * 4, call);
        }
        if (wid == 0) STAMP(4);
        BAR();
        u32x4 am; asm volatile("ds_read_b128 %0, %1\n\ts_waitcnt lgkmcnt(0)" : "=&v"(am) : "v"(lds0 + F_MISC + 16) : "memory");
        const float amax = __uint_as_float(am.x);
        const float inv = amax > 0.f ? 1.0f / (amax / 448.0f) : 0.f;
        const u32 nanm = am.y ? 0x7f7f7f7fu : 0u;
        if (remote) {
            if (STAGD > 0 && idx > STAGW) {
                const u64 t0 = __builtin_amdgcn_s_memrealtime(); const u32 dl = ((idx - STAGW) * STAGD * n) >> 16;
                while (__builtin_amdgcn_s_memrealtime() - t0 < dl) __builtin_amdgcn_s_sleep(1);
            }
            unsigned char* dst = pws + WS_PAY + (size_t)(sslot * 4 + wid) * PAY_PITCH + lane * 16;
            u32 la = sbase + lane * 64;
            for (u32 e = lane * 16; e < n; e += 1024, dst += 1024, la += 4096) {
                f32x4 x0, x1, x2, x3;
                asm volatile("ds_read_b128 %0, %4\n\tds_read_b128 %1, %4 offset:16\n\tds_read_b128 %2, %4 offset:32\n\tds_read_b128 %3, %4 offset:48\n\ts_waitcnt lgkmcnt(0)"
                             : "=&v"(x0), "=&v"(x1), "=&v"(x2), "=&v"(x3) : "v"(la) : "memory");
                x0 *= inv; x1 *= inv; x2 *= inv; x3 *= inv;
                u32x4 o; o.x = q4(x0.x, x0.y, x0.z, x0.w) | nanm; o.y = q4(x1.x, x1.y, x1.z, x1.w) | nanm; o.z = q4(x2.x, x2.y, x2.z, x2.w) | nanm; o.w = q4(x3.x, x3.y, x3.z, x3.w) | nanm;
                asm volatile("global_store_dwordx4 %0, %1, off sc0 sc1\n\ts_nop 1" :: "v"(dst), "v"(o) : "memory");
            }
            lds_store(lds0 + F_MISC + 32 + wid * 4, 1);
            if (wid == 0) STAMP(5);
            asm volatile("s_waitcnt vmcnt(0)" ::: "memory");
            if (lane == 0) st_sys32(pws + WS_PFLAG + (sslot * 4 + wid) * 4, call);
            if (wid == 0) STAMP(6);
        } else {
            unsigned char* dst = rout + (size_t)(lane & 15) * NC + wid * 64 + (lane >> 4) * 16;
            const bool odd = (lane >> 4) & 1;
#define RQ(c, t) (q4(__builtin_fmaxf(c[4 * (t)], 0.f) * inv, __builtin_fmaxf(c[4 * (t) + 1], 0.f) * inv, __builtin_fmaxf(c[4 * (t) + 2], 0.f) * inv, __builtin_fmaxf(c[4 * (t) + 3], 0.f) * inv) | nanm)
#define QS(c, m) { const u32 w0 = RQ(c, 0), w1 = RQ(c, 1), w2 = RQ(c, 2), w3 = RQ(c, 3); \
            u32x4 o; o.x = odd ? w1 : w0; o.y = odd ? w0 : w1; o.z = odd ? w3 : w2; o.w = odd ? w2 : w3; \
            *(GAS u32x4*)(dst + (size_t)(m) * 16 * NC) = o; }
            QS(c0, 0) QS(c1, 1) QS(c2, 2) QS(c3, 3) QS(c4, 4) QS(c5, 5) QS(c6, 6) QS(c7, 7)
        }
    } else {
        const u32 lane = threadIdx.x & 63, pw = wid - 4;
        __builtin_amdgcn_s_setprio(PPRIO);
        u32 call = 1;
        {
            const u32x4 ra = make_rsrc(p.x, ML * KD * 2), rb = make_rsrc(p.w, (u32)KD * ND * 2);
            const u32 rw = RW, rx = RX;
            const u32 a0 = ((lane >> 3) * (KD * 2) + (((lane & 7) ^ (lane >> 4)) << 4)) ^ ((rx & 1) << 6), a1 = a0 ^ 64;
            const u32 b0 = ((lane >> 5) * (WPAIR * ND * 2) + (((lane & 31) ^ (lane >> 5)) << 4)) ^ ((rw & 1) << 5) ^ ((rw >> 2) << 7), b1 = b0 ^ 32, b2 = b0 ^ 128, b3 = b0 ^ 160;
            const u32 px = ROTX, pq = ROTW;
            const u32 sa0 = rfl((rowblk * BM + px * 32 + rx * 8) * KD * 2), sb0 = rfl(rw * (WPAIR == 1 ? 2 : 1) * ND * 2 + (WPAIR <= 8 ? pq * 16 : WPAIR == 16 ? (pq >> 1) * 32 + (pq & 1) * 8 : pq * 8) * ND * 2 + (grp * NC + ct * BN) * 2);
            const u32 ma0 = rfl(lds0 + A_BASE + px * 4096 + rx * 1024), mb0 = rfl(lds0 + B_BASE + pq * 8192 + rw * 1024), mae = ma0 + 3 * A_ST;
            const u32 ssa = 128, ssb = 64 * ND * 2;
            u32 ma = ma0, mb = mb0, t, t2, n, tv = 0, tw = 0;
            const u32 kx = rfl(KX), kp = rfl((KP) + (LAG));
            u32 kk = rfl(KP);
            u32 sa = sa0 + (((kk ^ kx) - kp) & 63) * ssa, sb = sb0 + (((kk ^ kx) - kp) & 63) * ssb;
            asm volatile(
      "s_cmp_lg_u32 %[pw], 3\n"
      "s_cbranch_scc1 L_s_%=\n"
      "s_mov_b64 exec, 1\n"
      "global_atomic_add %[call], %[ca], %[call], off sc0\n"
      "s_mov_b64 exec, -1\n"
      "L_s_%=:\n"
      "s_xor_b32 m0, %[ma], 0\n"
      "s_xor_b32 %[t], %[sa], 0x0\n"
      "buffer_load_dwordx4 %[a0], %[ra], %[t] offen  lds\n"
      "s_xor_b32 m0, %[ma], 1024\n"
      "s_xor_b32 %[t], %[sa], 0x10000\n"
      "buffer_load_dwordx4 %[a1], %[ra], %[t] offen  lds\n"
      "s_xor_b32 m0, %[ma], 2048\n"
      "s_xor_b32 %[t], %[sa], 0x20000\n"
      "buffer_load_dwordx4 %[a0], %[ra], %[t] offen  lds\n"
      "s_xor_b32 m0, %[ma], 3072\n"
      "s_xor_b32 %[t], %[sa], 0x30000\n"
      "buffer_load_dwordx4 %[a1], %[ra], %[t] offen  lds\n"
      "s_xor_b32 m0, %[mb], 0\n"
      "s_xor_b32 %[t], %[sb], 0x0\n"
      "buffer_load_dwordx4 %[b0], %[rb], %[t] offen  lds\n"
      "s_xor_b32 m0, %[mb], 1024\n"
      "s_xor_b32 %[t], %[sb], 0x10000\n"
      "buffer_load_dwordx4 %[b1], %[rb], %[t] offen  lds\n"
      "s_xor_b32 m0, %[mb], 2048\n"
      "s_xor_b32 %[t], %[sb], 0x20000\n"
      "buffer_load_dwordx4 %[b0], %[rb], %[t] offen  lds\n"
      "s_xor_b32 m0, %[mb], 3072\n"
      "s_xor_b32 %[t], %[sb], 0x30000\n"
      "buffer_load_dwordx4 %[b1], %[rb], %[t] offen  lds\n"
      "s_xor_b32 m0, %[mb], 4096\n"
      "s_xor_b32 %[t], %[sb], 0x40000\n"
      "buffer_load_dwordx4 %[b2], %[rb], %[t] offen  lds\n"
      "s_xor_b32 m0, %[mb], 5120\n"
      "s_xor_b32 %[t], %[sb], 0x50000\n"
      "buffer_load_dwordx4 %[b3], %[rb], %[t] offen  lds\n"
      "s_xor_b32 m0, %[mb], 6144\n"
      "s_xor_b32 %[t], %[sb], 0x60000\n"
      "buffer_load_dwordx4 %[b2], %[rb], %[t] offen  lds\n"
      "s_xor_b32 m0, %[mb], 7168\n"
      "s_xor_b32 %[t], %[sb], 0x70000\n"
      "buffer_load_dwordx4 %[b3], %[rb], %[t] offen  lds\n"
      "s_add_u32 %[kk], %[kk], 1\n"
      "s_xor_b32 %[t], %[kk], %[kx]\n"
      "s_sub_u32 %[t], %[t], %[kp]\n"
      "s_and_b32 %[t], %[t], 63\n"
      "s_mul_i32 %[t2], %[t], %[ssa]\n"
      "s_add_u32 %[sa], %[sa0], %[t2]\n"
      "s_mul_i32 %[t2], %[t], %[ssb]\n"
      "s_add_u32 %[sb], %[sb0], %[t2]\n"
      "s_add_u32 %[ma], %[ma], 16384\n"
      "s_add_u32 %[mb], %[mb], 32768\n"
      "s_cmp_eq_u32 %[ma], %[mae]\n"
      "s_cselect_b32 %[ma], %[ma0], %[ma]\n"
      "s_cselect_b32 %[mb], %[mb0], %[mb]\n"
      "s_xor_b32 m0, %[ma], 0\n"
      "s_xor_b32 %[t], %[sa], 0x0\n"
      "buffer_load_dwordx4 %[a0], %[ra], %[t] offen  lds\n"
      "s_xor_b32 m0, %[ma], 1024\n"
      "s_xor_b32 %[t], %[sa], 0x10000\n"
      "buffer_load_dwordx4 %[a1], %[ra], %[t] offen  lds\n"
      "s_xor_b32 m0, %[ma], 2048\n"
      "s_xor_b32 %[t], %[sa], 0x20000\n"
      "buffer_load_dwordx4 %[a0], %[ra], %[t] offen  lds\n"
      "s_xor_b32 m0, %[ma], 3072\n"
      "s_xor_b32 %[t], %[sa], 0x30000\n"
      "buffer_load_dwordx4 %[a1], %[ra], %[t] offen  lds\n"
      "s_xor_b32 m0, %[mb], 0\n"
      "s_xor_b32 %[t], %[sb], 0x0\n"
      "buffer_load_dwordx4 %[b0], %[rb], %[t] offen  lds\n"
      "s_xor_b32 m0, %[mb], 1024\n"
      "s_xor_b32 %[t], %[sb], 0x10000\n"
      "buffer_load_dwordx4 %[b1], %[rb], %[t] offen  lds\n"
      "s_xor_b32 m0, %[mb], 2048\n"
      "s_xor_b32 %[t], %[sb], 0x20000\n"
      "buffer_load_dwordx4 %[b0], %[rb], %[t] offen  lds\n"
      "s_xor_b32 m0, %[mb], 3072\n"
      "s_xor_b32 %[t], %[sb], 0x30000\n"
      "buffer_load_dwordx4 %[b1], %[rb], %[t] offen  lds\n"
      "s_xor_b32 m0, %[mb], 4096\n"
      "s_xor_b32 %[t], %[sb], 0x40000\n"
      "buffer_load_dwordx4 %[b2], %[rb], %[t] offen  lds\n"
      "s_xor_b32 m0, %[mb], 5120\n"
      "s_xor_b32 %[t], %[sb], 0x50000\n"
      "buffer_load_dwordx4 %[b3], %[rb], %[t] offen  lds\n"
      "s_xor_b32 m0, %[mb], 6144\n"
      "s_xor_b32 %[t], %[sb], 0x60000\n"
      "buffer_load_dwordx4 %[b2], %[rb], %[t] offen  lds\n"
      "s_xor_b32 m0, %[mb], 7168\n"
      "s_xor_b32 %[t], %[sb], 0x70000\n"
      "buffer_load_dwordx4 %[b3], %[rb], %[t] offen  lds\n"
      "s_add_u32 %[kk], %[kk], 1\n"
      "s_xor_b32 %[t], %[kk], %[kx]\n"
      "s_sub_u32 %[t], %[t], %[kp]\n"
      "s_and_b32 %[t], %[t], 63\n"
      "s_mul_i32 %[t2], %[t], %[ssa]\n"
      "s_add_u32 %[sa], %[sa0], %[t2]\n"
      "s_mul_i32 %[t2], %[t], %[ssb]\n"
      "s_add_u32 %[sb], %[sb0], %[t2]\n"
      "s_add_u32 %[ma], %[ma], 16384\n"
      "s_add_u32 %[mb], %[mb], 32768\n"
      "s_cmp_eq_u32 %[ma], %[mae]\n"
      "s_cselect_b32 %[ma], %[ma0], %[ma]\n"
      "s_cselect_b32 %[mb], %[mb0], %[mb]\n"
      "s_xor_b32 m0, %[ma], 0\n"
      "s_xor_b32 %[t], %[sa], 0x0\n"
      "buffer_load_dwordx4 %[a0], %[ra], %[t] offen  lds\n"
      "s_xor_b32 m0, %[ma], 1024\n"
      "s_xor_b32 %[t], %[sa], 0x10000\n"
      "buffer_load_dwordx4 %[a1], %[ra], %[t] offen  lds\n"
      "s_xor_b32 m0, %[ma], 2048\n"
      "s_xor_b32 %[t], %[sa], 0x20000\n"
      "buffer_load_dwordx4 %[a0], %[ra], %[t] offen  lds\n"
      "s_xor_b32 m0, %[ma], 3072\n"
      "s_xor_b32 %[t], %[sa], 0x30000\n"
      "buffer_load_dwordx4 %[a1], %[ra], %[t] offen  lds\n"
      "s_xor_b32 m0, %[mb], 0\n"
      "s_xor_b32 %[t], %[sb], 0x0\n"
      "buffer_load_dwordx4 %[b0], %[rb], %[t] offen  lds\n"
      "s_xor_b32 m0, %[mb], 1024\n"
      "s_xor_b32 %[t], %[sb], 0x10000\n"
      "buffer_load_dwordx4 %[b1], %[rb], %[t] offen  lds\n"
      "s_xor_b32 m0, %[mb], 2048\n"
      "s_xor_b32 %[t], %[sb], 0x20000\n"
      "buffer_load_dwordx4 %[b0], %[rb], %[t] offen  lds\n"
      "s_xor_b32 m0, %[mb], 3072\n"
      "s_xor_b32 %[t], %[sb], 0x30000\n"
      "buffer_load_dwordx4 %[b1], %[rb], %[t] offen  lds\n"
      "s_xor_b32 m0, %[mb], 4096\n"
      "s_xor_b32 %[t], %[sb], 0x40000\n"
      "buffer_load_dwordx4 %[b2], %[rb], %[t] offen  lds\n"
      "s_xor_b32 m0, %[mb], 5120\n"
      "s_xor_b32 %[t], %[sb], 0x50000\n"
      "buffer_load_dwordx4 %[b3], %[rb], %[t] offen  lds\n"
      "s_xor_b32 m0, %[mb], 6144\n"
      "s_xor_b32 %[t], %[sb], 0x60000\n"
      "buffer_load_dwordx4 %[b2], %[rb], %[t] offen  lds\n"
      "s_xor_b32 m0, %[mb], 7168\n"
      "s_xor_b32 %[t], %[sb], 0x70000\n"
      "buffer_load_dwordx4 %[b3], %[rb], %[t] offen  lds\n"
      "s_add_u32 %[kk], %[kk], 1\n"
      "s_xor_b32 %[t], %[kk], %[kx]\n"
      "s_sub_u32 %[t], %[t], %[kp]\n"
      "s_and_b32 %[t], %[t], 63\n"
      "s_mul_i32 %[t2], %[t], %[ssa]\n"
      "s_add_u32 %[sa], %[sa0], %[t2]\n"
      "s_mul_i32 %[t2], %[t], %[ssb]\n"
      "s_add_u32 %[sb], %[sb0], %[t2]\n"
      "s_add_u32 %[ma], %[ma], 16384\n"
      "s_add_u32 %[mb], %[mb], 32768\n"
      "s_cmp_eq_u32 %[ma], %[mae]\n"
      "s_cselect_b32 %[ma], %[ma0], %[ma]\n"
      "s_cselect_b32 %[mb], %[mb0], %[mb]\n"
      "s_waitcnt vmcnt(24)\n"
      "s_barrier\n"
      "s_waitcnt vmcnt(12)\n"
      "s_barrier\n"
      "s_mov_b32 %[n], 3\n"
      "L_p_%=:\n"
      "s_xor_b32 m0, %[ma], 0\n"
      "s_xor_b32 %[t], %[sa], 0x0\n"
      "buffer_load_dwordx4 %[a0], %[ra], %[t] offen  lds\n"
      "s_xor_b32 m0, %[ma], 1024\n"
      "s_xor_b32 %[t], %[sa], 0x10000\n"
      "buffer_load_dwordx4 %[a1], %[ra], %[t] offen  lds\n"
      "s_xor_b32 m0, %[ma], 2048\n"
      "s_xor_b32 %[t], %[sa], 0x20000\n"
      "buffer_load_dwordx4 %[a0], %[ra], %[t] offen  lds\n"
      "s_xor_b32 m0, %[ma], 3072\n"
      "s_xor_b32 %[t], %[sa], 0x30000\n"
      "buffer_load_dwordx4 %[a1], %[ra], %[t] offen  lds\n"
      "s_xor_b32 m0, %[mb], 0\n"
      "s_xor_b32 %[t], %[sb], 0x0\n"
      "buffer_load_dwordx4 %[b0], %[rb], %[t] offen  lds\n"
      "s_xor_b32 m0, %[mb], 1024\n"
      "s_xor_b32 %[t], %[sb], 0x10000\n"
      "buffer_load_dwordx4 %[b1], %[rb], %[t] offen  lds\n"
      "s_xor_b32 m0, %[mb], 2048\n"
      "s_xor_b32 %[t], %[sb], 0x20000\n"
      "buffer_load_dwordx4 %[b0], %[rb], %[t] offen  lds\n"
      "s_xor_b32 m0, %[mb], 3072\n"
      "s_xor_b32 %[t], %[sb], 0x30000\n"
      "buffer_load_dwordx4 %[b1], %[rb], %[t] offen  lds\n"
      "s_xor_b32 m0, %[mb], 4096\n"
      "s_xor_b32 %[t], %[sb], 0x40000\n"
      "buffer_load_dwordx4 %[b2], %[rb], %[t] offen  lds\n"
      "s_xor_b32 m0, %[mb], 5120\n"
      "s_xor_b32 %[t], %[sb], 0x50000\n"
      "buffer_load_dwordx4 %[b3], %[rb], %[t] offen  lds\n"
      "s_xor_b32 m0, %[mb], 6144\n"
      "s_xor_b32 %[t], %[sb], 0x60000\n"
      "buffer_load_dwordx4 %[b2], %[rb], %[t] offen  lds\n"
      "s_xor_b32 m0, %[mb], 7168\n"
      "s_xor_b32 %[t], %[sb], 0x70000\n"
      "buffer_load_dwordx4 %[b3], %[rb], %[t] offen  lds\n"
      "s_add_u32 %[kk], %[kk], 1\n"
      "s_xor_b32 %[t], %[kk], %[kx]\n"
      "s_sub_u32 %[t], %[t], %[kp]\n"
      "s_and_b32 %[t], %[t], 63\n"
      "s_mul_i32 %[t2], %[t], %[ssa]\n"
      "s_add_u32 %[sa], %[sa0], %[t2]\n"
      "s_mul_i32 %[t2], %[t], %[ssb]\n"
      "s_add_u32 %[sb], %[sb0], %[t2]\n"
      "s_add_u32 %[ma], %[ma], 16384\n"
      "s_add_u32 %[mb], %[mb], 32768\n"
      "s_cmp_eq_u32 %[ma], %[mae]\n"
      "s_cselect_b32 %[ma], %[ma0], %[ma]\n"
      "s_cselect_b32 %[mb], %[mb0], %[mb]\n"
      "s_add_u32 %[n], %[n], 1\n"
      "s_waitcnt vmcnt(12)\n"
      "s_cmp_lt_u32 %[n], 64\n"
      "s_barrier\n"
      "s_cbranch_scc1 L_p_%=\n"
      "s_waitcnt vmcnt(0)\n"
      "s_barrier\n"
      "s_barrier\n"
                : [t] "=&s"(t), [t2] "=&s"(t2), [kk] "+&s"(kk), [n] "=&s"(n), [sa] "+&s"(sa), [sb] "+&s"(sb), [ma] "+&s"(ma), [mb] "+&s"(mb), [call] "+&v"(call), [tv] "+&v"(tv), [tw] "+&v"(tw)
                : [a0] "v"(a0), [a1] "v"(a1), [b0] "v"(b0), [b1] "v"(b1), [b2] "v"(b2), [b3] "v"(b3), [ra] "s"(ra), [rb] "s"(rb),
                  [ssa] "s"(ssa), [ssb] "s"(ssb), [sa0] "s"(sa0), [sb0] "s"(sb0), [kx] "s"(kx), [kp] "s"(kp), [ma0] "s"(ma0), [mb0] "s"(mb0), [mae] "s"(mae), [pw] "s"(pw), [ca] "v"(myws + WS_CALL + b * 4)
                : "memory", "scc", "s40", "s41");
        }
        if (wid == 5) STAMP(1);
        if (wid == 4) {
            const u32 k = lane & 15, sh = lane >> 4; u32 sel = 0, r = sh;
#pragma unroll
            for (int j = 0; j < 4; ++j) { const u32 kj = (k >> j) & 1; sel |= (kj ? r : 12u) << (8 * j); r += kj; }
            lds_store(lds0 + F_LUT + lane * 4, sel);
        }
        if (wid < 7) {
            BAR();
            asm volatile("ds_read_b32 %0, %1\n\ts_waitcnt lgkmcnt(0)" : "=&v"(call) : "v"(lds0 + F_MISC + 24) : "memory");
            call = rfl(call);
            BAR();
        } else {
            call = rfl(call) + 1;
            if (lane < 8) lds_store(lds0 + F_MISC + 24 + lane * 4, lane == 0 ? call : 0u);
            BAR();
            u32x4 wm; asm volatile("ds_read_b128 %0, %1\n\ts_waitcnt lgkmcnt(0)" : "=&v"(wm) : "v"(lds0 + F_MISC) : "memory");
            const u32 tm = max(max(wm.x, wm.y), max(wm.z, wm.w));
            bool bad = false;
            if (lane == 0) st_sys64(myws + WS_LMAX + (grp * 32 + idx) * 8, ((u64)call << 32) | tm);
            if (idx == 0) {
                const u32 gm = sweep(myws + WS_LMAX + grp * 256, 32, call, myws + WS_TMO, lane, bad);
                if (!bad && lane < NR) st_sys64(p.ws[lane] + WS_GMAX + (call & 1) * 512 + (p.rank * 8 + grp) * 8, ((u64)call << 32) | gm);
            }
            const u32 am = sweep(myws + WS_GMAX + (call & 1) * 512, 64, call, myws + WS_TMO, lane, bad);
            if (lane == 0) { lds_store(lds0 + F_MISC + 16, am); lds_store(lds0 + F_MISC + 20, bad ? 1u : 0u); }
            STAMP(3);
            BAR();
        }
        if (remote) {
            bool bad = wait_flag(myws + WS_HFLAG + (rslot * 4 + pw) * 4, call, myws + WS_TMO, lane);
            u32x4 d;
            asm volatile("global_load_dwordx4 %0, %1, off sc0 sc1\n\ts_waitcnt vmcnt(0)" : "=&v"(d) : "v"(myws + WS_HDR + (size_t)(((call & 1) * 256 + rslot) * 4 + pw) * 1024 + lane * 16) : "memory");
            const u32 k[4] = {~d.x, ~d.y, ~d.z, ~d.w};
            const u32 cnt = 1 + __builtin_popcount(k[0]) + __builtin_popcount(k[1]) + __builtin_popcount(k[2]) + __builtin_popcount(k[3]);
            const u32 inc = wave_scan(cnt);
            const u32 n = __builtin_amdgcn_readlane(inc, 63);
            const u32 sbase = lds0 + pw * ST_PITCH;
            u32 o = sbase + inc - cnt, ad[32], se[32];
#pragma unroll
            for (int i = 0; i < 32; ++i) {
                const u32 nb = (k[i >> 3] >> ((i & 7) * 4)) & 15;
                se[i] = *(const LAS u32*)(uintptr_t)(lds0 + F_LUT + (nb + (o & 3) * 16) * 4);
                ad[i] = (o & ~3u) + ((o >> 2) & 1) * (PAY_PITCH - 4);
                o += __builtin_popcount(nb);
            }
            const bool odd = (lane >> 4) & 1;
#pragma unroll
            for (int i = 0; i < 32; i += 2) {
                const u32 s0 = se[i], s1 = se[i + 1], a0 = ad[i], a1 = ad[i + 1];
                se[i] = odd ? s1 : s0; se[i + 1] = odd ? s0 : s1; ad[i] = odd ? a1 : a0; ad[i + 1] = odd ? a0 : a1;
            }
#pragma unroll
            for (int i = 0; i < 32; ++i) asm volatile("" : "+v"(se[i]), "+v"(ad[i]));
            if (wid == 4) STAMP(8);
            bad |= wait_flag(myws + WS_PFLAG + (rslot * 4 + pw) * 4, call, myws + WS_TMO, lane);
            if (wid == 4) STAMP(9);
            for (;;) {
                u32 f; asm volatile("ds_read_b32 %0, %1\n\ts_waitcnt lgkmcnt(0)" : "=&v"(f) : "v"(lds0 + F_MISC + 32 + pw * 4) : "memory");
                if (rfl(f)) break;
                __builtin_amdgcn_s_sleep(1);
            }
            const unsigned char* src = myws + WS_PAY + (size_t)(rslot * 4 + pw) * PAY_PITCH + lane * 16;
            for (u32 u = 0; u < n; u += 1024)
                asm volatile("s_mov_b32 m0, %2\n\tglobal_load_lds_dwordx4 %0, off sc0 sc1\n\ts_add_u32 m0, %2, %3\n\tglobal_load_lds_dwordx4 %1, off sc0 sc1"
                             :: "v"(src + u), "v"(src + u + 4), "s"(sbase + u), "n"(PAY_PITCH) : "memory", "scc");
            asm volatile("s_waitcnt vmcnt(0)" ::: "memory");
            const u32 nanm = bad ? 0x7f7f7f7fu : 0u;
            unsigned char* dst = rout + (size_t)(lane & 15) * NC + pw * 64 + (lane >> 4) * 16;
            u32x2 v[32];
#pragma unroll
            for (int i = 0; i < 32; ++i) asm volatile("ds_read_b64 %0, %1" : "=&v"(v[i]) : "v"(ad[i]) : "memory");
#pragma unroll
            for (int m = 0; m < 8; ++m) {
                asm volatile("s_waitcnt lgkmcnt(%4)" : "+v"(v[4 * m]), "+v"(v[4 * m + 1]), "+v"(v[4 * m + 2]), "+v"(v[4 * m + 3]) : "n"(m < 4 ? 15 : 28 - 4 * m) : "memory");
                u32x4 w;
                w.x = __builtin_amdgcn_perm(v[4 * m].y, v[4 * m].x, se[4 * m]) | nanm; w.y = __builtin_amdgcn_perm(v[4 * m + 1].y, v[4 * m + 1].x, se[4 * m + 1]) | nanm;
                w.z = __builtin_amdgcn_perm(v[4 * m + 2].y, v[4 * m + 2].x, se[4 * m + 2]) | nanm; w.w = __builtin_amdgcn_perm(v[4 * m + 3].y, v[4 * m + 3].x, se[4 * m + 3]) | nanm;
                *(GAS u32x4*)(dst + (size_t)m * 16 * NC) = w;
            }
            if (wid == 4) STAMP(7);
        }
    }
}

extern "C" void kernel_launch(int rank, int nranks, ncclComm_t comm, hipStream_t stream,
                              void* const* d_in, const int* in_sizes, int n_in,
                              void* d_out, int out_size, const int* in_dtypes, int out_dtype,
                              void* const* d_out_all, void* const* d_ws_all, size_t ws_bytes) {
    if (nranks != NR || n_in < 2 || in_sizes[0] != ML * KD || in_sizes[1] != KD * ND || out_size != NR * ML * NC || ws_bytes < WS_NEED) {
        fprintf(stderr, "kernel_launch: unexpected shapes\n"); return;
    }
    static bool attr[NR] = {};
    if (!attr[rank]) { HIP_CHECK(hipFuncSetAttribute((const void*)fused_kernel, hipFuncAttributeMaxDynamicSharedMemorySize, LDS_BYTES)); attr[rank] = true; }
    Params p;
    p.x = d_in[0]; p.w = d_in[1]; p.rank = rank; p.pad = 0;
    for (int i = 0; i < NR; ++i) { p.out[i] = (unsigned char*)d_out_all[i]; p.ws[i] = (unsigned char*)d_ws_all[i]; }
    hipLaunchKernelGGL(fused_kernel, dim3(NWG), dim3(512), LDS_BYTES, stream, p);
    HIP_CHECK(hipGetLastError());
}
```

```cpp
#include <hip/hip_runtime.h>
#include <rccl/rccl.h>
#include <cstdio>
#include <cstdint>

#define HIP_CHECK(expr) do { hipError_t _e = (expr); if (_e != hipSuccess) { \
    fprintf(stderr, "HIP error %s at %s:%d\n", hipGetErrorString(_e), __FILE__, __LINE__); \
    return; } } while (0)

typedef unsigned u32;
typedef unsigned long long u64;
typedef float f32x16 __attribute__((ext_vector_type(16)));
typedef u32 u32x4 __attribute__((ext_vector_type(4)));
typedef u32 u32x2 __attribute__((ext_vector_type(2)));
typedef u32 u32x3 __attribute__((ext_vector_type(3)));
typedef float f32x4 __attribute__((ext_vector_type(4)));
typedef float f32x2 __attribute__((ext_vector_type(2)));
typedef u32 u32x16 __attribute__((ext_vector_type(16)));
#define LAS __attribute__((address_space(3)))
#define GAS __attribute__((address_space(1)))
#define DI __device__ __forceinline__

constexpr int NR = 8, ML = 512, KD = 4096, ND = 16384, NC = 2048;
constexpr int BM = 128, BN = 256, NT = 64;
constexpr int NWG = 256;
constexpr u32 A_BASE = 0, A_ST = 16384, B_BASE = 49152, B_ST = 32768;
constexpr u32 F_DIG = 133632, F_BIT = 159936, F_LUT = 163328, F_MISC = F_LUT + 256, LDS_BYTES = F_MISC + 256;
constexpr u32 ST_HALF = 16640, ST_PITCH = 2 * ST_HALF;
constexpr u32 RC_HALF = 4352;
constexpr size_t WS_CALL = 0;
constexpr size_t WS_TMO = 4096;
constexpr size_t WS_LMAX = 8192;
constexpr size_t WS_GMAX = 49152;
constexpr size_t WS_FLAG = 16384;
constexpr size_t WS_DBG = 65536;
constexpr size_t WS_DBG2 = WS_DBG + 256 * 64 * 8;
constexpr size_t WS_OFF = 1u << 20;
constexpr size_t WS_CLS = 5u << 20;
constexpr u32 OFF_SLOT = 4096, CLS_SLOT = 5120;
constexpr size_t WS_NEED = WS_CLS + (size_t)2 * 8 * 32 * 4 * CLS_SLOT;
static_assert(WS_FLAG + 1024 * 32 <= WS_GMAX && WS_GMAX + 16384 <= WS_DBG && WS_DBG2 + 256 * 4 * 128 * 4 <= WS_OFF, "debug area");
constexpr long long DEADLINE = 100000000;

struct Params {
    const void* x; const void* w;
    unsigned char* out[NR];
    unsigned char* ws[NR];
    int rank; int pad;
};

DI u32 rfl(u32 v) { return __builtin_amdgcn_readfirstlane(v); }
DI u32x4 make_rsrc(const void* p, u32 bytes) {
    u64 a = (u64)p; u32x4 r; r.x = rfl((u32)a); r.y = rfl((u32)(a >> 32)); r.z = bytes; r.w = 0x00020000u; return r;
}
DI void lds_store(u32 addr, u32 v) { asm volatile("ds_write_b32 %0, %1" :: "v"(addr), "v"(v) : "memory"); }
DI u32 lds_min4(u32 addr) {
    u32x4 d; asm volatile("ds_read_b128 %0, %1\n\ts_waitcnt lgkmcnt(0)" : "=&v"(d) : "v"(addr) : "memory");
    return rfl(min(min(d.x, d.y), min(d.z, d.w)));
}
#define BAR() do { asm volatile("s_waitcnt lgkmcnt(0)\n\ts_barrier" ::: "memory"); } while (0)
#define NSTAMP 20
#define LOCX p.rank
#define OUTMOD "sc0 sc1"
#define TSDUMP 0
#define CPRIO 2
#define PPRIO 0
#define MAP 9
#define ROTX pw
#define ROTW pw
#define WPAIR 1
#define LAG 0
#define RW ((rowblk >> 1) * 4)
#define RX ((cj >> 2) * 2)
#define KX (63 ^ ((rowblk ^ (cj >> 1)) & 1))
#define KP (cj & 1)
#define PIPE 0
#define PSTAG "16"
#define TRN 0
#define PSLEEP 1
#define STAMP(k) do {} while (0)
#define FLUSH() do {} while (0)

DI u64 ld_sys64(const unsigned char* p) { return __hip_atomic_load((const GAS u64*)p, __ATOMIC_RELAXED, __HIP_MEMORY_SCOPE_SYSTEM); }
DI void st_sys64(unsigned char* p, u64 v) { __hip_atomic_store((GAS u64*)p, v, __ATOMIC_RELAXED, __HIP_MEMORY_SCOPE_SYSTEM); }
DI u32 ld_sys32(const unsigned char* p) { return __hip_atomic_load((const GAS u32*)p, __ATOMIC_RELAXED, __HIP_MEMORY_SCOPE_SYSTEM); }
DI void st_sys32(unsigned char* p, u32 v) { __hip_atomic_store((GAS u32*)p, v, __ATOMIC_RELAXED, __HIP_MEMORY_SCOPE_SYSTEM); }
DI u32 ld_agt32(const unsigned char* p) { return __hip_atomic_load((const GAS u32*)p, __ATOMIC_RELAXED, __HIP_MEMORY_SCOPE_AGENT); }
DI void st_agt32(unsigned char* p, u32 v) { __hip_atomic_store((GAS u32*)p, v, __ATOMIC_RELAXED, __HIP_MEMORY_SCOPE_AGENT); }
DI u32 wave_umax(u32 v) {
    v = max(v, (u32)__builtin_amdgcn_update_dpp(0u, v, 0x111, 0xf, 0xf, false));
    v = max(v, (u32)__builtin_amdgcn_update_dpp(0u, v, 0x112, 0xf, 0xf, false));
    v = max(v, (u32)__builtin_amdgcn_update_dpp(0u, v, 0x114, 0xf, 0xf, false));
    v = max(v, (u32)__builtin_amdgcn_update_dpp(0u, v, 0x118, 0xf, 0xf, false));
    v = max(v, (u32)__builtin_amdgcn_update_dpp(0u, v, 0x142, 0xa, 0xf, false));
    v = max(v, (u32)__builtin_amdgcn_update_dpp(0u, v, 0x143, 0xc, 0xf, false));
    return __builtin_amdgcn_readlane(v, 63);
}
DI u32 wave_scan(u32 v) {
    v += __builtin_amdgcn_update_dpp(0u, v, 0x111, 0xf, 0xf, false);
    v += __builtin_amdgcn_update_dpp(0u, v, 0x112, 0xf, 0xf, false);
    v += __builtin_amdgcn_update_dpp(0u, v, 0x114, 0xf, 0xf, false);
    v += __builtin_amdgcn_update_dpp(0u, v, 0x118, 0xf, 0xf, false);
    v += __builtin_amdgcn_update_dpp(0u, v, 0x142, 0xa, 0xf, false);
    v += __builtin_amdgcn_update_dpp(0u, v, 0x143, 0xc, 0xf, false);
    return v;
}
DI bool expired(u64 t0, u32 t, u32 call, unsigned char* tmo, int lane) {
    if (rfl(t) == call) return true;
    if ((long long)(__builtin_amdgcn_s_memrealtime() - t0) > DEADLINE) { if (lane == 0) st_agt32(tmo, call); return true; }
    __builtin_amdgcn_s_sleep(PSLEEP);
    return false;
}
constexpr u32 NOTYET = 0xffffffffu;
DI u32 lds_word(u32 addr) { u32 v; asm volatile("ds_read_b32 %0, %1\n\ts_waitcnt lgkmcnt(0)" : "=&v"(v) : "v"(addr) : "memory"); return rfl(v); }
DI u32 inv_scale(u32 mx) { const float amax = __uint_as_float(mx); return rfl(__float_as_uint(amax > 0.f ? 1.0f / (amax / 448.0f) : 0.f)); }
DI u32 scale_from(u32x2 g, u32 iw, u32 call, u32 lds0) {
    if (iw == NOTYET && __all(g.y == call)) { iw = inv_scale(wave_umax(g.x)); lds_store(lds0 + F_MISC + 16, iw); }
    return iw;
}
DI u32 class_shift(float inv, u32 lo) {
    const float t = inv * __uint_as_float(lo);
    return (120u - ((u32)__builtin_amdgcn_cvt_pk_fp8_f32(t, t, 0, false) & 255u)) * 0x00010001u;
}

#define INJ(s)
__global__ __launch_bounds__(512, 2) void fused_kernel(Params p) {
    extern __shared__ __attribute__((aligned(16))) unsigned char smem[];
    const int wid = rfl(threadIdx.x >> 6);
    const int b = blockIdx.x, xcd = b & 7, rowblk = (b >> 3) & 3, cj = b >> 5;
    const int ct = cj, grp = (xcd + 1 + p.rank) & 7;
    const int idx = ct * 4 + rowblk;
    const u32 lds0 = (u32)(uintptr_t)(LAS unsigned char*)smem;
    unsigned char* const myws = p.ws[p.rank];
    if (wid == 0) STAMP(0);
    unsigned char* const pws = p.ws[grp]; const u32 ssrc = p.rank;
    unsigned char* const rout = p.out[p.rank] + (size_t)(grp * ML + rowblk * BM) * NC + ct * BN;
    const bool remote = grp != p.rank;

    if (wid < 4) {
        f32x16 c0, c1, c2, c3, c4, c5, c6, c7; u32 slowc; u64 tc0 = 0;
        u32x4 d; u32 call, wm;
        if (wid == 0) {
            const u32 lane = threadIdx.x & 63, k = lane & 15; u32 sel = 0, r = (lane >> 4) + 1;
#pragma unroll
            for (int j = 0; j < 4; ++j) { const u32 kj = (k >> (3 - j)) & 1; sel |= (kj ? r : 12u) << (8 * j); r += kj; }
            lds_store(lds0 + F_LUT + lane * 4, sel);
            if (lane < 13) lds_store(lds0 + F_MISC + (lane < 5 ? lane * 4 : 12 + lane * 4), lane < 5 ? NOTYET : 0);
        }
        {
            const u32 lane = threadIdx.x & 63, r = lane & 15, g = lane >> 4, q = r >> 2, pp = lane & 3;
            const u32 a0 = lds0 + A_BASE + r * 128 + ((g ^ (r >> 1)) << 4), a1 = a0 ^ 64;
            const u32 bb = lds0 + B_BASE + (8 * g + q) * 512, F = q + 8 * (g & 1), cb = 8 * wid + 2 * pp, h = (pp & 1) * 8;
            const u32 b0 = bb + ((cb ^ F) << 4) + h, b1 = b0 ^ 8, b2 = b0 ^ 16, b3 = b0 ^ 24;
            u32 si, sst, sdA, sdB; slowc = 0;
            __builtin_amdgcn_s_setprio(CPRIO);
            asm volatile(
      "v_mov_b32 v16, %[a0]\n"
      "v_mov_b32 v17, %[a1]\n"
      "v_mov_b32 v18, %[b0]\n"
      "v_mov_b32 v19, %[b1]\n"
      "v_mov_b32 v20, %[b2]\n"
      "v_mov_b32 v21, %[b3]\n"
      "v_mov_b32 v128, 0\n"
      "v_mov_b32 v129, 0\n"
      "v_mov_b32 v130, 0\n"
      "v_mov_b32 v131, 0\n"
      "v_mov_b32 v132, 0\n"
      "v_mov_b32 v133, 0\n"
      "v_mov_b32 v134, 0\n"
      "v_mov_b32 v135, 0\n"
      "v_mov_b32 v136, 0\n"
      "v_mov_b32 v137, 0\n"
      "v_mov_b32 v138, 0\n"
      "v_mov_b32 v139, 0\n"
      "v_mov_b32 v140, 0\n"
      "v_mov_b32 v141, 0\n"
      "v_mov_b32 v142, 0\n"
      "v_mov_b32 v143, 0\n"
      "v_mov_b32 v144, 0\n"
      "v_mov_b32 v145, 0\n"
      "v_mov_b32 v146, 0\n"
      "v_mov_b32 v147, 0\n"
      "v_mov_b32 v148, 0\n"
      "v_mov_b32 v149, 0\n"
      "v_mov_b32 v150, 0\n"
      "v_mov_b32 v151, 0\n"
      "v_mov_b32 v152, 0\n"
      "v_mov_b32 v153, 0\n"
      "v_mov_b32 v154, 0\n"
      "v_mov_b32 v155, 0\n"
      "v_mov_b32 v156, 0\n"
      "v_mov_b32 v157, 0\n"
      "v_mov_b32 v158, 0\n"
      "v_mov_b32 v159, 0\n"
      "v_mov_b32 v160, 0\n"
      "v_mov_b32 v161, 0\n"
      "v_mov_b32 v162, 0\n"
      "v_mov_b32 v163, 0\n"
      "v_mov_b32 v164, 0\n"
      "v_mov_b32 v165, 0\n"
      "v_mov_b32 v166, 0\n"
      "v_mov_b32 v167, 0\n"
      "v_mov_b32 v168, 0\n"
      "v_mov_b32 v169, 0\n"
      "v_mov_b32 v170, 0\n"
      "v_mov_b32 v171, 0\n"
      "v_mov_b32 v172, 0\n"
      "v_mov_b32 v173, 0\n"
      "v_mov_b32 v174, 0\n"
      "v_mov_b32 v175, 0\n"
      "v_mov_b32 v176, 0\n"
      "v_mov_b32 v177, 0\n"
      "v_mov_b32 v178, 0\n"
      "v_mov_b32 v179, 0\n"
      "v_mov_b32 v180, 0\n"
      "v_mov_b32 v181, 0\n"
      "v_mov_b32 v182, 0\n"
      "v_mov_b32 v183, 0\n"
      "v_mov_b32 v184, 0\n"
      "v_mov_b32 v185, 0\n"
      "v_mov_b32 v186, 0\n"
      "v_mov_b32 v187, 0\n"
      "v_mov_b32 v188, 0\n"
      "v_mov_b32 v189, 0\n"
      "v_mov_b32 v190, 0\n"
      "v_mov_b32 v191, 0\n"
      "v_mov_b32 v192, 0\n"
      "v_mov_b32 v193, 0\n"
      "v_mov_b32 v194, 0\n"
      "v_mov_b32 v195, 0\n"
      "v_mov_b32 v196, 0\n"
      "v_mov_b32 v197, 0\n"
      "v_mov_b32 v198, 0\n"
      "v_mov_b32 v199, 0\n"
      "v_mov_b32 v200, 0\n"
      "v_mov_b32 v201, 0\n"
      "v_mov_b32 v202, 0\n"
      "v_mov_b32 v203, 0\n"
      "v_mov_b32 v204, 0\n"
      "v_mov_b32 v205, 0\n"
      "v_mov_b32 v206, 0\n"
      "v_mov_b32 v207, 0\n"
      "v_mov_b32 v208, 0\n"
      "v_mov_b32 v209, 0\n"
      "v_mov_b32 v210, 0\n"
      "v_mov_b32 v211, 0\n"
      "v_mov_b32 v212, 0\n"
      "v_mov_b32 v213, 0\n"
      "v_mov_b32 v214, 0\n"
      "v_mov_b32 v215, 0\n"
      "v_mov_b32 v216, 0\n"
      "v_mov_b32 v217, 0\n"
      "v_mov_b32 v218, 0\n"
      "v_mov_b32 v219, 0\n"
      "v_mov_b32 v220, 0\n"
      "v_mov_b32 v221, 0\n"
      "v_mov_b32 v222, 0\n"
      "v_mov_b32 v223, 0\n"
      "v_mov_b32 v224, 0\n"
      "v_mov_b32 v225, 0\n"
      "v_mov_b32 v226, 0\n"
      "v_mov_b32 v227, 0\n"
      "v_mov_b32 v228, 0\n"
      "v_mov_b32 v229, 0\n"
      "v_mov_b32 v230, 0\n"
      "v_mov_b32 v231, 0\n"
      "v_mov_b32 v232, 0\n"
      "v_mov_b32 v233, 0\n"
      "v_mov_b32 v234, 0\n"
      "v_mov_b32 v235, 0\n"
      "v_mov_b32 v236, 0\n"
      "v_mov_b32 v237, 0\n"
      "v_mov_b32 v238, 0\n"
      "v_mov_b32 v239, 0\n"
      "v_mov_b32 v240, 0\n"
      "v_mov_b32 v241, 0\n"
      "v_mov_b32 v242, 0\n"
      "v_mov_b32 v243, 0\n"
      "v_mov_b32 v244, 0\n"
      "v_mov_b32 v245, 0\n"
      "v_mov_b32 v246, 0\n"
      "v_mov_b32 v247, 0\n"
      "v_mov_b32 v248, 0\n"
      "v_mov_b32 v249, 0\n"
      "v_mov_b32 v250, 0\n"
      "v_mov_b32 v251, 0\n"
      "v_mov_b32 v252, 0\n"
      "v_mov_b32 v253, 0\n"
      "v_mov_b32 v254, 0\n"
      "v_mov_b32 v255, 0\n"
      "s_mov_b32 %[i], 0\n"
      "s_mov_b32 %[st], 0\n"
      "s_waitcnt lgkmcnt(0)\n"
      "s_barrier\n"
      "ds_read_b64_tr_b16 v[64:65], v18 offset:0\n"
      "ds_read_b64_tr_b16 v[66:67], v18 offset:2048\n"
      "ds_read_b128 v[32:35], v16 offset:0\n"
      "ds_read_b64_tr_b16 v[68:69], v19 offset:0\n"
      "ds_read_b64_tr_b16 v[70:71], v19 offset:2048\n"
      "ds_read_b64_tr_b16 v[72:73], v20 offset:0\n"
      "ds_read_b64_tr_b16 v[74:75], v20 offset:2048\n"
      "ds_read_b64_tr_b16 v[76:77], v21 offset:0\n"
      "ds_read_b64_tr_b16 v[78:79], v21 offset:2048\n"
      "ds_read_b128 v[36:39], v16 offset:2048\n"
      "ds_read_b128 v[40:43], v16 offset:4096\n"
      "ds_read_b128 v[44:47], v16 offset:6144\n"
      "ds_read_b128 v[48:51], v16 offset:8192\n"
      "ds_read_b128 v[52:55], v16 offset:10240\n"
      "ds_read_b128 v[56:59], v16 offset:12288\n"
      "ds_read_b128 v[60:63], v16 offset:14336\n"
      "s_waitcnt lgkmcnt(0)\n"
      ".p2align 4\n"
      "L_loop_%=:\n"
      "s_waitcnt lgkmcnt(0)\n"
      "v_mfma_f32_16x16x32_bf16 v[128:131], v[64:67], v[32:35], v[128:131]\n"
      "ds_read_b64_tr_b16 v[112:113], v18 offset:16384\n"
      "v_mfma_f32_16x16x32_bf16 v[132:135], v[68:71], v[32:35], v[132:135]\n"
      "ds_read_b64_tr_b16 v[114:115], v18 offset:18432\n"
      "s_cmp_eq_u32 %[st], 2\n"
      "v_mfma_f32_16x16x32_bf16 v[136:139], v[72:75], v[32:35], v[136:139]\n"
      "ds_read_b128 v[80:83], v17 offset:0\n"
      "s_cselect_b32 %[dA], %[cm], %[cp]\n"
      "v_mfma_f32_16x16x32_bf16 v[140:143], v[76:79], v[32:35], v[140:143]\n"
      "ds_read_b64_tr_b16 v[116:117], v19 offset:16384\n"
      "s_cselect_b32 %[st], -1, %[st]\n"
      "v_mfma_f32_16x16x32_bf16 v[156:159], v[76:79], v[36:39], v[156:159]\n"
      "ds_read_b64_tr_b16 v[118:119], v19 offset:18432\n"
      "s_add_u32 %[st], %[st], 1\n"
      "v_mfma_f32_16x16x32_bf16 v[152:155], v[72:75], v[36:39], v[152:155]\n"
      "ds_read_b64_tr_b16 v[120:121], v20 offset:16384\n"
      "s_lshl_b32 %[dB], %[dA], 1\n"
      "v_mfma_f32_16x16x32_bf16 v[148:151], v[68:71], v[36:39], v[148:151]\n"
      "ds_read_b64_tr_b16 v[122:123], v20 offset:18432\n"
      "v_mfma_f32_16x16x32_bf16 v[144:147], v[64:67], v[36:39], v[144:147]\n"
      "ds_read_b64_tr_b16 v[124:125], v21 offset:16384\n"
      "v_mfma_f32_16x16x32_bf16 v[160:163], v[64:67], v[40:43], v[160:163]\n"
      "ds_read_b64_tr_b16 v[126:127], v21 offset:18432\n"
      "v_mfma_f32_16x16x32_bf16 v[164:167], v[68:71], v[40:43], v[164:167]\n"
      "ds_read_b128 v[84:87], v17 offset:2048\n"
      "v_mfma_f32_16x16x32_bf16 v[168:171], v[72:75], v[40:43], v[168:171]\n"
      "ds_read_b128 v[88:91], v17 offset:4096\n"
      "v_mfma_f32_16x16x32_bf16 v[172:175], v[76:79], v[40:43], v[172:175]\n"
      "ds_read_b128 v[92:95], v17 offset:6144\n"
      "v_mfma_f32_16x16x32_bf16 v[188:191], v[76:79], v[44:47], v[188:191]\n"
      "ds_read_b128 v[96:99], v17 offset:8192\n"
      "v_mfma_f32_16x16x32_bf16 v[184:187], v[72:75], v[44:47], v[184:187]\n"
      "ds_read_b128 v[100:103], v17 offset:10240\n"
      "v_mfma_f32_16x16x32_bf16 v[180:183], v[68:71], v[44:47], v[180:183]\n"
      "ds_read_b128 v[104:107], v17 offset:12288\n"
      "v_mfma_f32_16x16x32_bf16 v[176:179], v[64:67], v[44:47], v[176:179]\n"
      "ds_read_b128 v[108:111], v17 offset:14336\n"
      "v_mfma_f32_16x16x32_bf16 v[192:195], v[64:67], v[48:51], v[192:195]\n"
      "v_add_u32 v16, %[dA], v16\n"
      "v_mfma_f32_16x16x32_bf16 v[196:199], v[68:71], v[48:51], v[196:199]\n"
      "v_add_u32 v17, %[dA], v17\n"
      "v_mfma_f32_16x16x32_bf16 v[200:203], v[72:75], v[48:51], v[200:203]\n"
      "v_add_u32 v18, %[dB], v18\n"
      "v_mfma_f32_16x16x32_bf16 v[204:207], v[76:79], v[48:51], v[204:207]\n"
      "v_add_u32 v19, %[dB], v19\n"
      "v_mfma_f32_16x16x32_bf16 v[220:223], v[76:79], v[52:55], v[220:223]\n"
      "v_add_u32 v20, %[dB], v20\n"
      "v_mfma_f32_16x16x32_bf16 v[216:219], v[72:75], v[52:55], v[216:219]\n"
      "v_add_u32 v21, %[dB], v21\n"
      "v_mfma_f32_16x16x32_bf16 v[212:215], v[68:71], v[52:55], v[212:215]\n"
      "v_mfma_f32_16x16x32_bf16 v[208:211], v[64:67], v[52:55], v[208:211]\n"
      "v_mfma_f32_16x16x32_bf16 v[224:227], v[64:67], v[56:59], v[224:227]\n"
      "s_waitcnt lgkmcnt(0)\n"
      "s_barrier\n"
      "v_mfma_f32_16x16x32_bf16 v[228:231], v[68:71], v[56:59], v[228:231]\n"
      "v_mfma_f32_16x16x32_bf16 v[232:235], v[72:75], v[56:59], v[232:235]\n"
      "v_mfma_f32_16x16x32_bf16 v[236:239], v[76:79], v[56:59], v[236:239]\n"
      "v_mfma_f32_16x16x32_bf16 v[252:255], v[76:79], v[60:63], v[252:255]\n"
      "v_mfma_f32_16x16x32_bf16 v[248:251], v[72:75], v[60:63], v[248:251]\n"
      "v_mfma_f32_16x16x32_bf16 v[244:247], v[68:71], v[60:63], v[244:247]\n"
      "v_mfma_f32_16x16x32_bf16 v[240:243], v[64:67], v[60:63], v[240:243]\n"
      "v_mfma_f32_16x16x32_bf16 v[128:131], v[112:115], v[80:83], v[128:131]\n"
      "ds_read_b64_tr_b16 v[64:65], v18 offset:0\n"
      "v_mfma_f32_16x16x32_bf16 v[132:135], v[116:119], v[80:83], v[132:135]\n"
      "ds_read_b64_tr_b16 v[66:67], v18 offset:2048\n"
      "v_mfma_f32_16x16x32_bf16 v[136:139], v[120:123], v[80:83], v[136:139]\n"
      "ds_read_b128 v[32:35], v16 offset:0\n"
      "v_mfma_f32_16x16x32_bf16 v[140:143], v[124:127], v[80:83], v[140:143]\n"
      "ds_read_b64_tr_b16 v[68:69], v19 offset:0\n"
      "v_mfma_f32_16x16x32_bf16 v[156:159], v[124:127], v[84:87], v[156:159]\n"
      "ds_read_b64_tr_b16 v[70:71], v19 offset:2048\n"
      "v_mfma_f32_16x16x32_bf16 v[152:155], v[120:123], v[84:87], v[152:155]\n"
      "ds_read_b64_tr_b16 v[72:73], v20 offset:0\n"
      "v_mfma_f32_16x16x32_bf16 v[148:151], v[116:119], v[84:87], v[148:151]\n"
      "ds_read_b64_tr_b16 v[74:75], v20 offset:2048\n"
      "v_mfma_f32_16x16x32_bf16 v[144:147], v[112:115], v[84:87], v[144:147]\n"
      "ds_read_b64_tr_b16 v[76:77], v21 offset:0\n"
      "v_mfma_f32_16x16x32_bf16 v[160:163], v[112:115], v[88:91], v[160:163]\n"
      "ds_read_b64_tr_b16 v[78:79], v21 offset:2048\n"
      "v_mfma_f32_16x16x32_bf16 v[164:167], v[116:119], v[88:91], v[164:167]\n"
      "ds_read_b128 v[36:39], v16 offset:2048\n"
      "v_mfma_f32_16x16x32_bf16 v[168:171], v[120:123], v[88:91], v[168:171]\n"
      "ds_read_b128 v[40:43], v16 offset:4096\n"
      "v_mfma_f32_16x16x32_bf16 v[172:175], v[124:127], v[88:91], v[172:175]\n"
      "ds_read_b128 v[44:47], v16 offset:6144\n"
      "v_mfma_f32_16x16x32_bf16 v[188:191], v[124:127], v[92:95], v[188:191]\n"
      "ds_read_b128 v[48:51], v16 offset:8192\n"
      "v_mfma_f32_16x16x32_bf16 v[184:187], v[120:123], v[92:95], v[184:187]\n"
      "ds_read_b128 v[52:55], v16 offset:10240\n"
      "v_mfma_f32_16x16x32_bf16 v[180:183], v[116:119], v[92:95], v[180:183]\n"
      "ds_read_b128 v[56:59], v16 offset:12288\n"
      "v_mfma_f32_16x16x32_bf16 v[176:179], v[112:115], v[92:95], v[176:179]\n"
      "ds_read_b128 v[60:63], v16 offset:14336\n"
      "v_mfma_f32_16x16x32_bf16 v[192:195], v[112:115], v[96:99], v[192:195]\n"
      "v_mfma_f32_16x16x32_bf16 v[196:199], v[116:119], v[96:99], v[196:199]\n"
      "v_mfma_f32_16x16x32_bf16 v[200:203], v[120:123], v[96:99], v[200:203]\n"
      "v_mfma_f32_16x16x32_bf16 v[204:207], v[124:127], v[96:99], v[204:207]\n"
      "v_mfma_f32_16x16x32_bf16 v[220:223], v[124:127], v[100:103], v[220:223]\n"
      "v_mfma_f32_16x16x32_bf16 v[216:219], v[120:123], v[100:103], v[216:219]\n"
      "v_mfma_f32_16x16x32_bf16 v[212:215], v[116:119], v[100:103], v[212:215]\n"
      "v_mfma_f32_16x16x32_bf16 v[208:211], v[112:115], v[100:103], v[208:211]\n"
      "v_mfma_f32_16x16x32_bf16 v[224:227], v[112:115], v[104:107], v[224:227]\n"
      "v_mfma_f32_16x16x32_bf16 v[228:231], v[116:119], v[104:107], v[228:231]\n"
      "v_mfma_f32_16x16x32_bf16 v[232:235], v[120:123], v[104:107], v[232:235]\n"
      "v_mfma_f32_16x16x32_bf16 v[236:239], v[124:127], v[104:107], v[236:239]\n"
      "v_mfma_f32_16x16x32_bf16 v[252:255], v[124:127], v[108:111], v[252:255]\n"
      "s_add_u32 %[i], %[i], 1\n"
      "v_mfma_f32_16x16x32_bf16 v[248:251], v[120:123], v[108:111], v[248:251]\n"
      "s_cmp_lt_u32 %[i], 63\n"
      "v_mfma_f32_16x16x32_bf16 v[244:247], v[116:119], v[108:111], v[244:247]\n"
      "v_mfma_f32_16x16x32_bf16 v[240:243], v[112:115], v[108:111], v[240:243]\n"
      "s_cbranch_scc1 L_loop_%=\n"
      "s_waitcnt lgkmcnt(0)\n"
      "v_mfma_f32_16x16x32_bf16 v[128:131], v[64:67], v[32:35], v[128:131]\n"
      "ds_read_b64_tr_b16 v[112:113], v18 offset:16384\n"
      "v_mfma_f32_16x16x32_bf16 v[132:135], v[68:71], v[32:35], v[132:135]\n"
      "ds_read_b64_tr_b16 v[114:115], v18 offset:18432\n"
      "v_mfma_f32_16x16x32_bf16 v[136:139], v[72:75], v[32:35], v[136:139]\n"
      "ds_read_b128 v[80:83], v17 offset:0\n"
      "v_mfma_f32_16x16x32_bf16 v[140:143], v[76:79], v[32:35], v[140:143]\n"
      "ds_read_b64_tr_b16 v[116:117], v19 offset:16384\n"
      "v_mfma_f32_16x16x32_bf16 v[156:159], v[76:79], v[36:39], v[156:159]\n"
      "ds_read_b64_tr_b16 v[118:119], v19 offset:18432\n"
      "v_mfma_f32_16x16x32_bf16 v[152:155], v[72:75], v[36:39], v[152:155]\n"
      "ds_read_b64_tr_b16 v[120:121], v20 offset:16384\n"
      "v_mfma_f32_16x16x32_bf16 v[148:151], v[68:71], v[36:39], v[148:151]\n"
      "ds_read_b64_tr_b16 v[122:123], v20 offset:18432\n"
      "v_mfma_f32_16x16x32_bf16 v[144:147], v[64:67], v[36:39], v[144:147]\n"
      "ds_read_b64_tr_b16 v[124:125], v21 offset:16384\n"
      "v_mfma_f32_16x16x32_bf16 v[160:163], v[64:67], v[40:43], v[160:163]\n"
      "ds_read_b64_tr_b16 v[126:127], v21 offset:18432\n"
      "v_mfma_f32_16x16x32_bf16 v[164:167], v[68:71], v[40:43], v[164:167]\n"
      "ds_read_b128 v[84:87], v17 offset:2048\n"
      "v_mfma_f32_16x16x32_bf16 v[168:171], v[72:75], v[40:43], v[168:171]\n"
      "ds_read_b128 v[88:91], v17 offset:4096\n"
      "v_mfma_f32_16x16x32_bf16 v[172:175], v[76:79], v[40:43], v[172:175]\n"
      "ds_read_b128 v[92:95], v17 offset:6144\n"
      "v_mfma_f32_16x16x32_bf16 v[188:191], v[76:79], v[44:47], v[188:191]\n"
      "ds_read_b128 v[96:99], v17 offset:8192\n"
      "v_mfma_f32_16x16x32_bf16 v[184:187], v[72:75], v[44:47], v[184:187]\n"
      "ds_read_b128 v[100:103], v17 offset:10240\n"
      "v_mfma_f32_16x16x32_bf16 v[180:183], v[68:71], v[44:47], v[180:183]\n"
      "ds_read_b128 v[104:107], v17 offset:12288\n"
      "v_mfma_f32_16x16x32_bf16 v[176:179], v[64:67], v[44:47], v[176:179]\n"
      "ds_read_b128 v[108:111], v17 offset:14336\n"
      "v_mfma_f32_16x16x32_bf16 v[192:195], v[64:67], v[48:51], v[192:195]\n"
      "v_mfma_f32_16x16x32_bf16 v[196:199], v[68:71], v[48:51], v[196:199]\n"
      "v_mfma_f32_16x16x32_bf16 v[200:203], v[72:75], v[48:51], v[200:203]\n"
      "v_mfma_f32_16x16x32_bf16 v[204:207], v[76:79], v[48:51], v[204:207]\n"
      "v_mfma_f32_16x16x32_bf16 v[220:223], v[76:79], v[52:55], v[220:223]\n"
      "v_mfma_f32_16x16x32_bf16 v[216:219], v[72:75], v[52:55], v[216:219]\n"
      "v_mfma_f32_16x16x32_bf16 v[212:215], v[68:71], v[52:55], v[212:215]\n"
      "v_mfma_f32_16x16x32_bf16 v[208:211], v[64:67], v[52:55], v[208:211]\n"
      "v_mfma_f32_16x16x32_bf16 v[224:227], v[64:67], v[56:59], v[224:227]\n"
      "s_waitcnt lgkmcnt(0)\n"
      "s_barrier\n"
      "v_mfma_f32_16x16x32_bf16 v[228:231], v[68:71], v[56:59], v[228:231]\n"
      "ds_read_b32 %[cl], %[ca]\n"
      "v_mfma_f32_16x16x32_bf16 v[232:235], v[72:75], v[56:59], v[232:235]\n"
      "v_mfma_f32_16x16x32_bf16 v[236:239], v[76:79], v[56:59], v[236:239]\n"
      "v_mfma_f32_16x16x32_bf16 v[252:255], v[76:79], v[60:63], v[252:255]\n"
      "v_mfma_f32_16x16x32_bf16 v[248:251], v[72:75], v[60:63], v[248:251]\n"
      "v_mfma_f32_16x16x32_bf16 v[244:247], v[68:71], v[60:63], v[244:247]\n"
      "v_mfma_f32_16x16x32_bf16 v[240:243], v[64:67], v[60:63], v[240:243]\n"
      "v_mfma_f32_16x16x32_bf16 v[128:131], v[112:115], v[80:83], v[128:131]\n"
      "v_mfma_f32_16x16x32_bf16 v[132:135], v[116:119], v[80:83], v[132:135]\n"
      "v_mfma_f32_16x16x32_bf16 v[136:139], v[120:123], v[80:83], v[136:139]\n"
      "v_mfma_f32_16x16x32_bf16 v[140:143], v[124:127], v[80:83], v[140:143]\n"
      "v_mfma_f32_16x16x32_bf16 v[156:159], v[124:127], v[84:87], v[156:159]\n"
      "v_mfma_f32_16x16x32_bf16 v[152:155], v[120:123], v[84:87], v[152:155]\n"
      "v_mfma_f32_16x16x32_bf16 v[148:151], v[116:119], v[84:87], v[148:151]\n"
      "v_mfma_f32_16x16x32_bf16 v[144:147], v[112:115], v[84:87], v[144:147]\n"
      "v_mfma_f32_16x16x32_bf16 v[160:163], v[112:115], v[88:91], v[160:163]\n"
      "v_max3_f32 v22, v128, v129, 0\n"
      "v_max3_f32 v23, v130, v131, 0\n"
      "v_lshrrev_b32 v24, 31, v128\n"
      "v_mfma_f32_16x16x32_bf16 v[164:167], v[116:119], v[88:91], v[164:167]\n"
      "v_max3_f32 v22, v132, v133, v22\n"
      "v_max3_f32 v23, v134, v135, v23\n"
      "v_alignbit_b32 v24, v24, v129, 31\n"
      "v_mfma_f32_16x16x32_bf16 v[168:171], v[120:123], v[88:91], v[168:171]\n"
      "v_max3_f32 v22, v136, v137, v22\n"
      "v_max3_f32 v23, v138, v139, v23\n"
      "v_alignbit_b32 v24, v24, v130, 31\n"
      "v_mfma_f32_16x16x32_bf16 v[172:175], v[124:127], v[88:91], v[172:175]\n"
      "v_max3_f32 v22, v140, v141, v22\n"
      "v_max3_f32 v23, v142, v143, v23\n"
      "v_alignbit_b32 v24, v24, v131, 31\n"
      "v_mfma_f32_16x16x32_bf16 v[188:191], v[124:127], v[92:95], v[188:191]\n"
      "v_max3_f32 v22, v144, v145, v22\n"
      "v_max3_f32 v23, v146, v147, v23\n"
      "v_alignbit_b32 v24, v24, v132, 31\n"
      "v_mfma_f32_16x16x32_bf16 v[184:187], v[120:123], v[92:95], v[184:187]\n"
      "v_max3_f32 v22, v148, v149, v22\n"
      "v_max3_f32 v23, v150, v151, v23\n"
      "v_alignbit_b32 v24, v24, v133, 31\n"
      "v_mfma_f32_16x16x32_bf16 v[180:183], v[116:119], v[92:95], v[180:183]\n"
      "v_max3_f32 v22, v152, v153, v22\n"
      "v_max3_f32 v23, v154, v155, v23\n"
      "v_alignbit_b32 v24, v24, v134, 31\n"
      "v_mfma_f32_16x16x32_bf16 v[176:179], v[112:115], v[92:95], v[176:179]\n"
      "v_max3_f32 v22, v156, v157, v22\n"
      "v_max3_f32 v23, v158, v159, v23\n"
      "v_alignbit_b32 v24, v24, v135, 31\n"
      "v_mfma_f32_16x16x32_bf16 v[192:195], v[112:115], v[96:99], v[192:195]\n"
      "v_max3_f32 v22, v160, v161, v22\n"
      "v_max3_f32 v23, v162, v163, v23\n"
      "v_alignbit_b32 v24, v24, v136, 31\n"
      "v_mfma_f32_16x16x32_bf16 v[196:199], v[116:119], v[96:99], v[196:199]\n"
      "v_max3_f32 v22, v164, v165, v22\n"
      "v_max3_f32 v23, v166, v167, v23\n"
      "v_alignbit_b32 v24, v24, v137, 31\n"
      "v_mfma_f32_16x16x32_bf16 v[200:203], v[120:123], v[96:99], v[200:203]\n"
      "v_max3_f32 v22, v168, v169, v22\n"
      "v_max3_f32 v23, v170, v171, v23\n"
      "v_alignbit_b32 v24, v24, v138, 31\n"
      "v_mfma_f32_16x16x32_bf16 v[204:207], v[124:127], v[96:99], v[204:207]\n"
      "v_max3_f32 v22, v172, v173, v22\n"
      "v_max3_f32 v23, v174, v175, v23\n"
      "v_alignbit_b32 v24, v24, v139, 31\n"
      "v_mfma_f32_16x16x32_bf16 v[220:223], v[124:127], v[100:103], v[220:223]\n"
      "v_max3_f32 v22, v176, v177, v22\n"
      "v_max3_f32 v23, v178, v179, v23\n"
      "v_alignbit_b32 v24, v24, v140, 31\n"
      "v_mfma_f32_16x16x32_bf16 v[216:219], v[120:123], v[100:103], v[216:219]\n"
      "v_max3_f32 v22, v180, v181, v22\n"
      "v_max3_f32 v23, v182, v183, v23\n"
      "v_alignbit_b32 v24, v24, v141, 31\n"
      "v_mfma_f32_16x16x32_bf16 v[212:215], v[116:119], v[100:103], v[212:215]\n"
      "v_max3_f32 v22, v184, v185, v22\n"
      "v_max3_f32 v23, v186, v187, v23\n"
      "v_alignbit_b32 v24, v24, v142, 31\n"
      "v_mfma_f32_16x16x32_bf16 v[208:211], v[112:115], v[100:103], v[208:211]\n"
      "v_max3_f32 v22, v188, v189, v22\n"
      "v_max3_f32 v23, v190, v191, v23\n"
      "v_alignbit_b32 v24, v24, v143, 31\n"
      "v_mfma_f32_16x16x32_bf16 v[224:227], v[112:115], v[104:107], v[224:227]\n"
      "v_max3_f32 v22, v192, v193, v22\n"
      "v_max3_f32 v23, v194, v195, v23\n"
      "v_alignbit_b32 v24, v24, v144, 31\n"
      "v_mfma_f32_16x16x32_bf16 v[228:231], v[116:119], v[104:107], v[228:231]\n"
      "v_max3_f32 v22, v196, v197, v22\n"
      "v_max3_f32 v23, v198, v199, v23\n"
      "v_alignbit_b32 v24, v24, v145, 31\n"
      "v_mfma_f32_16x16x32_bf16 v[232:235], v[120:123], v[104:107], v[232:235]\n"
      "v_max3_f32 v22, v200, v201, v22\n"
      "v_max3_f32 v23, v202, v203, v23\n"
      "v_alignbit_b32 v24, v24, v146, 31\n"
      "v_mfma_f32_16x16x32_bf16 v[236:239], v[124:127], v[104:107], v[236:239]\n"
      "v_max3_f32 v22, v204, v205, v22\n"
      "v_max3_f32 v23, v206, v207, v23\n"
      "v_alignbit_b32 v24, v24, v147, 31\n"
      "v_mfma_f32_16x16x32_bf16 v[252:255], v[124:127], v[108:111], v[252:255]\n"
      "v_max3_f32 v22, v208, v209, v22\n"
      "v_max3_f32 v23, v210, v211, v23\n"
      "v_alignbit_b32 v24, v24, v148, 31\n"
      "v_mfma_f32_16x16x32_bf16 v[248:251], v[120:123], v[108:111], v[248:251]\n"
      "v_max3_f32 v22, v212, v213, v22\n"
      "v_max3_f32 v23, v214, v215, v23\n"
      "v_alignbit_b32 v24, v24, v149, 31\n"
      "v_mfma_f32_16x16x32_bf16 v[244:247], v[116:119], v[108:111], v[244:247]\n"
      "v_max3_f32 v22, v216, v217, v22\n"
      "v_max3_f32 v23, v218, v219, v23\n"
      "v_alignbit_b32 v24, v24, v150, 31\n"
      "v_mfma_f32_16x16x32_bf16 v[240:243], v[112:115], v[108:111], v[240:243]\n"
      "v_max3_f32 v22, v220, v221, v22\n"
      "v_max3_f32 v23, v222, v223, v23\n"
      "v_alignbit_b32 v24, v24, v151, 31\n"
      "v_max3_f32 v22, v224, v225, v22\n"
      "v_max3_f32 v23, v226, v227, v23\n"
      "v_max3_f32 v22, v228, v229, v22\n"
      "v_max3_f32 v23, v230, v231, v23\n"
      "v_max3_f32 v22, v232, v233, v22\n"
      "v_max3_f32 v23, v234, v235, v23\n"
      "v_max3_f32 v22, v236, v237, v22\n"
      "v_max3_f32 v23, v238, v239, v23\n"
      "v_alignbit_b32 v24, v24, v152, 31\n"
      "v_alignbit_b32 v24, v24, v153, 31\n"
      "v_alignbit_b32 v24, v24, v154, 31\n"
      "v_alignbit_b32 v24, v24, v155, 31\n"
      "v_max3_f32 v23, v254, v255, v23\n"
      "v_max3_f32 v22, v252, v253, v22\n"
      "v_max3_f32 v23, v250, v251, v23\n"
      "v_max3_f32 v22, v248, v249, v22\n"
      "v_max3_f32 v23, v246, v247, v23\n"
      "v_max3_f32 v22, v244, v245, v22\n"
      "v_alignbit_b32 v24, v24, v156, 31\n"
      "v_alignbit_b32 v24, v24, v157, 31\n"
      "v_alignbit_b32 v24, v24, v158, 31\n"
      "v_alignbit_b32 v24, v24, v159, 31\n"
      "v_lshrrev_b32 v25, 31, v160\n"
      "v_alignbit_b32 v25, v25, v161, 31\n"
      "v_max3_f32 v23, v242, v243, v23\n"
      "v_max3_f32 v22, v240, v241, v22\n"
      "v_max_u32 v22, v22, v23\n"
      "v_alignbit_b32 v25, v25, v162, 31\n"
      "v_alignbit_b32 v25, v25, v163, 31\n"
      "v_max_u32_dpp v22, v22, v22 row_shr:1 row_mask:0xf bank_mask:0xf\n"
      "v_alignbit_b32 v25, v25, v164, 31\n"
      "v_alignbit_b32 v25, v25, v165, 31\n"
      "v_max_u32_dpp v22, v22, v22 row_shr:2 row_mask:0xf bank_mask:0xf\n"
      "v_alignbit_b32 v25, v25, v166, 31\n"
      "v_alignbit_b32 v25, v25, v167, 31\n"
      "v_max_u32_dpp v22, v22, v22 row_shr:4 row_mask:0xf bank_mask:0xf\n"
      "v_alignbit_b32 v25, v25, v168, 31\n"
      "v_alignbit_b32 v25, v25, v169, 31\n"
      "v_max_u32_dpp v22, v22, v22 row_shr:8 row_mask:0xf bank_mask:0xf\n"
      "v_alignbit_b32 v25, v25, v170, 31\n"
      "v_alignbit_b32 v25, v25, v171, 31\n"
      "v_max_u32_dpp v22, v22, v22 row_bcast:15 row_mask:0xa bank_mask:0xf\n"
      "v_alignbit_b32 v25, v25, v172, 31\n"
      "v_alignbit_b32 v25, v25, v173, 31\n"
      "v_max_u32_dpp v22, v22, v22 row_bcast:31 row_mask:0xc bank_mask:0xf\n"
      "v_alignbit_b32 v25, v25, v174, 31\n"
      "v_alignbit_b32 v25, v25, v175, 31\n"
      "v_readlane_b32 %[wm], v22, 63\n"
      "v_alignbit_b32 v25, v25, v176, 31\n"
      "v_alignbit_b32 v25, v25, v177, 31\n"
      "v_mov_b32 v23, %[wm]\n"
      "v_alignbit_b32 v25, v25, v178, 31\n"
      "v_alignbit_b32 v25, v25, v179, 31\n"
      "ds_write_b32 %[wa], v23\n"
      "v_alignbit_b32 v25, v25, v180, 31\n"
      "v_alignbit_b32 v25, v25, v181, 31\n"
      "v_alignbit_b32 v25, v25, v182, 31\n"
      "v_alignbit_b32 v25, v25, v183, 31\n"
      "v_alignbit_b32 v25, v25, v184, 31\n"
      "v_alignbit_b32 v25, v25, v185, 31\n"
      "v_alignbit_b32 v25, v25, v186, 31\n"
      "v_alignbit_b32 v25, v25, v187, 31\n"
      "v_alignbit_b32 v25, v25, v188, 31\n"
      "v_alignbit_b32 v25, v25, v189, 31\n"
      "v_alignbit_b32 v25, v25, v190, 31\n"
      "v_alignbit_b32 v25, v25, v191, 31\n"
      "v_lshrrev_b32 v26, 31, v192\n"
      "v_alignbit_b32 v26, v26, v193, 31\n"
      "v_alignbit_b32 v26, v26, v194, 31\n"
      "v_alignbit_b32 v26, v26, v195, 31\n"
      "v_alignbit_b32 v26, v26, v196, 31\n"
      "v_alignbit_b32 v26, v26, v197, 31\n"
      "v_alignbit_b32 v26, v26, v198, 31\n"
      "v_alignbit_b32 v26, v26, v199, 31\n"
      "v_alignbit_b32 v26, v26, v200, 31\n"
      "v_alignbit_b32 v26, v26, v201, 31\n"
      "v_alignbit_b32 v26, v26, v202, 31\n"
      "v_alignbit_b32 v26, v26, v203, 31\n"
      "v_alignbit_b32 v26, v26, v204, 31\n"
      "v_alignbit_b32 v26, v26, v205, 31\n"
      "v_alignbit_b32 v26, v26, v206, 31\n"
      "v_alignbit_b32 v26, v26, v207, 31\n"
      "v_alignbit_b32 v26, v26, v208, 31\n"
      "v_alignbit_b32 v26, v26, v209, 31\n"
      "v_alignbit_b32 v26, v26, v210, 31\n"
      "v_alignbit_b32 v26, v26, v211, 31\n"
      "v_alignbit_b32 v26, v26, v212, 31\n"
      "v_alignbit_b32 v26, v26, v213, 31\n"
      "v_alignbit_b32 v26, v26, v214, 31\n"
      "v_alignbit_b32 v26, v26, v215, 31\n"
      "v_alignbit_b32 v26, v26, v216, 31\n"
      "v_alignbit_b32 v26, v26, v217, 31\n"
      "v_alignbit_b32 v26, v26, v218, 31\n"
      "v_alignbit_b32 v26, v26, v219, 31\n"
      "v_alignbit_b32 v26, v26, v220, 31\n"
      "v_alignbit_b32 v26, v26, v221, 31\n"
      "v_alignbit_b32 v26, v26, v222, 31\n"
      "v_alignbit_b32 v26, v26, v223, 31\n"
      "v_lshrrev_b32 v27, 31, v224\n"
      "v_alignbit_b32 v27, v27, v225, 31\n"
      "v_alignbit_b32 v27, v27, v226, 31\n"
      "v_alignbit_b32 v27, v27, v227, 31\n"
      "v_alignbit_b32 v27, v27, v228, 31\n"
      "v_alignbit_b32 v27, v27, v229, 31\n"
      "v_alignbit_b32 v27, v27, v230, 31\n"
      "v_alignbit_b32 v27, v27, v231, 31\n"
      "v_alignbit_b32 v27, v27, v232, 31\n"
      "v_alignbit_b32 v27, v27, v233, 31\n"
      "v_alignbit_b32 v27, v27, v234, 31\n"
      "v_alignbit_b32 v27, v27, v235, 31\n"
      "v_alignbit_b32 v27, v27, v236, 31\n"
      "v_alignbit_b32 v27, v27, v237, 31\n"
      "v_alignbit_b32 v27, v27, v238, 31\n"
      "v_alignbit_b32 v27, v27, v239, 31\n"
      "v_alignbit_b32 v27, v27, v240, 31\n"
      "v_alignbit_b32 v27, v27, v241, 31\n"
      "v_alignbit_b32 v27, v27, v242, 31\n"
      "v_alignbit_b32 v27, v27, v243, 31\n"
      "v_alignbit_b32 v27, v27, v244, 31\n"
      "v_alignbit_b32 v27, v27, v245, 31\n"
      "v_alignbit_b32 v27, v27, v246, 31\n"
      "v_alignbit_b32 v27, v27, v247, 31\n"
      "v_alignbit_b32 v27, v27, v248, 31\n"
      "v_alignbit_b32 v27, v27, v249, 31\n"
      "v_alignbit_b32 v27, v27, v250, 31\n"
      "v_alignbit_b32 v27, v27, v251, 31\n"
      "v_alignbit_b32 v27, v27, v252, 31\n"
      "v_alignbit_b32 v27, v27, v253, 31\n"
      "v_alignbit_b32 v27, v27, v254, 31\n"
      "v_alignbit_b32 v27, v27, v255, 31\n"
      "s_waitcnt lgkmcnt(0)\n"
                : "=&{v[128:143]}"(c0), "=&{v[144:159]}"(c1), "=&{v[160:175]}"(c2), "=&{v[176:191]}"(c3),
                  "=&{v[192:207]}"(c4), "=&{v[208:223]}"(c5), "=&{v[224:239]}"(c6), "=&{v[240:255]}"(c7),
                  "=&{v[24:27]}"(d), [cl] "=&v"(call), [wm] "=&s"(wm), [i] "=&s"(si), [st] "=&s"(sst), [dA] "=&s"(sdA), [dB] "=&s"(sdB)
                : [a0] "v"(a0), [a1] "v"(a1), [b0] "v"(b0), [b1] "v"(b1), [b2] "v"(b2), [b3] "v"(b3), [ca] "v"(lds0 + F_MISC + 24), [wa] "v"(lds0 + F_MISC + wid * 4),
                  [cm] "s"(0xffff8000u), [cp] "s"(16384u)
                : "v16", "v17", "v18", "v19", "v20", "v21", "v22", "v23", "v28", "v29", "v30", "v31", "v32", "v33", "v34", "v35", "v36", "v37", "v38", "v39", "v40", "v41", "v42", "v43", "v44", "v45", "v46", "v47", "v48", "v49", "v50", "v51", "v52", "v53", "v54", "v55", "v56", "v57", "v58", "v59", "v60", "v61", "v62", "v63", "v64", "v65", "v66", "v67", "v68", "v69", "v70", "v71", "v72", "v73", "v74", "v75", "v76", "v77", "v78", "v79", "v80", "v81", "v82", "v83", "v84", "v85", "v86", "v87", "v88", "v89", "v90", "v91", "v92", "v93", "v94", "v95", "v96", "v97", "v98", "v99", "v100", "v101", "v102", "v103", "v104", "v105", "v106", "v107", "v108", "v109", "v110", "v111", "v112", "v113", "v114", "v115", "v116", "v117", "v118", "v119", "v120", "v121", "v122", "v123", "v124", "v125", "v126", "v127", "vcc", "scc", "memory");
        }
        if (wid == 0) STAMP(2);
        u32 tz = threadIdx.x; asm volatile("" : "+v"(tz));
        const u32 lane = tz & 63;
        call = rfl(call);
        unsigned char* const dcl0 = pws + WS_CLS + (size_t)(((ssrc * 2 + (call & 1)) * 32 + idx) * 4 + wid) * CLS_SLOT + 1024;
        unsigned char* const dcls = dcl0 + lane * 16;
        const u32 sbase = lds0 + 512 + wid * ST_PITCH;
        const u32 cnt = (64 - __builtin_popcount(d.x) - __builtin_popcount(d.y)) | (64 - __builtin_popcount(d.z) - __builtin_popcount(d.w)) << 16;
        const u32 inc = wave_scan(cnt), tot = __builtin_amdgcn_readlane(inc, 63), nA = tot & 0xffff, nB = tot >> 16;
        u32 A = sbase + ((inc - cnt) & 0xffff) * 4, B = sbase + ST_HALF + ((inc - cnt) >> 16) * 4;
        d ^= 0x55555555u;
        u32 zm = min(min(d.x, d.y), min(d.z, d.w));
        const u64 k0 = __ballot(lane * 32 < nA), k1 = __ballot(lane * 32 + 2048 < nA), k2 = __ballot(lane * 32 < nB), k3 = __ballot(lane * 32 + 2048 < nB);
        const u32 rot = ((lane >> 1) & 7) ^ ((lane >> 4) & 3);
        u32x16 e0, e1, o0, o1;
        const float top = __uint_as_float(max(wm, 15u << 23)), cs = 492.f * __builtin_amdgcn_rcpf(top);
        const u32 l0 = __float_as_uint(top * 0.504065f) - 32;
        const u32 ef = rfl(l0) + __builtin_ctzll(__ballot(__uint_as_float(l0 + lane) * cs >= 248.f));
        const f32x2 cs2 = {cs, cs};
        u32x2 fv = {ef, call & 0x3fffffffu}; u32 sn; u64 xm;
        if (wid == 0 && remote) STAMP(18);
        INJ(2)
#define ACC_IO "+{v[128:143]}"(c0), "+{v[144:159]}"(c1), "+{v[160:175]}"(c2), "+{v[176:191]}"(c3), "+{v[192:207]}"(c4), "+{v[208:223]}"(c5), "+{v[224:239]}"(c6), "+{v[240:255]}"(c7)
        asm volatile(
      "s_cmp_lg_u32 %[rem], 0\n"
      "s_cbranch_scc1 L_e1_%=\n"
      "v_max_f32 v128, 0, v128\n"
      "v_max_f32 v129, 0, v129\n"
      "v_max_f32 v130, 0, v130\n"
      "v_max_f32 v131, 0, v131\n"
      "v_max_f32 v132, 0, v132\n"
      "v_max_f32 v133, 0, v133\n"
      "v_max_f32 v134, 0, v134\n"
      "v_max_f32 v135, 0, v135\n"
      "v_max_f32 v136, 0, v136\n"
      "v_max_f32 v137, 0, v137\n"
      "v_max_f32 v138, 0, v138\n"
      "v_max_f32 v139, 0, v139\n"
      "v_max_f32 v140, 0, v140\n"
      "v_max_f32 v141, 0, v141\n"
      "v_max_f32 v142, 0, v142\n"
      "v_max_f32 v143, 0, v143\n"
      "v_max_f32 v144, 0, v144\n"
      "v_max_f32 v145, 0, v145\n"
      "v_max_f32 v146, 0, v146\n"
      "v_max_f32 v147, 0, v147\n"
      "v_max_f32 v148, 0, v148\n"
      "v_max_f32 v149, 0, v149\n"
      "v_max_f32 v150, 0, v150\n"
      "v_max_f32 v151, 0, v151\n"
      "v_max_f32 v152, 0, v152\n"
      "v_max_f32 v153, 0, v153\n"
      "v_max_f32 v154, 0, v154\n"
      "v_max_f32 v155, 0, v155\n"
      "v_max_f32 v156, 0, v156\n"
      "v_max_f32 v157, 0, v157\n"
      "v_max_f32 v158, 0, v158\n"
      "v_max_f32 v159, 0, v159\n"
      "v_max_f32 v160, 0, v160\n"
      "v_max_f32 v161, 0, v161\n"
      "v_max_f32 v162, 0, v162\n"
      "v_max_f32 v163, 0, v163\n"
      "v_max_f32 v164, 0, v164\n"
      "v_max_f32 v165, 0, v165\n"
      "v_max_f32 v166, 0, v166\n"
      "v_max_f32 v167, 0, v167\n"
      "v_max_f32 v168, 0, v168\n"
      "v_max_f32 v169, 0, v169\n"
      "v_max_f32 v170, 0, v170\n"
      "v_max_f32 v171, 0, v171\n"
      "v_max_f32 v172, 0, v172\n"
      "v_max_f32 v173, 0, v173\n"
      "v_max_f32 v174, 0, v174\n"
      "v_max_f32 v175, 0, v175\n"
      "v_max_f32 v176, 0, v176\n"
      "v_max_f32 v177, 0, v177\n"
      "v_max_f32 v178, 0, v178\n"
      "v_max_f32 v179, 0, v179\n"
      "v_max_f32 v180, 0, v180\n"
      "v_max_f32 v181, 0, v181\n"
      "v_max_f32 v182, 0, v182\n"
      "v_max_f32 v183, 0, v183\n"
      "v_max_f32 v184, 0, v184\n"
      "v_max_f32 v185, 0, v185\n"
      "v_max_f32 v186, 0, v186\n"
      "v_max_f32 v187, 0, v187\n"
      "v_max_f32 v188, 0, v188\n"
      "v_max_f32 v189, 0, v189\n"
      "v_max_f32 v190, 0, v190\n"
      "v_max_f32 v191, 0, v191\n"
      "v_max_f32 v192, 0, v192\n"
      "v_max_f32 v193, 0, v193\n"
      "v_max_f32 v194, 0, v194\n"
      "v_max_f32 v195, 0, v195\n"
      "v_max_f32 v196, 0, v196\n"
      "v_max_f32 v197, 0, v197\n"
      "v_max_f32 v198, 0, v198\n"
      "v_max_f32 v199, 0, v199\n"
      "v_max_f32 v200, 0, v200\n"
      "v_max_f32 v201, 0, v201\n"
      "v_max_f32 v202, 0, v202\n"
      "v_max_f32 v203, 0, v203\n"
      "v_max_f32 v204, 0, v204\n"
      "v_max_f32 v205, 0, v205\n"
      "v_max_f32 v206, 0, v206\n"
      "v_max_f32 v207, 0, v207\n"
      "v_max_f32 v208, 0, v208\n"
      "v_max_f32 v209, 0, v209\n"
      "v_max_f32 v210, 0, v210\n"
      "v_max_f32 v211, 0, v211\n"
      "v_max_f32 v212, 0, v212\n"
      "v_max_f32 v213, 0, v213\n"
      "v_max_f32 v214, 0, v214\n"
      "v_max_f32 v215, 0, v215\n"
      "v_max_f32 v216, 0, v216\n"
      "v_max_f32 v217, 0, v217\n"
      "v_max_f32 v218, 0, v218\n"
      "v_max_f32 v219, 0, v219\n"
      "v_max_f32 v220, 0, v220\n"
      "v_max_f32 v221, 0, v221\n"
      "v_max_f32 v222, 0, v222\n"
      "v_max_f32 v223, 0, v223\n"
      "v_max_f32 v224, 0, v224\n"
      "v_max_f32 v225, 0, v225\n"
      "v_max_f32 v226, 0, v226\n"
      "v_max_f32 v227, 0, v227\n"
      "v_max_f32 v228, 0, v228\n"
      "v_max_f32 v229, 0, v229\n"
      "v_max_f32 v230, 0, v230\n"
      "v_max_f32 v231, 0, v231\n"
      "v_max_f32 v232, 0, v232\n"
      "v_max_f32 v233, 0, v233\n"
      "v_max_f32 v234, 0, v234\n"
      "v_max_f32 v235, 0, v235\n"
      "v_max_f32 v236, 0, v236\n"
      "v_max_f32 v237, 0, v237\n"
      "v_max_f32 v238, 0, v238\n"
      "v_max_f32 v239, 0, v239\n"
      "v_max_f32 v240, 0, v240\n"
      "v_max_f32 v241, 0, v241\n"
      "v_max_f32 v242, 0, v242\n"
      "v_max_f32 v243, 0, v243\n"
      "v_max_f32 v244, 0, v244\n"
      "v_max_f32 v245, 0, v245\n"
      "v_max_f32 v246, 0, v246\n"
      "v_max_f32 v247, 0, v247\n"
      "v_max_f32 v248, 0, v248\n"
      "v_max_f32 v249, 0, v249\n"
      "v_max_f32 v250, 0, v250\n"
      "v_max_f32 v251, 0, v251\n"
      "v_max_f32 v252, 0, v252\n"
      "v_max_f32 v253, 0, v253\n"
      "v_max_f32 v254, 0, v254\n"
      "v_max_f32 v255, 0, v255\n"
      "s_mov_b64 exec, %[odd]\n"
      "v_swap_b32 v128, v132\n"
      "v_swap_b32 v129, v133\n"
      "v_swap_b32 v130, v134\n"
      "v_swap_b32 v131, v135\n"
      "v_swap_b32 v136, v140\n"
      "v_swap_b32 v137, v141\n"
      "v_swap_b32 v138, v142\n"
      "v_swap_b32 v139, v143\n"
      "v_swap_b32 v144, v148\n"
      "v_swap_b32 v145, v149\n"
      "v_swap_b32 v146, v150\n"
      "v_swap_b32 v147, v151\n"
      "v_swap_b32 v152, v156\n"
      "v_swap_b32 v153, v157\n"
      "v_swap_b32 v154, v158\n"
      "v_swap_b32 v155, v159\n"
      "v_swap_b32 v160, v164\n"
      "v_swap_b32 v161, v165\n"
      "v_swap_b32 v162, v166\n"
      "v_swap_b32 v163, v167\n"
      "v_swap_b32 v168, v172\n"
      "v_swap_b32 v169, v173\n"
      "v_swap_b32 v170, v174\n"
      "v_swap_b32 v171, v175\n"
      "v_swap_b32 v176, v180\n"
      "v_swap_b32 v177, v181\n"
      "v_swap_b32 v178, v182\n"
      "v_swap_b32 v179, v183\n"
      "v_swap_b32 v184, v188\n"
      "v_swap_b32 v185, v189\n"
      "v_swap_b32 v186, v190\n"
      "v_swap_b32 v187, v191\n"
      "v_swap_b32 v192, v196\n"
      "v_swap_b32 v193, v197\n"
      "v_swap_b32 v194, v198\n"
      "v_swap_b32 v195, v199\n"
      "v_swap_b32 v200, v204\n"
      "v_swap_b32 v201, v205\n"
      "v_swap_b32 v202, v206\n"
      "v_swap_b32 v203, v207\n"
      "v_swap_b32 v208, v212\n"
      "v_swap_b32 v209, v213\n"
      "v_swap_b32 v210, v214\n"
      "v_swap_b32 v211, v215\n"
      "v_swap_b32 v216, v220\n"
      "v_swap_b32 v217, v221\n"
      "v_swap_b32 v218, v222\n"
      "v_swap_b32 v219, v223\n"
      "v_swap_b32 v224, v228\n"
      "v_swap_b32 v225, v229\n"
      "v_swap_b32 v226, v230\n"
      "v_swap_b32 v227, v231\n"
      "v_swap_b32 v232, v236\n"
      "v_swap_b32 v233, v237\n"
      "v_swap_b32 v234, v238\n"
      "v_swap_b32 v235, v239\n"
      "v_swap_b32 v240, v244\n"
      "v_swap_b32 v241, v245\n"
      "v_swap_b32 v242, v246\n"
      "v_swap_b32 v243, v247\n"
      "v_swap_b32 v248, v252\n"
      "v_swap_b32 v249, v253\n"
      "v_swap_b32 v250, v254\n"
      "v_swap_b32 v251, v255\n"
      "s_mov_b64 exec, -1\n"
      "s_branch L_e0_%=\n"
      "L_e1_%=:\n"
      "global_store_dwordx4 %[dst], v[24:27], off offset:-1024 sc0 sc1\n"
      "ds_write_b32 %[pa], %[pz]\n"
      "v_ashrrev_i32 v28, 31, v128\n"
      "v_and_or_b32 v40, v128, %[sb], %[A]\n"
      "v_lshl_add_u32 %[A], v28, 2, %[A]\n"
      "v_ashrrev_i32 v29, 31, v129\n"
      "v_and_or_b32 v41, v129, %[sb], %[A]\n"
      "v_lshl_add_u32 %[A], v29, 2, %[A]\n"
      "v_ashrrev_i32 v28, 31, v130\n"
      "v_and_or_b32 v42, v130, %[sb], %[A]\n"
      "v_lshl_add_u32 %[A], v28, 2, %[A]\n"
      "v_ashrrev_i32 v29, 31, v131\n"
      "v_and_or_b32 v43, v131, %[sb], %[A]\n"
      "v_lshl_add_u32 %[A], v29, 2, %[A]\n"
      "v_ashrrev_i32 v28, 31, v132\n"
      "v_and_or_b32 v44, v132, %[sb], %[A]\n"
      "v_lshl_add_u32 %[A], v28, 2, %[A]\n"
      "v_ashrrev_i32 v29, 31, v133\n"
      "v_and_or_b32 v45, v133, %[sb], %[A]\n"
      "v_lshl_add_u32 %[A], v29, 2, %[A]\n"
      "v_ashrrev_i32 v28, 31, v134\n"
      "v_and_or_b32 v46, v134, %[sb], %[A]\n"
      "v_lshl_add_u32 %[A], v28, 2, %[A]\n"
      "v_ashrrev_i32 v29, 31, v135\n"
      "v_and_or_b32 v47, v135, %[sb], %[A]\n"
      "v_lshl_add_u32 %[A], v29, 2, %[A]\n"
      "ds_write_b32 v40, v128 offset:0\n"
      "ds_write_b32 v41, v129 offset:4\n"
      "ds_write_b32 v42, v130 offset:8\n"
      "ds_write_b32 v43, v131 offset:12\n"
      "ds_write_b32 v44, v132 offset:16\n"
      "ds_write_b32 v45, v133 offset:20\n"
      "ds_write_b32 v46, v134 offset:24\n"
      "ds_write_b32 v47, v135 offset:28\n"
      "v_ashrrev_i32 v28, 31, v136\n"
      "v_and_or_b32 v40, v136, %[sb], %[A]\n"
      "v_lshl_add_u32 %[A], v28, 2, %[A]\n"
      "v_ashrrev_i32 v29, 31, v137\n"
      "v_and_or_b32 v41, v137, %[sb], %[A]\n"
      "v_lshl_add_u32 %[A], v29, 2, %[A]\n"
      "v_ashrrev_i32 v28, 31, v138\n"
      "v_and_or_b32 v42, v138, %[sb], %[A]\n"
      "v_lshl_add_u32 %[A], v28, 2, %[A]\n"
      "v_ashrrev_i32 v29, 31, v139\n"
      "v_and_or_b32 v43, v139, %[sb], %[A]\n"
      "v_lshl_add_u32 %[A], v29, 2, %[A]\n"
      "v_ashrrev_i32 v28, 31, v140\n"
      "v_and_or_b32 v44, v140, %[sb], %[A]\n"
      "v_lshl_add_u32 %[A], v28, 2, %[A]\n"
      "v_ashrrev_i32 v29, 31, v141\n"
      "v_and_or_b32 v45, v141, %[sb], %[A]\n"
      "v_lshl_add_u32 %[A], v29, 2, %[A]\n"
      "v_ashrrev_i32 v28, 31, v142\n"
      "v_and_or_b32 v46, v142, %[sb], %[A]\n"
      "v_lshl_add_u32 %[A], v28, 2, %[A]\n"
      "v_ashrrev_i32 v29, 31, v143\n"
      "v_and_or_b32 v47, v143, %[sb], %[A]\n"
      "v_lshl_add_u32 %[A], v29, 2, %[A]\n"
      "ds_write_b32 v40, v136 offset:32\n"
      "ds_write_b32 v41, v137 offset:36\n"
      "ds_write_b32 v42, v138 offset:40\n"
      "ds_write_b32 v43, v139 offset:44\n"
      "ds_write_b32 v44, v140 offset:48\n"
      "ds_write_b32 v45, v141 offset:52\n"
      "ds_write_b32 v46, v142 offset:56\n"
      "ds_write_b32 v47, v143 offset:60\n"
      "v_ashrrev_i32 v28, 31, v144\n"
      "v_and_or_b32 v40, v144, %[sb], %[A]\n"
      "v_lshl_add_u32 %[A], v28, 2, %[A]\n"
      "v_ashrrev_i32 v29, 31, v145\n"
      "v_and_or_b32 v41, v145, %[sb], %[A]\n"
      "v_lshl_add_u32 %[A], v29, 2, %[A]\n"
      "v_ashrrev_i32 v28, 31, v146\n"
      "v_and_or_b32 v42, v146, %[sb], %[A]\n"
      "v_lshl_add_u32 %[A], v28, 2, %[A]\n"
      "v_ashrrev_i32 v29, 31, v147\n"
      "v_and_or_b32 v43, v147, %[sb], %[A]\n"
      "v_lshl_add_u32 %[A], v29, 2, %[A]\n"
      "v_ashrrev_i32 v28, 31, v148\n"
      "v_and_or_b32 v44, v148, %[sb], %[A]\n"
      "v_lshl_add_u32 %[A], v28, 2, %[A]\n"
      "v_ashrrev_i32 v29, 31, v149\n"
      "v_and_or_b32 v45, v149, %[sb], %[A]\n"
      "v_lshl_add_u32 %[A], v29, 2, %[A]\n"
      "v_ashrrev_i32 v28, 31, v150\n"
      "v_and_or_b32 v46, v150, %[sb], %[A]\n"
      "v_lshl_add_u32 %[A], v28, 2, %[A]\n"
      "v_ashrrev_i32 v29, 31, v151\n"
      "v_and_or_b32 v47, v151, %[sb], %[A]\n"
      "v_lshl_add_u32 %[A], v29, 2, %[A]\n"
      "ds_write_b32 v40, v144 offset:64\n"
      "ds_write_b32 v41, v145 offset:68\n"
      "ds_write_b32 v42, v146 offset:72\n"
      "ds_write_b32 v43, v147 offset:76\n"
      "ds_write_b32 v44, v148 offset:80\n"
      "ds_write_b32 v45, v149 offset:84\n"
      "ds_write_b32 v46, v150 offset:88\n"
      "ds_write_b32 v47, v151 offset:92\n"
      "v_ashrrev_i32 v28, 31, v152\n"
      "v_and_or_b32 v40, v152, %[sb], %[A]\n"
      "v_lshl_add_u32 %[A], v28, 2, %[A]\n"
      "v_ashrrev_i32 v29, 31, v153\n"
      "v_and_or_b32 v41, v153, %[sb], %[A]\n"
      "v_lshl_add_u32 %[A], v29, 2, %[A]\n"
      "v_ashrrev_i32 v28, 31, v154\n"
      "v_and_or_b32 v42, v154, %[sb], %[A]\n"
      "v_lshl_add_u32 %[A], v28, 2, %[A]\n"
      "v_ashrrev_i32 v29, 31, v155\n"
      "v_and_or_b32 v43, v155, %[sb], %[A]\n"
      "v_lshl_add_u32 %[A], v29, 2, %[A]\n"
      "v_ashrrev_i32 v28, 31, v156\n"
      "v_and_or_b32 v44, v156, %[sb], %[A]\n"
      "v_lshl_add_u32 %[A], v28, 2, %[A]\n"
      "v_ashrrev_i32 v29, 31, v157\n"
      "v_and_or_b32 v45, v157, %[sb], %[A]\n"
      "v_lshl_add_u32 %[A], v29, 2, %[A]\n"
      "v_ashrrev_i32 v28, 31, v158\n"
      "v_and_or_b32 v46, v158, %[sb], %[A]\n"
      "v_lshl_add_u32 %[A], v28, 2, %[A]\n"
      "v_ashrrev_i32 v29, 31, v159\n"
      "v_and_or_b32 v47, v159, %[sb], %[A]\n"
      "v_lshl_add_u32 %[A], v29, 2, %[A]\n"
      "ds_write_b32 v40, v152 offset:96\n"
      "ds_write_b32 v41, v153 offset:100\n"
      "ds_write_b32 v42, v154 offset:104\n"
      "ds_write_b32 v43, v155 offset:108\n"
      "ds_write_b32 v44, v156 offset:112\n"
      "ds_write_b32 v45, v157 offset:116\n"
      "ds_write_b32 v46, v158 offset:120\n"
      "ds_write_b32 v47, v159 offset:124\n"
      "v_ashrrev_i32 v28, 31, v160\n"
      "v_and_or_b32 v40, v160, %[sb], %[A]\n"
      "v_lshl_add_u32 %[A], v28, 2, %[A]\n"
      "v_ashrrev_i32 v29, 31, v161\n"
      "v_and_or_b32 v41, v161, %[sb], %[A]\n"
      "v_lshl_add_u32 %[A], v29, 2, %[A]\n"
      "v_ashrrev_i32 v28, 31, v162\n"
      "v_and_or_b32 v42, v162, %[sb], %[A]\n"
      "v_lshl_add_u32 %[A], v28, 2, %[A]\n"
      "v_ashrrev_i32 v29, 31, v163\n"
      "v_and_or_b32 v43, v163, %[sb], %[A]\n"
      "v_lshl_add_u32 %[A], v29, 2, %[A]\n"
      "v_ashrrev_i32 v28, 31, v164\n"
      "v_and_or_b32 v44, v164, %[sb], %[A]\n"
      "v_lshl_add_u32 %[A], v28, 2, %[A]\n"
      "v_ashrrev_i32 v29, 31, v165\n"
      "v_and_or_b32 v45, v165, %[sb], %[A]\n"
      "v_lshl_add_u32 %[A], v29, 2, %[A]\n"
      "v_ashrrev_i32 v28, 31, v166\n"
      "v_and_or_b32 v46, v166, %[sb], %[A]\n"
      "v_lshl_add_u32 %[A], v28, 2, %[A]\n"
      "v_ashrrev_i32 v29, 31, v167\n"
      "v_and_or_b32 v47, v167, %[sb], %[A]\n"
      "v_lshl_add_u32 %[A], v29, 2, %[A]\n"
      "ds_write_b32 v40, v160 offset:128\n"
      "ds_write_b32 v41, v161 offset:132\n"
      "ds_write_b32 v42, v162 offset:136\n"
      "ds_write_b32 v43, v163 offset:140\n"
      "ds_write_b32 v44, v164 offset:144\n"
      "ds_write_b32 v45, v165 offset:148\n"
      "ds_write_b32 v46, v166 offset:152\n"
      "ds_write_b32 v47, v167 offset:156\n"
      "v_ashrrev_i32 v28, 31, v168\n"
      "v_and_or_b32 v40, v168, %[sb], %[A]\n"
      "v_lshl_add_u32 %[A], v28, 2, %[A]\n"
      "v_ashrrev_i32 v29, 31, v169\n"
      "v_and_or_b32 v41, v169, %[sb], %[A]\n"
      "v_lshl_add_u32 %[A], v29, 2, %[A]\n"
      "v_ashrrev_i32 v28, 31, v170\n"
      "v_and_or_b32 v42, v170, %[sb], %[A]\n"
      "v_lshl_add_u32 %[A], v28, 2, %[A]\n"
      "v_ashrrev_i32 v29, 31, v171\n"
      "v_and_or_b32 v43, v171, %[sb], %[A]\n"
      "v_lshl_add_u32 %[A], v29, 2, %[A]\n"
      "v_ashrrev_i32 v28, 31, v172\n"
      "v_and_or_b32 v44, v172, %[sb], %[A]\n"
      "v_lshl_add_u32 %[A], v28, 2, %[A]\n"
      "v_ashrrev_i32 v29, 31, v173\n"
      "v_and_or_b32 v45, v173, %[sb], %[A]\n"
      "v_lshl_add_u32 %[A], v29, 2, %[A]\n"
      "v_ashrrev_i32 v28, 31, v174\n"
      "v_and_or_b32 v46, v174, %[sb], %[A]\n"
      "v_lshl_add_u32 %[A], v28, 2, %[A]\n"
      "v_ashrrev_i32 v29, 31, v175\n"
      "v_and_or_b32 v47, v175, %[sb], %[A]\n"
      "v_lshl_add_u32 %[A], v29, 2, %[A]\n"
      "ds_write_b32 v40, v168 offset:160\n"
      "ds_write_b32 v41, v169 offset:164\n"
      "ds_write_b32 v42, v170 offset:168\n"
      "ds_write_b32 v43, v171 offset:172\n"
      "ds_write_b32 v44, v172 offset:176\n"
      "ds_write_b32 v45, v173 offset:180\n"
      "ds_write_b32 v46, v174 offset:184\n"
      "ds_write_b32 v47, v175 offset:188\n"
      "v_ashrrev_i32 v28, 31, v176\n"
      "v_and_or_b32 v40, v176, %[sb], %[A]\n"
      "v_lshl_add_u32 %[A], v28, 2, %[A]\n"
      "v_ashrrev_i32 v29, 31, v177\n"
      "v_and_or_b32 v41, v177, %[sb], %[A]\n"
      "v_lshl_add_u32 %[A], v29, 2, %[A]\n"
      "v_ashrrev_i32 v28, 31, v178\n"
      "v_and_or_b32 v42, v178, %[sb], %[A]\n"
      "v_lshl_add_u32 %[A], v28, 2, %[A]\n"
      "v_ashrrev_i32 v29, 31, v179\n"
      "v_and_or_b32 v43, v179, %[sb], %[A]\n"
      "v_lshl_add_u32 %[A], v29, 2, %[A]\n"
      "v_ashrrev_i32 v28, 31, v180\n"
      "v_and_or_b32 v44, v180, %[sb], %[A]\n"
      "v_lshl_add_u32 %[A], v28, 2, %[A]\n"
      "v_ashrrev_i32 v29, 31, v181\n"
      "v_and_or_b32 v45, v181, %[sb], %[A]\n"
      "v_lshl_add_u32 %[A], v29, 2, %[A]\n"
      "v_ashrrev_i32 v28, 31, v182\n"
      "v_and_or_b32 v46, v182, %[sb], %[A]\n"
      "v_lshl_add_u32 %[A], v28, 2, %[A]\n"
      "v_ashrrev_i32 v29, 31, v183\n"
      "v_and_or_b32 v47, v183, %[sb], %[A]\n"
      "v_lshl_add_u32 %[A], v29, 2, %[A]\n"
      "ds_write_b32 v40, v176 offset:192\n"
      "ds_write_b32 v41, v177 offset:196\n"
      "ds_write_b32 v42, v178 offset:200\n"
      "ds_write_b32 v43, v179 offset:204\n"
      "ds_write_b32 v44, v180 offset:208\n"
      "ds_write_b32 v45, v181 offset:212\n"
      "ds_write_b32 v46, v182 offset:216\n"
      "ds_write_b32 v47, v183 offset:220\n"
      "v_ashrrev_i32 v28, 31, v184\n"
      "v_and_or_b32 v40, v184, %[sb], %[A]\n"
      "v_lshl_add_u32 %[A], v28, 2, %[A]\n"
      "v_ashrrev_i32 v29, 31, v185\n"
      "v_and_or_b32 v41, v185, %[sb], %[A]\n"
      "v_lshl_add_u32 %[A], v29, 2, %[A]\n"
      "v_ashrrev_i32 v28, 31, v186\n"
      "v_and_or_b32 v42, v186, %[sb], %[A]\n"
      "v_lshl_add_u32 %[A], v28, 2, %[A]\n"
      "v_ashrrev_i32 v29, 31, v187\n"
      "v_and_or_b32 v43, v187, %[sb], %[A]\n"
      "v_lshl_add_u32 %[A], v29, 2, %[A]\n"
      "v_ashrrev_i32 v28, 31, v188\n"
      "v_and_or_b32 v44, v188, %[sb], %[A]\n"
      "v_lshl_add_u32 %[A], v28, 2, %[A]\n"
      "v_ashrrev_i32 v29, 31, v189\n"
      "v_and_or_b32 v45, v189, %[sb], %[A]\n"
      "v_lshl_add_u32 %[A], v29, 2, %[A]\n"
      "v_ashrrev_i32 v28, 31, v190\n"
      "v_and_or_b32 v46, v190, %[sb], %[A]\n"
      "v_lshl_add_u32 %[A], v28, 2, %[A]\n"
      "v_ashrrev_i32 v29, 31, v191\n"
      "v_and_or_b32 v47, v191, %[sb], %[A]\n"
      "v_lshl_add_u32 %[A], v29, 2, %[A]\n"
      "ds_write_b32 v40, v184 offset:224\n"
      "ds_write_b32 v41, v185 offset:228\n"
      "ds_write_b32 v42, v186 offset:232\n"
      "ds_write_b32 v43, v187 offset:236\n"
      "ds_write_b32 v44, v188 offset:240\n"
      "ds_write_b32 v45, v189 offset:244\n"
      "ds_write_b32 v46, v190 offset:248\n"
      "ds_write_b32 v47, v191 offset:252\n"
      "v_mov_b32 v32, %[la]\n"
      "v_xor_b32 v33, 16, %[la]\n"
      "v_xor_b32 v34, 32, %[la]\n"
      "v_xor_b32 v35, 48, %[la]\n"
      "v_xor_b32 v36, 64, %[la]\n"
      "v_xor_b32 v37, 80, %[la]\n"
      "v_xor_b32 v38, 96, %[la]\n"
      "v_xor_b32 v39, 112, %[la]\n"
      "ds_read_b128 v[128:131], v32 offset:0\n"
      "ds_read_b128 v[132:135], v33 offset:0\n"
      "ds_read_b128 v[136:139], v34 offset:0\n"
      "ds_read_b128 v[140:143], v35 offset:0\n"
      "ds_read_b128 v[144:147], v36 offset:0\n"
      "ds_read_b128 v[148:151], v37 offset:0\n"
      "ds_read_b128 v[152:155], v38 offset:0\n"
      "ds_read_b128 v[156:159], v39 offset:0\n"
      "v_ashrrev_i32 v28, 31, v192\n"
      "v_and_or_b32 v40, v192, %[sb], %[B]\n"
      "v_lshl_add_u32 %[B], v28, 2, %[B]\n"
      "v_ashrrev_i32 v29, 31, v193\n"
      "v_and_or_b32 v41, v193, %[sb], %[B]\n"
      "v_lshl_add_u32 %[B], v29, 2, %[B]\n"
      "v_ashrrev_i32 v28, 31, v194\n"
      "v_and_or_b32 v42, v194, %[sb], %[B]\n"
      "v_lshl_add_u32 %[B], v28, 2, %[B]\n"
      "v_ashrrev_i32 v29, 31, v195\n"
      "v_and_or_b32 v43, v195, %[sb], %[B]\n"
      "v_lshl_add_u32 %[B], v29, 2, %[B]\n"
      "v_ashrrev_i32 v28, 31, v196\n"
      "v_and_or_b32 v44, v196, %[sb], %[B]\n"
      "v_lshl_add_u32 %[B], v28, 2, %[B]\n"
      "v_ashrrev_i32 v29, 31, v197\n"
      "v_and_or_b32 v45, v197, %[sb], %[B]\n"
      "v_lshl_add_u32 %[B], v29, 2, %[B]\n"
      "v_ashrrev_i32 v28, 31, v198\n"
      "v_and_or_b32 v46, v198, %[sb], %[B]\n"
      "v_lshl_add_u32 %[B], v28, 2, %[B]\n"
      "v_ashrrev_i32 v29, 31, v199\n"
      "v_and_or_b32 v47, v199, %[sb], %[B]\n"
      "v_lshl_add_u32 %[B], v29, 2, %[B]\n"
      "ds_write_b32 v40, v192 offset:0\n"
      "ds_write_b32 v41, v193 offset:4\n"
      "ds_write_b32 v42, v194 offset:8\n"
      "ds_write_b32 v43, v195 offset:12\n"
      "ds_write_b32 v44, v196 offset:16\n"
      "ds_write_b32 v45, v197 offset:20\n"
      "ds_write_b32 v46, v198 offset:24\n"
      "ds_write_b32 v47, v199 offset:28\n"
      "s_waitcnt lgkmcnt(8)\n"
      "v_pk_mul_f32 v[96:97], v[128:129], %[sc]\n"
      "v_pk_mul_f32 v[98:99], v[130:131], %[sc]\n"
      "v_pk_mul_f32 v[100:101], v[132:133], %[sc]\n"
      "v_pk_mul_f32 v[102:103], v[134:135], %[sc]\n"
      "v_pk_mul_f32 v[104:105], v[136:137], %[sc]\n"
      "v_pk_mul_f32 v[106:107], v[138:139], %[sc]\n"
      "v_pk_mul_f32 v[108:109], v[140:141], %[sc]\n"
      "v_pk_mul_f32 v[110:111], v[142:143], %[sc]\n"
      "v_pk_mul_f32 v[112:113], v[144:145], %[sc]\n"
      "v_pk_mul_f32 v[114:115], v[146:147], %[sc]\n"
      "v_pk_mul_f32 v[116:117], v[148:149], %[sc]\n"
      "v_pk_mul_f32 v[118:119], v[150:151], %[sc]\n"
      "v_pk_mul_f32 v[120:121], v[152:153], %[sc]\n"
      "v_pk_mul_f32 v[122:123], v[154:155], %[sc]\n"
      "v_pk_mul_f32 v[124:125], v[156:157], %[sc]\n"
      "v_pk_mul_f32 v[126:127], v[158:159], %[sc]\n"
      "v_cvt_pk_fp8_f32 v64, v96, v97\n"
      "v_cvt_pk_fp8_f32 v65, v100, v101\n"
      "v_cvt_pk_fp8_f32 v66, v104, v105\n"
      "v_cvt_pk_fp8_f32 v67, v108, v109\n"
      "v_cvt_pk_fp8_f32 v68, v112, v113\n"
      "v_cvt_pk_fp8_f32 v69, v116, v117\n"
      "v_cvt_pk_fp8_f32 v70, v120, v121\n"
      "v_cvt_pk_fp8_f32 v71, v124, v125\n"
      "v_cvt_pk_fp8_f32 v64, v98, v99 op_sel:[0,0,1]\n"
      "v_cvt_pk_fp8_f32 v65, v102, v103 op_sel:[0,0,1]\n"
      "v_cvt_pk_fp8_f32 v66, v106, v107 op_sel:[0,0,1]\n"
      "v_cvt_pk_fp8_f32 v67, v110, v111 op_sel:[0,0,1]\n"
      "v_cvt_pk_fp8_f32 v68, v114, v115 op_sel:[0,0,1]\n"
      "v_cvt_pk_fp8_f32 v69, v118, v119 op_sel:[0,0,1]\n"
      "v_cvt_pk_fp8_f32 v70, v122, v123 op_sel:[0,0,1]\n"
      "v_cvt_pk_fp8_f32 v71, v126, v127 op_sel:[0,0,1]\n"
      "v_lshrrev_b32 v48, 3, v64\n"
      "v_lshrrev_b32 v50, 3, v66\n"
      "v_lshrrev_b32 v52, 3, v68\n"
      "v_lshrrev_b32 v54, 3, v70\n"
      "v_lshlrev_b32 v49, 1, v65\n"
      "v_lshlrev_b32 v51, 1, v67\n"
      "v_lshlrev_b32 v53, 1, v69\n"
      "v_lshlrev_b32 v55, 1, v71\n"
      "v_bitop3_b32 v56, %[m0f], v48, v49 bitop3:0xc5\n"
      "v_bitop3_b32 v57, %[m0f], v50, v51 bitop3:0xc5\n"
      "v_bitop3_b32 v58, %[m0f], v52, v53 bitop3:0xc5\n"
      "v_bitop3_b32 v59, %[m0f], v54, v55 bitop3:0xc5\n"
      "v_mov_b32 v48, 0\n"
      "s_mov_b64 exec, %[k0]\n"
      "v_lshrrev_b32 v53, 3, v56\n"
      "v_lshrrev_b32 v54, 2, v57\n"
      "v_lshrrev_b32 v55, 1, v58\n"
      "v_lshlrev_b32 v50, 3, v59\n"
      "v_lshlrev_b32 v51, 2, v59\n"
      "v_lshlrev_b32 v52, 1, v59\n"
      "v_bfi_b32 v56, %[m8], v50, v56\n"
      "v_bfi_b32 v57, %[m8], v51, v57\n"
      "v_bfi_b32 v58, %[m8], v52, v58\n"
      "s_waitcnt vmcnt(0)\n"
      "global_store_dwordx3 %[d3], v[56:58], off offset:0 sc0 sc1\n"
      "v_min3_u32 %[zm], %[zm], v56, v57\n"
      "v_min_u32 %[zm], %[zm], v58\n"
      "v_bfi_b32 v55, %[m8], v59, v55\n"
      "v_bfi_b32 v54, %[mc], v55, v54\n"
      "v_bfi_b32 v53, %[mE], v54, v53\n"
      "v_xor_b32 v48, %[m0f], v53\n"
      "s_mov_b64 exec, -1\n"
      "v_bcnt_u32_b32 v49, v48, 0\n"
      "v_bcnt_u32_b32 v51, v48, 0\n"
      "s_nop 0\n"
      "v_add_u32_dpp v49, v49, v49 row_shr:1 row_mask:0xf bank_mask:0xf\n"
      "s_nop 1\n"
      "v_add_u32_dpp v49, v49, v49 row_shr:2 row_mask:0xf bank_mask:0xf\n"
      "s_nop 1\n"
      "v_add_u32_dpp v49, v49, v49 row_shr:4 row_mask:0xf bank_mask:0xf\n"
      "s_nop 1\n"
      "v_add_u32_dpp v49, v49, v49 row_shr:8 row_mask:0xf bank_mask:0xf\n"
      "s_nop 1\n"
      "v_add_u32_dpp v49, v49, v49 row_bcast:15 row_mask:0xa bank_mask:0xf\n"
      "s_nop 1\n"
      "v_add_u32_dpp v49, v49, v49 row_bcast:31 row_mask:0xc bank_mask:0xf\n"
      "s_nop 1\n"
      "v_readlane_b32 %[sn], v49, 63\n"
      "v_sub_u32 v50, v49, v51\n"
      "v_lshl_add_u32 v50, v50, 1, %[lh]\n"
      "s_cmp_gt_u32 %[sn], 95\n"
      "s_cbranch_scc1 L_d0_%=\n"
      "s_or_b32 vcc_lo, %[sn], 0x8000\n"
      "v_mov_b32 v52, vcc_lo\n"
      "s_mov_b64 exec, 1\n"
      "ds_write_b16 %[lb], v52 offset:0\n"
      "s_mov_b64 exec, -1\n"
      "s_add_u32 %[sn], %[sn], 2\n"
      "s_lshr_b32 %[sn], %[sn], 1\n"
      "s_bfm_b64 %[xm], %[sn], 0\n"
      "L_a0_%=:\n"
      "v_cmp_ne_u32 vcc, 0, v48\n"
      "s_cbranch_vccz L_b0_%=\n"
      "s_mov_b64 exec, vcc\n"
      "v_ffbl_b32 v51, v48\n"
      "v_add_u32 v52, -1, v48\n"
      "v_or_b32 v51, v51, %[pb]\n"
      "v_and_b32 v48, v48, v52\n"
      "ds_write_b16 v50, v51 offset:0\n"
      "v_add_u32 v50, 2, v50\n"
      "s_branch L_a0_%=\n"
      "L_b0_%=:\n"
      "s_mov_b64 exec, -1\n"
      "ds_read_b32 v52, %[lb] offset:0\n"
      "s_branch L_c0_%=\n"
      "L_d0_%=:\n"
      "s_mov_b64 %[xm], %[k0]\n"
      "v_xor_b32 v52, 0x55555555, v48\n"
      "v_or_b32 v31, 0x40000000, v31\n"
      "s_mov_b64 exec, %[k0]\n"
      "v_min_u32 %[zm], %[zm], v52\n"
      "s_mov_b64 exec, -1\n"
      "L_c0_%=:\n"
      "v_ashrrev_i32 v28, 31, v200\n"
      "v_and_or_b32 v40, v200, %[sb], %[B]\n"
      "v_lshl_add_u32 %[B], v28, 2, %[B]\n"
      "v_ashrrev_i32 v29, 31, v201\n"
      "v_and_or_b32 v41, v201, %[sb], %[B]\n"
      "v_lshl_add_u32 %[B], v29, 2, %[B]\n"
      "v_ashrrev_i32 v28, 31, v202\n"
      "v_and_or_b32 v42, v202, %[sb], %[B]\n"
      "v_lshl_add_u32 %[B], v28, 2, %[B]\n"
      "v_ashrrev_i32 v29, 31, v203\n"
      "v_and_or_b32 v43, v203, %[sb], %[B]\n"
      "v_lshl_add_u32 %[B], v29, 2, %[B]\n"
      "v_ashrrev_i32 v28, 31, v204\n"
      "v_and_or_b32 v44, v204, %[sb], %[B]\n"
      "v_lshl_add_u32 %[B], v28, 2, %[B]\n"
      "v_ashrrev_i32 v29, 31, v205\n"
      "v_and_or_b32 v45, v205, %[sb], %[B]\n"
      "v_lshl_add_u32 %[B], v29, 2, %[B]\n"
      "v_ashrrev_i32 v28, 31, v206\n"
      "v_and_or_b32 v46, v206, %[sb], %[B]\n"
      "v_lshl_add_u32 %[B], v28, 2, %[B]\n"
      "v_ashrrev_i32 v29, 31, v207\n"
      "v_and_or_b32 v47, v207, %[sb], %[B]\n"
      "v_lshl_add_u32 %[B], v29, 2, %[B]\n"
      "ds_write_b32 v40, v200 offset:32\n"
      "ds_write_b32 v41, v201 offset:36\n"
      "ds_write_b32 v42, v202 offset:40\n"
      "ds_write_b32 v43, v203 offset:44\n"
      "ds_write_b32 v44, v204 offset:48\n"
      "ds_write_b32 v45, v205 offset:52\n"
      "ds_write_b32 v46, v206 offset:56\n"
      "ds_write_b32 v47, v207 offset:60\n"
      "v_ashrrev_i32 v28, 31, v208\n"
      "v_and_or_b32 v40, v208, %[sb], %[B]\n"
      "v_lshl_add_u32 %[B], v28, 2, %[B]\n"
      "v_ashrrev_i32 v29, 31, v209\n"
      "v_and_or_b32 v41, v209, %[sb], %[B]\n"
      "v_lshl_add_u32 %[B], v29, 2, %[B]\n"
      "v_ashrrev_i32 v28, 31, v210\n"
      "v_and_or_b32 v42, v210, %[sb], %[B]\n"
      "v_lshl_add_u32 %[B], v28, 2, %[B]\n"
      "v_ashrrev_i32 v29, 31, v211\n"
      "v_and_or_b32 v43, v211, %[sb], %[B]\n"
      "v_lshl_add_u32 %[B], v29, 2, %[B]\n"
      "v_ashrrev_i32 v28, 31, v212\n"
      "v_and_or_b32 v44, v212, %[sb], %[B]\n"
      "v_lshl_add_u32 %[B], v28, 2, %[B]\n"
      "v_ashrrev_i32 v29, 31, v213\n"
      "v_and_or_b32 v45, v213, %[sb], %[B]\n"
      "v_lshl_add_u32 %[B], v29, 2, %[B]\n"
      "v_ashrrev_i32 v28, 31, v214\n"
      "v_and_or_b32 v46, v214, %[sb], %[B]\n"
      "v_lshl_add_u32 %[B], v28, 2, %[B]\n"
      "v_ashrrev_i32 v29, 31, v215\n"
      "v_and_or_b32 v47, v215, %[sb], %[B]\n"
      "v_lshl_add_u32 %[B], v29, 2, %[B]\n"
      "ds_write_b32 v40, v208 offset:64\n"
      "ds_write_b32 v41, v209 offset:68\n"
      "ds_write_b32 v42, v210 offset:72\n"
      "ds_write_b32 v43, v211 offset:76\n"
      "ds_write_b32 v44, v212 offset:80\n"
      "ds_write_b32 v45, v213 offset:84\n"
      "ds_write_b32 v46, v214 offset:88\n"
      "ds_write_b32 v47, v215 offset:92\n"
      "s_mov_b64 exec, %[xm]\n"
      "s_waitcnt lgkmcnt(15)\n"
      "global_store_dword %[lb], v52, %[s4] offset:768 sc0 sc1\n"
      "s_mov_b64 exec, -1\n"
      "v_ashrrev_i32 v28, 31, v216\n"
      "v_and_or_b32 v40, v216, %[sb], %[B]\n"
      "v_lshl_add_u32 %[B], v28, 2, %[B]\n"
      "v_ashrrev_i32 v29, 31, v217\n"
      "v_and_or_b32 v41, v217, %[sb], %[B]\n"
      "v_lshl_add_u32 %[B], v29, 2, %[B]\n"
      "v_ashrrev_i32 v28, 31, v218\n"
      "v_and_or_b32 v42, v218, %[sb], %[B]\n"
      "v_lshl_add_u32 %[B], v28, 2, %[B]\n"
      "v_ashrrev_i32 v29, 31, v219\n"
      "v_and_or_b32 v43, v219, %[sb], %[B]\n"
      "v_lshl_add_u32 %[B], v29, 2, %[B]\n"
      "v_ashrrev_i32 v28, 31, v220\n"
      "v_and_or_b32 v44, v220, %[sb], %[B]\n"
      "v_lshl_add_u32 %[B], v28, 2, %[B]\n"
      "v_ashrrev_i32 v29, 31, v221\n"
      "v_and_or_b32 v45, v221, %[sb], %[B]\n"
      "v_lshl_add_u32 %[B], v29, 2, %[B]\n"
      "v_ashrrev_i32 v28, 31, v222\n"
      "v_and_or_b32 v46, v222, %[sb], %[B]\n"
      "v_lshl_add_u32 %[B], v28, 2, %[B]\n"
      "v_ashrrev_i32 v29, 31, v223\n"
      "v_and_or_b32 v47, v223, %[sb], %[B]\n"
      "v_lshl_add_u32 %[B], v29, 2, %[B]\n"
      "ds_write_b32 v40, v216 offset:96\n"
      "ds_write_b32 v41, v217 offset:100\n"
      "ds_write_b32 v42, v218 offset:104\n"
      "ds_write_b32 v43, v219 offset:108\n"
      "ds_write_b32 v44, v220 offset:112\n"
      "ds_write_b32 v45, v221 offset:116\n"
      "ds_write_b32 v46, v222 offset:120\n"
      "ds_write_b32 v47, v223 offset:124\n"
      "v_ashrrev_i32 v28, 31, v224\n"
      "v_and_or_b32 v40, v224, %[sb], %[B]\n"
      "v_lshl_add_u32 %[B], v28, 2, %[B]\n"
      "v_ashrrev_i32 v29, 31, v225\n"
      "v_and_or_b32 v41, v225, %[sb], %[B]\n"
      "v_lshl_add_u32 %[B], v29, 2, %[B]\n"
      "v_ashrrev_i32 v28, 31, v226\n"
      "v_and_or_b32 v42, v226, %[sb], %[B]\n"
      "v_lshl_add_u32 %[B], v28, 2, %[B]\n"
      "v_ashrrev_i32 v29, 31, v227\n"
      "v_and_or_b32 v43, v227, %[sb], %[B]\n"
      "v_lshl_add_u32 %[B], v29, 2, %[B]\n"
      "v_ashrrev_i32 v28, 31, v228\n"
      "v_and_or_b32 v44, v228, %[sb], %[B]\n"
      "v_lshl_add_u32 %[B], v28, 2, %[B]\n"
      "v_ashrrev_i32 v29, 31, v229\n"
      "v_and_or_b32 v45, v229, %[sb], %[B]\n"
      "v_lshl_add_u32 %[B], v29, 2, %[B]\n"
      "v_ashrrev_i32 v28, 31, v230\n"
      "v_and_or_b32 v46, v230, %[sb], %[B]\n"
      "v_lshl_add_u32 %[B], v28, 2, %[B]\n"
      "v_ashrrev_i32 v29, 31, v231\n"
      "v_and_or_b32 v47, v231, %[sb], %[B]\n"
      "v_lshl_add_u32 %[B], v29, 2, %[B]\n"
      "ds_write_b32 v40, v224 offset:128\n"
      "ds_write_b32 v41, v225 offset:132\n"
      "ds_write_b32 v42, v226 offset:136\n"
      "ds_write_b32 v43, v227 offset:140\n"
      "ds_write_b32 v44, v228 offset:144\n"
      "ds_write_b32 v45, v229 offset:148\n"
      "ds_write_b32 v46, v230 offset:152\n"
      "ds_write_b32 v47, v231 offset:156\n"
      "v_ashrrev_i32 v28, 31, v232\n"
      "v_and_or_b32 v40, v232, %[sb], %[B]\n"
      "v_lshl_add_u32 %[B], v28, 2, %[B]\n"
      "v_ashrrev_i32 v29, 31, v233\n"
      "v_and_or_b32 v41, v233, %[sb], %[B]\n"
      "v_lshl_add_u32 %[B], v29, 2, %[B]\n"
      "v_ashrrev_i32 v28, 31, v234\n"
      "v_and_or_b32 v42, v234, %[sb], %[B]\n"
      "v_lshl_add_u32 %[B], v28, 2, %[B]\n"
      "v_ashrrev_i32 v29, 31, v235\n"
      "v_and_or_b32 v43, v235, %[sb], %[B]\n"
      "v_lshl_add_u32 %[B], v29, 2, %[B]\n"
      "v_ashrrev_i32 v28, 31, v236\n"
      "v_and_or_b32 v44, v236, %[sb], %[B]\n"
      "v_lshl_add_u32 %[B], v28, 2, %[B]\n"
      "v_ashrrev_i32 v29, 31, v237\n"
      "v_and_or_b32 v45, v237, %[sb], %[B]\n"
      "v_lshl_add_u32 %[B], v29, 2, %[B]\n"
      "v_ashrrev_i32 v28, 31, v238\n"
      "v_and_or_b32 v46, v238, %[sb], %[B]\n"
      "v_lshl_add_u32 %[B], v28, 2, %[B]\n"
      "v_ashrrev_i32 v29, 31, v239\n"
      "v_and_or_b32 v47, v239, %[sb], %[B]\n"
      "v_lshl_add_u32 %[B], v29, 2, %[B]\n"
      "ds_write_b32 v40, v232 offset:160\n"
      "ds_write_b32 v41, v233 offset:164\n"
      "ds_write_b32 v42, v234 offset:168\n"
      "ds_write_b32 v43, v235 offset:172\n"
      "ds_write_b32 v44, v236 offset:176\n"
      "ds_write_b32 v45, v237 offset:180\n"
      "ds_write_b32 v46, v238 offset:184\n"
      "ds_write_b32 v47, v239 offset:188\n"
      "v_ashrrev_i32 v28, 31, v240\n"
      "v_and_or_b32 v40, v240, %[sb], %[B]\n"
      "v_lshl_add_u32 %[B], v28, 2, %[B]\n"
      "v_ashrrev_i32 v29, 31, v241\n"
      "v_and_or_b32 v41, v241, %[sb], %[B]\n"
      "v_lshl_add_u32 %[B], v29, 2, %[B]\n"
      "v_ashrrev_i32 v28, 31, v242\n"
      "v_and_or_b32 v42, v242, %[sb], %[B]\n"
      "v_lshl_add_u32 %[B], v28, 2, %[B]\n"
      "v_ashrrev_i32 v29, 31, v243\n"
      "v_and_or_b32 v43, v243, %[sb], %[B]\n"
      "v_lshl_add_u32 %[B], v29, 2, %[B]\n"
      "v_ashrrev_i32 v28, 31, v244\n"
      "v_and_or_b32 v44, v244, %[sb], %[B]\n"
      "v_lshl_add_u32 %[B], v28, 2, %[B]\n"
      "v_ashrrev_i32 v29, 31, v245\n"
      "v_and_or_b32 v45, v245, %[sb], %[B]\n"
      "v_lshl_add_u32 %[B], v29, 2, %[B]\n"
      "v_ashrrev_i32 v28, 31, v246\n"
      "v_and_or_b32 v46, v246, %[sb], %[B]\n"
      "v_lshl_add_u32 %[B], v28, 2, %[B]\n"
      "v_ashrrev_i32 v29, 31, v247\n"
      "v_and_or_b32 v47, v247, %[sb], %[B]\n"
      "v_lshl_add_u32 %[B], v29, 2, %[B]\n"
      "ds_write_b32 v40, v240 offset:192\n"
      "ds_write_b32 v41, v241 offset:196\n"
      "ds_write_b32 v42, v242 offset:200\n"
      "ds_write_b32 v43, v243 offset:204\n"
      "ds_write_b32 v44, v244 offset:208\n"
      "ds_write_b32 v45, v245 offset:212\n"
      "ds_write_b32 v46, v246 offset:216\n"
      "ds_write_b32 v47, v247 offset:220\n"
      "v_ashrrev_i32 v28, 31, v248\n"
      "v_and_or_b32 v40, v248, %[sb], %[B]\n"
      "v_lshl_add_u32 %[B], v28, 2, %[B]\n"
      "v_ashrrev_i32 v29, 31, v249\n"
      "v_and_or_b32 v41, v249, %[sb], %[B]\n"
      "v_lshl_add_u32 %[B], v29, 2, %[B]\n"
      "v_ashrrev_i32 v28, 31, v250\n"
      "v_and_or_b32 v42, v250, %[sb], %[B]\n"
      "v_lshl_add_u32 %[B], v28, 2, %[B]\n"
      "v_ashrrev_i32 v29, 31, v251\n"
      "v_and_or_b32 v43, v251, %[sb], %[B]\n"
      "v_lshl_add_u32 %[B], v29, 2, %[B]\n"
      "v_ashrrev_i32 v28, 31, v252\n"
      "v_and_or_b32 v44, v252, %[sb], %[B]\n"
      "v_lshl_add_u32 %[B], v28, 2, %[B]\n"
      "v_ashrrev_i32 v29, 31, v253\n"
      "v_and_or_b32 v45, v253, %[sb], %[B]\n"
      "v_lshl_add_u32 %[B], v29, 2, %[B]\n"
      "v_ashrrev_i32 v28, 31, v254\n"
      "v_and_or_b32 v46, v254, %[sb], %[B]\n"
      "v_lshl_add_u32 %[B], v28, 2, %[B]\n"
      "v_ashrrev_i32 v29, 31, v255\n"
      "v_and_or_b32 v47, v255, %[sb], %[B]\n"
      "v_lshl_add_u32 %[B], v29, 2, %[B]\n"
      "ds_write_b32 v40, v248 offset:224\n"
      "ds_write_b32 v41, v249 offset:228\n"
      "ds_write_b32 v42, v250 offset:232\n"
      "ds_write_b32 v43, v251 offset:236\n"
      "ds_write_b32 v44, v252 offset:240\n"
      "ds_write_b32 v45, v253 offset:244\n"
      "ds_write_b32 v46, v254 offset:248\n"
      "ds_write_b32 v47, v255 offset:252\n"
      "ds_read_b128 v[192:195], v32 offset:16640\n"
      "ds_read_b128 v[196:199], v33 offset:16640\n"
      "ds_read_b128 v[200:203], v34 offset:16640\n"
      "ds_read_b128 v[204:207], v35 offset:16640\n"
      "ds_read_b128 v[208:211], v36 offset:16640\n"
      "ds_read_b128 v[212:215], v37 offset:16640\n"
      "ds_read_b128 v[216:219], v38 offset:16640\n"
      "ds_read_b128 v[220:223], v39 offset:16640\n"
      "s_cmp_gt_u32 %[nA], 2048\n"
      "s_cbranch_scc0 L_e2_%=\n"
      "ds_read_b128 v[160:163], v32 offset:8192\n"
      "ds_read_b128 v[164:167], v33 offset:8192\n"
      "ds_read_b128 v[168:171], v34 offset:8192\n"
      "ds_read_b128 v[172:175], v35 offset:8192\n"
      "ds_read_b128 v[176:179], v36 offset:8192\n"
      "ds_read_b128 v[180:183], v37 offset:8192\n"
      "ds_read_b128 v[184:187], v38 offset:8192\n"
      "ds_read_b128 v[188:191], v39 offset:8192\n"
      "s_waitcnt lgkmcnt(8)\n"
      "v_pk_mul_f32 v[96:97], v[192:193], %[sc]\n"
      "v_pk_mul_f32 v[98:99], v[194:195], %[sc]\n"
      "v_pk_mul_f32 v[100:101], v[196:197], %[sc]\n"
      "v_pk_mul_f32 v[102:103], v[198:199], %[sc]\n"
      "v_pk_mul_f32 v[104:105], v[200:201], %[sc]\n"
      "v_pk_mul_f32 v[106:107], v[202:203], %[sc]\n"
      "v_pk_mul_f32 v[108:109], v[204:205], %[sc]\n"
      "v_pk_mul_f32 v[110:111], v[206:207], %[sc]\n"
      "v_pk_mul_f32 v[112:113], v[208:209], %[sc]\n"
      "v_pk_mul_f32 v[114:115], v[210:211], %[sc]\n"
      "v_pk_mul_f32 v[116:117], v[212:213], %[sc]\n"
      "v_pk_mul_f32 v[118:119], v[214:215], %[sc]\n"
      "v_pk_mul_f32 v[120:121], v[216:217], %[sc]\n"
      "v_pk_mul_f32 v[122:123], v[218:219], %[sc]\n"
      "v_pk_mul_f32 v[124:125], v[220:221], %[sc]\n"
      "v_pk_mul_f32 v[126:127], v[222:223], %[sc]\n"
      "v_cvt_pk_fp8_f32 v80, v96, v97\n"
      "v_cvt_pk_fp8_f32 v81, v100, v101\n"
      "v_cvt_pk_fp8_f32 v82, v104, v105\n"
      "v_cvt_pk_fp8_f32 v83, v108, v109\n"
      "v_cvt_pk_fp8_f32 v84, v112, v113\n"
      "v_cvt_pk_fp8_f32 v85, v116, v117\n"
      "v_cvt_pk_fp8_f32 v86, v120, v121\n"
      "v_cvt_pk_fp8_f32 v87, v124, v125\n"
      "v_cvt_pk_fp8_f32 v80, v98, v99 op_sel:[0,0,1]\n"
      "v_cvt_pk_fp8_f32 v81, v102, v103 op_sel:[0,0,1]\n"
      "v_cvt_pk_fp8_f32 v82, v106, v107 op_sel:[0,0,1]\n"
      "v_cvt_pk_fp8_f32 v83, v110, v111 op_sel:[0,0,1]\n"
      "v_cvt_pk_fp8_f32 v84, v114, v115 op_sel:[0,0,1]\n"
      "v_cvt_pk_fp8_f32 v85, v118, v119 op_sel:[0,0,1]\n"
      "v_cvt_pk_fp8_f32 v86, v122, v123 op_sel:[0,0,1]\n"
      "v_cvt_pk_fp8_f32 v87, v126, v127 op_sel:[0,0,1]\n"
      "v_lshrrev_b32 v48, 3, v80\n"
      "v_lshrrev_b32 v50, 3, v82\n"
      "v_lshrrev_b32 v52, 3, v84\n"
      "v_lshrrev_b32 v54, 3, v86\n"
      "v_lshlrev_b32 v49, 1, v81\n"
      "v_lshlrev_b32 v51, 1, v83\n"
      "v_lshlrev_b32 v53, 1, v85\n"
      "v_lshlrev_b32 v55, 1, v87\n"
      "v_bitop3_b32 v60, %[m0f], v48, v49 bitop3:0xc5\n"
      "v_bitop3_b32 v61, %[m0f], v50, v51 bitop3:0xc5\n"
      "v_bitop3_b32 v62, %[m0f], v52, v53 bitop3:0xc5\n"
      "v_bitop3_b32 v63, %[m0f], v54, v55 bitop3:0xc5\n"
      "v_mov_b32 v48, 0\n"
      "s_mov_b64 exec, %[k2]\n"
      "v_lshrrev_b32 v53, 3, v60\n"
      "v_lshrrev_b32 v54, 2, v61\n"
      "v_lshrrev_b32 v55, 1, v62\n"
      "v_lshlrev_b32 v50, 3, v63\n"
      "v_lshlrev_b32 v51, 2, v63\n"
      "v_lshlrev_b32 v52, 1, v63\n"
      "v_bfi_b32 v60, %[m8], v50, v60\n"
      "v_bfi_b32 v61, %[m8], v51, v61\n"
      "v_bfi_b32 v62, %[m8], v52, v62\n"
      "s_waitcnt vmcnt(0)\n"
      "global_store_dwordx3 %[d3], v[60:62], off offset:2048 sc0 sc1\n"
      "v_min3_u32 %[zm], %[zm], v60, v61\n"
      "v_min_u32 %[zm], %[zm], v62\n"
      "v_bfi_b32 v55, %[m8], v63, v55\n"
      "v_bfi_b32 v54, %[mc], v55, v54\n"
      "v_bfi_b32 v53, %[mE], v54, v53\n"
      "v_xor_b32 v48, %[m0f], v53\n"
      "s_mov_b64 exec, -1\n"
      "v_bcnt_u32_b32 v49, v48, 0\n"
      "v_bcnt_u32_b32 v51, v48, 0\n"
      "s_nop 0\n"
      "v_add_u32_dpp v49, v49, v49 row_shr:1 row_mask:0xf bank_mask:0xf\n"
      "s_nop 1\n"
      "v_add_u32_dpp v49, v49, v49 row_shr:2 row_mask:0xf bank_mask:0xf\n"
      "s_nop 1\n"
      "v_add_u32_dpp v49, v49, v49 row_shr:4 row_mask:0xf bank_mask:0xf\n"
      "s_nop 1\n"
      "v_add_u32_dpp v49, v49, v49 row_shr:8 row_mask:0xf bank_mask:0xf\n"
      "s_nop 1\n"
      "v_add_u32_dpp v49, v49, v49 row_bcast:15 row_mask:0xa bank_mask:0xf\n"
      "s_nop 1\n"
      "v_add_u32_dpp v49, v49, v49 row_bcast:31 row_mask:0xc bank_mask:0xf\n"
      "s_nop 1\n"
      "v_readlane_b32 %[sn], v49, 63\n"
      "v_sub_u32 v50, v49, v51\n"
      "v_lshl_add_u32 v50, v50, 1, %[lh]\n"
      "s_cmp_gt_u32 %[sn], 95\n"
      "s_cbranch_scc1 L_d1_%=\n"
      "s_or_b32 vcc_lo, %[sn], 0x8000\n"
      "v_mov_b32 v52, vcc_lo\n"
      "s_mov_b64 exec, 1\n"
      "ds_write_b16 %[lb], v52 offset:16640\n"
      "s_mov_b64 exec, -1\n"
      "s_add_u32 %[sn], %[sn], 2\n"
      "s_lshr_b32 %[sn], %[sn], 1\n"
      "s_bfm_b64 %[xm], %[sn], 0\n"
      "L_a1_%=:\n"
      "v_cmp_ne_u32 vcc, 0, v48\n"
      "s_cbranch_vccz L_b1_%=\n"
      "s_mov_b64 exec, vcc\n"
      "v_ffbl_b32 v51, v48\n"
      "v_add_u32 v52, -1, v48\n"
      "v_or_b32 v51, v51, %[pb]\n"
      "v_and_b32 v48, v48, v52\n"
      "ds_write_b16 v50, v51 offset:16640\n"
      "v_add_u32 v50, 2, v50\n"
      "s_branch L_a1_%=\n"
      "L_b1_%=:\n"
      "s_mov_b64 exec, -1\n"
      "ds_read_b32 v52, %[lb] offset:16640\n"
      "s_branch L_c1_%=\n"
      "L_d1_%=:\n"
      "s_mov_b64 %[xm], %[k2]\n"
      "v_xor_b32 v52, 0x55555555, v48\n"
      "v_or_b32 v31, 0x80000000, v31\n"
      "s_mov_b64 exec, %[k2]\n"
      "v_min_u32 %[zm], %[zm], v52\n"
      "s_mov_b64 exec, -1\n"
      "L_c1_%=:\n"
      "s_mov_b64 exec, %[xm]\n"
      "s_waitcnt lgkmcnt(0)\n"
      "global_store_dword %[lb], v52, %[s4] offset:2816 sc0 sc1\n"
      "s_mov_b64 exec, -1\n"
      "s_waitcnt lgkmcnt(0)\n"
      "v_pk_mul_f32 v[96:97], v[160:161], %[sc]\n"
      "v_pk_mul_f32 v[98:99], v[162:163], %[sc]\n"
      "v_pk_mul_f32 v[100:101], v[164:165], %[sc]\n"
      "v_pk_mul_f32 v[102:103], v[166:167], %[sc]\n"
      "v_pk_mul_f32 v[104:105], v[168:169], %[sc]\n"
      "v_pk_mul_f32 v[106:107], v[170:171], %[sc]\n"
      "v_pk_mul_f32 v[108:109], v[172:173], %[sc]\n"
      "v_pk_mul_f32 v[110:111], v[174:175], %[sc]\n"
      "v_pk_mul_f32 v[112:113], v[176:177], %[sc]\n"
      "v_pk_mul_f32 v[114:115], v[178:179], %[sc]\n"
      "v_pk_mul_f32 v[116:117], v[180:181], %[sc]\n"
      "v_pk_mul_f32 v[118:119], v[182:183], %[sc]\n"
      "v_pk_mul_f32 v[120:121], v[184:185], %[sc]\n"
      "v_pk_mul_f32 v[122:123], v[186:187], %[sc]\n"
      "v_pk_mul_f32 v[124:125], v[188:189], %[sc]\n"
      "v_pk_mul_f32 v[126:127], v[190:191], %[sc]\n"
      "v_cvt_pk_fp8_f32 v72, v96, v97\n"
      "v_cvt_pk_fp8_f32 v73, v100, v101\n"
      "v_cvt_pk_fp8_f32 v74, v104, v105\n"
      "v_cvt_pk_fp8_f32 v75, v108, v109\n"
      "v_cvt_pk_fp8_f32 v76, v112, v113\n"
      "v_cvt_pk_fp8_f32 v77, v116, v117\n"
      "v_cvt_pk_fp8_f32 v78, v120, v121\n"
      "v_cvt_pk_fp8_f32 v79, v124, v125\n"
      "v_cvt_pk_fp8_f32 v72, v98, v99 op_sel:[0,0,1]\n"
      "v_cvt_pk_fp8_f32 v73, v102, v103 op_sel:[0,0,1]\n"
      "v_cvt_pk_fp8_f32 v74, v106, v107 op_sel:[0,0,1]\n"
      "v_cvt_pk_fp8_f32 v75, v110, v111 op_sel:[0,0,1]\n"
      "v_cvt_pk_fp8_f32 v76, v114, v115 op_sel:[0,0,1]\n"
      "v_cvt_pk_fp8_f32 v77, v118, v119 op_sel:[0,0,1]\n"
      "v_cvt_pk_fp8_f32 v78, v122, v123 op_sel:[0,0,1]\n"
      "v_cvt_pk_fp8_f32 v79, v126, v127 op_sel:[0,0,1]\n"
      "v_lshrrev_b32 v48, 3, v72\n"
      "v_lshrrev_b32 v50, 3, v74\n"
      "v_lshrrev_b32 v52, 3, v76\n"
      "v_lshrrev_b32 v54, 3, v78\n"
      "v_lshlrev_b32 v49, 1, v73\n"
      "v_lshlrev_b32 v51, 1, v75\n"
      "v_lshlrev_b32 v53, 1, v77\n"
      "v_lshlrev_b32 v55, 1, v79\n"
      "v_bitop3_b32 v56, %[m0f], v48, v49 bitop3:0xc5\n"
      "v_bitop3_b32 v57, %[m0f], v50, v51 bitop3:0xc5\n"
      "v_bitop3_b32 v58, %[m0f], v52, v53 bitop3:0xc5\n"
      "v_bitop3_b32 v59, %[m0f], v54, v55 bitop3:0xc5\n"
      "s_mov_b64 exec, %[k1]\n"
      "global_store_dwordx4 %[dst], v[56:59], off offset:1024 sc0 sc1\n"
      "v_min3_u32 %[zm], %[zm], v56, v57\n"
      "v_min3_u32 %[zm], %[zm], v58, v59\n"
      "s_mov_b64 exec, -1\n"
      "s_branch L_e3_%=\n"
      "L_e2_%=:\n"
      "s_waitcnt lgkmcnt(0)\n"
      "v_pk_mul_f32 v[96:97], v[192:193], %[sc]\n"
      "v_pk_mul_f32 v[98:99], v[194:195], %[sc]\n"
      "v_pk_mul_f32 v[100:101], v[196:197], %[sc]\n"
      "v_pk_mul_f32 v[102:103], v[198:199], %[sc]\n"
      "v_pk_mul_f32 v[104:105], v[200:201], %[sc]\n"
      "v_pk_mul_f32 v[106:107], v[202:203], %[sc]\n"
      "v_pk_mul_f32 v[108:109], v[204:205], %[sc]\n"
      "v_pk_mul_f32 v[110:111], v[206:207], %[sc]\n"
      "v_pk_mul_f32 v[112:113], v[208:209], %[sc]\n"
      "v_pk_mul_f32 v[114:115], v[210:211], %[sc]\n"
      "v_pk_mul_f32 v[116:117], v[212:213], %[sc]\n"
      "v_pk_mul_f32 v[118:119], v[214:215], %[sc]\n"
      "v_pk_mul_f32 v[120:121], v[216:217], %[sc]\n"
      "v_pk_mul_f32 v[122:123], v[218:219], %[sc]\n"
      "v_pk_mul_f32 v[124:125], v[220:221], %[sc]\n"
      "v_pk_mul_f32 v[126:127], v[222:223], %[sc]\n"
      "v_cvt_pk_fp8_f32 v80, v96, v97\n"
      "v_cvt_pk_fp8_f32 v81, v100, v101\n"
      "v_cvt_pk_fp8_f32 v82, v104, v105\n"
      "v_cvt_pk_fp8_f32 v83, v108, v109\n"
      "v_cvt_pk_fp8_f32 v84, v112, v113\n"
      "v_cvt_pk_fp8_f32 v85, v116, v117\n"
      "v_cvt_pk_fp8_f32 v86, v120, v121\n"
      "v_cvt_pk_fp8_f32 v87, v124, v125\n"
      "v_cvt_pk_fp8_f32 v80, v98, v99 op_sel:[0,0,1]\n"
      "v_cvt_pk_fp8_f32 v81, v102, v103 op_sel:[0,0,1]\n"
      "v_cvt_pk_fp8_f32 v82, v106, v107 op_sel:[0,0,1]\n"
      "v_cvt_pk_fp8_f32 v83, v110, v111 op_sel:[0,0,1]\n"
      "v_cvt_pk_fp8_f32 v84, v114, v115 op_sel:[0,0,1]\n"
      "v_cvt_pk_fp8_f32 v85, v118, v119 op_sel:[0,0,1]\n"
      "v_cvt_pk_fp8_f32 v86, v122, v123 op_sel:[0,0,1]\n"
      "v_cvt_pk_fp8_f32 v87, v126, v127 op_sel:[0,0,1]\n"
      "v_lshrrev_b32 v48, 3, v80\n"
      "v_lshrrev_b32 v50, 3, v82\n"
      "v_lshrrev_b32 v52, 3, v84\n"
      "v_lshrrev_b32 v54, 3, v86\n"
      "v_lshlrev_b32 v49, 1, v81\n"
      "v_lshlrev_b32 v51, 1, v83\n"
      "v_lshlrev_b32 v53, 1, v85\n"
      "v_lshlrev_b32 v55, 1, v87\n"
      "v_bitop3_b32 v60, %[m0f], v48, v49 bitop3:0xc5\n"
      "v_bitop3_b32 v61, %[m0f], v50, v51 bitop3:0xc5\n"
      "v_bitop3_b32 v62, %[m0f], v52, v53 bitop3:0xc5\n"
      "v_bitop3_b32 v63, %[m0f], v54, v55 bitop3:0xc5\n"
      "v_mov_b32 v48, 0\n"
      "s_mov_b64 exec, %[k2]\n"
      "v_lshrrev_b32 v53, 3, v60\n"
      "v_lshrrev_b32 v54, 2, v61\n"
      "v_lshrrev_b32 v55, 1, v62\n"
      "v_lshlrev_b32 v50, 3, v63\n"
      "v_lshlrev_b32 v51, 2, v63\n"
      "v_lshlrev_b32 v52, 1, v63\n"
      "v_bfi_b32 v60, %[m8], v50, v60\n"
      "v_bfi_b32 v61, %[m8], v51, v61\n"
      "v_bfi_b32 v62, %[m8], v52, v62\n"
      "s_waitcnt vmcnt(0)\n"
      "global_store_dwordx3 %[d3], v[60:62], off offset:2048 sc0 sc1\n"
      "v_min3_u32 %[zm], %[zm], v60, v61\n"
      "v_min_u32 %[zm], %[zm], v62\n"
      "v_bfi_b32 v55, %[m8], v63, v55\n"
      "v_bfi_b32 v54, %[mc], v55, v54\n"
      "v_bfi_b32 v53, %[mE], v54, v53\n"
      "v_xor_b32 v48, %[m0f], v53\n"
      "s_mov_b64 exec, -1\n"
      "v_bcnt_u32_b32 v49, v48, 0\n"
      "v_bcnt_u32_b32 v51, v48, 0\n"
      "s_nop 0\n"
      "v_add_u32_dpp v49, v49, v49 row_shr:1 row_mask:0xf bank_mask:0xf\n"
      "s_nop 1\n"
      "v_add_u32_dpp v49, v49, v49 row_shr:2 row_mask:0xf bank_mask:0xf\n"
      "s_nop 1\n"
      "v_add_u32_dpp v49, v49, v49 row_shr:4 row_mask:0xf bank_mask:0xf\n"
      "s_nop 1\n"
      "v_add_u32_dpp v49, v49, v49 row_shr:8 row_mask:0xf bank_mask:0xf\n"
      "s_nop 1\n"
      "v_add_u32_dpp v49, v49, v49 row_bcast:15 row_mask:0xa bank_mask:0xf\n"
      "s_nop 1\n"
      "v_add_u32_dpp v49, v49, v49 row_bcast:31 row_mask:0xc bank_mask:0xf\n"
      "s_nop 1\n"
      "v_readlane_b32 %[sn], v49, 63\n"
      "v_sub_u32 v50, v49, v51\n"
      "v_lshl_add_u32 v50, v50, 1, %[lh]\n"
      "s_cmp_gt_u32 %[sn], 95\n"
      "s_cbranch_scc1 L_d2_%=\n"
      "s_or_b32 vcc_lo, %[sn], 0x8000\n"
      "v_mov_b32 v52, vcc_lo\n"
      "s_mov_b64 exec, 1\n"
      "ds_write_b16 %[lb], v52 offset:16640\n"
      "s_mov_b64 exec, -1\n"
      "s_add_u32 %[sn], %[sn], 2\n"
      "s_lshr_b32 %[sn], %[sn], 1\n"
      "s_bfm_b64 %[xm], %[sn], 0\n"
      "L_a2_%=:\n"
      "v_cmp_ne_u32 vcc, 0, v48\n"
      "s_cbranch_vccz L_b2_%=\n"
      "s_mov_b64 exec, vcc\n"
      "v_ffbl_b32 v51, v48\n"
      "v_add_u32 v52, -1, v48\n"
      "v_or_b32 v51, v51, %[pb]\n"
      "v_and_b32 v48, v48, v52\n"
      "ds_write_b16 v50, v51 offset:16640\n"
      "v_add_u32 v50, 2, v50\n"
      "s_branch L_a2_%=\n"
      "L_b2_%=:\n"
      "s_mov_b64 exec, -1\n"
      "ds_read_b32 v52, %[lb] offset:16640\n"
      "s_branch L_c2_%=\n"
      "L_d2_%=:\n"
      "s_mov_b64 %[xm], %[k2]\n"
      "v_xor_b32 v52, 0x55555555, v48\n"
      "v_or_b32 v31, 0x80000000, v31\n"
      "s_mov_b64 exec, %[k2]\n"
      "v_min_u32 %[zm], %[zm], v52\n"
      "s_mov_b64 exec, -1\n"
      "L_c2_%=:\n"
      "s_mov_b64 exec, %[xm]\n"
      "s_waitcnt lgkmcnt(0)\n"
      "global_store_dword %[lb], v52, %[s4] offset:2816 sc0 sc1\n"
      "s_mov_b64 exec, -1\n"
      "L_e3_%=:\n"
      "s_cmp_gt_u32 %[nB], 2048\n"
      "s_cbranch_scc0 L_e9_%=\n"
      "ds_read_b128 v[224:227], v32 offset:24832\n"
      "ds_read_b128 v[228:231], v33 offset:24832\n"
      "ds_read_b128 v[232:235], v34 offset:24832\n"
      "ds_read_b128 v[236:239], v35 offset:24832\n"
      "ds_read_b128 v[240:243], v36 offset:24832\n"
      "ds_read_b128 v[244:247], v37 offset:24832\n"
      "ds_read_b128 v[248:251], v38 offset:24832\n"
      "ds_read_b128 v[252:255], v39 offset:24832\n"
      "s_waitcnt lgkmcnt(0)\n"
      "v_pk_mul_f32 v[96:97], v[224:225], %[sc]\n"
      "v_pk_mul_f32 v[98:99], v[226:227], %[sc]\n"
      "v_pk_mul_f32 v[100:101], v[228:229], %[sc]\n"
      "v_pk_mul_f32 v[102:103], v[230:231], %[sc]\n"
      "v_pk_mul_f32 v[104:105], v[232:233], %[sc]\n"
      "v_pk_mul_f32 v[106:107], v[234:235], %[sc]\n"
      "v_pk_mul_f32 v[108:109], v[236:237], %[sc]\n"
      "v_pk_mul_f32 v[110:111], v[238:239], %[sc]\n"
      "v_pk_mul_f32 v[112:113], v[240:241], %[sc]\n"
      "v_pk_mul_f32 v[114:115], v[242:243], %[sc]\n"
      "v_pk_mul_f32 v[116:117], v[244:245], %[sc]\n"
      "v_pk_mul_f32 v[118:119], v[246:247], %[sc]\n"
      "v_pk_mul_f32 v[120:121], v[248:249], %[sc]\n"
      "v_pk_mul_f32 v[122:123], v[250:251], %[sc]\n"
      "v_pk_mul_f32 v[124:125], v[252:253], %[sc]\n"
      "v_pk_mul_f32 v[126:127], v[254:255], %[sc]\n"
      "v_cvt_pk_fp8_f32 v88, v96, v97\n"
      "v_cvt_pk_fp8_f32 v89, v100, v101\n"
      "v_cvt_pk_fp8_f32 v90, v104, v105\n"
      "v_cvt_pk_fp8_f32 v91, v108, v109\n"
      "v_cvt_pk_fp8_f32 v92, v112, v113\n"
      "v_cvt_pk_fp8_f32 v93, v116, v117\n"
      "v_cvt_pk_fp8_f32 v94, v120, v121\n"
      "v_cvt_pk_fp8_f32 v95, v124, v125\n"
      "v_cvt_pk_fp8_f32 v88, v98, v99 op_sel:[0,0,1]\n"
      "v_cvt_pk_fp8_f32 v89, v102, v103 op_sel:[0,0,1]\n"
      "v_cvt_pk_fp8_f32 v90, v106, v107 op_sel:[0,0,1]\n"
      "v_cvt_pk_fp8_f32 v91, v110, v111 op_sel:[0,0,1]\n"
      "v_cvt_pk_fp8_f32 v92, v114, v115 op_sel:[0,0,1]\n"
      "v_cvt_pk_fp8_f32 v93, v118, v119 op_sel:[0,0,1]\n"
      "v_cvt_pk_fp8_f32 v94, v122, v123 op_sel:[0,0,1]\n"
      "v_cvt_pk_fp8_f32 v95, v126, v127 op_sel:[0,0,1]\n"
      "v_lshrrev_b32 v48, 3, v88\n"
      "v_lshrrev_b32 v50, 3, v90\n"
      "v_lshrrev_b32 v52, 3, v92\n"
      "v_lshrrev_b32 v54, 3, v94\n"
      "v_lshlrev_b32 v49, 1, v89\n"
      "v_lshlrev_b32 v51, 1, v91\n"
      "v_lshlrev_b32 v53, 1, v93\n"
      "v_lshlrev_b32 v55, 1, v95\n"
      "v_bitop3_b32 v60, %[m0f], v48, v49 bitop3:0xc5\n"
      "v_bitop3_b32 v61, %[m0f], v50, v51 bitop3:0xc5\n"
      "v_bitop3_b32 v62, %[m0f], v52, v53 bitop3:0xc5\n"
      "v_bitop3_b32 v63, %[m0f], v54, v55 bitop3:0xc5\n"
      "s_mov_b64 exec, %[k3]\n"
      "global_store_dwordx4 %[dst], v[60:63], off offset:3072 sc0 sc1\n"
      "v_min3_u32 %[zm], %[zm], v60, v61\n"
      "v_min3_u32 %[zm], %[zm], v62, v63\n"
      "s_mov_b64 exec, -1\n"
      "L_e9_%=:\n"
      "v_cmp_eq_u32 vcc, 0, %[zm]\n"
      "s_cbranch_vccz L_e7_%=\n"
      "s_waitcnt vmcnt(0)\n"
      "v_or_b32 v30, 0x80000000, v30\n"
      "L_e7_%=:\n"
      "s_mov_b64 exec, 1\n"
      "global_store_dwordx2 %[lb], v[30:31], %[fa] sc0 sc1\n"
      "s_mov_b64 exec, -1\n"
      "v_and_b32 v96, %[mo], v64\n"
      "v_and_b32 v64, %[me], v64\n"
      "v_and_b32 v97, %[mo], v65\n"
      "v_and_b32 v65, %[me], v65\n"
      "v_and_b32 v98, %[mo], v66\n"
      "v_and_b32 v66, %[me], v66\n"
      "v_and_b32 v99, %[mo], v67\n"
      "v_and_b32 v67, %[me], v67\n"
      "v_and_b32 v100, %[mo], v68\n"
      "v_and_b32 v68, %[me], v68\n"
      "v_and_b32 v101, %[mo], v69\n"
      "v_and_b32 v69, %[me], v69\n"
      "v_and_b32 v102, %[mo], v70\n"
      "v_and_b32 v70, %[me], v70\n"
      "v_and_b32 v103, %[mo], v71\n"
      "v_and_b32 v71, %[me], v71\n"
      "v_and_b32 v104, %[mo], v72\n"
      "v_and_b32 v72, %[me], v72\n"
      "v_and_b32 v105, %[mo], v73\n"
      "v_and_b32 v73, %[me], v73\n"
      "v_and_b32 v106, %[mo], v74\n"
      "v_and_b32 v74, %[me], v74\n"
      "v_and_b32 v107, %[mo], v75\n"
      "v_and_b32 v75, %[me], v75\n"
      "v_and_b32 v108, %[mo], v76\n"
      "v_and_b32 v76, %[me], v76\n"
      "v_and_b32 v109, %[mo], v77\n"
      "v_and_b32 v77, %[me], v77\n"
      "v_and_b32 v110, %[mo], v78\n"
      "v_and_b32 v78, %[me], v78\n"
      "v_and_b32 v111, %[mo], v79\n"
      "v_and_b32 v79, %[me], v79\n"
      "v_and_b32 v112, %[mo], v80\n"
      "v_and_b32 v80, %[me], v80\n"
      "v_and_b32 v113, %[mo], v81\n"
      "v_and_b32 v81, %[me], v81\n"
      "v_and_b32 v114, %[mo], v82\n"
      "v_and_b32 v82, %[me], v82\n"
      "v_and_b32 v115, %[mo], v83\n"
      "v_and_b32 v83, %[me], v83\n"
      "v_and_b32 v116, %[mo], v84\n"
      "v_and_b32 v84, %[me], v84\n"
      "v_and_b32 v117, %[mo], v85\n"
      "v_and_b32 v85, %[me], v85\n"
      "v_and_b32 v118, %[mo], v86\n"
      "v_and_b32 v86, %[me], v86\n"
      "v_and_b32 v119, %[mo], v87\n"
      "v_and_b32 v87, %[me], v87\n"
      "v_and_b32 v120, %[mo], v88\n"
      "v_and_b32 v88, %[me], v88\n"
      "v_and_b32 v121, %[mo], v89\n"
      "v_and_b32 v89, %[me], v89\n"
      "v_and_b32 v122, %[mo], v90\n"
      "v_and_b32 v90, %[me], v90\n"
      "v_and_b32 v123, %[mo], v91\n"
      "v_and_b32 v91, %[me], v91\n"
      "v_and_b32 v124, %[mo], v92\n"
      "v_and_b32 v92, %[me], v92\n"
      "v_and_b32 v125, %[mo], v93\n"
      "v_and_b32 v93, %[me], v93\n"
      "v_and_b32 v126, %[mo], v94\n"
      "v_and_b32 v94, %[me], v94\n"
      "v_and_b32 v127, %[mo], v95\n"
      "v_and_b32 v95, %[me], v95\n"
      "L_e0_%=:\n"
            : ACC_IO, "=&{v[64:79]}"(e0), "=&{v[80:95]}"(e1), "=&{v[96:111]}"(o0), "=&{v[112:127]}"(o1), [zm] "+&v"(zm), "+{v[30:31]}"(fv), [A] "+&v"(A), [B] "+&v"(B), [sn] "=&s"(sn), [xm] "=&s"(xm)
            : "{v[24:27]}"(d), [la] "v"(sbase + lane * 128 + rot * 16), [dst] "v"(dcls), [fa] "s"(pws + WS_FLAG + ((ssrc * 32 + idx) * 4 + wid) * 32 + (call & 1) * 16 - sbase), [nA] "s"(nA), [nB] "s"(nB), [rem] "s"(rfl(grp ^ p.rank)), [odd] "s"(0xffff0000ffff0000ull),
              [k0] "s"(k0), [k1] "s"(k1), [k2] "s"(k2), [k3] "s"(k3), [m0f] "s"(0x0f0f0f0fu), [me] "s"(0x00780078u), [mo] "s"(0x78007800u), [sb] "s"(0x80000000u),
              [pa] "v"(sbase + (lane < 32 ? nA + lane : ST_HALF / 4 + nB + lane - 32) * 4), [pz] "v"(top), [sc] "v"(cs2),
              [d3] "v"(dcls - lane * 4), [s4] "s"(dcl0 - sbase), [m8] "s"(0x88888888u), [mc] "s"(0xccccccccu), [mE] "s"(0xeeeeeeeeu),
              [lh] "s"(rfl(sbase + 2)), [lb] "v"(sbase + lane * 4), [pb] "v"(0x8000u | lane << 5)
            : "v28", "v29", "v32", "v33", "v34", "v35", "v36", "v37", "v38", "v39", "v40", "v41", "v42", "v43", "v44", "v45", "v46", "v47", "v48", "v49", "v50", "v51", "v52", "v53", "v54", "v55", "v56", "v57", "v58", "v59", "v60", "v61", "v62", "v63", "vcc", "scc", "memory");
        if (wid == 0 && remote) STAMP(5);
        lds_store(lds0 + F_MISC + 32 + wid * 4, 1);
        u32 iw = lds_word(lds0 + F_MISC + 16);
        if (wid == 0 && iw == NOTYET) {
            const u64 gs = (u64)(myws + WS_GMAX + (call & 1) * 8192 + (b & 31) * 256);
            const u64 t0 = __builtin_amdgcn_s_memrealtime();
            for (;;) {
                u32x16 g;
                asm volatile("s_load_dwordx16 %0, %2, 0x0 glc\n\tds_read_b32 %1, %3\n\ts_waitcnt lgkmcnt(0)"
                             : "=&s"(g), "=&v"(iw) : "s"(gs), "v"(lds0 + F_MISC + 16) : "memory");
                iw = rfl(iw);
                if (iw != NOTYET) break;
                u32 miss = 0, mx = 0;
#pragma unroll
                for (int r = 0; r < 8; ++r) { miss |= g[2 * r + 1] ^ call; mx = max(mx, g[2 * r]); }
                if (!miss) { iw = inv_scale(mx); lds_store(lds0 + F_MISC + 16, iw); break; }
                u32 t = 0;
                if ((long long)(__builtin_amdgcn_s_memrealtime() - t0) > 5000)
                    asm volatile("global_load_dword %0, %1, off sc1\n\ts_waitcnt vmcnt(0)" : "=&v"(t) : "v"(myws + WS_TMO) : "memory");
                if (expired(t0, t, call, myws + WS_TMO, lane)) { iw = 0x7fc00000u; lds_store(lds0 + F_MISC + 16, iw); break; }
            }
            STAMP(3);
        }
        while (iw == NOTYET) { __builtin_amdgcn_s_sleep(1); iw = lds_word(lds0 + F_MISC + 16); }
        const float inv = __uint_as_float(iw);
        if (wid == 0) STAMP(11);
        INJ(3)
        const u32 sh = class_shift(inv, ef);
        unsigned char* const dl = pws + WS_OFF + (size_t)((ssrc * 32 + idx) * 4 + wid) * OFF_SLOT + lane * 16;
        const u64 j0 = __ballot(lane < 48 ? lane * 128 < nA * 3 + 96 : lane < 52 && (lane - 48) * 512 < nA), j2 = __ballot(lane < 48 ? lane * 128 < nB * 3 + 96 : lane < 52 && (lane - 48) * 512 < nB);
        const f32x2 iv = {inv, inv};
        asm volatile(
      "s_cmp_eq_u32 %[md], 1\n"
      "s_cbranch_scc1 L_l1_%=\n"
      "s_cmp_eq_u32 %[md], 0\n"
      "s_cbranch_scc1 L_l9_%=\n"
      "v_pk_mul_f32 v[128:129], v[128:129], %[iv]\n"
      "v_pk_mul_f32 v[130:131], v[130:131], %[iv]\n"
      "v_pk_mul_f32 v[132:133], v[132:133], %[iv]\n"
      "v_pk_mul_f32 v[134:135], v[134:135], %[iv]\n"
      "v_pk_mul_f32 v[136:137], v[136:137], %[iv]\n"
      "v_pk_mul_f32 v[138:139], v[138:139], %[iv]\n"
      "v_pk_mul_f32 v[140:141], v[140:141], %[iv]\n"
      "v_pk_mul_f32 v[142:143], v[142:143], %[iv]\n"
      "v_cvt_pk_fp8_f32 v48, v128, v129\n"
      "v_cvt_pk_fp8_f32 v49, v132, v133\n"
      "v_cvt_pk_fp8_f32 v50, v136, v137\n"
      "v_cvt_pk_fp8_f32 v51, v140, v141\n"
      "v_cvt_pk_fp8_f32 v48, v130, v131 op_sel:[0,0,1]\n"
      "v_cvt_pk_fp8_f32 v49, v134, v135 op_sel:[0,0,1]\n"
      "v_cvt_pk_fp8_f32 v50, v138, v139 op_sel:[0,0,1]\n"
      "v_cvt_pk_fp8_f32 v51, v142, v143 op_sel:[0,0,1]\n"
      "v_add_u32 v32, 0x0, %[lo]\n"
      "global_store_dwordx4 v32, v[48:51], %[ro] sc0 sc1\n"
      "v_pk_mul_f32 v[144:145], v[144:145], %[iv]\n"
      "v_pk_mul_f32 v[146:147], v[146:147], %[iv]\n"
      "v_pk_mul_f32 v[148:149], v[148:149], %[iv]\n"
      "v_pk_mul_f32 v[150:151], v[150:151], %[iv]\n"
      "v_pk_mul_f32 v[152:153], v[152:153], %[iv]\n"
      "v_pk_mul_f32 v[154:155], v[154:155], %[iv]\n"
      "v_pk_mul_f32 v[156:157], v[156:157], %[iv]\n"
      "v_pk_mul_f32 v[158:159], v[158:159], %[iv]\n"
      "v_cvt_pk_fp8_f32 v52, v144, v145\n"
      "v_cvt_pk_fp8_f32 v53, v148, v149\n"
      "v_cvt_pk_fp8_f32 v54, v152, v153\n"
      "v_cvt_pk_fp8_f32 v55, v156, v157\n"
      "v_cvt_pk_fp8_f32 v52, v146, v147 op_sel:[0,0,1]\n"
      "v_cvt_pk_fp8_f32 v53, v150, v151 op_sel:[0,0,1]\n"
      "v_cvt_pk_fp8_f32 v54, v154, v155 op_sel:[0,0,1]\n"
      "v_cvt_pk_fp8_f32 v55, v158, v159 op_sel:[0,0,1]\n"
      "v_add_u32 v33, 0x8000, %[lo]\n"
      "global_store_dwordx4 v33, v[52:55], %[ro] sc0 sc1\n"
      "v_pk_mul_f32 v[160:161], v[160:161], %[iv]\n"
      "v_pk_mul_f32 v[162:163], v[162:163], %[iv]\n"
      "v_pk_mul_f32 v[164:165], v[164:165], %[iv]\n"
      "v_pk_mul_f32 v[166:167], v[166:167], %[iv]\n"
      "v_pk_mul_f32 v[168:169], v[168:169], %[iv]\n"
      "v_pk_mul_f32 v[170:171], v[170:171], %[iv]\n"
      "v_pk_mul_f32 v[172:173], v[172:173], %[iv]\n"
      "v_pk_mul_f32 v[174:175], v[174:175], %[iv]\n"
      "v_cvt_pk_fp8_f32 v56, v160, v161\n"
      "v_cvt_pk_fp8_f32 v57, v164, v165\n"
      "v_cvt_pk_fp8_f32 v58, v168, v169\n"
      "v_cvt_pk_fp8_f32 v59, v172, v173\n"
      "v_cvt_pk_fp8_f32 v56, v162, v163 op_sel:[0,0,1]\n"
      "v_cvt_pk_fp8_f32 v57, v166, v167 op_sel:[0,0,1]\n"
      "v_cvt_pk_fp8_f32 v58, v170, v171 op_sel:[0,0,1]\n"
      "v_cvt_pk_fp8_f32 v59, v174, v175 op_sel:[0,0,1]\n"
      "v_add_u32 v34, 0x10000, %[lo]\n"
      "global_store_dwordx4 v34, v[56:59], %[ro] sc0 sc1\n"
      "v_pk_mul_f32 v[176:177], v[176:177], %[iv]\n"
      "v_pk_mul_f32 v[178:179], v[178:179], %[iv]\n"
      "v_pk_mul_f32 v[180:181], v[180:181], %[iv]\n"
      "v_pk_mul_f32 v[182:183], v[182:183], %[iv]\n"
      "v_pk_mul_f32 v[184:185], v[184:185], %[iv]\n"
      "v_pk_mul_f32 v[186:187], v[186:187], %[iv]\n"
      "v_pk_mul_f32 v[188:189], v[188:189], %[iv]\n"
      "v_pk_mul_f32 v[190:191], v[190:191], %[iv]\n"
      "v_cvt_pk_fp8_f32 v60, v176, v177\n"
      "v_cvt_pk_fp8_f32 v61, v180, v181\n"
      "v_cvt_pk_fp8_f32 v62, v184, v185\n"
      "v_cvt_pk_fp8_f32 v63, v188, v189\n"
      "v_cvt_pk_fp8_f32 v60, v178, v179 op_sel:[0,0,1]\n"
      "v_cvt_pk_fp8_f32 v61, v182, v183 op_sel:[0,0,1]\n"
      "v_cvt_pk_fp8_f32 v62, v186, v187 op_sel:[0,0,1]\n"
      "v_cvt_pk_fp8_f32 v63, v190, v191 op_sel:[0,0,1]\n"
      "v_add_u32 v35, 0x18000, %[lo]\n"
      "global_store_dwordx4 v35, v[60:63], %[ro] sc0 sc1\n"
      "v_pk_mul_f32 v[192:193], v[192:193], %[iv]\n"
      "v_pk_mul_f32 v[194:195], v[194:195], %[iv]\n"
      "v_pk_mul_f32 v[196:197], v[196:197], %[iv]\n"
      "v_pk_mul_f32 v[198:199], v[198:199], %[iv]\n"
      "v_pk_mul_f32 v[200:201], v[200:201], %[iv]\n"
      "v_pk_mul_f32 v[202:203], v[202:203], %[iv]\n"
      "v_pk_mul_f32 v[204:205], v[204:205], %[iv]\n"
      "v_pk_mul_f32 v[206:207], v[206:207], %[iv]\n"
      "v_cvt_pk_fp8_f32 v48, v192, v193\n"
      "v_cvt_pk_fp8_f32 v49, v196, v197\n"
      "v_cvt_pk_fp8_f32 v50, v200, v201\n"
      "v_cvt_pk_fp8_f32 v51, v204, v205\n"
      "v_cvt_pk_fp8_f32 v48, v194, v195 op_sel:[0,0,1]\n"
      "v_cvt_pk_fp8_f32 v49, v198, v199 op_sel:[0,0,1]\n"
      "v_cvt_pk_fp8_f32 v50, v202, v203 op_sel:[0,0,1]\n"
      "v_cvt_pk_fp8_f32 v51, v206, v207 op_sel:[0,0,1]\n"
      "v_add_u32 v36, 0x20000, %[lo]\n"
      "global_store_dwordx4 v36, v[48:51], %[ro] sc0 sc1\n"
      "v_pk_mul_f32 v[208:209], v[208:209], %[iv]\n"
      "v_pk_mul_f32 v[210:211], v[210:211], %[iv]\n"
      "v_pk_mul_f32 v[212:213], v[212:213], %[iv]\n"
      "v_pk_mul_f32 v[214:215], v[214:215], %[iv]\n"
      "v_pk_mul_f32 v[216:217], v[216:217], %[iv]\n"
      "v_pk_mul_f32 v[218:219], v[218:219], %[iv]\n"
      "v_pk_mul_f32 v[220:221], v[220:221], %[iv]\n"
      "v_pk_mul_f32 v[222:223], v[222:223], %[iv]\n"
      "v_cvt_pk_fp8_f32 v52, v208, v209\n"
      "v_cvt_pk_fp8_f32 v53, v212, v213\n"
      "v_cvt_pk_fp8_f32 v54, v216, v217\n"
      "v_cvt_pk_fp8_f32 v55, v220, v221\n"
      "v_cvt_pk_fp8_f32 v52, v210, v211 op_sel:[0,0,1]\n"
      "v_cvt_pk_fp8_f32 v53, v214, v215 op_sel:[0,0,1]\n"
      "v_cvt_pk_fp8_f32 v54, v218, v219 op_sel:[0,0,1]\n"
      "v_cvt_pk_fp8_f32 v55, v222, v223 op_sel:[0,0,1]\n"
      "v_add_u32 v37, 0x28000, %[lo]\n"
      "global_store_dwordx4 v37, v[52:55], %[ro] sc0 sc1\n"
      "v_pk_mul_f32 v[224:225], v[224:225], %[iv]\n"
      "v_pk_mul_f32 v[226:227], v[226:227], %[iv]\n"
      "v_pk_mul_f32 v[228:229], v[228:229], %[iv]\n"
      "v_pk_mul_f32 v[230:231], v[230:231], %[iv]\n"
      "v_pk_mul_f32 v[232:233], v[232:233], %[iv]\n"
      "v_pk_mul_f32 v[234:235], v[234:235], %[iv]\n"
      "v_pk_mul_f32 v[236:237], v[236:237], %[iv]\n"
      "v_pk_mul_f32 v[238:239], v[238:239], %[iv]\n"
      "v_cvt_pk_fp8_f32 v56, v224, v225\n"
      "v_cvt_pk_fp8_f32 v57, v228, v229\n"
      "v_cvt_pk_fp8_f32 v58, v232, v233\n"
      "v_cvt_pk_fp8_f32 v59, v236, v237\n"
      "v_cvt_pk_fp8_f32 v56, v226, v227 op_sel:[0,0,1]\n"
      "v_cvt_pk_fp8_f32 v57, v230, v231 op_sel:[0,0,1]\n"
      "v_cvt_pk_fp8_f32 v58, v234, v235 op_sel:[0,0,1]\n"
      "v_cvt_pk_fp8_f32 v59, v238, v239 op_sel:[0,0,1]\n"
      "v_add_u32 v38, 0x30000, %[lo]\n"
      "global_store_dwordx4 v38, v[56:59], %[ro] sc0 sc1\n"
      "v_pk_mul_f32 v[240:241], v[240:241], %[iv]\n"
      "v_pk_mul_f32 v[242:243], v[242:243], %[iv]\n"
      "v_pk_mul_f32 v[244:245], v[244:245], %[iv]\n"
      "v_pk_mul_f32 v[246:247], v[246:247], %[iv]\n"
      "v_pk_mul_f32 v[248:249], v[248:249], %[iv]\n"
      "v_pk_mul_f32 v[250:251], v[250:251], %[iv]\n"
      "v_pk_mul_f32 v[252:253], v[252:253], %[iv]\n"
      "v_pk_mul_f32 v[254:255], v[254:255], %[iv]\n"
      "v_cvt_pk_fp8_f32 v60, v240, v241\n"
      "v_cvt_pk_fp8_f32 v61, v244, v245\n"
      "v_cvt_pk_fp8_f32 v62, v248, v249\n"
      "v_cvt_pk_fp8_f32 v63, v252, v253\n"
      "v_cvt_pk_fp8_f32 v60, v242, v243 op_sel:[0,0,1]\n"
      "v_cvt_pk_fp8_f32 v61, v246, v247 op_sel:[0,0,1]\n"
      "v_cvt_pk_fp8_f32 v62, v250, v251 op_sel:[0,0,1]\n"
      "v_cvt_pk_fp8_f32 v63, v254, v255 op_sel:[0,0,1]\n"
      "v_add_u32 v39, 0x38000, %[lo]\n"
      "global_store_dwordx4 v39, v[60:63], %[ro] sc0 sc1\n"
      "s_branch L_l9_%=\n"
      "L_l1_%=:\n"
      "v_pk_mul_f32 v[128:129], v[128:129], %[iv]\n"
      "v_pk_mul_f32 v[130:131], v[130:131], %[iv]\n"
      "v_pk_mul_f32 v[132:133], v[132:133], %[iv]\n"
      "v_pk_mul_f32 v[134:135], v[134:135], %[iv]\n"
      "v_pk_mul_f32 v[136:137], v[136:137], %[iv]\n"
      "v_pk_mul_f32 v[138:139], v[138:139], %[iv]\n"
      "v_pk_mul_f32 v[140:141], v[140:141], %[iv]\n"
      "v_pk_mul_f32 v[142:143], v[142:143], %[iv]\n"
      "v_pk_mul_f32 v[144:145], v[144:145], %[iv]\n"
      "v_pk_mul_f32 v[146:147], v[146:147], %[iv]\n"
      "v_pk_mul_f32 v[148:149], v[148:149], %[iv]\n"
      "v_pk_mul_f32 v[150:151], v[150:151], %[iv]\n"
      "v_pk_mul_f32 v[152:153], v[152:153], %[iv]\n"
      "v_pk_mul_f32 v[154:155], v[154:155], %[iv]\n"
      "v_pk_mul_f32 v[156:157], v[156:157], %[iv]\n"
      "v_pk_mul_f32 v[158:159], v[158:159], %[iv]\n"
      "v_cvt_pk_fp8_f32 v48, v128, v129\n"
      "v_cvt_pk_fp8_f32 v49, v132, v133\n"
      "v_cvt_pk_fp8_f32 v50, v136, v137\n"
      "v_cvt_pk_fp8_f32 v51, v140, v141\n"
      "v_cvt_pk_fp8_f32 v52, v144, v145\n"
      "v_cvt_pk_fp8_f32 v53, v148, v149\n"
      "v_cvt_pk_fp8_f32 v54, v152, v153\n"
      "v_cvt_pk_fp8_f32 v55, v156, v157\n"
      "v_cvt_pk_fp8_f32 v48, v130, v131 op_sel:[0,0,1]\n"
      "v_cvt_pk_fp8_f32 v49, v134, v135 op_sel:[0,0,1]\n"
      "v_cvt_pk_fp8_f32 v50, v138, v139 op_sel:[0,0,1]\n"
      "v_cvt_pk_fp8_f32 v51, v142, v143 op_sel:[0,0,1]\n"
      "v_cvt_pk_fp8_f32 v52, v146, v147 op_sel:[0,0,1]\n"
      "v_cvt_pk_fp8_f32 v53, v150, v151 op_sel:[0,0,1]\n"
      "v_cvt_pk_fp8_f32 v54, v154, v155 op_sel:[0,0,1]\n"
      "v_cvt_pk_fp8_f32 v55, v158, v159 op_sel:[0,0,1]\n"
      "v_pk_sub_u16 v64, v64, %[se] clamp\n"
      "v_pk_sub_u16 v65, v65, %[se] clamp\n"
      "v_pk_sub_u16 v66, v66, %[se] clamp\n"
      "v_pk_sub_u16 v67, v67, %[se] clamp\n"
      "v_pk_sub_u16 v68, v68, %[se] clamp\n"
      "v_pk_sub_u16 v69, v69, %[se] clamp\n"
      "v_pk_sub_u16 v70, v70, %[se] clamp\n"
      "v_pk_sub_u16 v71, v71, %[se] clamp\n"
      "v_pk_sub_u16 v96, v96, %[so] clamp\n"
      "v_pk_sub_u16 v97, v97, %[so] clamp\n"
      "v_pk_sub_u16 v98, v98, %[so] clamp\n"
      "v_pk_sub_u16 v99, v99, %[so] clamp\n"
      "v_pk_sub_u16 v100, v100, %[so] clamp\n"
      "v_pk_sub_u16 v101, v101, %[so] clamp\n"
      "v_pk_sub_u16 v102, v102, %[so] clamp\n"
      "v_pk_sub_u16 v103, v103, %[so] clamp\n"
      "v_add3_u32 v64, v64, v96, %[m1]\n"
      "v_add3_u32 v65, v65, v97, %[m1]\n"
      "v_add3_u32 v66, v66, v98, %[m1]\n"
      "v_add3_u32 v67, v67, v99, %[m1]\n"
      "v_add3_u32 v68, v68, v100, %[m1]\n"
      "v_add3_u32 v69, v69, v101, %[m1]\n"
      "v_add3_u32 v70, v70, v102, %[m1]\n"
      "v_add3_u32 v71, v71, v103, %[m1]\n"
      "v_sub_u32 v48, v48, v64\n"
      "v_sub_u32 v49, v49, v65\n"
      "v_sub_u32 v50, v50, v66\n"
      "v_sub_u32 v51, v51, v67\n"
      "v_sub_u32 v52, v52, v68\n"
      "v_sub_u32 v53, v53, v69\n"
      "v_sub_u32 v54, v54, v70\n"
      "v_sub_u32 v55, v55, v71\n"
      "v_mul_u32_u24_sdwa v96, v48, %[c3] dst_sel:DWORD dst_unused:UNUSED_PAD src0_sel:BYTE_3 src1_sel:DWORD\n"
      "v_mul_u32_u24_sdwa v97, v49, %[c3] dst_sel:DWORD dst_unused:UNUSED_PAD src0_sel:BYTE_3 src1_sel:DWORD\n"
      "v_mul_u32_u24_sdwa v98, v50, %[c3] dst_sel:DWORD dst_unused:UNUSED_PAD src0_sel:BYTE_3 src1_sel:DWORD\n"
      "v_mul_u32_u24_sdwa v99, v51, %[c3] dst_sel:DWORD dst_unused:UNUSED_PAD src0_sel:BYTE_3 src1_sel:DWORD\n"
      "v_mul_u32_u24_sdwa v100, v52, %[c3] dst_sel:DWORD dst_unused:UNUSED_PAD src0_sel:BYTE_3 src1_sel:DWORD\n"
      "v_mul_u32_u24_sdwa v101, v53, %[c3] dst_sel:DWORD dst_unused:UNUSED_PAD src0_sel:BYTE_3 src1_sel:DWORD\n"
      "v_mul_u32_u24_sdwa v102, v54, %[c3] dst_sel:DWORD dst_unused:UNUSED_PAD src0_sel:BYTE_3 src1_sel:DWORD\n"
      "v_mul_u32_u24_sdwa v103, v55, %[c3] dst_sel:DWORD dst_unused:UNUSED_PAD src0_sel:BYTE_3 src1_sel:DWORD\n"
      "v_dot4_u32_u8 v96, v48, %[c9], v96\n"
      "v_dot4_u32_u8 v97, v49, %[c9], v97\n"
      "v_dot4_u32_u8 v98, v50, %[c9], v98\n"
      "v_dot4_u32_u8 v99, v51, %[c9], v99\n"
      "v_dot4_u32_u8 v100, v52, %[c9], v100\n"
      "v_dot4_u32_u8 v101, v53, %[c9], v101\n"
      "v_dot4_u32_u8 v102, v54, %[c9], v102\n"
      "v_dot4_u32_u8 v103, v55, %[c9], v103\n"
      "v_lshl_or_b32 v56, v97, 16, v96\n"
      "v_lshl_or_b32 v57, v99, 16, v98\n"
      "v_lshl_or_b32 v58, v101, 16, v100\n"
      "v_lshl_or_b32 v59, v103, 16, v102\n"
      "v_lshlrev_b32 v48, 13, v59\n"
      "v_lshlrev_b32 v49, 10, v59\n"
      "v_lshlrev_b32 v50, 7, v59\n"
      "v_bfe_u32 v51, v59, 9, 4\n"
      "v_lshrrev_b32 v52, 21, v59\n"
      "v_and_or_b32 v56, v48, %[mh], v56\n"
      "v_and_or_b32 v57, v49, %[mh], v57\n"
      "v_and_or_b32 v58, v50, %[mh], v58\n"
      "v_and_or_b32 v59, v52, %[mf], v51\n"
      "v_add_u32 v59, 16, v59\n"
      "s_mov_b64 exec, %[k0]\n"
      "s_waitcnt vmcnt(0)\n"
      "global_store_dwordx3 %[d3], v[56:58], off offset:0 sc0 sc1\n"
      "global_store_byte %[d1], v59, off offset:0 sc0 sc1\n"
      "s_mov_b64 exec, -1\n"
      "s_cmp_gt_u32 %[nA], 2048\n"
      "s_cbranch_scc0 L_l2_%=\n"
      "v_pk_mul_f32 v[160:161], v[160:161], %[iv]\n"
      "v_pk_mul_f32 v[162:163], v[162:163], %[iv]\n"
      "v_pk_mul_f32 v[164:165], v[164:165], %[iv]\n"
      "v_pk_mul_f32 v[166:167], v[166:167], %[iv]\n"
      "v_pk_mul_f32 v[168:169], v[168:169], %[iv]\n"
      "v_pk_mul_f32 v[170:171], v[170:171], %[iv]\n"
      "v_pk_mul_f32 v[172:173], v[172:173], %[iv]\n"
      "v_pk_mul_f32 v[174:175], v[174:175], %[iv]\n"
      "v_pk_mul_f32 v[176:177], v[176:177], %[iv]\n"
      "v_pk_mul_f32 v[178:179], v[178:179], %[iv]\n"
      "v_pk_mul_f32 v[180:181], v[180:181], %[iv]\n"
      "v_pk_mul_f32 v[182:183], v[182:183], %[iv]\n"
      "v_pk_mul_f32 v[184:185], v[184:185], %[iv]\n"
      "v_pk_mul_f32 v[186:187], v[186:187], %[iv]\n"
      "v_pk_mul_f32 v[188:189], v[188:189], %[iv]\n"
      "v_pk_mul_f32 v[190:191], v[190:191], %[iv]\n"
      "v_cvt_pk_fp8_f32 v48, v160, v161\n"
      "v_cvt_pk_fp8_f32 v49, v164, v165\n"
      "v_cvt_pk_fp8_f32 v50, v168, v169\n"
      "v_cvt_pk_fp8_f32 v51, v172, v173\n"
      "v_cvt_pk_fp8_f32 v52, v176, v177\n"
      "v_cvt_pk_fp8_f32 v53, v180, v181\n"
      "v_cvt_pk_fp8_f32 v54, v184, v185\n"
      "v_cvt_pk_fp8_f32 v55, v188, v189\n"
      "v_cvt_pk_fp8_f32 v48, v162, v163 op_sel:[0,0,1]\n"
      "v_cvt_pk_fp8_f32 v49, v166, v167 op_sel:[0,0,1]\n"
      "v_cvt_pk_fp8_f32 v50, v170, v171 op_sel:[0,0,1]\n"
      "v_cvt_pk_fp8_f32 v51, v174, v175 op_sel:[0,0,1]\n"
      "v_cvt_pk_fp8_f32 v52, v178, v179 op_sel:[0,0,1]\n"
      "v_cvt_pk_fp8_f32 v53, v182, v183 op_sel:[0,0,1]\n"
      "v_cvt_pk_fp8_f32 v54, v186, v187 op_sel:[0,0,1]\n"
      "v_cvt_pk_fp8_f32 v55, v190, v191 op_sel:[0,0,1]\n"
      "v_pk_sub_u16 v72, v72, %[se] clamp\n"
      "v_pk_sub_u16 v73, v73, %[se] clamp\n"
      "v_pk_sub_u16 v74, v74, %[se] clamp\n"
      "v_pk_sub_u16 v75, v75, %[se] clamp\n"
      "v_pk_sub_u16 v76, v76, %[se] clamp\n"
      "v_pk_sub_u16 v77, v77, %[se] clamp\n"
      "v_pk_sub_u16 v78, v78, %[se] clamp\n"
      "v_pk_sub_u16 v79, v79, %[se] clamp\n"
      "v_pk_sub_u16 v104, v104, %[so] clamp\n"
      "v_pk_sub_u16 v105, v105, %[so] clamp\n"
      "v_pk_sub_u16 v106, v106, %[so] clamp\n"
      "v_pk_sub_u16 v107, v107, %[so] clamp\n"
      "v_pk_sub_u16 v108, v108, %[so] clamp\n"
      "v_pk_sub_u16 v109, v109, %[so] clamp\n"
      "v_pk_sub_u16 v110, v110, %[so] clamp\n"
      "v_pk_sub_u16 v111, v111, %[so] clamp\n"
      "v_add3_u32 v72, v72, v104, %[m1]\n"
      "v_add3_u32 v73, v73, v105, %[m1]\n"
      "v_add3_u32 v74, v74, v106, %[m1]\n"
      "v_add3_u32 v75, v75, v107, %[m1]\n"
      "v_add3_u32 v76, v76, v108, %[m1]\n"
      "v_add3_u32 v77, v77, v109, %[m1]\n"
      "v_add3_u32 v78, v78, v110, %[m1]\n"
      "v_add3_u32 v79, v79, v111, %[m1]\n"
      "v_sub_u32 v48, v48, v72\n"
      "v_sub_u32 v49, v49, v73\n"
      "v_sub_u32 v50, v50, v74\n"
      "v_sub_u32 v51, v51, v75\n"
      "v_sub_u32 v52, v52, v76\n"
      "v_sub_u32 v53, v53, v77\n"
      "v_sub_u32 v54, v54, v78\n"
      "v_sub_u32 v55, v55, v79\n"
      "v_mul_u32_u24_sdwa v104, v48, %[c3] dst_sel:DWORD dst_unused:UNUSED_PAD src0_sel:BYTE_3 src1_sel:DWORD\n"
      "v_mul_u32_u24_sdwa v105, v49, %[c3] dst_sel:DWORD dst_unused:UNUSED_PAD src0_sel:BYTE_3 src1_sel:DWORD\n"
      "v_mul_u32_u24_sdwa v106, v50, %[c3] dst_sel:DWORD dst_unused:UNUSED_PAD src0_sel:BYTE_3 src1_sel:DWORD\n"
      "v_mul_u32_u24_sdwa v107, v51, %[c3] dst_sel:DWORD dst_unused:UNUSED_PAD src0_sel:BYTE_3 src1_sel:DWORD\n"
      "v_mul_u32_u24_sdwa v108, v52, %[c3] dst_sel:DWORD dst_unused:UNUSED_PAD src0_sel:BYTE_3 src1_sel:DWORD\n"
      "v_mul_u32_u24_sdwa v109, v53, %[c3] dst_sel:DWORD dst_unused:UNUSED_PAD src0_sel:BYTE_3 src1_sel:DWORD\n"
      "v_mul_u32_u24_sdwa v110, v54, %[c3] dst_sel:DWORD dst_unused:UNUSED_PAD src0_sel:BYTE_3 src1_sel:DWORD\n"
      "v_mul_u32_u24_sdwa v111, v55, %[c3] dst_sel:DWORD dst_unused:UNUSED_PAD src0_sel:BYTE_3 src1_sel:DWORD\n"
      "v_dot4_u32_u8 v104, v48, %[c9], v104\n"
      "v_dot4_u32_u8 v105, v49, %[c9], v105\n"
      "v_dot4_u32_u8 v106, v50, %[c9], v106\n"
      "v_dot4_u32_u8 v107, v51, %[c9], v107\n"
      "v_dot4_u32_u8 v108, v52, %[c9], v108\n"
      "v_dot4_u32_u8 v109, v53, %[c9], v109\n"
      "v_dot4_u32_u8 v110, v54, %[c9], v110\n"
      "v_dot4_u32_u8 v111, v55, %[c9], v111\n"
      "v_lshl_or_b32 v56, v105, 16, v104\n"
      "v_lshl_or_b32 v57, v107, 16, v106\n"
      "v_lshl_or_b32 v58, v109, 16, v108\n"
      "v_lshl_or_b32 v59, v111, 16, v110\n"
      "v_lshlrev_b32 v48, 13, v59\n"
      "v_lshlrev_b32 v49, 10, v59\n"
      "v_lshlrev_b32 v50, 7, v59\n"
      "v_bfe_u32 v51, v59, 9, 4\n"
      "v_lshrrev_b32 v52, 21, v59\n"
      "v_and_or_b32 v56, v48, %[mh], v56\n"
      "v_and_or_b32 v57, v49, %[mh], v57\n"
      "v_and_or_b32 v58, v50, %[mh], v58\n"
      "v_and_or_b32 v59, v52, %[mf], v51\n"
      "v_add_u32 v59, 16, v59\n"
      "s_mov_b64 exec, %[k1]\n"
      "global_store_dwordx4 %[dst], v[56:59], off offset:1024 sc0 sc1\n"
      "s_mov_b64 exec, -1\n"
      "L_l2_%=:\n"
      "v_pk_mul_f32 v[192:193], v[192:193], %[iv]\n"
      "v_pk_mul_f32 v[194:195], v[194:195], %[iv]\n"
      "v_pk_mul_f32 v[196:197], v[196:197], %[iv]\n"
      "v_pk_mul_f32 v[198:199], v[198:199], %[iv]\n"
      "v_pk_mul_f32 v[200:201], v[200:201], %[iv]\n"
      "v_pk_mul_f32 v[202:203], v[202:203], %[iv]\n"
      "v_pk_mul_f32 v[204:205], v[204:205], %[iv]\n"
      "v_pk_mul_f32 v[206:207], v[206:207], %[iv]\n"
      "v_pk_mul_f32 v[208:209], v[208:209], %[iv]\n"
      "v_pk_mul_f32 v[210:211], v[210:211], %[iv]\n"
      "v_pk_mul_f32 v[212:213], v[212:213], %[iv]\n"
      "v_pk_mul_f32 v[214:215], v[214:215], %[iv]\n"
      "v_pk_mul_f32 v[216:217], v[216:217], %[iv]\n"
      "v_pk_mul_f32 v[218:219], v[218:219], %[iv]\n"
      "v_pk_mul_f32 v[220:221], v[220:221], %[iv]\n"
      "v_pk_mul_f32 v[222:223], v[222:223], %[iv]\n"
      "v_cvt_pk_fp8_f32 v48, v192, v193\n"
      "v_cvt_pk_fp8_f32 v49, v196, v197\n"
      "v_cvt_pk_fp8_f32 v50, v200, v201\n"
      "v_cvt_pk_fp8_f32 v51, v204, v205\n"
      "v_cvt_pk_fp8_f32 v52, v208, v209\n"
      "v_cvt_pk_fp8_f32 v53, v212, v213\n"
      "v_cvt_pk_fp8_f32 v54, v216, v217\n"
      "v_cvt_pk_fp8_f32 v55, v220, v221\n"
      "v_cvt_pk_fp8_f32 v48, v194, v195 op_sel:[0,0,1]\n"
      "v_cvt_pk_fp8_f32 v49, v198, v199 op_sel:[0,0,1]\n"
      "v_cvt_pk_fp8_f32 v50, v202, v203 op_sel:[0,0,1]\n"
      "v_cvt_pk_fp8_f32 v51, v206, v207 op_sel:[0,0,1]\n"
      "v_cvt_pk_fp8_f32 v52, v210, v211 op_sel:[0,0,1]\n"
      "v_cvt_pk_fp8_f32 v53, v214, v215 op_sel:[0,0,1]\n"
      "v_cvt_pk_fp8_f32 v54, v218, v219 op_sel:[0,0,1]\n"
      "v_cvt_pk_fp8_f32 v55, v222, v223 op_sel:[0,0,1]\n"
      "v_pk_sub_u16 v80, v80, %[se] clamp\n"
      "v_pk_sub_u16 v81, v81, %[se] clamp\n"
      "v_pk_sub_u16 v82, v82, %[se] clamp\n"
      "v_pk_sub_u16 v83, v83, %[se] clamp\n"
      "v_pk_sub_u16 v84, v84, %[se] clamp\n"
      "v_pk_sub_u16 v85, v85, %[se] clamp\n"
      "v_pk_sub_u16 v86, v86, %[se] clamp\n"
      "v_pk_sub_u16 v87, v87, %[se] clamp\n"
      "v_pk_sub_u16 v112, v112, %[so] clamp\n"
      "v_pk_sub_u16 v113, v113, %[so] clamp\n"
      "v_pk_sub_u16 v114, v114, %[so] clamp\n"
      "v_pk_sub_u16 v115, v115, %[so] clamp\n"
      "v_pk_sub_u16 v116, v116, %[so] clamp\n"
      "v_pk_sub_u16 v117, v117, %[so] clamp\n"
      "v_pk_sub_u16 v118, v118, %[so] clamp\n"
      "v_pk_sub_u16 v119, v119, %[so] clamp\n"
      "v_add3_u32 v80, v80, v112, %[m1]\n"
      "v_add3_u32 v81, v81, v113, %[m1]\n"
      "v_add3_u32 v82, v82, v114, %[m1]\n"
      "v_add3_u32 v83, v83, v115, %[m1]\n"
      "v_add3_u32 v84, v84, v116, %[m1]\n"
      "v_add3_u32 v85, v85, v117, %[m1]\n"
      "v_add3_u32 v86, v86, v118, %[m1]\n"
      "v_add3_u32 v87, v87, v119, %[m1]\n"
      "v_sub_u32 v48, v48, v80\n"
      "v_sub_u32 v49, v49, v81\n"
      "v_sub_u32 v50, v50, v82\n"
      "v_sub_u32 v51, v51, v83\n"
      "v_sub_u32 v52, v52, v84\n"
      "v_sub_u32 v53, v53, v85\n"
      "v_sub_u32 v54, v54, v86\n"
      "v_sub_u32 v55, v55, v87\n"
      "v_mul_u32_u24_sdwa v112, v48, %[c3] dst_sel:DWORD dst_unused:UNUSED_PAD src0_sel:BYTE_3 src1_sel:DWORD\n"
      "v_mul_u32_u24_sdwa v113, v49, %[c3] dst_sel:DWORD dst_unused:UNUSED_PAD src0_sel:BYTE_3 src1_sel:DWORD\n"
      "v_mul_u32_u24_sdwa v114, v50, %[c3] dst_sel:DWORD dst_unused:UNUSED_PAD src0_sel:BYTE_3 src1_sel:DWORD\n"
      "v_mul_u32_u24_sdwa v115, v51, %[c3] dst_sel:DWORD dst_unused:UNUSED_PAD src0_sel:BYTE_3 src1_sel:DWORD\n"
      "v_mul_u32_u24_sdwa v116, v52, %[c3] dst_sel:DWORD dst_unused:UNUSED_PAD src0_sel:BYTE_3 src1_sel:DWORD\n"
      "v_mul_u32_u24_sdwa v117, v53, %[c3] dst_sel:DWORD dst_unused:UNUSED_PAD src0_sel:BYTE_3 src1_sel:DWORD\n"
      "v_mul_u32_u24_sdwa v118, v54, %[c3] dst_sel:DWORD dst_unused:UNUSED_PAD src0_sel:BYTE_3 src1_sel:DWORD\n"
      "v_mul_u32_u24_sdwa v119, v55, %[c3] dst_sel:DWORD dst_unused:UNUSED_PAD src0_sel:BYTE_3 src1_sel:DWORD\n"
      "v_dot4_u32_u8 v112, v48, %[c9], v112\n"
      "v_dot4_u32_u8 v113, v49, %[c9], v113\n"
      "v_dot4_u32_u8 v114, v50, %[c9], v114\n"
      "v_dot4_u32_u8 v115, v51, %[c9], v115\n"
      "v_dot4_u32_u8 v116, v52, %[c9], v116\n"
      "v_dot4_u32_u8 v117, v53, %[c9], v117\n"
      "v_dot4_u32_u8 v118, v54, %[c9], v118\n"
      "v_dot4_u32_u8 v119, v55, %[c9], v119\n"
      "v_lshl_or_b32 v60, v113, 16, v112\n"
      "v_lshl_or_b32 v61, v115, 16, v114\n"
      "v_lshl_or_b32 v62, v117, 16, v116\n"
      "v_lshl_or_b32 v63, v119, 16, v118\n"
      "v_lshlrev_b32 v48, 13, v63\n"
      "v_lshlrev_b32 v49, 10, v63\n"
      "v_lshlrev_b32 v50, 7, v63\n"
      "v_bfe_u32 v51, v63, 9, 4\n"
      "v_lshrrev_b32 v52, 21, v63\n"
      "v_and_or_b32 v60, v48, %[mh], v60\n"
      "v_and_or_b32 v61, v49, %[mh], v61\n"
      "v_and_or_b32 v62, v50, %[mh], v62\n"
      "v_and_or_b32 v63, v52, %[mf], v51\n"
      "v_add_u32 v63, 16, v63\n"
      "s_mov_b64 exec, %[k2]\n"
      "s_waitcnt vmcnt(0)\n"
      "global_store_dwordx3 %[d3], v[60:62], off offset:2048 sc0 sc1\n"
      "global_store_byte %[d1], v63, off offset:2048 sc0 sc1\n"
      "s_mov_b64 exec, -1\n"
      "s_cmp_gt_u32 %[nB], 2048\n"
      "s_cbranch_scc0 L_l9_%=\n"
      "v_pk_mul_f32 v[224:225], v[224:225], %[iv]\n"
      "v_pk_mul_f32 v[226:227], v[226:227], %[iv]\n"
      "v_pk_mul_f32 v[228:229], v[228:229], %[iv]\n"
      "v_pk_mul_f32 v[230:231], v[230:231], %[iv]\n"
      "v_pk_mul_f32 v[232:233], v[232:233], %[iv]\n"
      "v_pk_mul_f32 v[234:235], v[234:235], %[iv]\n"
      "v_pk_mul_f32 v[236:237], v[236:237], %[iv]\n"
      "v_pk_mul_f32 v[238:239], v[238:239], %[iv]\n"
      "v_pk_mul_f32 v[240:241], v[240:241], %[iv]\n"
      "v_pk_mul_f32 v[242:243], v[242:243], %[iv]\n"
      "v_pk_mul_f32 v[244:245], v[244:245], %[iv]\n"
      "v_pk_mul_f32 v[246:247], v[246:247], %[iv]\n"
      "v_pk_mul_f32 v[248:249], v[248:249], %[iv]\n"
      "v_pk_mul_f32 v[250:251], v[250:251], %[iv]\n"
      "v_pk_mul_f32 v[252:253], v[252:253], %[iv]\n"
      "v_pk_mul_f32 v[254:255], v[254:255], %[iv]\n"
      "v_cvt_pk_fp8_f32 v48, v224, v225\n"
      "v_cvt_pk_fp8_f32 v49, v228, v229\n"
      "v_cvt_pk_fp8_f32 v50, v232, v233\n"
      "v_cvt_pk_fp8_f32 v51, v236, v237\n"
      "v_cvt_pk_fp8_f32 v52, v240, v241\n"
      "v_cvt_pk_fp8_f32 v53, v244, v245\n"
      "v_cvt_pk_fp8_f32 v54, v248, v249\n"
      "v_cvt_pk_fp8_f32 v55, v252, v253\n"
      "v_cvt_pk_fp8_f32 v48, v226, v227 op_sel:[0,0,1]\n"
      "v_cvt_pk_fp8_f32 v49, v230, v231 op_sel:[0,0,1]\n"
      "v_cvt_pk_fp8_f32 v50, v234, v235 op_sel:[0,0,1]\n"
      "v_cvt_pk_fp8_f32 v51, v238, v239 op_sel:[0,0,1]\n"
      "v_cvt_pk_fp8_f32 v52, v242, v243 op_sel:[0,0,1]\n"
      "v_cvt_pk_fp8_f32 v53, v246, v247 op_sel:[0,0,1]\n"
      "v_cvt_pk_fp8_f32 v54, v250, v251 op_sel:[0,0,1]\n"
      "v_cvt_pk_fp8_f32 v55, v254, v255 op_sel:[0,0,1]\n"
      "v_pk_sub_u16 v88, v88, %[se] clamp\n"
      "v_pk_sub_u16 v89, v89, %[se] clamp\n"
      "v_pk_sub_u16 v90, v90, %[se] clamp\n"
      "v_pk_sub_u16 v91, v91, %[se] clamp\n"
      "v_pk_sub_u16 v92, v92, %[se] clamp\n"
      "v_pk_sub_u16 v93, v93, %[se] clamp\n"
      "v_pk_sub_u16 v94, v94, %[se] clamp\n"
      "v_pk_sub_u16 v95, v95, %[se] clamp\n"
      "v_pk_sub_u16 v120, v120, %[so] clamp\n"
      "v_pk_sub_u16 v121, v121, %[so] clamp\n"
      "v_pk_sub_u16 v122, v122, %[so] clamp\n"
      "v_pk_sub_u16 v123, v123, %[so] clamp\n"
      "v_pk_sub_u16 v124, v124, %[so] clamp\n"
      "v_pk_sub_u16 v125, v125, %[so] clamp\n"
      "v_pk_sub_u16 v126, v126, %[so] clamp\n"
      "v_pk_sub_u16 v127, v127, %[so] clamp\n"
      "v_add3_u32 v88, v88, v120, %[m1]\n"
      "v_add3_u32 v89, v89, v121, %[m1]\n"
      "v_add3_u32 v90, v90, v122, %[m1]\n"
      "v_add3_u32 v91, v91, v123, %[m1]\n"
      "v_add3_u32 v92, v92, v124, %[m1]\n"
      "v_add3_u32 v93, v93, v125, %[m1]\n"
      "v_add3_u32 v94, v94, v126, %[m1]\n"
      "v_add3_u32 v95, v95, v127, %[m1]\n"
      "v_sub_u32 v48, v48, v88\n"
      "v_sub_u32 v49, v49, v89\n"
      "v_sub_u32 v50, v50, v90\n"
      "v_sub_u32 v51, v51, v91\n"
      "v_sub_u32 v52, v52, v92\n"
      "v_sub_u32 v53, v53, v93\n"
      "v_sub_u32 v54, v54, v94\n"
      "v_sub_u32 v55, v55, v95\n"
      "v_mul_u32_u24_sdwa v120, v48, %[c3] dst_sel:DWORD dst_unused:UNUSED_PAD src0_sel:BYTE_3 src1_sel:DWORD\n"
      "v_mul_u32_u24_sdwa v121, v49, %[c3] dst_sel:DWORD dst_unused:UNUSED_PAD src0_sel:BYTE_3 src1_sel:DWORD\n"
      "v_mul_u32_u24_sdwa v122, v50, %[c3] dst_sel:DWORD dst_unused:UNUSED_PAD src0_sel:BYTE_3 src1_sel:DWORD\n"
      "v_mul_u32_u24_sdwa v123, v51, %[c3] dst_sel:DWORD dst_unused:UNUSED_PAD src0_sel:BYTE_3 src1_sel:DWORD\n"
      "v_mul_u32_u24_sdwa v124, v52, %[c3] dst_sel:DWORD dst_unused:UNUSED_PAD src0_sel:BYTE_3 src1_sel:DWORD\n"
      "v_mul_u32_u24_sdwa v125, v53, %[c3] dst_sel:DWORD dst_unused:UNUSED_PAD src0_sel:BYTE_3 src1_sel:DWORD\n"
      "v_mul_u32_u24_sdwa v126, v54, %[c3] dst_sel:DWORD dst_unused:UNUSED_PAD src0_sel:BYTE_3 src1_sel:DWORD\n"
      "v_mul_u32_u24_sdwa v127, v55, %[c3] dst_sel:DWORD dst_unused:UNUSED_PAD src0_sel:BYTE_3 src1_sel:DWORD\n"
      "v_dot4_u32_u8 v120, v48, %[c9], v120\n"
      "v_dot4_u32_u8 v121, v49, %[c9], v121\n"
      "v_dot4_u32_u8 v122, v50, %[c9], v122\n"
      "v_dot4_u32_u8 v123, v51, %[c9], v123\n"
      "v_dot4_u32_u8 v124, v52, %[c9], v124\n"
      "v_dot4_u32_u8 v125, v53, %[c9], v125\n"
      "v_dot4_u32_u8 v126, v54, %[c9], v126\n"
      "v_dot4_u32_u8 v127, v55, %[c9], v127\n"
      "v_lshl_or_b32 v60, v121, 16, v120\n"
      "v_lshl_or_b32 v61, v123, 16, v122\n"
      "v_lshl_or_b32 v62, v125, 16, v124\n"
      "v_lshl_or_b32 v63, v127, 16, v126\n"
      "v_lshlrev_b32 v48, 13, v63\n"
      "v_lshlrev_b32 v49, 10, v63\n"
      "v_lshlrev_b32 v50, 7, v63\n"
      "v_bfe_u32 v51, v63, 9, 4\n"
      "v_lshrrev_b32 v52, 21, v63\n"
      "v_and_or_b32 v60, v48, %[mh], v60\n"
      "v_and_or_b32 v61, v49, %[mh], v61\n"
      "v_and_or_b32 v62, v50, %[mh], v62\n"
      "v_and_or_b32 v63, v52, %[mf], v51\n"
      "v_add_u32 v63, 16, v63\n"
      "s_mov_b64 exec, %[k3]\n"
      "global_store_dwordx4 %[dst], v[60:63], off offset:3072 sc0 sc1\n"
      "s_mov_b64 exec, -1\n"
      "L_l9_%=:\n"
            : ACC_IO, "+{v[64:79]}"(e0), "+{v[80:95]}"(e1), "+{v[96:111]}"(o0), "+{v[112:127]}"(o1)
            : [iv] "v"(iv), [se] "v"(sh), [so] "v"(sh << 8), [dst] "v"(dl), [nA] "s"(nA), [nB] "s"(nB), [m1] "s"(0xffffffffu),
              [c3] "s"(729u), [c9] "s"(0x00510901u), [mh] "s"(0xe000e000u), [mf] "s"(0xf0u),
              [d3] "v"(dl - lane * 4), [d1] "v"(dl + 768 - lane * 15),
              [md] "s"(rfl(remote ? (__float_as_uint(inv) == 0x7fc00000u ? 0u : 1u) : 2u)), [ro] "s"(rout), [lo] "v"((lane & 15) * NC + wid * 64 + (lane >> 4) * 16),
              [k0] "s"(k0), [k1] "s"(k1), [k2] "s"(k2), [k3] "s"(k3)
            : "v28", "v29", "v32", "v33", "v34", "v35", "v36", "v37", "v38", "v39", "v40", "v41", "v42", "v43", "v44", "v45", "v46", "v47", "v48", "v49", "v50", "v51", "v52", "v53", "v54", "v55", "v56", "v57", "v58", "v59", "v60", "v61", "v62", "v63", "scc", "memory");
        if (wid == 0) STAMP(6);
    } else {
        const u32 lane = threadIdx.x & 63, pw = wid - 4;
        __builtin_amdgcn_s_setprio(PPRIO);
        unsigned char* const roff = myws + WS_OFF + (size_t)((grp * 32 + idx) * 4 + pw) * OFF_SLOT + lane * 16;
        const u32x4 zero4 = {0, 0, 0, 0};
        u32 call = 1;
        {
            const u32x4 ra = make_rsrc(p.x, ML * KD * 2), rb = make_rsrc(p.w, (u32)KD * ND * 2);
            const u32 rw = RW, rx = RX;
            const u32 a0 = ((lane >> 3) * (KD * 2) + (((lane & 7) ^ (lane >> 4)) << 4)) ^ ((rx & 1) << 6), a1 = a0 ^ 64;
            const u32 b0 = ((lane >> 5) * (WPAIR * ND * 2) + (((lane & 31) ^ (lane >> 5)) << 4)) ^ ((rw & 1) << 5) ^ ((rw >> 2) << 7), b1 = b0 ^ 32, b2 = b0 ^ 128, b3 = b0 ^ 160;
            const u32 px = ROTX, pq = ROTW;
            const u32 sa0 = rfl((rowblk * BM + px * 32 + rx * 8) * KD * 2), sb0 = rfl(rw * (WPAIR == 1 ? 2 : 1) * ND * 2 + (WPAIR <= 8 ? pq * 16 : WPAIR == 16 ? (pq >> 1) * 32 + (pq & 1) * 8 : pq * 8) * ND * 2 + (grp * NC + ct * BN) * 2);
            const u32 ma0 = rfl(lds0 + A_BASE + px * 4096 + rx * 1024), mb0 = rfl(lds0 + B_BASE + pq * 8192 + rw * 1024), mae = ma0 + 3 * A_ST;
            const u32 ssa = 128, ssb = 64 * ND * 2;
            u32 ma = ma0, mb = mb0, t, t2, n, tv = 0, tw = 0;
            const u32 kx = rfl(KX), kp = rfl((KP) + (LAG));
            u32 kk = rfl(KP);
            u32 sa = sa0 + (((kk ^ kx) - kp) & 63) * ssa, sb = sb0 + (((kk ^ kx) - kp) & 63) * ssb;
            asm volatile(
      "s_cmp_lg_u32 %[pw], 3\n"
      "s_cbranch_scc1 L_s_%=\n"
      "s_mov_b64 exec, 1\n"
      "global_atomic_add %[call], %[ca], %[call], off sc0\n"
      "s_mov_b64 exec, -1\n"
      "L_s_%=:\n"
      "s_xor_b32 m0, %[ma], 0\n"
      "s_xor_b32 %[t], %[sa], 0x0\n"
      "buffer_load_dwordx4 %[a0], %[ra], %[t] offen  lds\n"
      "s_xor_b32 m0, %[ma], 1024\n"
      "s_xor_b32 %[t], %[sa], 0x10000\n"
      "buffer_load_dwordx4 %[a1], %[ra], %[t] offen  lds\n"
      "s_xor_b32 m0, %[ma], 2048\n"
      "s_xor_b32 %[t], %[sa], 0x20000\n"
      "buffer_load_dwordx4 %[a0], %[ra], %[t] offen  lds\n"
      "s_xor_b32 m0, %[ma], 3072\n"
      "s_xor_b32 %[t], %[sa], 0x30000\n"
      "buffer_load_dwordx4 %[a1], %[ra], %[t] offen  lds\n"
      "s_xor_b32 m0, %[mb], 0\n"
      "s_xor_b32 %[t], %[sb], 0x0\n"
      "buffer_load_dwordx4 %[b0], %[rb], %[t] offen  lds\n"
      "s_xor_b32 m0, %[mb], 1024\n"
      "s_xor_b32 %[t], %[sb], 0x10000\n"
      "buffer_load_dwordx4 %[b1], %[rb], %[t] offen  lds\n"
      "s_xor_b32 m0, %[mb], 2048\n"
      "s_xor_b32 %[t], %[sb], 0x20000\n"
      "buffer_load_dwordx4 %[b0], %[rb], %[t] offen  lds\n"
      "s_xor_b32 m0, %[mb], 3072\n"
      "s_xor_b32 %[t], %[sb], 0x30000\n"
      "buffer_load_dwordx4 %[b1], %[rb], %[t] offen  lds\n"
      "s_xor_b32 m0, %[mb], 4096\n"
      "s_xor_b32 %[t], %[sb], 0x40000\n"
      "buffer_load_dwordx4 %[b2], %[rb], %[t] offen  lds\n"
      "s_xor_b32 m0, %[mb], 5120\n"
      "s_xor_b32 %[t], %[sb], 0x50000\n"
      "buffer_load_dwordx4 %[b3], %[rb], %[t] offen  lds\n"
      "s_xor_b32 m0, %[mb], 6144\n"
      "s_xor_b32 %[t], %[sb], 0x60000\n"
      "buffer_load_dwordx4 %[b2], %[rb], %[t] offen  lds\n"
      "s_xor_b32 m0, %[mb], 7168\n"
      "s_xor_b32 %[t], %[sb], 0x70000\n"
      "buffer_load_dwordx4 %[b3], %[rb], %[t] offen  lds\n"
      "s_add_u32 %[kk], %[kk], 1\n"
      "s_xor_b32 %[t], %[kk], %[kx]\n"
      "s_sub_u32 %[t], %[t], %[kp]\n"
      "s_and_b32 %[t], %[t], 63\n"
      "s_mul_i32 %[t2], %[t], %[ssa]\n"
      "s_add_u32 %[sa], %[sa0], %[t2]\n"
      "s_mul_i32 %[t2], %[t], %[ssb]\n"
      "s_add_u32 %[sb], %[sb0], %[t2]\n"
      "s_add_u32 %[ma], %[ma], 16384\n"
      "s_add_u32 %[mb], %[mb], 32768\n"
      "s_cmp_eq_u32 %[ma], %[mae]\n"
      "s_cselect_b32 %[ma], %[ma0], %[ma]\n"
      "s_cselect_b32 %[mb], %[mb0], %[mb]\n"
      "s_xor_b32 m0, %[ma], 0\n"
      "s_xor_b32 %[t], %[sa], 0x0\n"
      "buffer_load_dwordx4 %[a0], %[ra], %[t] offen  lds\n"
      "s_xor_b32 m0, %[ma], 1024\n"
      "s_xor_b32 %[t], %[sa], 0x10000\n"
      "buffer_load_dwordx4 %[a1], %[ra], %[t] offen  lds\n"
      "s_xor_b32 m0, %[ma], 2048\n"
      "s_xor_b32 %[t], %[sa], 0x20000\n"
      "buffer_load_dwordx4 %[a0], %[ra], %[t] offen  lds\n"
      "s_xor_b32 m0, %[ma], 3072\n"
      "s_xor_b32 %[t], %[sa], 0x30000\n"
      "buffer_load_dwordx4 %[a1], %[ra], %[t] offen  lds\n"
      "s_xor_b32 m0, %[mb], 0\n"
      "s_xor_b32 %[t], %[sb], 0x0\n"
      "buffer_load_dwordx4 %[b0], %[rb], %[t] offen  lds\n"
      "s_xor_b32 m0, %[mb], 1024\n"
      "s_xor_b32 %[t], %[sb], 0x10000\n"
      "buffer_load_dwordx4 %[b1], %[rb], %[t] offen  lds\n"
      "s_xor_b32 m0, %[mb], 2048\n"
      "s_xor_b32 %[t], %[sb], 0x20000\n"
      "buffer_load_dwordx4 %[b0], %[rb], %[t] offen  lds\n"
      "s_xor_b32 m0, %[mb], 3072\n"
      "s_xor_b32 %[t], %[sb], 0x30000\n"
      "buffer_load_dwordx4 %[b1], %[rb], %[t] offen  lds\n"
      "s_xor_b32 m0, %[mb], 4096\n"
      "s_xor_b32 %[t], %[sb], 0x40000\n"
      "buffer_load_dwordx4 %[b2], %[rb], %[t] offen  lds\n"
      "s_xor_b32 m0, %[mb], 5120\n"
      "s_xor_b32 %[t], %[sb], 0x50000\n"
      "buffer_load_dwordx4 %[b3], %[rb], %[t] offen  lds\n"
      "s_xor_b32 m0, %[mb], 6144\n"
      "s_xor_b32 %[t], %[sb], 0x60000\n"
      "buffer_load_dwordx4 %[b2], %[rb], %[t] offen  lds\n"
      "s_xor_b32 m0, %[mb], 7168\n"
      "s_xor_b32 %[t], %[sb], 0x70000\n"
      "buffer_load_dwordx4 %[b3], %[rb], %[t] offen  lds\n"
      "s_add_u32 %[kk], %[kk], 1\n"
      "s_xor_b32 %[t], %[kk], %[kx]\n"
      "s_sub_u32 %[t], %[t], %[kp]\n"
      "s_and_b32 %[t], %[t], 63\n"
      "s_mul_i32 %[t2], %[t], %[ssa]\n"
      "s_add_u32 %[sa], %[sa0], %[t2]\n"
      "s_mul_i32 %[t2], %[t], %[ssb]\n"
      "s_add_u32 %[sb], %[sb0], %[t2]\n"
      "s_add_u32 %[ma], %[ma], 16384\n"
      "s_add_u32 %[mb], %[mb], 32768\n"
      "s_cmp_eq_u32 %[ma], %[mae]\n"
      "s_cselect_b32 %[ma], %[ma0], %[ma]\n"
      "s_cselect_b32 %[mb], %[mb0], %[mb]\n"
      "s_xor_b32 m0, %[ma], 0\n"
      "s_xor_b32 %[t], %[sa], 0x0\n"
      "buffer_load_dwordx4 %[a0], %[ra], %[t] offen  lds\n"
      "s_xor_b32 m0, %[ma], 1024\n"
      "s_xor_b32 %[t], %[sa], 0x10000\n"
      "buffer_load_dwordx4 %[a1], %[ra], %[t] offen  lds\n"
      "s_xor_b32 m0, %[ma], 2048\n"
      "s_xor_b32 %[t], %[sa], 0x20000\n"
      "buffer_load_dwordx4 %[a0], %[ra], %[t] offen  lds\n"
      "s_xor_b32 m0, %[ma], 3072\n"
      "s_xor_b32 %[t], %[sa], 0x30000\n"
      "buffer_load_dwordx4 %[a1], %[ra], %[t] offen  lds\n"
      "s_xor_b32 m0, %[mb], 0\n"
      "s_xor_b32 %[t], %[sb], 0x0\n"
      "buffer_load_dwordx4 %[b0], %[rb], %[t] offen  lds\n"
      "s_xor_b32 m0, %[mb], 1024\n"
      "s_xor_b32 %[t], %[sb], 0x10000\n"
      "buffer_load_dwordx4 %[b1], %[rb], %[t] offen  lds\n"
      "s_xor_b32 m0, %[mb], 2048\n"
      "s_xor_b32 %[t], %[sb], 0x20000\n"
      "buffer_load_dwordx4 %[b0], %[rb], %[t] offen  lds\n"
      "s_xor_b32 m0, %[mb], 3072\n"
      "s_xor_b32 %[t], %[sb], 0x30000\n"
      "buffer_load_dwordx4 %[b1], %[rb], %[t] offen  lds\n"
      "s_xor_b32 m0, %[mb], 4096\n"
      "s_xor_b32 %[t], %[sb], 0x40000\n"
      "buffer_load_dwordx4 %[b2], %[rb], %[t] offen  lds\n"
      "s_xor_b32 m0, %[mb], 5120\n"
      "s_xor_b32 %[t], %[sb], 0x50000\n"
      "buffer_load_dwordx4 %[b3], %[rb], %[t] offen  lds\n"
      "s_xor_b32 m0, %[mb], 6144\n"
      "s_xor_b32 %[t], %[sb], 0x60000\n"
      "buffer_load_dwordx4 %[b2], %[rb], %[t] offen  lds\n"
      "s_xor_b32 m0, %[mb], 7168\n"
      "s_xor_b32 %[t], %[sb], 0x70000\n"
      "buffer_load_dwordx4 %[b3], %[rb], %[t] offen  lds\n"
      "s_add_u32 %[kk], %[kk], 1\n"
      "s_xor_b32 %[t], %[kk], %[kx]\n"
      "s_sub_u32 %[t], %[t], %[kp]\n"
      "s_and_b32 %[t], %[t], 63\n"
      "s_mul_i32 %[t2], %[t], %[ssa]\n"
      "s_add_u32 %[sa], %[sa0], %[t2]\n"
      "s_mul_i32 %[t2], %[t], %[ssb]\n"
      "s_add_u32 %[sb], %[sb0], %[t2]\n"
      "s_add_u32 %[ma], %[ma], 16384\n"
      "s_add_u32 %[mb], %[mb], 32768\n"
      "s_cmp_eq_u32 %[ma], %[mae]\n"
      "s_cselect_b32 %[ma], %[ma0], %[ma]\n"
      "s_cselect_b32 %[mb], %[mb0], %[mb]\n"
      "s_waitcnt vmcnt(24)\n"
      "s_cmp_lg_u32 %[pw], 3\n"
      "s_cbranch_scc1 L_c_%=\n"
      "s_mov_b64 exec, 1\n"
      "v_add_u32 %[call], 1, %[call]\n"
      "ds_write_b32 %[cw], %[call]\n"
      "s_mov_b64 exec, -1\n"
      "s_waitcnt lgkmcnt(0)\n"
      "L_c_%=:\n"
      "s_barrier\n"
      "s_waitcnt vmcnt(12)\n"
      "s_barrier\n"
      "s_mov_b32 %[n], 3\n"
      "L_p_%=:\n"
      "s_xor_b32 m0, %[ma], 0\n"
      "s_xor_b32 %[t], %[sa], 0x0\n"
      "buffer_load_dwordx4 %[a0], %[ra], %[t] offen  lds\n"
      "s_xor_b32 m0, %[ma], 1024\n"
      "s_xor_b32 %[t], %[sa], 0x10000\n"
      "buffer_load_dwordx4 %[a1], %[ra], %[t] offen  lds\n"
      "s_xor_b32 m0, %[ma], 2048\n"
      "s_xor_b32 %[t], %[sa], 0x20000\n"
      "buffer_load_dwordx4 %[a0], %[ra], %[t] offen  lds\n"
      "s_xor_b32 m0, %[ma], 3072\n"
      "s_xor_b32 %[t], %[sa], 0x30000\n"
      "buffer_load_dwordx4 %[a1], %[ra], %[t] offen  lds\n"
      "s_xor_b32 m0, %[mb], 0\n"
      "s_xor_b32 %[t], %[sb], 0x0\n"
      "buffer_load_dwordx4 %[b0], %[rb], %[t] offen  lds\n"
      "s_xor_b32 m0, %[mb], 1024\n"
      "s_xor_b32 %[t], %[sb], 0x10000\n"
      "buffer_load_dwordx4 %[b1], %[rb], %[t] offen  lds\n"
      "s_xor_b32 m0, %[mb], 2048\n"
      "s_xor_b32 %[t], %[sb], 0x20000\n"
      "buffer_load_dwordx4 %[b0], %[rb], %[t] offen  lds\n"
      "s_xor_b32 m0, %[mb], 3072\n"
      "s_xor_b32 %[t], %[sb], 0x30000\n"
      "buffer_load_dwordx4 %[b1], %[rb], %[t] offen  lds\n"
      "s_xor_b32 m0, %[mb], 4096\n"
      "s_xor_b32 %[t], %[sb], 0x40000\n"
      "buffer_load_dwordx4 %[b2], %[rb], %[t] offen  lds\n"
      "s_xor_b32 m0, %[mb], 5120\n"
      "s_xor_b32 %[t], %[sb], 0x50000\n"
      "buffer_load_dwordx4 %[b3], %[rb], %[t] offen  lds\n"
      "s_xor_b32 m0, %[mb], 6144\n"
      "s_xor_b32 %[t], %[sb], 0x60000\n"
      "buffer_load_dwordx4 %[b2], %[rb], %[t] offen  lds\n"
      "s_xor_b32 m0, %[mb], 7168\n"
      "s_xor_b32 %[t], %[sb], 0x70000\n"
      "buffer_load_dwordx4 %[b3], %[rb], %[t] offen  lds\n"
      "s_add_u32 %[kk], %[kk], 1\n"
      "s_xor_b32 %[t], %[kk], %[kx]\n"
      "s_sub_u32 %[t], %[t], %[kp]\n"
      "s_and_b32 %[t], %[t], 63\n"
      "s_mul_i32 %[t2], %[t], %[ssa]\n"
      "s_add_u32 %[sa], %[sa0], %[t2]\n"
      "s_mul_i32 %[t2], %[t], %[ssb]\n"
      "s_add_u32 %[sb], %[sb0], %[t2]\n"
      "s_add_u32 %[ma], %[ma], 16384\n"
      "s_add_u32 %[mb], %[mb], 32768\n"
      "s_cmp_eq_u32 %[ma], %[mae]\n"
      "s_cselect_b32 %[ma], %[ma0], %[ma]\n"
      "s_cselect_b32 %[mb], %[mb0], %[mb]\n"
      "s_add_u32 %[n], %[n], 1\n"
      "s_waitcnt vmcnt(12)\n"
      "s_cmp_lt_u32 %[n], 64\n"
      "s_barrier\n"
      "s_cbranch_scc1 L_p_%=\n"
      "global_store_dwordx4 %[za], %[z], off offset:0 sc0 sc1\n"
      "global_store_dwordx4 %[za], %[z], off offset:1024 sc0 sc1\n"
      "global_store_dwordx4 %[za], %[z], off offset:2048 sc0 sc1\n"
      "global_store_dwordx4 %[za], %[z], off offset:3072 sc0 sc1\n"
      "ds_read_b32 %[call], %[cw]\n"
      "s_waitcnt vmcnt(4) lgkmcnt(0)\n"
      "s_barrier\n"
      "s_waitcnt vmcnt(0)\n"
      "s_barrier\n"
                : [t] "=&s"(t), [t2] "=&s"(t2), [kk] "+&s"(kk), [n] "=&s"(n), [sa] "+&s"(sa), [sb] "+&s"(sb), [ma] "+&s"(ma), [mb] "+&s"(mb), [call] "+&v"(call), [tv] "+&v"(tv), [tw] "+&v"(tw)
                : [a0] "v"(a0), [a1] "v"(a1), [b0] "v"(b0), [b1] "v"(b1), [b2] "v"(b2), [b3] "v"(b3), [ra] "s"(ra), [rb] "s"(rb),
                  [ssa] "s"(ssa), [ssb] "s"(ssb), [sa0] "s"(sa0), [sb0] "s"(sb0), [kx] "s"(kx), [kp] "s"(kp), [ma0] "s"(ma0), [mb0] "s"(mb0), [mae] "s"(mae), [pw] "s"(pw), [ca] "v"(myws + WS_CALL + b * 4),
                  [za] "v"(roff), [z] "v"(zero4), [cw] "v"(lds0 + F_MISC + 24)
                : "memory", "scc", "s40", "s41");
        }
        if (wid == 5) STAMP(1);
        call = rfl(call);
        const bool leader = !remote && idx == 0 && wid == 6;
        unsigned char* gm = myws;
        if (leader) gm = p.ws[lane >> 3] + WS_GMAX + (lane & 7) * 1024 + p.rank * 8;
        bool bad = false;
        if (wid == 7) {
            u32x4 wm;
            for (;;) {
                asm volatile("ds_read_b128 %0, %1\n\ts_waitcnt lgkmcnt(0)" : "=&v"(wm) : "v"(lds0 + F_MISC) : "memory");
                wm.x = rfl(max(max(wm.x, wm.y), max(wm.z, wm.w)));
                if (wm.x != NOTYET) break;
                __builtin_amdgcn_s_sleep(1);
            }
            if (lane == 0) st_sys64(myws + WS_LMAX + b * 8, ((u64)call << 32) | wm.x);
            STAMP(15);
        }
        if (leader) {
            const u64 t0 = __builtin_amdgcn_s_memrealtime();
            for (;;) {
                u32x4 u, v; u32 t;
                asm volatile("global_load_dwordx4 %0, %3, off sc0 sc1\n\tglobal_load_dwordx4 %1, %3, off offset:16 sc0 sc1\n\tglobal_load_dword %2, %4, off sc1\n\ts_waitcnt vmcnt(0)"
                             : "=&v"(u), "=&v"(v), "=&v"(t) : "v"(myws + WS_LMAX + lane * 32), "v"(myws + WS_TMO) : "memory");
                if (__all(u.y == call && u.w == call && v.y == call && v.w == call)) {
                    STAMP(14);
                    const u64 gv = ((u64)call << 32) | wave_umax(max(max(u.x, u.z), max(v.x, v.z)));
#pragma unroll
                    for (int r = 0; r < 4; ++r) st_sys64(gm + (call & 1) * 8192 + r * 256, gv);
                    break;
                }
                if (expired(t0, t, call, myws + WS_TMO, lane)) break;
            }
        }
        if (remote) {
            unsigned char* const rcls = myws + WS_CLS + (size_t)(((grp * 2 + (call & 1)) * 32 + idx) * 4 + pw) * CLS_SLOT + 1024 + lane * 16;
            const unsigned char* const rf = myws + WS_FLAG + ((grp * 32 + idx) * 4 + pw) * 32 + (call & 1) * 16;
            const unsigned char* const gq = myws + WS_GMAX + (call & 1) * 8192 + (b & 31) * 256 + (lane & 7) * 8;
            if (pw * 64 + lane < 243) {
                const u32 i = pw * 64 + lane, s = i / 81, r = i - s * 81, w0 = (r % 9) | (r / 9) << 8 | s << 16;
#pragma unroll
                for (int j = 0; j < 27; ++j) asm volatile("ds_write_b32 %0, %1 offset:%2" :: "v"(lds0 + F_DIG + i * 4), "v"(w0 + ((j % 3 * 3) << 16 | (j / 3) << 24)), "n"(j * 972) : "memory");
            }
            asm volatile("s_waitcnt lgkmcnt(0)" ::: "memory");
            lds_store(lds0 + F_MISC + 48 + pw * 4, 1);
            const unsigned char* const r3 = roff - lane * 4; const unsigned char* const r1 = roff + 768 - lane * 15;
            INJ(0)
            u64 t0 = __builtin_amdgcn_s_memrealtime();
            bool landed = false;
            u32 inv = NOTYET;
            const unsigned char* const c3 = rcls - lane * 4; const unsigned char* const c4 = rcls + 768 - lane * 12;
            const u32 c30 = call & 0x3fffffffu;
            u32x4 d, e[4], l1, l3; u32x3 l0, l2, e0, e2; u32 b0, b2, x0, x2; u32x2 f;
            e[1] = zero4; e[3] = zero4;
            for (;;) {
                u32 t, iw; u32x2 g;
                asm volatile("global_load_dwordx4 %0, %[a], off offset:-1024 sc0 sc1\n\tglobal_load_dwordx3 %1, %[c3], off sc0 sc1\n\tglobal_load_dwordx3 %2, %[c3], off offset:2048 sc0 sc1\n\t"
                             "global_load_dword %9, %[c4], off sc0 sc1\n\tglobal_load_dword %10, %[c4], off offset:2048 sc0 sc1\n\t"
                             "s_mov_b64 exec, 15\n\tglobal_load_dwordx4 %3, %[a], off offset:1024 sc0 sc1\n\tglobal_load_dwordx4 %8, %[a], off offset:3072 sc0 sc1\n\ts_mov_b64 exec, -1\n\t"
                             "global_load_dwordx2 %4, %[f], off sc0 sc1\n\tglobal_load_dwordx2 %5, %[g], off sc0 sc1\n\tglobal_load_dword %6, %[t], off sc1\n\ts_waitcnt vmcnt(0)\n\t"
                             "ds_read_b32 %7, %[i]\n\ts_waitcnt lgkmcnt(0)"
                             : "=&v"(d), "=&v"(e0), "=&v"(e2), "+&v"(e[1]), "=&v"(f), "=&v"(g), "=&v"(t), "=&v"(iw), "+&v"(e[3]), "=&v"(x0), "=&v"(x2)
                             : [a] "v"(rcls), [c3] "v"(c3), [c4] "v"(c4), [f] "v"(rf), [g] "v"(gq), [t] "v"(myws + WS_TMO), [i] "v"(lds0 + F_MISC + 16) : "memory");
                inv = scale_from(g, rfl(iw), call, lds0);
                if (landed || __all(min(min(d.x, d.y), min(d.z, d.w)) != 0)) break;
                landed = (rfl(f.y) & 0x3fffffffu) == c30 && (rfl(f.x) >> 31);
                if (!landed && expired(t0, t, call, myws + WS_TMO, lane)) { bad = true; break; }
            }
            if (wid == 4) STAMP(10);
            const u32 k[4] = {d.x ^ 0xaaaaaaaau, d.y ^ 0xaaaaaaaau, d.z ^ 0xaaaaaaaau, d.w ^ 0xaaaaaaaau};
            const u32 cnt = (__builtin_popcount(k[0]) + __builtin_popcount(k[1])) | (__builtin_popcount(k[2]) + __builtin_popcount(k[3])) << 16;
            const u32 inc = wave_scan(cnt), tot = __builtin_amdgcn_readlane(inc, 63), n[2] = {tot & 0xffff, tot >> 16};
            const u32 sbase = lds0 + 512 + pw * ST_PITCH;
            u32 ad[34], se[32];
#pragma unroll
            for (int h = 0; h < 2; ++h) {
                u32 o = sbase + (h ? RC_HALF + ((inc - cnt) >> 16) : (inc - cnt) & 0xffff) - 1;
                ad[h * 17] = o & ~3u;
#pragma unroll
                for (int i = h * 16; i < h * 16 + 16; ++i) {
                    const u32 nb = (k[i >> 3] >> (28 - (i & 7) * 4)) & 15;
                    se[i] = *(const LAS u32*)(uintptr_t)(lds0 + F_LUT + (nb + (o & 3) * 16) * 4);
                    o += __builtin_popcount(nb);
                    ad[i + 1 + h] = o & ~3u;
                }
            }
            const u32 rot = ((lane >> 1) & 7) ^ ((lane >> 4) & 3);
            u32 wa[8];
#pragma unroll
            for (int q = 0; q < 8; ++q) wa[q] = sbase + lane * 32 + ((q ^ rot) << 2);
            const u64 k0 = __ballot(lane * 32 < n[0]), k1 = __ballot(lane * 32 + 2048 < n[0]), k2 = __ballot(lane * 32 < n[1]), k3 = __ballot(lane * 32 + 2048 < n[1]);
            unsigned char* dst = rout + (size_t)(lane & 15) * NC + pw * 64 + (lane >> 4) * 16;
#pragma unroll
            for (int j = 0; j < 4; ++j) {
                const u32 out = lane * 32 + (j & 1) * 2048 >= n[j >> 1];
                if (j == 0) e0 |= out; else if (j == 2) e2 |= out; else e[j] |= out;
                if (j == 0) { l0 = u32x3{out, out, out}; b0 = out; } else if (j == 1) l1 = u32x4{out, out, out, out}; else if (j == 2) { l2 = u32x3{out, out, out}; b2 = out; } else l3 = u32x4{out, out, out, out};
            }
#pragma unroll
            for (int i = 0; i < 32; ++i) asm volatile("" : "+v"(se[i]), "+v"(ad[i]));
#pragma unroll
            for (int q = 0; q < 8; ++q) asm volatile("" : "+v"(wa[q]));
            asm volatile("" : "+v"(ad[32]), "+v"(ad[33]), "+v"(dst));
#define LOADE "s_mov_b64 exec, %[k0]\n\tglobal_load_dwordx3 %0, %[c3], off sc0 sc1\n\ts_mov_b64 exec, %[k1]\n\tglobal_load_dwordx4 %1, %[a], off offset:1024 sc0 sc1\n\t" \
                    "s_mov_b64 exec, %[k2]\n\tglobal_load_dwordx3 %2, %[c3], off offset:2048 sc0 sc1\n\ts_mov_b64 exec, %[k3]\n\tglobal_load_dwordx4 %3, %[a], off offset:3072 sc0 sc1\n\t"
#define LOADA(k) "s_mov_b64 exec, %[" k "0]\n\tglobal_load_dwordx3 %[l0], %[r3], off sc0 sc1\n\tglobal_load_ubyte %[b0], %[r1], off sc0 sc1\n\ts_mov_b64 exec, %[" k "1]\n\tglobal_load_dwordx4 %[l1], %[b], off offset:1024 sc0 sc1\n\t"
#define LOADB(k) "s_mov_b64 exec, %[" k "2]\n\tglobal_load_dwordx3 %[l2], %[r3], off offset:2048 sc0 sc1\n\tglobal_load_ubyte %[b2], %[r1], off offset:2048 sc0 sc1\n\ts_mov_b64 exec, %[" k "3]\n\tglobal_load_dwordx4 %[l3], %[b], off offset:3072 sc0 sc1\n\ts_mov_b64 exec, -1\n\t"
#define LA_IO [l0] "+&v"(l0), [b0] "+&v"(b0), [l1] "+&v"(l1)
#define LB_IO [l2] "+&v"(l2), [b2] "+&v"(b2), [l3] "+&v"(l3)
#define MINA min(min(min(l0.x, l0.y), min(l0.z, b0)), MIN4(l1))
#define MINB min(min(min(l2.x, l2.y), min(l2.z, b2)), MIN4(l3))
#define MIN4(v) min(min((v).x, (v).y), min((v).z, (v).w))
            auto bits_in = [&](u32 x, u32 all, u32 m) { const u32 h = rfl(x) & 0xffff; return x | (all ? lane * 32 >= m : h && lane * 2 > (h & 0x7fff)); };
#define EARLY_IN __all(min(min(min(min(e0.x, e0.y), min(e0.z, e2.x)), min(min(e2.y, e2.z), min(MIN4(e[1]), MIN4(e[3])))), min(bits_in(x0, rfl(f.y) >> 30 & 1, n[0]), bits_in(x2, rfl(f.y) >> 31, n[1]))) != 0)
            bool have = (rfl(f.y) & 0x3fffffffu) == c30 && EARLY_IN;
            while (!bad && !have) {
                u32 t, iw; u32x2 g;
                const u64 on = inv == NOTYET ? 0 : ~0ull;
                asm volatile(LOADE LOADA("q") LOADB("q") "global_load_dword %8, %[c4], off sc0 sc1\n\tglobal_load_dword %9, %[c4], off offset:2048 sc0 sc1\n\t"
                             "global_load_dwordx2 %4, %[f], off sc0 sc1\n\tglobal_load_dwordx2 %5, %[g], off sc0 sc1\n\tglobal_load_dword %6, %[t], off sc1\n\ts_waitcnt vmcnt(0)\n\t"
                             "ds_read_b32 %7, %[i]\n\ts_waitcnt lgkmcnt(0)"
                             : "+&v"(e0), "+&v"(e[1]), "+&v"(e2), "+&v"(e[3]), "=&v"(f), "=&v"(g), "=&v"(t), "=&v"(iw), "=&v"(x0), "=&v"(x2), LA_IO, LB_IO
                             : [a] "v"(rcls), [c3] "v"(c3), [c4] "v"(c4), [b] "v"(roff), [r3] "v"(r3), [r1] "v"(r1), [f] "v"(rf), [g] "v"(gq), [t] "v"(myws + WS_TMO), [i] "v"(lds0 + F_MISC + 16),
                               [k0] "s"(k0), [k1] "s"(k1), [k2] "s"(k2), [k3] "s"(k3), [q0] "s"(k0 & on), [q1] "s"(k1 & on), [q2] "s"(k2 & on), [q3] "s"(k3 & on) : "memory");
                inv = scale_from(g, rfl(iw), call, lds0);
                const bool ok = (rfl(f.y) & 0x3fffffffu) == c30;
                have = ok && (landed || EARLY_IN);
                if (have) break;
                landed = ok && (rfl(f.x) >> 31);
                if (!landed) bad = expired(t0, t, call, myws + WS_TMO, lane);
            }
            asm volatile("global_store_dwordx4 %[a], %[z], off offset:-1024 sc0 sc1\n\t"
                         "global_store_dwordx4 %[a], %[z], off sc0 sc1\n\tglobal_store_dwordx4 %[a], %[z], off offset:2048 sc0 sc1\n\ts_mov_b64 exec, %[k1]\n\tglobal_store_dwordx4 %[a], %[z], off offset:1024 sc0 sc1\n\t"
                         "s_mov_b64 exec, %[k3]\n\tglobal_store_dwordx4 %[a], %[z], off offset:3072 sc0 sc1\n\ts_mov_b64 exec, -1\n\ts_nop 1"
                         :: [a] "v"(rcls), [z] "v"(zero4), [k1] "s"(k1), [k3] "s"(k3) : "memory");
#pragma unroll
            for (int h = 0; h < 2; ++h) {
                const u32 x = h ? x2 : x0; const u32x3 w = h ? e2 : e0;
                u32 g = x ^ 0x55555555u;
                if (!(rfl(f.y) >> (30 + h) & 1)) {
                    const u32 m = rfl(x) & 0x7fff, pl = lds0 + F_BIT + (pw * 2 + h) * 256;
                    asm volatile("ds_write_b32 %1, %2\n\tds_or_b32 %3, %4\n\tds_or_b32 %5, %6\n\tds_read_b32 %0, %1\n\ts_waitcnt lgkmcnt(0)"
                                 : "=&v"(g) : "v"(pl + lane * 4), "v"(0u), "v"(pl + (x >> 3 & 0xfc)), "v"(lane && lane * 2 <= m ? 1u << (x & 31) : 0u),
                                   "v"(pl + (x >> 19 & 0xfc)), "v"(lane * 2 < m ? 1u << (x >> 16 & 31) : 0u) : "memory");
                }
                g ^= 0x0f0f0f0fu;
                e[2 * h] = u32x4{(w.x & 0x77777777u) | (g << 3 & 0x88888888u), (w.y & 0x77777777u) | (g << 2 & 0x88888888u), (w.z & 0x77777777u) | (g << 1 & 0x88888888u),
                                 (w.x >> 3 & 0x11111111u) | (w.y >> 2 & 0x22222222u) | (w.z >> 1 & 0x44444444u) | (g & 0x88888888u)};
            }
            if (wid == 4) STAMP(8);
            while (!bad && inv == NOTYET) {
                u32 t, iw; u32x2 g;
                asm volatile("global_load_dwordx2 %0, %3, off sc0 sc1\n\tglobal_load_dword %1, %4, off sc1\n\ts_waitcnt vmcnt(0)\n\tds_read_b32 %2, %5\n\ts_waitcnt lgkmcnt(0)"
                             : "=&v"(g), "=&v"(t), "=&v"(iw) : "v"(gq), "v"(myws + WS_TMO), "v"(lds0 + F_MISC + 16) : "memory");
                inv = scale_from(g, rfl(iw), call, lds0);
                if (inv == NOTYET) bad = expired(t0, t, call, myws + WS_TMO, lane);
            }
            bad |= inv == 0x7fc00000u;
            typedef unsigned short u16x2 __attribute__((ext_vector_type(2)));
            const u32 sh = class_shift(__uint_as_float(inv), f.x & 0x7fffffffu);
            u32 fc[32];
#pragma unroll
            for (int j = 0; j < 4; ++j) {
                if (!(j & 1) || n[j >> 1] > 2048) {
#pragma unroll
                    for (int i = j * 8; i < j * 8 + 8; ++i) {
                        const u32 x = (i & 1) ? ~e[i >> 3][(i >> 1) & 3] >> 1 : e[i >> 3][(i >> 1) & 3] << 3;
                        const u16x2 lo = __builtin_elementwise_sub_sat(__builtin_bit_cast(u16x2, x & 0x00780078u), __builtin_bit_cast(u16x2, sh));
                        const u16x2 hi = __builtin_elementwise_sub_sat(__builtin_bit_cast(u16x2, x & 0x78007800u), __builtin_bit_cast(u16x2, sh << 8));
                        fc[i] = __builtin_bit_cast(u32, lo) + __builtin_bit_cast(u32, hi);
                        asm volatile("" : "+v"(fc[i]));
                    }
                } else {
#pragma unroll
                    for (int i = j * 8; i < j * 8 + 8; ++i) asm volatile("" : "=v"(fc[i]));
                }
            }
            while (lds_word(lds0 + F_MISC + 32 + pw * 4) == 0 || lds_min4(lds0 + F_MISC + 48) == 0) __builtin_amdgcn_s_sleep(1);
            if (wid == 4) STAMP(12);
            INJ(1)
            t0 = __builtin_amdgcn_s_memrealtime();
            while (!bad && !__all(MINA != 0)) {
                u32 t;
                asm volatile(LOADA("k") LOADB("k") "global_load_dword %0, %[t], off sc1\n\ts_waitcnt vmcnt(0)"
                             : "=&v"(t), LA_IO, LB_IO
                             : [b] "v"(roff), [r3] "v"(r3), [r1] "v"(r1), [t] "v"(myws + WS_TMO), [k0] "s"(k0), [k1] "s"(k1), [k2] "s"(k2), [k3] "s"(k3) : "memory");
                if (__all(MINA != 0)) break;
                bad = expired(t0, t, call, myws + WS_TMO, lane);
            }
            if (wid == 4) STAMP(16);
            const bool odd = (lane >> 4) & 1;
#pragma unroll
            for (int h = 0; h < 2; ++h) {
                if (h) {
                    while (!bad && !__all(MINB != 0)) {
                        u32 t;
                        asm volatile(LOADB("k") "global_load_dword %0, %[t], off sc1\n\ts_waitcnt vmcnt(0)"
                                     : "=&v"(t), LB_IO : [b] "v"(roff), [r3] "v"(r3), [r1] "v"(r1), [t] "v"(myws + WS_TMO), [k2] "s"(k2), [k3] "s"(k3) : "memory");
                        if (__all(MINB != 0)) break;
                        bad = expired(t0, t, call, myws + WS_TMO, lane);
                    }
                    if (wid == 4) STAMP(9);
                }
                if (bad) {
#pragma unroll 1
                    for (int m = 0; m < 8; ++m) *(GAS u32x4*)(dst + (size_t)m * 16 * NC) = u32x4{0x7f7f7f7fu, 0x7f7f7f7fu, 0x7f7f7f7fu, 0x7f7f7f7fu};
                    return;
                }
#pragma unroll
                for (int j = 2 * h; j < 2 * h + 2; ++j) {
                    if (!(j & 1) || n[h] > 2048) {
                        const u32x4 z = j == 0 ? u32x4{l0.x, l0.y, l0.z, b0} : j == 1 ? l1 : j == 2 ? u32x4{l2.x, l2.y, l2.z, b2} : l3;
                        const u32 pk[4] = {z.x, z.y, z.z, (z.x >> 13 & 0x00070007u) | (z.y >> 10 & 0x00380038u) | (z.z >> 7 & 0x01c001c0u) | (z.w & 15) << 9 | ((z.w - 16) & 0xf0) << 21};
                        u32 cd[8];
#pragma unroll
                        for (int q = 0; q < 8; ++q) cd[q] = lds0 + F_DIG - 4 + (pk[q >> 1] >> (q & 1) * 16 & 0x1fff) * 4;
                        asm volatile("ds_read_b32 %0, %0\n\tds_read_b32 %1, %1\n\tds_read_b32 %2, %2\n\tds_read_b32 %3, %3\n\tds_read_b32 %4, %4\n\tds_read_b32 %5, %5\n\tds_read_b32 %6, %6\n\tds_read_b32 %7, %7\n\ts_waitcnt lgkmcnt(0)"
                                     : "+v"(cd[0]), "+v"(cd[1]), "+v"(cd[2]), "+v"(cd[3]), "+v"(cd[4]), "+v"(cd[5]), "+v"(cd[6]), "+v"(cd[7]) :: "memory");
#pragma unroll
                        for (int q = 0; q < 8; ++q) asm volatile("ds_write_b32 %0, %1 offset:%2" :: "v"(wa[q]), "v"(cd[q] + fc[j * 8 + q]), "n"(h * RC_HALF + (j & 1) * 2048) : "memory");
                    }
                }
                u32 v[17];
#pragma unroll
                for (int i = 0; i < 17; ++i) asm volatile("ds_read_b32 %0, %1" : "=&v"(v[i]) : "v"(ad[h * 17 + i]) : "memory");
#pragma unroll
                for (int m = 0; m < 4; ++m) {
                    const u32* const s = se + h * 16 + 4 * m;
                    asm volatile("s_waitcnt lgkmcnt(%5)" : "+v"(v[4 * m]), "+v"(v[4 * m + 1]), "+v"(v[4 * m + 2]), "+v"(v[4 * m + 3]), "+v"(v[4 * m + 4]) : "n"(12 - 4 * m) : "memory");
                    const u32 w0 = __builtin_amdgcn_perm(v[4 * m + 1], v[4 * m], s[0]), w1 = __builtin_amdgcn_perm(v[4 * m + 2], v[4 * m + 1], s[1]);
                    const u32 w2 = __builtin_amdgcn_perm(v[4 * m + 3], v[4 * m + 2], s[2]), w3 = __builtin_amdgcn_perm(v[4 * m + 4], v[4 * m + 3], s[3]);
                    u32x4 w; w.x = odd ? w1 : w0; w.y = odd ? w0 : w1; w.z = odd ? w3 : w2; w.w = odd ? w2 : w3;
                    asm volatile("global_store_dwordx4 %0, %1, off " OUTMOD "\n\ts_nop 1" :: "v"(dst + (size_t)(h * 4 + m) * 16 * NC), "v"(w) : "memory");
                }
            }
            if (wid == 4) STAMP(7);
        }
    }
    FLUSH();
}

extern "C" void kernel_launch(int rank, int nranks, ncclComm_t comm, hipStream_t stream,
                              void* const* d_in, const int* in_sizes, int n_in,
                              void* d_out, int out_size, const int* in_dtypes, int out_dtype,
                              void* const* d_out_all, void* const* d_ws_all, size_t ws_bytes) {
    if (nranks != NR || n_in < 2 || in_sizes[0] != ML * KD || in_sizes[1] != KD * ND || out_size != NR * ML * NC || ws_bytes < WS_NEED) {
        fprintf(stderr, "kernel_launch: unexpected shapes\n"); return;
    }
    static bool attr[NR] = {};
    if (!attr[rank]) { HIP_CHECK(hipFuncSetAttribute((const void*)fused_kernel, hipFuncAttributeMaxDynamicSharedMemorySize, LDS_BYTES)); attr[rank] = true; }
    Params p;
    p.x = d_in[0]; p.w = d_in[1]; p.rank = rank; p.pad = 0;
    for (int i = 0; i < NR; ++i) { p.out[i] = (unsigned char*)d_out_all[i]; p.ws[i] = (unsigned char*)d_ws_all[i]; }
    hipLaunchKernelGGL(fused_kernel, dim3(NWG), dim3(512), LDS_BYTES, stream, p);
    HIP_CHECK(hipGetLastError());
}
```
